```python
import numpy as np

kernel_name = "gpu_gemm_ar_m4096_k4096_n4096_bf16_silu"


ABSMAX_RTOL = 0.008

COMPUTE_DTYPE = "bfloat16"
EPILOGUE = "silu"
IN_QUANT = None
OUT_QUANT = None
IN_DTYPES = {"x": "bfloat16", "w_mat": "bfloat16"}
WORLD_SIZE = 8

def _to_bf16(x):
    u = np.ascontiguousarray(x, dtype=np.float32).view(np.uint32)
    return ((u + 0x7FFF + ((u >> 16) & 1)) & 0xFFFF0000).view(np.float32)

def setup_inputs():

    x = _to_bf16(np.random.randn(4096, 4096).astype(np.float32))
    w_mat = _to_bf16((np.random.randn(4096, 4096) / np.sqrt(4096)).astype(np.float32))
    return {"x": x, "w_mat": w_mat}

def _epilogue(y):
    return (y / (1.0 + np.exp(-np.clip(y, -60.0, 60.0)))).astype(np.float32)

def reference(x, w_mat):
    y = (x @ w_mat).astype(np.float32)
    return _epilogue(y)
```

```cpp
#include <hip/hip_runtime.h>
#include <rccl/rccl.h>
#include <cstdio>

#define HIP_CHECK(expr) do { hipError_t _e = (expr); if (_e != hipSuccess) { \
    fprintf(stderr, "HIP error %s at %s:%d\n", hipGetErrorString(_e), __FILE__, __LINE__); \
    return; } } while (0)

namespace {

typedef unsigned int u32;
typedef unsigned long long u64;
typedef u32 u32x4 __attribute__((ext_vector_type(4)));
typedef u32 u32x2 __attribute__((ext_vector_type(2)));
typedef __bf16 bf16x2 __attribute__((ext_vector_type(2)));
typedef __bf16 bf16x4 __attribute__((ext_vector_type(4)));
typedef __bf16 bf16x8 __attribute__((ext_vector_type(8)));
typedef float f32x16 __attribute__((ext_vector_type(16)));
#define LAS __attribute__((address_space(3)))
#define GAS __attribute__((address_space(1)))

constexpr int NR = 8;
constexpr u32 X_BYTES = 4096u * 512u * 2u;
constexpr u32 W_BYTES = 512u * 4096u * 2u;
constexpr u32 OUT_BYTES = 4096u * 4096u * 2u;

constexpr u32 BLK = 16384;
constexpr u32 XW_OFF = 0;
constexpr u32 XW_SRC = 8 * 16 * BLK, XW_KT = 16 * BLK;
constexpr u32 WW_OFF = XW_OFF + NR * XW_SRC;
constexpr u32 WW_SRC = 8 * 8 * BLK, WW_KT = 8 * BLK;
constexpr u32 DATA_BYTES = WW_OFF + NR * WW_SRC;
constexpr u32 FX_OFF = DATA_BYTES;
constexpr u32 FW_OFF = FX_OFF + NR * 8 * 16 * 2 * 4;
constexpr u32 FT_OFF = FW_OFF + NR * 8 * 8 * 2 * 4;
constexpr u32 CTR_OFF = FT_OFF + NR * 128 * 4;
constexpr u32 TMO_OFF = CTR_OFF + 256 * 4;
constexpr u32 TS_OFF = TMO_OFF + 1024;
constexpr u32 WS_USED = TS_OFF + 256 * 16 * 8;

constexpr u64 DEADLINE_TICKS = 30000000ull;

constexpr int PUSH_WGS = 8, GEMM_WGS = 16, PULL_WGS = 4;
constexpr int GRID = 8 * (PUSH_WGS + GEMM_WGS + PULL_WGS);
constexpr int THREADS = 512;
constexpr int NSTAGE = 4;
constexpr u32 STAGE = 2 * BLK;
constexpr u32 LDS_BYTES = NSTAGE * STAGE + 64;
constexpr int AUX_SYS = 17;

struct Params {
  const char* x;
  const char* w;
  char* ws[NR];
  char* out[NR];
  int rank;
};

__device__ __forceinline__ __amdgpu_buffer_rsrc_t mk_rsrc(const void* p, u32 n) {
  const u64 a = (u64)p;
  const u64 u = ((u64)(u32)__builtin_amdgcn_readfirstlane((u32)(a >> 32)) << 32) | (u32)__builtin_amdgcn_readfirstlane((u32)a);
  return __builtin_amdgcn_make_buffer_rsrc((void*)u, 0, n, 0x00020000);
}
__device__ __forceinline__ u32 ld_sys(const char* p) {
  return __hip_atomic_load((const GAS u32*)p, __ATOMIC_RELAXED, __HIP_MEMORY_SCOPE_SYSTEM);
}
__device__ __forceinline__ void st_sys(char* p, u32 v) {
  __hip_atomic_store((GAS u32*)p, v, __ATOMIC_RELAXED, __HIP_MEMORY_SCOPE_SYSTEM);
}
#define WAIT_VM(n) asm volatile("s_waitcnt vmcnt(%0)" ::"n"(n) : "memory")

__device__ __forceinline__ bool wave_wait(const char* flag, bool act, int need, u32 seq, char* tmo) {
  const u64 t0 = __builtin_amdgcn_s_memrealtime();
  for (;;) {
    const bool ok = act && ld_sys(flag) >= seq;
    if (__popcll(__ballot(ok)) >= need) return false;
    const u32 t = __hip_atomic_load((const GAS u32*)tmo, __ATOMIC_RELAXED, __HIP_MEMORY_SCOPE_AGENT);
    if (__builtin_amdgcn_readfirstlane(t) == seq || __builtin_amdgcn_s_memrealtime() - t0 > DEADLINE_TICKS) {
      __hip_atomic_store((GAS u32*)tmo, seq, __ATOMIC_RELAXED, __HIP_MEMORY_SCOPE_AGENT);
      return true;
    }
    __builtin_amdgcn_s_sleep(2);
  }
}

#define STAMP(i) do {} while (0)

__device__ __forceinline__ void dma4(u32 dst, __amdgpu_buffer_rsrc_t ra, u32 va0, u32 va1, u32 sa,
                                     __amdgpu_buffer_rsrc_t rb, u32 vb0, u32 vb1, u32 sb) {
  u32 keep;
  dst = __builtin_amdgcn_readfirstlane(dst);
  sa = __builtin_amdgcn_readfirstlane(sa);
  sb = __builtin_amdgcn_readfirstlane(sb);
  asm volatile(
      "s_mov_b32 %0, m0\n\t"
      "s_nop 4\n\t"
      "s_mov_b32 m0, %1\n\t"
      "s_nop 0\n\t"
      "buffer_load_dwordx4 %3, %5, %6 offen lds\n\t"
      "buffer_load_dwordx4 %4, %5, %6 offen offset:1024 lds\n\t"
      "s_mov_b32 m0, %2\n\t"
      "s_nop 0\n\t"
      "buffer_load_dwordx4 %7, %9, %10 offen lds\n\t"
      "buffer_load_dwordx4 %8, %9, %10 offen offset:1024 lds\n\t"
      "s_mov_b32 m0, %0"
      : "=&s"(keep)
      : "s"(dst), "s"(dst + BLK), "v"(va0), "v"(va1), "s"(ra), "s"(sa), "v"(vb0), "v"(vb1), "s"(rb), "s"(sb)
      : "memory");
}

template <int I, int N, class F>
__device__ __forceinline__ void static_for(F&& f) {
  if constexpr (I < N) { f(std::integral_constant<int, I>{}); static_for<I + 1, N>(f); }
}

__device__ __forceinline__ void push_role(const Params& p, u32 peer, int cw, int lane, u32 seqv) {
  constexpr int D = 4, NITEM = 48;
  char* wsp = p.ws[peer];
  const auto rx = mk_rsrc(p.x, X_BYTES);
  const auto rw = mk_rsrc(p.w, W_BYTES);
  const auto rd = mk_rsrc(wsp, DATA_BYTES);
  const u32 rank = p.rank;
  const u32 half = cw & 1, blk = cw >> 1;
  const bool isw = blk >= 16;
  const u32 vx = (lane >> 3) * 1024 + (lane & 7) * 16;
  const u32 vw = (lane >> 4) * 8192 + (lane & 15) * 16;
  const u32 vd = lane * 16;
  const u32 prow = (peer >> 2) * 2048, pcol = (peer & 3) * 1024;
  const u32 idxA = (blk & 15) >> 1, ktA = blk & 1, ktB = blk & 3, rbB = 8 + (blk >> 2);
  const u32 sA0 = isw ? (ktA * 64 + half * 32) * 8192 + (pcol + idxA * 128) * 2
                      : (prow + idxA * 128 + half * 64) * 1024 + ktA * 128;
  const u32 sAstep = isw ? 128 * 8192 : 256, sAitem = isw ? 4 * 8192 : 8 * 1024;
  const u32 dA0 = (isw ? WW_OFF + rank * WW_SRC + ktA * WW_KT : XW_OFF + rank * XW_SRC + ktA * XW_KT) + idxA * BLK + half * 8192;
  const u32 dAstep = isw ? 2 * WW_KT : 2 * XW_KT;
  const u32 fA0 = isw ? FW_OFF + (((rank * 8 + ktA) * 8 + idxA) * 2 + half) * 4
                      : FX_OFF + (((rank * 8 + ktA) * 16 + idxA) * 2 + half) * 4;
  const u32 fAstep = isw ? 2 * 8 * 2 * 4 : 2 * 16 * 2 * 4;
  const u32 sB0 = (prow + rbB * 128 + half * 64) * 1024 + ktB * 128;
  const u32 dB0 = XW_OFF + rank * XW_SRC + ktB * XW_KT + rbB * BLK + half * 8192;
  const u32 fB0 = FX_OFF + (((rank * 8 + ktB) * 16 + rbB) * 2 + half) * 4;
  const auto rA = isw ? rw : rx;
  const u32 vA = isw ? vw : vx;

  auto load = [&](auto ic) -> u32x4 {
    constexpr int i = decltype(ic)::value, s = i / 8, j = i % 8;
    if constexpr (s < 4) return __builtin_amdgcn_raw_buffer_load_b128(rA, vA, sA0 + s * sAstep + j * sAitem, 0);
    else return __builtin_amdgcn_raw_buffer_load_b128(rx, vx, sB0 + (s - 4) * 512 + j * 8192, 0);
  };
  auto flag = [&](auto sc) {
    constexpr int s = decltype(sc)::value;
    const u32 off = s < 4 ? fA0 + s * fAstep : fB0 + (s - 4) * (4 * 16 * 2 * 4);
    if (lane == 0) st_sys(wsp + off, seqv + 1);
  };
  u32x4 v[D];
  static_for<0, D>([&](auto ic) { v[decltype(ic)::value] = load(ic); });
  static_for<0, NITEM>([&](auto ic) {
    constexpr int i = decltype(ic)::value, s = i / 8, j = i % 8;
    if constexpr (i >= 8 && j == D - 1) {
      WAIT_VM(2 * D - 1);
      flag(std::integral_constant<int, s - 1>{});
    }
    const u32 doff = s < 4 ? dA0 + s * dAstep + j * 1024 : dB0 + (s - 4) * 4 * XW_KT + j * 1024;
    __builtin_amdgcn_raw_buffer_store_b128(v[i % D], rd, vd, doff, AUX_SYS);
    if constexpr (i + D < NITEM) v[i % D] = load(std::integral_constant<int, i + D>{});
  });
  WAIT_VM(0);
  flag(std::integral_constant<int, 5>{});
}

__device__ __forceinline__ void pull_role(const Params& p, u32 peer, int pw, int lane, u32 seq) {
  constexpr int D = 4, NPW = PULL_WGS * 8, TPR = NPW / 4, ROUNDS = 128 / TPR;
  char* ws = p.ws[p.rank];
  char* tmo = ws + TMO_OFF;
  const auto rs = mk_rsrc(p.out[peer], OUT_BYTES);
  const auto rd = mk_rsrc(p.out[p.rank], OUT_BYTES);
  const u32 vo = (lane >> 4) * 8192 + (lane & 15) * 16;
  const u32 base = ((peer >> 2) * 2048 + (pw & 3) * 32) * 8192 + (peer & 3) * 2048;
  const char* ft = ws + FT_OFF + peer * 128 * 4;
  auto toff = [&](int r) -> u32 {
    const u32 tile = r * TPR + (pw >> 2);
    return base + (tile >> 3) * 128 * 8192 + (tile & 7) * 256;
  };
  u32 bad = 0;
  {
    const u32 blk = lane >> 1;
    const char* f = ws + FX_OFF + (((peer * 8 + 4 + (blk & 3)) * 16 + 8 + (blk >> 2)) * 2 + (lane & 1)) * 4;
    if (wave_wait(f, true, 1, seq, tmo)) bad = 0x7FC07FC0u;
    if (wave_wait(ft + (pw >> 2) * 4, true, 64, seq, tmo)) bad = 0x7FC07FC0u;
  }
  STAMP(1);
  u32x4 v[D];
  u32 cur = toff(0);
#pragma unroll
  for (int j = 0; j < D; ++j) v[j] = __builtin_amdgcn_raw_buffer_load_b128(rs, vo, cur + j * 4 * 8192, AUX_SYS);
  for (int r = 0; r + 1 < ROUNDS; ++r) {
    const u32 nxt = toff(r + 1);
    const char* fn = ft + ((r + 1) * TPR + (pw >> 2)) * 4;
    u32 fv = 0, badn = bad;
#pragma unroll
    for (int j = 0; j < 8; ++j) {
      if (j == 8 - D) {
        if (__builtin_amdgcn_readfirstlane(fv) < seq)
          if (wave_wait(fn, true, 64, seq, tmo)) badn = 0x7FC07FC0u;
      }
      __builtin_amdgcn_raw_buffer_store_b128(v[j % D] | bad, rd, vo, cur + j * 4 * 8192, AUX_SYS);
      if (j + D < 8) v[j % D] = __builtin_amdgcn_raw_buffer_load_b128(rs, vo, cur + (j + D) * 4 * 8192, AUX_SYS);
      else v[j % D] = __builtin_amdgcn_raw_buffer_load_b128(rs, vo, nxt + (j + D - 8) * 4 * 8192, AUX_SYS);
      if (j == 0) fv = ld_sys(fn);
    }
    cur = nxt;
    bad = badn;
  }
#pragma unroll
  for (int j = 0; j < 8; ++j) {
    __builtin_amdgcn_raw_buffer_store_b128(v[j % D] | bad, rd, vo, cur + j * 4 * 8192, AUX_SYS);
    if (j + D < 8) v[j % D] = __builtin_amdgcn_raw_buffer_load_b128(rs, vo, cur + (j + D) * 4 * 8192, AUX_SYS);
  }
}

__device__ __forceinline__ void gemm_role(const Params& p, u32 rb, u32 cb, u32 seq, LAS char* lds) {
  const int tid = threadIdx.x, lane = tid & 63;
  const int wid = __builtin_amdgcn_readfirstlane(tid >> 6);
  const u32 rank = p.rank;
  char* ws = p.ws[rank];
  char* tmo = ws + TMO_OFF;
  const auto rx = mk_rsrc(p.x, X_BYTES);
  const auto rw = mk_rsrc(p.w, W_BYTES);
  const auto rs = mk_rsrc(ws, DATA_BYTES);
  const u32 row0 = (rank >> 2) * 2048 + rb * 128, col0 = (rank & 3) * 1024 + cb * 128;

  u32 vao[2], vbo[2], vaw[2], vbw[2];
#pragma unroll
  for (int q = 0; q < 2; ++q) {
    const u32 ra = wid * 16 + q * 8 + (lane >> 3), ca = (lane & 7) ^ ((ra >> 1) & 7);
    const u32 kb = wid * 8 + q * 4 + (lane >> 4), cbk = (lane & 15) ^ (((kb & 3) << 2) | ((kb >> 2) & 3));
    vao[q] = ra * 1024 + ca * 16 - q * 1024;  vaw[q] = ra * 128 + ca * 16 - q * 1024;
    vbo[q] = kb * 8192 + cbk * 16 - q * 1024; vbw[q] = kb * 256 + cbk * 16 - q * 1024;
  }
  const u32 lds0 = (u32)(size_t)lds + wid * 2048;
  auto issue = [&](int t) {
    const u32 dst = lds0 + (t & (NSTAGE - 1)) * STAGE;
    if (t < 8) {
      dma4(dst, rx, vao[0], vao[1], row0 * 1024 + t * 128, rw, vbo[0], vbo[1], t * 64 * 8192 + col0 * 2);
    } else {
      const u32 u = t - 8, g = u / 14, v = u - g * 14;
      const u32 src = rank ^ ((v >> 1) + 1), kt = 2 * g + (v & 1);
      dma4(dst, rs, vaw[0], vaw[1], XW_OFF + src * XW_SRC + kt * XW_KT + rb * BLK,
           rs, vbw[0], vbw[1], WW_OFF + src * WW_SRC + kt * WW_KT + cb * BLK);
    }
  };
  const u32 wm = wid & 1, wn = wid >> 1, h = lane >> 5;
  u32 xo[4], wo[2];
  {
    const u32 r = wm * 64 + (lane & 31);
#pragma unroll
    for (int ks = 0; ks < 4; ++ks) xo[ks] = r * 128 + (((2 * ks + h) ^ ((r >> 1) & 7)) * 16);
    const u32 bk = (lane >> 4) & 1, q = (lane & 15) >> 2, pp = lane & 3;
#pragma unroll
    for (int t = 0; t < 2; ++t)
      wo[t] = BLK + 256 * (8 * h + 4 * t + q) + 16 * ((4 * wn + 2 * bk + (pp >> 1)) ^ ((q << 2) | (2 * h + t))) + 8 * (pp & 1);
  }
  const bool fact = lane < 56;
  const u32 fsrc = rank ^ (((lane >> 3) + 1) & 7), fkt = (lane >> 1) & 1;
  const bool fisw = (lane >> 2) & 1;
  const u32 foff = fisw ? FW_OFF + (((fsrc * 8 + fkt) * 8 + cb) * 2 + (lane & 1)) * 4
                        : FX_OFF + (((fsrc * 8 + fkt) * 16 + rb) * 2 + (lane & 1)) * 4;
  const u32 fstep = fisw ? 2 * 8 * 2 * 4 : 2 * 16 * 2 * 4;
  LAS u32* lbad = (LAS u32*)(lds + NSTAGE * STAGE);
  if (tid == 0) *lbad = 0;

  f32x16 acc[2] = {};
  issue(0); issue(1); issue(2);
  int next_wait = 8, grp = 0;
  for (int t = 0; t < 64; ++t) {
    const int tn = t + NSTAGE - 1;
    if (tn == next_wait && tn < 64) {
      if (wid == 0) {
        if (wave_wait(ws + foff + grp * fstep, fact, 56, seq, tmo)) *lbad = 1;
        __builtin_amdgcn_fence(__ATOMIC_ACQUIRE, "");
        WAIT_VM(0);
        STAMP(1 + grp);
      }
      next_wait += 14; ++grp;
    }
    if (t < 62) WAIT_VM(8); else if (t == 62) WAIT_VM(4); else WAIT_VM(0);
    asm volatile("s_waitcnt lgkmcnt(0)" ::: "memory");
    __builtin_amdgcn_s_barrier();
    if (tn < 64) issue(tn);
    const LAS char* st = lds + (t & (NSTAGE - 1)) * STAGE;
#pragma unroll
    for (int ks = 0; ks < 4; ++ks) {
      const bf16x4 w0 = __builtin_amdgcn_ds_read_tr16_b64_v4bf16((LAS bf16x4*)(st + wo[0] + ks * 4096));
      const bf16x4 w1 = __builtin_amdgcn_ds_read_tr16_b64_v4bf16((LAS bf16x4*)(st + wo[1] + ks * 4096));
      const bf16x8 wf = __builtin_shufflevector(w0, w1, 0, 1, 2, 3, 4, 5, 6, 7);
      const bf16x8 x0 = *(const LAS bf16x8*)(st + xo[ks]);
      const bf16x8 x1 = *(const LAS bf16x8*)(st + xo[ks] + 4096);
      acc[0] = __builtin_amdgcn_mfma_f32_32x32x16_bf16(wf, x0, acc[0], 0, 0, 0);
      acc[1] = __builtin_amdgcn_mfma_f32_32x32x16_bf16(wf, x1, acc[1], 0, 0, 0);
    }
  }
  STAMP(5);
  __syncthreads();
  constexpr u32 EROW = 272;
#pragma unroll
  for (int mi = 0; mi < 2; ++mi) {
    const u32 m = wm * 64 + mi * 32 + (lane & 31);
#pragma unroll
    for (int g = 0; g < 4; ++g) {
      float y[4];
#pragma unroll
      for (int e = 0; e < 4; ++e) {
        const float a = acc[mi][4 * g + e];
        const float c = fminf(fmaxf(a, -60.f), 60.f);
        y[e] = a * __builtin_amdgcn_rcpf(1.f + __expf(-c));
      }
      bf16x4 o = {(__bf16)y[0], (__bf16)y[1], (__bf16)y[2], (__bf16)y[3]};
      *(LAS bf16x4*)(lds + m * EROW + (wn * 32 + 8 * g + 4 * h) * 2) = o;
    }
  }
  __syncthreads();
  const u32 bad = *lbad ? 0x7FC07FC0u : 0u;
  const auto ro = mk_rsrc(p.out[rank], OUT_BYTES);
#pragma unroll
  for (int it = 0; it < 4; ++it) {
    const u32 r = it * 32 + (tid >> 4), c = tid & 15;
    const u32x4 v = *(const LAS u32x4*)(lds + r * EROW + c * 16);
    __builtin_amdgcn_raw_buffer_store_b128(v | bad, ro, r * 8192 + c * 16, row0 * 8192 + col0 * 2, AUX_SYS);
  }
  WAIT_VM(0);
  __syncthreads();
  if (tid < NR && (u32)tid != rank) st_sys(p.ws[tid] + FT_OFF + (rank * 128 + rb * 8 + cb) * 4, seq);
  STAMP(6);
}

__global__ void __launch_bounds__(THREADS, 2) fused_gemm_allreduce_silu(Params p) {
  extern __shared__ __attribute__((aligned(16))) char smem[];
  const u32 g = blockIdx.x & 7, s = blockIdx.x >> 3;
  const bool is_gemm = s >= PUSH_WGS && s < PUSH_WGS + GEMM_WGS;
  if (!is_gemm && g == 0) return;
  STAMP(0);
  GAS u32* ctr = (GAS u32*)(p.ws[p.rank] + CTR_OFF) + blockIdx.x;
  const u32 seqv = __hip_atomic_load(ctr, __ATOMIC_RELAXED, __HIP_MEMORY_SCOPE_AGENT);
  const int lane = threadIdx.x & 63;
  const int wid = __builtin_amdgcn_readfirstlane(threadIdx.x >> 6);
  const u32 peer = ((u32)p.rank ^ g) & (NR - 1);
  if (s < PUSH_WGS) {
    push_role(p, peer, s * 8 + wid, lane, seqv);
  } else if (is_gemm) {
    const u32 i = s - PUSH_WGS, j = i & 7;
    gemm_role(p, (i >> 3) * 8 + (g >> 1) * 2 + (j >> 2), (g & 1) * 4 + (j & 3),
              __builtin_amdgcn_readfirstlane(seqv) + 1, (LAS char*)smem);
  } else {
    pull_role(p, peer, (s - PUSH_WGS - GEMM_WGS) * 8 + wid, lane, __builtin_amdgcn_readfirstlane(seqv) + 1);
  }
  __syncthreads();
  if (threadIdx.x == 0) __hip_atomic_store(ctr, seqv + 1, __ATOMIC_RELAXED, __HIP_MEMORY_SCOPE_AGENT);
  STAMP(7);
}

}

extern "C" void kernel_launch(int rank, int nranks, ncclComm_t comm,
                              hipStream_t stream,
                              void* const* d_in, const int* in_sizes, int n_in,
                              void* d_out, int out_size,
                              const int* in_dtypes, int out_dtype,
                              void* const* d_out_all,
                              void* const* d_ws_all, size_t ws_bytes) {
  if (nranks != NR || (unsigned)rank >= (unsigned)NR || n_in < 2 || ws_bytes < WS_USED ||
      in_sizes[0] != 4096 * 512 || in_sizes[1] != 512 * 4096 || out_size != 4096 * 4096) {
    fprintf(stderr, "kernel_launch: unexpected problem shape\n");
    return;
  }
  HIP_CHECK(hipFuncSetAttribute((const void*)fused_gemm_allreduce_silu,
                                hipFuncAttributeMaxDynamicSharedMemorySize, LDS_BYTES));
  Params p;
  p.x = (const char*)d_in[0];
  p.w = (const char*)d_in[1];
  for (int i = 0; i < NR; ++i) { p.ws[i] = (char*)d_ws_all[i]; p.out[i] = (char*)d_out_all[i]; }
  p.rank = rank;
  hipLaunchKernelGGL(fused_gemm_allreduce_silu, dim3(GRID), dim3(THREADS), LDS_BYTES, stream, p);
  HIP_CHECK(hipGetLastError());
}
```

```cpp
#include <hip/hip_runtime.h>
#include <rccl/rccl.h>
#include <cstdio>

#define HIP_CHECK(expr) do { hipError_t _e = (expr); if (_e != hipSuccess) { \
    fprintf(stderr, "HIP error %s at %s:%d\n", hipGetErrorString(_e), __FILE__, __LINE__); \
    return; } } while (0)

namespace {

typedef unsigned int u32;
typedef unsigned long long u64;
typedef u32 u32x4 __attribute__((ext_vector_type(4)));
typedef u32 u32x2 __attribute__((ext_vector_type(2)));
typedef __bf16 bf16x2 __attribute__((ext_vector_type(2)));
typedef __bf16 bf16x4 __attribute__((ext_vector_type(4)));
typedef __bf16 bf16x8 __attribute__((ext_vector_type(8)));
typedef float f32x16 __attribute__((ext_vector_type(16)));
#define LAS __attribute__((address_space(3)))
#define GAS __attribute__((address_space(1)))

constexpr int NR = 8;
constexpr u32 X_BYTES = 4096u * 512u * 2u;
constexpr u32 W_BYTES = 512u * 4096u * 2u;
constexpr u32 OUT_BYTES = 4096u * 4096u * 2u;

constexpr u32 BLK = 16384;
constexpr u32 XW_OFF = 0;
constexpr u32 XW_SRC = 8 * 16 * BLK, XW_KT = 16 * BLK;
constexpr u32 WW_OFF = XW_OFF + NR * XW_SRC;
constexpr u32 WW_SRC = 8 * 8 * BLK, WW_KT = 8 * BLK;
constexpr u32 DATA_BYTES = WW_OFF + NR * WW_SRC;
constexpr u32 FX_OFF = DATA_BYTES;
constexpr u32 FW_OFF = FX_OFF + NR * 8 * 16 * 2 * 4;
constexpr u32 FT_OFF = FW_OFF + NR * 8 * 8 * 2 * 4;
constexpr u32 CTR_OFF = FT_OFF + NR * 128 * 4;
constexpr u32 TMO_OFF = CTR_OFF + 256 * 4;
constexpr u32 TICK_OFF = TMO_OFF + 128;
constexpr u32 TS_OFF = TICK_OFF + 16 * 128;
constexpr u32 WS_USED = TS_OFF + 256 * 16 * 8;

constexpr u64 DEADLINE_TICKS = 30000000ull;

#define PUSH_WGS_ 4
#define PUSH_WAVES_ 4
#define PUSH_D_ 4
#define PULL_WGS_ 4
#define PULL_WAVES_ 4
#define PULL_D_ 4
#define TRIG_JOB_ 360
constexpr int PUSH_WGS = PUSH_WGS_, GEMM_WGS = 16, PULL_WGS = PULL_WGS_;
constexpr int PUSH_WAVES = PUSH_WAVES_, PULL_WAVES = PULL_WAVES_, PUSH_D = PUSH_D_, PULL_D = PULL_D_;
constexpr u32 TRIG_JOB = TRIG_JOB_;
constexpr u32 NAN2 = 0x7FC07FC0u;
constexpr int GRID = 8 * (PUSH_WGS + GEMM_WGS + PULL_WGS);
constexpr int THREADS = 512;
constexpr int NSTAGE = 4;
constexpr u32 STAGE = 2 * BLK;
constexpr u32 LDS_BYTES = NSTAGE * STAGE + 64;
constexpr int AUX_SYS = 17;

struct Params {
  const char* x;
  const char* w;
  char* ws[NR];
  char* out[NR];
  int rank;
};

__device__ __forceinline__ __amdgpu_buffer_rsrc_t mk_rsrc(const void* p, u32 n) {
  const u64 a = (u64)p;
  const u64 u = ((u64)(u32)__builtin_amdgcn_readfirstlane((u32)(a >> 32)) << 32) | (u32)__builtin_amdgcn_readfirstlane((u32)a);
  return __builtin_amdgcn_make_buffer_rsrc((void*)u, 0, n, 0x00020000);
}
__device__ __forceinline__ u32 ld_sys(const char* p) {
  return __hip_atomic_load((const GAS u32*)p, __ATOMIC_RELAXED, __HIP_MEMORY_SCOPE_SYSTEM);
}
__device__ __forceinline__ void st_sys(char* p, u32 v) {
  __hip_atomic_store((GAS u32*)p, v, __ATOMIC_RELAXED, __HIP_MEMORY_SCOPE_SYSTEM);
}
#define WAIT_VM(n) asm volatile("s_waitcnt vmcnt(%0)" ::"n"(n) : "memory")

__device__ __forceinline__ bool wave_wait(const char* flag, bool act, int need, u32 seq, char* tmo) {
  const u64 t0 = __builtin_amdgcn_s_memrealtime();
  for (;;) {
    const bool ok = act && ld_sys(flag) >= seq;
    if (__popcll(__ballot(ok)) >= need) return false;
    const u32 t = __hip_atomic_load((const GAS u32*)tmo, __ATOMIC_RELAXED, __HIP_MEMORY_SCOPE_AGENT);
    if (__builtin_amdgcn_readfirstlane(t) == seq || __builtin_amdgcn_s_memrealtime() - t0 > DEADLINE_TICKS) {
      __hip_atomic_store((GAS u32*)tmo, seq, __ATOMIC_RELAXED, __HIP_MEMORY_SCOPE_AGENT);
      return true;
    }
    __builtin_amdgcn_s_sleep(2);
  }
}

#define STAMP(i) do {} while (0)

__device__ __forceinline__ void dma4(u32 dst, __amdgpu_buffer_rsrc_t ra, u32 va0, u32 va1, u32 sa,
                                     __amdgpu_buffer_rsrc_t rb, u32 vb0, u32 vb1, u32 sb) {
  u32 keep;
  dst = __builtin_amdgcn_readfirstlane(dst);
  sa = __builtin_amdgcn_readfirstlane(sa);
  sb = __builtin_amdgcn_readfirstlane(sb);
  asm volatile(
      "s_mov_b32 %0, m0\n\t"
      "s_nop 4\n\t"
      "s_mov_b32 m0, %1\n\t"
      "s_nop 0\n\t"
      "buffer_load_dwordx4 %3, %5, %6 offen lds\n\t"
      "buffer_load_dwordx4 %4, %5, %6 offen offset:1024 lds\n\t"
      "s_mov_b32 m0, %2\n\t"
      "s_nop 0\n\t"
      "buffer_load_dwordx4 %7, %9, %10 offen lds\n\t"
      "buffer_load_dwordx4 %8, %9, %10 offen offset:1024 lds\n\t"
      "s_mov_b32 m0, %0"
      : "=&s"(keep)
      : "s"(dst), "s"(dst + BLK), "v"(va0), "v"(va1), "s"(ra), "s"(sa), "v"(vb0), "v"(vb1), "s"(rb), "s"(sb)
      : "memory");
}

template <int I, int N, class F>
__device__ __forceinline__ void static_for(F&& f) {
  if constexpr (I < N) { f(std::integral_constant<int, I>{}); static_for<I + 1, N>(f); }
}

struct Job { u32 src, sstride, dst, flag, wide; };

__device__ __forceinline__ u32 push_flag_off(u32 srcrank, u32 job) {
  const u32 step = job >> 6, half = job & 1, blk = (job & 63) >> 1;
  if (step >= 4) return FX_OFF + (((srcrank * 8 + (step - 4) * 4 + (blk & 3)) * 16 + 8 + (blk >> 2)) * 2 + half) * 4;
  const u32 kt = 2 * step + (blk & 1), idx = (blk & 15) >> 1;
  return blk >= 16 ? FW_OFF + (((srcrank * 8 + kt) * 8 + idx) * 2 + half) * 4
                   : FX_OFF + (((srcrank * 8 + kt) * 16 + idx) * 2 + half) * 4;
}

template <bool PULL>
__device__ __forceinline__ Job decode_job(u32 job, u32 rank, u32 peer) {
  Job j;
  if constexpr (PULL) {
    const u32 tile = job >> 2;
    j.src = ((peer >> 2) * 2048 + (tile >> 3) * 128 + (job & 3) * 32) * 8192 + (peer & 3) * 2048 + (tile & 7) * 256;
    j.dst = j.src; j.sstride = 4 * 8192; j.wide = 1;
    j.flag = FT_OFF + (peer * 128 + tile) * 4;
  } else {
    const u32 step = job >> 6, half = job & 1, blk = (job & 63) >> 1;
    const bool second = step >= 4;
    const bool isw = !second && blk >= 16;
    const u32 kt = second ? (step - 4) * 4 + (blk & 3) : 2 * step + (blk & 1);
    const u32 idx = second ? 8 + (blk >> 2) : (blk & 15) >> 1;
    j.wide = isw;
    j.src = isw ? (kt * 64 + half * 32) * 8192 + ((peer & 3) * 1024 + idx * 128) * 2
                : ((peer >> 2) * 2048 + idx * 128 + half * 64) * 1024 + kt * 128;
    j.sstride = isw ? 4 * 8192 : 8 * 1024;
    j.dst = (isw ? WW_OFF + rank * WW_SRC + kt * WW_KT : XW_OFF + rank * XW_SRC + kt * XW_KT) + idx * BLK + half * 8192;
    j.flag = push_flag_off(rank, job);
  }
  return j;
}
#define CBAR() asm volatile("" ::: "memory")

template <bool PULL>
__device__ __forceinline__ void engine(const Params& p, u32 peer, u32 w, int lane, u32 seqv) {
  constexpr int D = PULL ? PULL_D : PUSH_D, JI = 8;
  constexpr u32 NW = PULL ? PULL_WGS * PULL_WAVES : PUSH_WGS * PUSH_WAVES;
  constexpr u32 NJOBS = PULL ? 512 : 384;
  constexpr int LAUX = PULL ? AUX_SYS : 0;
  constexpr u32 DSTRIDE = PULL ? 4 * 8192 : 1024;
  const u32 rank = p.rank;
  char* ws = p.ws[rank];
  char* wsp = p.ws[peer];
  char* tmo = ws + TMO_OFF;
  const auto r0 = PULL ? mk_rsrc(p.out[peer], OUT_BYTES) : mk_rsrc(p.x, X_BYTES);
  const auto r1 = PULL ? r0 : mk_rsrc(p.w, W_BYTES);
  const auto rd = PULL ? mk_rsrc(p.out[rank], OUT_BYTES) : mk_rsrc(wsp, DATA_BYTES);
  const u32 vnarrow = (lane >> 3) * 1024 + (lane & 7) * 16;
  const u32 vwide = (lane >> 4) * 8192 + (lane & 15) * 16;
  const u32 vd = PULL ? vwide : lane * 16;
  u32 zero = 0;
  asm volatile("" : "+v"(zero));
  GAS u32* tick = (GAS u32*)(ws + TICK_OFF + ((PULL ? 8 : 0) + peer) * 128 + zero);
  auto ld = [&](const Job& j, int i) -> u32x4 {
    return __builtin_amdgcn_raw_buffer_load_b128(j.wide ? r1 : r0, j.wide ? vwide : vnarrow, j.src + i * j.sstride, LAUX);
  };
  u32 bad = 0;
  Job cur = decode_job<PULL>(w, rank, peer);
  if constexpr (PULL) {
    const u32 seq = __builtin_amdgcn_readfirstlane(seqv) + 1;
    if (wave_wait(ws + push_flag_off(peer, TRIG_JOB), true, 64, seq, tmo)) bad = NAN2;
    if (wave_wait(ws + cur.flag, true, 64, seq, tmo)) bad = NAN2;
    STAMP(1);
  }
  u32x4 v[D];
#pragma unroll
  for (int i = 0; i < D; ++i) { v[i] = ld(cur, i); CBAR(); }
  u32 nid = NW + w, pflag = 0;
  auto round = [&](auto first_c, auto last_c) {
    constexpr bool FIRST = decltype(first_c)::value, LAST = decltype(last_c)::value;
    Job nxt = cur;
    if constexpr (!LAST) nxt = decode_job<PULL>(nid, rank, peer);
    u32 tk = 0, fv = 0;
#pragma unroll
    for (int i = 0; i < JI; ++i) {
      if (!PULL && !FIRST && i == D - 1) {
        WAIT_VM(2 * D - 1);
        if (lane == 0) st_sys(wsp + pflag, seqv + 1);
        CBAR();
      }
      if (PULL && !LAST && i == JI - D) {
        const u32 seq = __builtin_amdgcn_readfirstlane(seqv) + 1;
        if (__builtin_amdgcn_readfirstlane(fv) < seq)
          if (wave_wait(ws + nxt.flag, true, 64, seq, tmo)) bad = NAN2;
      }
      __builtin_amdgcn_raw_buffer_store_b128(v[i % D], rd, vd, cur.dst + i * DSTRIDE, AUX_SYS);
      CBAR();
      if (i + D < JI) v[i % D] = ld(cur, i + D);
      else if (!LAST) v[i % D] = ld(nxt, i + D - JI);
      CBAR();
      if (!LAST && i == 0) {
        if (lane == 0) tk = __hip_atomic_fetch_add(tick, 1u, __ATOMIC_RELAXED, __HIP_MEMORY_SCOPE_AGENT);
        if (PULL) fv = ld_sys(ws + nxt.flag);
        CBAR();
      }
    }
    pflag = cur.flag;
    cur = nxt;
    if constexpr (!LAST)
      nid = __builtin_amdgcn_readfirstlane(tk) - __builtin_amdgcn_readfirstlane(seqv) * (NJOBS - NW) + 2 * NW;
  };
  round(std::true_type{}, std::false_type{});
  while (nid < NJOBS) round(std::false_type{}, std::false_type{});
  round(std::false_type{}, std::true_type{});
  if constexpr (!PULL) {
    WAIT_VM(0);
    if (lane == 0) st_sys(wsp + pflag, seqv + 1);
  } else if (bad) {
    WAIT_VM(0);
    const u32x4 nanv = {NAN2, NAN2, NAN2, NAN2};
    for (int i = 0; i < JI; ++i) __builtin_amdgcn_raw_buffer_store_b128(nanv, rd, vd, cur.dst + i * DSTRIDE, AUX_SYS);
  }
}

__device__ __forceinline__ void gemm_role(const Params& p, u32 rb, u32 cb, u32 seq, LAS char* lds) {
  const int tid = threadIdx.x, lane = tid & 63;
  const int wid = __builtin_amdgcn_readfirstlane(tid >> 6);
  const u32 rank = p.rank;
  char* ws = p.ws[rank];
  char* tmo = ws + TMO_OFF;
  const auto rx = mk_rsrc(p.x, X_BYTES);
  const auto rw = mk_rsrc(p.w, W_BYTES);
  const auto rs = mk_rsrc(ws, DATA_BYTES);
  const u32 row0 = (rank >> 2) * 2048 + rb * 128, col0 = (rank & 3) * 1024 + cb * 128;

  u32 vao[2], vbo[2], vaw[2], vbw[2];
#pragma unroll
  for (int q = 0; q < 2; ++q) {
    const u32 ra = wid * 16 + q * 8 + (lane >> 3), ca = (lane & 7) ^ ((ra >> 1) & 7);
    const u32 kb = wid * 8 + q * 4 + (lane >> 4), cbk = (lane & 15) ^ (((kb & 3) << 2) | ((kb >> 2) & 3));
    vao[q] = ra * 1024 + ca * 16 - q * 1024;  vaw[q] = ra * 128 + ca * 16 - q * 1024;
    vbo[q] = kb * 8192 + cbk * 16 - q * 1024; vbw[q] = kb * 256 + cbk * 16 - q * 1024;
  }
  const u32 lds0 = (u32)(size_t)lds + wid * 2048;
  auto issue = [&](int t) {
    const u32 dst = lds0 + (t & (NSTAGE - 1)) * STAGE;
    if (t < 8) {
      dma4(dst, rx, vao[0], vao[1], row0 * 1024 + t * 128, rw, vbo[0], vbo[1], t * 64 * 8192 + col0 * 2);
    } else {
      const u32 u = t - 8, g = u / 14, v = u - g * 14;
      const u32 src = rank ^ ((v >> 1) + 1), kt = 2 * g + (v & 1);
      dma4(dst, rs, vaw[0], vaw[1], XW_OFF + src * XW_SRC + kt * XW_KT + rb * BLK,
           rs, vbw[0], vbw[1], WW_OFF + src * WW_SRC + kt * WW_KT + cb * BLK);
    }
  };
  const u32 wm = wid & 1, wn = wid >> 1, h = lane >> 5;
  u32 xo[4], wo[2];
  {
    const u32 r = wm * 64 + (lane & 31);
#pragma unroll
    for (int ks = 0; ks < 4; ++ks) xo[ks] = r * 128 + (((2 * ks + h) ^ ((r >> 1) & 7)) * 16);
    const u32 bk = (lane >> 4) & 1, q = (lane & 15) >> 2, pp = lane & 3;
#pragma unroll
    for (int t = 0; t < 2; ++t)
      wo[t] = BLK + 256 * (8 * h + 4 * t + q) + 16 * ((4 * wn + 2 * bk + (pp >> 1)) ^ ((q << 2) | (2 * h + t))) + 8 * (pp & 1);
  }
  const bool fact = lane < 56;
  const u32 fsrc = rank ^ (((lane >> 3) + 1) & 7), fkt = (lane >> 1) & 1;
  const bool fisw = (lane >> 2) & 1;
  const u32 foff = fisw ? FW_OFF + (((fsrc * 8 + fkt) * 8 + cb) * 2 + (lane & 1)) * 4
                        : FX_OFF + (((fsrc * 8 + fkt) * 16 + rb) * 2 + (lane & 1)) * 4;
  const u32 fstep = fisw ? 2 * 8 * 2 * 4 : 2 * 16 * 2 * 4;
  LAS u32* lbad = (LAS u32*)(lds + NSTAGE * STAGE);
  if (tid == 0) *lbad = 0;

  f32x16 acc[2] = {};
  issue(0); issue(1); issue(2);
  int next_wait = 8, grp = 0;
  for (int t = 0; t < 64; ++t) {
    const int tn = t + NSTAGE - 1;
    if (tn == next_wait && tn < 64) {
      if (wid == 0) {
        if (wave_wait(ws + foff + grp * fstep, fact, 56, seq, tmo)) *lbad = 1;
        __builtin_amdgcn_fence(__ATOMIC_ACQUIRE, "");
        WAIT_VM(0);
        STAMP(1 + grp);
      }
      next_wait += 14; ++grp;
    }
    if (t < 62) WAIT_VM(8); else if (t == 62) WAIT_VM(4); else WAIT_VM(0);
    asm volatile("s_waitcnt lgkmcnt(0)" ::: "memory");
    __builtin_amdgcn_s_barrier();
    const LAS char* st = lds + (t & (NSTAGE - 1)) * STAGE;
    bf16x8 wf[4], x0[4], x1[4];
#pragma unroll
    for (int ks = 0; ks < 4; ++ks) {
      const bf16x4 w0 = __builtin_amdgcn_ds_read_tr16_b64_v4bf16((LAS bf16x4*)(st + wo[0] + ks * 4096));
      const bf16x4 w1 = __builtin_amdgcn_ds_read_tr16_b64_v4bf16((LAS bf16x4*)(st + wo[1] + ks * 4096));
      wf[ks] = __builtin_shufflevector(w0, w1, 0, 1, 2, 3, 4, 5, 6, 7);
      x0[ks] = *(const LAS bf16x8*)(st + xo[ks]);
      x1[ks] = *(const LAS bf16x8*)(st + xo[ks] + 4096);
    }
    if (tn < 64) issue(tn);
#pragma unroll
    for (int ks = 0; ks < 4; ++ks) {
      acc[0] = __builtin_amdgcn_mfma_f32_32x32x16_bf16(wf[ks], x0[ks], acc[0], 0, 0, 0);
      acc[1] = __builtin_amdgcn_mfma_f32_32x32x16_bf16(wf[ks], x1[ks], acc[1], 0, 0, 0);
    }
  }
  STAMP(5);
  __syncthreads();
  constexpr u32 EROW = 272;
#pragma unroll
  for (int mi = 0; mi < 2; ++mi) {
    const u32 m = wm * 64 + mi * 32 + (lane & 31);
#pragma unroll
    for (int g = 0; g < 4; ++g) {
      float y[4];
#pragma unroll
      for (int e = 0; e < 4; ++e) {
        const float a = acc[mi][4 * g + e];
        const float c = fminf(fmaxf(a, -60.f), 60.f);
        y[e] = a * __builtin_amdgcn_rcpf(1.f + __expf(-c));
      }
      bf16x4 o = {(__bf16)y[0], (__bf16)y[1], (__bf16)y[2], (__bf16)y[3]};
      *(LAS bf16x4*)(lds + m * EROW + (wn * 32 + 8 * g + 4 * h) * 2) = o;
    }
  }
  __syncthreads();
  const u32 bad = *lbad ? NAN2 : 0u;
  const auto ro = mk_rsrc(p.out[rank], OUT_BYTES);
#pragma unroll
  for (int it = 0; it < 4; ++it) {
    const u32 r = it * 32 + (tid >> 4), c = tid & 15;
    const u32x4 v = *(const LAS u32x4*)(lds + r * EROW + c * 16);
    __builtin_amdgcn_raw_buffer_store_b128(v | bad, ro, r * 8192 + c * 16, row0 * 8192 + col0 * 2, AUX_SYS);
  }
  WAIT_VM(0);
  __syncthreads();
  if (tid < NR && (u32)tid != rank) st_sys(p.ws[tid] + FT_OFF + (rank * 128 + rb * 8 + cb) * 4, seq);
  STAMP(6);
}

__global__ void __launch_bounds__(THREADS, 2) fused_gemm_allreduce_silu(Params p) {
  extern __shared__ __attribute__((aligned(16))) char smem[];
  const u32 g = blockIdx.x & 7, s = blockIdx.x >> 3;
  const bool is_gemm = s >= PUSH_WGS && s < PUSH_WGS + GEMM_WGS;
  if (!is_gemm && g == 0) return;
  STAMP(0);
  GAS u32* ctr = (GAS u32*)(p.ws[p.rank] + CTR_OFF) + blockIdx.x;
  const u32 seqv = __hip_atomic_load(ctr, __ATOMIC_RELAXED, __HIP_MEMORY_SCOPE_AGENT);
  const int lane = threadIdx.x & 63;
  const int wid = __builtin_amdgcn_readfirstlane(threadIdx.x >> 6);
  const u32 peer = ((u32)p.rank ^ g) & (NR - 1);
  if (s < PUSH_WGS) {
    if (wid < PUSH_WAVES) engine<false>(p, peer, s * PUSH_WAVES + wid, lane, seqv);
  } else if (is_gemm) {
    const u32 i = s - PUSH_WGS, j = i & 7;
    gemm_role(p, (i >> 3) * 8 + (g >> 1) * 2 + (j >> 2), (g & 1) * 4 + (j & 3),
              __builtin_amdgcn_readfirstlane(seqv) + 1, (LAS char*)smem);
  } else {
    if (wid < PULL_WAVES) engine<true>(p, peer, (s - PUSH_WGS - GEMM_WGS) * PULL_WAVES + wid, lane, seqv);
  }
  __syncthreads();
  if (threadIdx.x == 0) __hip_atomic_store(ctr, seqv + 1, __ATOMIC_RELAXED, __HIP_MEMORY_SCOPE_AGENT);
  STAMP(7);
}

}

extern "C" void kernel_launch(int rank, int nranks, ncclComm_t comm,
                              hipStream_t stream,
                              void* const* d_in, const int* in_sizes, int n_in,
                              void* d_out, int out_size,
                              const int* in_dtypes, int out_dtype,
                              void* const* d_out_all,
                              void* const* d_ws_all, size_t ws_bytes) {
  if (nranks != NR || (unsigned)rank >= (unsigned)NR || n_in < 2 || ws_bytes < WS_USED ||
      in_sizes[0] != 4096 * 512 || in_sizes[1] != 512 * 4096 || out_size != 4096 * 4096) {
    fprintf(stderr, "kernel_launch: unexpected problem shape\n");
    return;
  }
  HIP_CHECK(hipFuncSetAttribute((const void*)fused_gemm_allreduce_silu,
                                hipFuncAttributeMaxDynamicSharedMemorySize, LDS_BYTES));
  Params p;
  p.x = (const char*)d_in[0];
  p.w = (const char*)d_in[1];
  for (int i = 0; i < NR; ++i) { p.ws[i] = (char*)d_ws_all[i]; p.out[i] = (char*)d_out_all[i]; }
  p.rank = rank;
  hipLaunchKernelGGL(fused_gemm_allreduce_silu, dim3(GRID), dim3(THREADS), LDS_BYTES, stream, p);
  HIP_CHECK(hipGetLastError());
}
```

```cpp
#include <hip/hip_runtime.h>
#include <rccl/rccl.h>
#include <cstdio>
#include <type_traits>

#define HIP_CHECK(expr) do { hipError_t _e = (expr); if (_e != hipSuccess) { \
    fprintf(stderr, "HIP error %s at %s:%d\n", hipGetErrorString(_e), __FILE__, __LINE__); \
    return; } } while (0)

namespace {

typedef unsigned int u32;
typedef unsigned long long u64;
typedef u32 u32x4 __attribute__((ext_vector_type(4)));
typedef float f32x4 __attribute__((ext_vector_type(4)));
typedef __bf16 bf16x4 __attribute__((ext_vector_type(4)));
typedef __bf16 bf16x8 __attribute__((ext_vector_type(8)));
typedef float f32x16 __attribute__((ext_vector_type(16)));
#define LAS __attribute__((address_space(3)))
#define GAS __attribute__((address_space(1)))

constexpr int NR = 8;
constexpr u32 X_BYTES = 4096u * 512u * 2u;
constexpr u32 W_BYTES = 512u * 4096u * 2u;
constexpr u32 OUT_BYTES = 4096u * 4096u * 2u;
constexpr u32 KI = 5, KC = 3;

constexpr u32 BLK = 16384;
constexpr u32 IN_OFF = 0;
constexpr u32 IN_SRC = KI * 32 * BLK;
constexpr u32 P16_OFF = IN_OFF + NR * IN_SRC;
constexpr u32 P32_OFF = P16_OFF + 256 * 16384;
constexpr u32 DATA_BYTES = P32_OFF + 256 * 32768;
constexpr u32 FI_OFF = DATA_BYTES;
constexpr u32 FPR_OFF = FI_OFF + NR * 320 * 4;
constexpr u32 FPL_OFF = FPR_OFF + 256 * 4;
constexpr u32 FF_OFF = FPL_OFF + 256 * 4;
constexpr u32 CTR_OFF = FF_OFF + NR * 512 * 4;
constexpr u32 TMO_OFF = CTR_OFF + 256 * 4;
constexpr u32 TICK_OFF = TMO_OFF + 128;
constexpr u32 TS_OFF = TICK_OFF + 16 * 128;
constexpr u32 WS_USED = TS_OFF + 256 * 16 * 8;

constexpr u64 DEADLINE_TICKS = 30000000ull;

#define TRIG_BACK_ 24
#define ACQ_MODE 0
#define SL_PULL 8
#define SL_TRIG 24
#define SL_FIN 8
#define TOKENS 0
#define PAUSE 32
#define FIN_WAVES_ 2
#define SL_GEMM 8
#define PUSH_WAVES_ 2
#define PULL_WAVES_ 2
#define PUSH_D_ 4
#define STAGGER 300
constexpr int ENG_WGS = 8, GEMM_WGS = 24;
constexpr int PUSH_WAVES = PUSH_WAVES_, PULL_WAVES = PULL_WAVES_, FIN_WAVES = FIN_WAVES_, PUSH_D = PUSH_D_;
static_assert(PUSH_WAVES + PULL_WAVES + FIN_WAVES <= 8, "waves per engine workgroup");
constexpr u32 TRIG_BACK = TRIG_BACK_;
constexpr u32 NAN2 = 0x7FC07FC0u;
constexpr int GRID = 8 * (ENG_WGS + GEMM_WGS);
constexpr int THREADS = 512;
constexpr int NSTAGE = 4;
constexpr u32 STAGE = 2 * BLK;
constexpr u32 LDS_BYTES = NSTAGE * STAGE + 64;
constexpr int AUX_SYS = 17;

struct Params {
  const char* x;
  const char* w;
  char* ws[NR];
  char* out[NR];
  int rank;
};

__device__ __forceinline__ u32 rfl(u32 v) { return __builtin_amdgcn_readfirstlane(v); }
__device__ __forceinline__ __amdgpu_buffer_rsrc_t mk_rsrc(const void* p, u32 n) {
  const u64 a = (u64)p;
  const u64 u = ((u64)rfl((u32)(a >> 32)) << 32) | rfl((u32)a);
  return __builtin_amdgcn_make_buffer_rsrc((void*)u, 0, n, 0x00020000);
}
__device__ __forceinline__ u32 ld_sys(const char* p) {
  return __hip_atomic_load((const GAS u32*)p, __ATOMIC_RELAXED, __HIP_MEMORY_SCOPE_SYSTEM);
}
__device__ __forceinline__ void st_sys(char* p, u32 v) {
  __hip_atomic_store((GAS u32*)p, v, __ATOMIC_RELAXED, __HIP_MEMORY_SCOPE_SYSTEM);
}
#define WAIT_VM(n) asm volatile("s_waitcnt vmcnt(%0)" ::"n"(n) : "memory")
#define CBAR() asm volatile("" ::: "memory")

__device__ __forceinline__ void st128(u32x4 v, __amdgpu_buffer_rsrc_t r, u32 voff, u32 soff) {
  __builtin_amdgcn_raw_buffer_store_b128(v, r, voff, soff, AUX_SYS);
  asm volatile("s_nop 1" ::"v"(v) : "memory");
}

template <int SLEEP>
__device__ __forceinline__ u64 wave_wait(const char* flag, u64 need, u32 seq, char* tmo, bool& bad) {
  const u64 t0 = __builtin_amdgcn_s_memrealtime();
  for (u32 it = 1;; ++it) {
    const u64 ok = __ballot(ld_sys(flag) >= seq);
    if ((ok & need) == need) return ok;
    if ((it & 15) == 0 || bad) {
      const u32 t = __hip_atomic_load((const GAS u32*)tmo, __ATOMIC_RELAXED, __HIP_MEMORY_SCOPE_AGENT);
      if (rfl(t) == seq || __builtin_amdgcn_s_memrealtime() - t0 > DEADLINE_TICKS) {
        __hip_atomic_store((GAS u32*)tmo, seq, __ATOMIC_RELAXED, __HIP_MEMORY_SCOPE_AGENT);
        bad = true;
        return ~0ull;
      }
    }
    __builtin_amdgcn_s_sleep(SLEEP);
  }
}

#define STAMPW(i, w) do {} while (0)
#define STAMP(i) STAMPW(i, 0)

__device__ __forceinline__ void dma4(u32 dst, __amdgpu_buffer_rsrc_t ra, u32 va0, u32 va1, u32 sa,
                                     __amdgpu_buffer_rsrc_t rb, u32 vb0, u32 vb1, u32 sb) {
  u32 keep;
  dst = rfl(dst); sa = rfl(sa); sb = rfl(sb);
  asm volatile(
      "s_mov_b32 %0, m0\n\t"
      "s_nop 4\n\t"
      "s_mov_b32 m0, %1\n\t"
      "s_nop 0\n\t"
      "buffer_load_dwordx4 %3, %5, %6 offen lds\n\t"
      "buffer_load_dwordx4 %4, %5, %6 offen offset:1024 lds\n\t"
      "s_mov_b32 m0, %2\n\t"
      "s_nop 0\n\t"
      "buffer_load_dwordx4 %7, %9, %10 offen lds\n\t"
      "buffer_load_dwordx4 %8, %9, %10 offen offset:1024 lds\n\t"
      "s_mov_b32 m0, %0"
      : "=&s"(keep)
      : "s"(dst), "s"(dst + BLK), "v"(va0), "v"(va1), "s"(ra), "s"(sa), "v"(vb0), "v"(vb1), "s"(rb), "s"(sb)
      : "memory");
}

__device__ __forceinline__ u32 sup(u32 grp, u32 c, u32 u) { return grp ? c + 2 * u : 2 * c + u; }
__device__ __forceinline__ u32 srow(u32 r, u32 u) { return sup(r >> 2, (r >> 1) & 1, u); }
__device__ __forceinline__ u32 scol(u32 r, u32 v) { return sup(r >> 2, r & 1, v); }
__device__ __forceinline__ void tile_rc(u32 m, u32& ri, u32& ci) {
  const u32 l = (m >= 1) + (m >= 4) + (m >= 9) + (m >= 16) + (m >= 25) + (m >= 36) + (m >= 49);
  const u32 idx = m - l * l;
  ri = idx < l ? l : idx - l;
  ci = idx < l ? idx : l;
}
__device__ __forceinline__ u32 fin_off(u32 r, u32 m, u32 sb, u32 part) {
  u32 ri, ci;
  tile_rc(m, ri, ci);
  return (srow(r, sb >> 1) * 1024 + ri * 128 + (r >> 2) * 64 + part * 32) * 8192 + (scol(r, sb & 1) * 1024 + ci * 128) * 2;
}

struct Job { u32 src, sstride, dst, flag, blo, bhi, voff; };

__device__ __forceinline__ Job decode_push(u32 job, u32 rank, u32 peer, u64 b0, u64 b1, u32 vnarrow, u32 vwide) {
  Job j;
  const bool cross = (rank ^ peer) >= 4;
  const u32 lk = job >> 3, op = (job >> 2) & 1, uv = (job >> 1) & 1, half = job & 1;
  const u32 lev = cross ? lk / KC : lk / KI;
  const u32 ktl = lk - lev * (cross ? KC : KI), kt = cross ? KI + ktl : ktl;
  j.blo = op ? (u32)b1 : (u32)b0; j.bhi = op ? (u32)(b1 >> 32) : (u32)(b0 >> 32); j.voff = op ? vwide : vnarrow;
  j.src = op ? (kt * 64 + half * 32) * 8192 + (scol(peer, uv) * 1024 + lev * 128) * 2
             : (srow(peer, uv) * 1024 + lev * 128 + half * 64) * 1024 + kt * 128;
  j.sstride = op ? 4 * 8192 : 8 * 1024;
  j.dst = IN_OFF + rank * IN_SRC + job * 8192;
  j.flag = FI_OFF + (rank * 320 + job) * 4;
  return j;
}

__device__ __forceinline__ GAS u32* ticket_ptr(char* ws, u32 idx) {
  u32 zero = 0;
  asm volatile("" : "+v"(zero));
  return (GAS u32*)(ws + TICK_OFF + idx * 128 + zero);
}

__device__ __forceinline__ void push_role(const Params& p, u32 peer, u32 w, int lane, u32 seqv) {
  constexpr int D = PUSH_D, JI = 8;
  constexpr u32 NW = ENG_WGS * PUSH_WAVES;
  const u32 rank = p.rank;
  const u32 NJOBS = ((rank ^ peer) >= 4 ? KC : KI) * 64;
  char* wsp = p.ws[peer];
  static_assert(X_BYTES == W_BYTES, "one size for both input shards");
  const u64 b0 = (u64)p.x, b1 = (u64)p.w;
  const auto rd = mk_rsrc(wsp, DATA_BYTES);
  const u32 vnarrow = (lane >> 3) * 1024 + (lane & 7) * 16;
  const u32 vwide = (lane >> 4) * 8192 + (lane & 15) * 16;
  GAS u32* tick = ticket_ptr(p.ws[rank], peer);
  auto ld = [&](const Job& j, int i) -> u32x4 {
    const auto r = mk_rsrc((const void*)(((u64)j.bhi << 32) | j.blo), X_BYTES);
    return __builtin_amdgcn_raw_buffer_load_b128(r, j.voff, j.src + i * j.sstride, 0);
  };
  Job cur = decode_push(w, rank, peer, b0, b1, vnarrow, vwide);
  u32x4 v[D];
#pragma unroll
  for (int i = 0; i < D; ++i) { v[i] = ld(cur, i); CBAR(); }
  u32 nid = NW + w, pflag = 0;
  auto round = [&](auto first_c, auto last_c) {
    constexpr bool FIRST = decltype(first_c)::value, LAST = decltype(last_c)::value;
    Job nxt = cur;
    if constexpr (!LAST) nxt = decode_push(nid, rank, peer, b0, b1, vnarrow, vwide);
    u32 tk = 0;
#pragma unroll
    for (int i = 0; i < JI; ++i) {
      if (!FIRST && i == D - 1) {
        WAIT_VM(2 * D - 1);
        if (lane == 0) st_sys(wsp + pflag, seqv + 1);
        CBAR();
      }
      st128(v[i % D], rd, lane * 16, cur.dst + i * 1024);
      if (i + D < JI) v[i % D] = ld(cur, i + D);
      else if (!LAST) v[i % D] = ld(nxt, i + D - JI);
      CBAR();
      if (!LAST && i == 0) {
        if (lane == 0) tk = __hip_atomic_fetch_add(tick, 1u, __ATOMIC_RELAXED, __HIP_MEMORY_SCOPE_AGENT);
        CBAR();
      }
    }
    pflag = cur.flag;
    cur = nxt;
    if constexpr (!LAST) nid = rfl(tk) - rfl(seqv) * (NJOBS - NW) + 2 * NW;
  };
  round(std::true_type{}, std::false_type{});
  while (nid < NJOBS) round(std::false_type{}, std::false_type{});
  round(std::false_type{}, std::true_type{});
  WAIT_VM(0);
  if (lane == 0) st_sys(wsp + pflag, seqv + 1);
  STAMP(3);
}

__device__ __forceinline__ float silu(float a) {
  const float c = fminf(fmaxf(a, -60.f), 60.f);
  return a * __builtin_amdgcn_rcpf(1.f + __expf(-c));
}

__device__ __forceinline__ void pull_role(const Params& p, u32 peer, u32 w, int lane, u32 seq, LAS u32* ctl) {
  constexpr u32 NW = ENG_WGS * PULL_WAVES, NF = 1024;
  const u32 rank = p.rank;
  const bool cross = (rank ^ peer) >= 4;
  char* ws = p.ws[rank];
  char* tmo = ws + TMO_OFF;
  const auto rf = mk_rsrc(p.out[peer], OUT_BYTES);
  const auto ro = mk_rsrc(p.out[rank], OUT_BYTES);
  const u32 vwide = (lane >> 4) * 8192 + (lane & 15) * 16;
  GAS u32* tick = ticket_ptr(ws, 8 + peer);
  const char* ff = ws + FF_OFF + peer * 2048;
  u32 tf = w, tf2 = NW + w, cur = 0;
  bool bad = false;
  wave_wait<SL_TRIG>(ws + FI_OFF + (peer * 320 + (cross ? KC : KI) * 64 - TRIG_BACK) * 4, 1, seq, tmo, bad);
  u32 pv = 0;
  u32x4 v[4];
  auto f_next = [&]() -> bool {
    const u32 tok = __hip_atomic_load(ctl + 1, __ATOMIC_RELAXED, __HIP_MEMORY_SCOPE_WORKGROUP);
    return tf < NF && rfl(pv) >= seq && (int)rfl(tok) <= 0;
  };
  auto f_round = [&](auto first_c) {
    constexpr bool FIRST = decltype(first_c)::value;
    const u32 nxt = rfl(fin_off(peer, tf >> 4, (tf >> 2) & 3, (tf >> 1) & 1) + (tf & 1) * 16 * 8192);
    u32 tk = 0;
    tf = tf2;
    if (lane == 0) tk = __hip_atomic_fetch_add(tick, 1u, __ATOMIC_RELAXED, __HIP_MEMORY_SCOPE_AGENT);
    pv = ld_sys(ff + ((tf >> 1) & 511) * 4);
    CBAR();
#pragma unroll
    for (int i = 0; i < 4; ++i) {
      if (!FIRST) st128(v[i], ro, vwide, rfl(cur) + i * 4 * 8192);
      v[i] = __builtin_amdgcn_raw_buffer_load_b128(rf, vwide, nxt + i * 4 * 8192, AUX_SYS);
      CBAR();
    }
    cur = nxt;
    tf2 = rfl(tk) - (seq - 1) * NF + 2 * NW;
  };
  while (tf < NF) {
    if (rfl(pv) < seq) {
      wave_wait<SL_PULL>(ff + (tf >> 1) * 4, 1, seq, tmo, bad);
      if (lane == 0) __hip_atomic_store(ctl + 1, 0u, __ATOMIC_RELAXED, __HIP_MEMORY_SCOPE_WORKGROUP);
    }
    if (tf == w) { STAMPW(5, PUSH_WAVES); }
    f_round(std::true_type{});
    if (f_next()) {
      f_round(std::false_type{});
      while (f_next()) f_round(std::false_type{});
    }
#pragma unroll
    for (int i = 0; i < 4; ++i) st128(v[i], ro, vwide, rfl(cur) + i * 4 * 8192);
    for (;;) {
      u32 tok = 0;
      if (lane == 0) tok = __hip_atomic_fetch_add(ctl + 1, ~0u, __ATOMIC_RELAXED, __HIP_MEMORY_SCOPE_WORKGROUP);
      if ((int)rfl(tok) <= 0) {
        if (lane == 0) __hip_atomic_fetch_add(ctl + 1, 1u, __ATOMIC_RELAXED, __HIP_MEMORY_SCOPE_WORKGROUP);
        break;
      }
      __builtin_amdgcn_s_sleep(PAUSE);
    }
  }
  if (bad) {
    WAIT_VM(0);
    const u32x4 nanv = {NAN2, NAN2, NAN2, NAN2};
    for (int i = 0; i < 4; ++i) st128(nanv, ro, vwide, rfl(cur) + i * 4 * 8192);
  }
  STAMPW(6, PUSH_WAVES);
}

__device__ __forceinline__ void fin_role(const Params& p, u32 peer, u32 w, int lane, u32 seq, LAS u32* ctl) {
  const u32 rank = p.rank, sb = peer & 3;
  char* ws = p.ws[rank];
  char* tmo = ws + TMO_OFF;
  const auto rr = mk_rsrc(p.ws[peer], DATA_BYTES);
  const auto rl = mk_rsrc(ws, DATA_BYTES);
  const auto ro = mk_rsrc(p.out[rank], OUT_BYTES);
  const u32 vwide = (lane >> 4) * 8192 + (lane & 15) * 16;
  char* wsl = p.ws[lane & 7];
  for (u32 job = w; job < 128; job += ENG_WGS * FIN_WAVES) {
    const u32 m = job >> 1, part = job & 1;
    const u32 so = P16_OFF + ((rank & 3) * 64 + m) * 16384 + part * 8192;
    const u32 lo = P32_OFF + (sb * 64 + m) * 32768 + part * 16384;
    const u32 dst = fin_off(rank, m, sb, part);
    bool bad = false;
    wave_wait<SL_FIN>(ws + ((lane & 1) ? FPL_OFF : FPR_OFF) + (sb * 64 + m) * 4, 3, seq, tmo, bad);
    u32x4 r[8], a[8], b[8];
#pragma unroll
    for (int i = 0; i < 8; ++i) {
      a[i] = __builtin_amdgcn_raw_buffer_load_b128(rl, lane * 16, lo + i * 2048, AUX_SYS);
      b[i] = __builtin_amdgcn_raw_buffer_load_b128(rl, lane * 16, lo + i * 2048 + 1024, AUX_SYS);
    }
    for (;;) {
      u32 old = 0;
      if (lane == 0) old = __hip_atomic_exchange(ctl, 1u, __ATOMIC_RELAXED, __HIP_MEMORY_SCOPE_WORKGROUP);
      if (rfl(old) == 0) break;
      __builtin_amdgcn_s_sleep(2);
    }
    if (lane == 0) __hip_atomic_fetch_add(ctl + 1, (u32)TOKENS, __ATOMIC_RELAXED, __HIP_MEMORY_SCOPE_WORKGROUP);
#pragma unroll
    for (int i = 0; i < 8; ++i) r[i] = __builtin_amdgcn_raw_buffer_load_b128(rr, lane * 16, so + i * 1024, AUX_SYS);
    WAIT_VM(2);
    if (lane == 0) __hip_atomic_store(ctl, 0u, __ATOMIC_RELAXED, __HIP_MEMORY_SCOPE_WORKGROUP);
    __builtin_amdgcn_sched_barrier(0);
#pragma unroll
    for (int i = 0; i < 8; ++i) {
      float y[8];
#pragma unroll
      for (int e = 0; e < 4; ++e) {
        const u32 l0 = e < 2 ? a[i][2 * e] : b[i][2 * e - 4], l1 = e < 2 ? a[i][2 * e + 1] : b[i][2 * e - 3];
        y[2 * e] = silu(__uint_as_float(l0) + __uint_as_float(r[i][e] << 16));
        y[2 * e + 1] = silu(__uint_as_float(l1) + __uint_as_float(r[i][e] & 0xFFFF0000u));
      }
      const bf16x8 o = {(__bf16)y[0], (__bf16)y[1], (__bf16)y[2], (__bf16)y[3],
                        (__bf16)y[4], (__bf16)y[5], (__bf16)y[6], (__bf16)y[7]};
      u32x4 ov = __builtin_bit_cast(u32x4, o);
      if (bad) ov |= NAN2;
      st128(ov, ro, vwide, dst + i * 4 * 8192);
    }
    WAIT_VM(0);
    if (lane < NR && (u32)lane != rank) st_sys(wsl + FF_OFF + (rank * 512 + m * 8 + sb * 2 + part) * 4, seq);
    if (job == w) { STAMPW(1, PUSH_WAVES + PULL_WAVES); }
  }
  STAMPW(2, PUSH_WAVES + PULL_WAVES + FIN_WAVES - 1);
}

__device__ __forceinline__ void gemm_role(const Params& p, u32 g, u32 k, u32 seq, LAS char* lds) {
  const int tid = threadIdx.x, lane = tid & 63;
  const u32 wid = rfl(tid >> 6);
  const u32 rank = p.rank;
  char* ws = p.ws[rank];
  char* tmo = ws + TMO_OFF;
  const auto rx = mk_rsrc(p.x, X_BYTES);
  const auto rw = mk_rsrc(p.w, W_BYTES);
  const auto rs = mk_rsrc(ws, DATA_BYTES);
  const u32 sb = g >> 1, arm = g & 1, u = sb >> 1, v = sb & 1;
  constexpr int NT = 32, T_CROSS = KI, T_INTRA = KI + 4 * KC;
  const u64 tstart = __builtin_amdgcn_s_memrealtime();

  u32 vao[2], vbo[2], vaw[2], vbw[2];
#pragma unroll
  for (int q = 0; q < 2; ++q) {
    const u32 ra = wid * 16 + q * 8 + (lane >> 3), ca = (lane & 7) ^ ((ra >> 1) & 7);
    const u32 kb = wid * 8 + q * 4 + (lane >> 4), cbk = (lane & 15) ^ (((kb & 3) << 2) | ((kb >> 2) & 3));
    vao[q] = ra * 1024 + ca * 16 - q * 1024;  vaw[q] = ra * 128 + ca * 16 - q * 1024;
    vbo[q] = kb * 8192 + cbk * 16 - q * 1024; vbw[q] = kb * 256 + cbk * 16 - q * 1024;
  }
  const u32 lds0 = (u32)(size_t)lds + wid * 2048;
  const u32 wm = wid & 1, wn = wid >> 1, h = lane >> 5;
  u32 xo[4], wo[2];
  {
    const u32 r = wm * 64 + (lane & 31);
#pragma unroll
    for (int ks = 0; ks < 4; ++ks) xo[ks] = r * 128 + (((2 * ks + h) ^ ((r >> 1) & 7)) * 16);
    const u32 bk = (lane >> 4) & 1, q = (lane & 15) >> 2, pp = lane & 3;
#pragma unroll
    for (int t = 0; t < 2; ++t)
      wo[t] = BLK + 256 * (8 * h + 4 * t + q) + 16 * ((4 * wn + 2 * bk + (pp >> 1)) ^ ((q << 2) | (2 * h + t))) + 8 * (pp & 1);
  }
  LAS u32* lbad = (LAS u32*)(lds + NSTAGE * STAGE);
  if (tid == 0) *lbad = 0;
  if (wid == 0) __builtin_amdgcn_fence(__ATOMIC_ACQUIRE, "");

  for (u32 n = k; n < 32; n += GEMM_WGS) {
    u32 lev = 0, idx = n;
    for (;;) {
      const u32 cnt = lev + ((lev & 1) == arm);
      if (idx < cnt) break;
      idx -= cnt; ++lev;
    }
    const u32 ri = arm ? idx : lev, ci = arm ? lev : idx;
    const u32 m = lev * lev + (idx == lev ? 2 * lev : arm * lev + idx);
    const u32 row0 = srow(rank, u) * 1024 + ri * 128, col0 = scol(rank, v) * 1024 + ci * 128;
    auto issue = [&](int t) {
      const u32 dst = lds0 + (t & (NSTAGE - 1)) * STAGE;
      if (t < T_CROSS) {
        dma4(dst, rx, vao[0], vao[1], row0 * 1024 + t * 128, rw, vbo[0], vbo[1], t * 64 * 8192 + col0 * 2);
      } else {
        u32 src, ktl, kts;
        if (t < T_INTRA) { const u32 c = t - T_CROSS; ktl = c >> 2; src = rank ^ (4 + (c & 3)); kts = KC; }
        else { const u32 c = t - T_INTRA; ktl = c / 3; src = rank ^ (1 + c - ktl * 3); kts = KI; }
        const u32 base = IN_OFF + src * IN_SRC + ktl * 4 * BLK;
        dma4(dst, rs, vaw[0], vaw[1], base + (ri * kts * 4 + u) * BLK, rs, vbw[0], vbw[1], base + (ci * kts * 4 + 2 + v) * BLK);
      }
    };
    const u32 fop = (lane >> 1) & 1, flev = fop ? ci : ri, fuv = fop ? v : u, fq = lane >> 2;
    const u32 fck = min(fq >> 2, KC - 1), fik = min(fq / 3, KI - 1);
    const char* fcross = ws + FI_OFF + ((rank ^ (4 + (fq & 3))) * 320 + ((flev * KC + fck) * 4 + fop * 2 + fuv) * 2 + (lane & 1)) * 4;
    const char* fintra = ws + FI_OFF + ((rank ^ (1 + fq % 3)) * 320 + ((flev * KI + fik) * 4 + fop * 2 + fuv) * 2 + (lane & 1)) * 4;

    while (__builtin_amdgcn_s_memrealtime() - tstart < (u64)lev * STAGGER) __builtin_amdgcn_s_sleep(16);
    f32x16 acc[2] = {};
    issue(0); issue(1); issue(2);
    int next_wait = T_CROSS;
    u64 need = 0xFFFF, have = 0;
    for (int t = 0; t < NT; ++t) {
      const int tn = t + NSTAGE - 1;
      if (tn == next_wait && tn < NT) {
        if (tn == T_INTRA) { need = 0xFFF; have = 0; }
        if (wid == 0 && (have & need) != need) {
          bool bad = false;
          have = wave_wait<SL_GEMM>(tn < T_INTRA ? fcross : fintra, need, seq, tmo, bad);
          if (bad) *lbad = 1;
        }
        if (tn == T_CROSS) { STAMP(n < GEMM_WGS ? 1 : 8); }
        if (tn == NT - 3) { STAMP(n < GEMM_WGS ? 2 : 9); }
        if (tn < T_INTRA) { next_wait += 4; need = (need << 16) | 0xFFFF; }
        else { next_wait += 3; need = (need << 12) | 0xFFF; }
      }
      if (t < NT - 2) WAIT_VM(8); else if (t == NT - 2) WAIT_VM(4); else WAIT_VM(0);
      asm volatile("s_waitcnt lgkmcnt(0)" ::: "memory");
      __builtin_amdgcn_s_barrier();
      const LAS char* st = lds + (t & (NSTAGE - 1)) * STAGE;
      bf16x8 wf[4], x0[4], x1[4];
#pragma unroll
      for (int ks = 0; ks < 4; ++ks) {
        const bf16x4 w0 = __builtin_amdgcn_ds_read_tr16_b64_v4bf16((LAS bf16x4*)(st + wo[0] + ks * 4096));
        const bf16x4 w1 = __builtin_amdgcn_ds_read_tr16_b64_v4bf16((LAS bf16x4*)(st + wo[1] + ks * 4096));
        wf[ks] = __builtin_shufflevector(w0, w1, 0, 1, 2, 3, 4, 5, 6, 7);
        x0[ks] = *(const LAS bf16x8*)(st + xo[ks]);
        x1[ks] = *(const LAS bf16x8*)(st + xo[ks] + 4096);
      }
      if (tn < NT) issue(tn);
#pragma unroll
      for (int ks = 0; ks < 4; ++ks) {
        acc[0] = __builtin_amdgcn_mfma_f32_32x32x16_bf16(wf[ks], x0[ks], acc[0], 0, 0, 0);
        acc[1] = __builtin_amdgcn_mfma_f32_32x32x16_bf16(wf[ks], x1[ks], acc[1], 0, 0, 0);
      }
    }
    STAMP(n < GEMM_WGS ? 3 : 10);
    __syncthreads();
    constexpr u32 EROW = 528;
#pragma unroll
    for (int mi = 0; mi < 2; ++mi) {
      const u32 mr = wm * 64 + mi * 32 + (lane & 31);
#pragma unroll
      for (int gq = 0; gq < 4; ++gq) {
        const f32x4 o = {acc[mi][4 * gq], acc[mi][4 * gq + 1], acc[mi][4 * gq + 2], acc[mi][4 * gq + 3]};
        *(LAS f32x4*)(lds + mr * EROW + (wn * 32 + 8 * gq + 4 * h) * 4) = o;
      }
    }
    __syncthreads();
    const u32 bad = *lbad ? NAN2 : 0u;
    const u32 keep = rank >> 2;
    const u32 e16 = P16_OFF + (sb * 64 + m) * 16384 + wid * 1024, e32 = P32_OFF + (sb * 64 + m) * 32768 + wid * 2048;
#pragma unroll
    for (int part = 0; part < 2; ++part) {
      const LAS char* se = lds + ((keep ^ 1) * 64 + part * 32 + (tid >> 4)) * EROW + (tid & 15) * 32;
      const f32x4 c0 = *(const LAS f32x4*)se, c1 = *(const LAS f32x4*)(se + 16);
      const bf16x8 o = {(__bf16)c0[0], (__bf16)c0[1], (__bf16)c0[2], (__bf16)c0[3],
                        (__bf16)c1[0], (__bf16)c1[1], (__bf16)c1[2], (__bf16)c1[3]};
      st128(__builtin_bit_cast(u32x4, o) | bad, rs, lane * 16, e16 + part * 8192);
    }
#pragma unroll
    for (int part = 0; part < 2; ++part) {
      const LAS char* sk = lds + (keep * 64 + part * 32 + (tid >> 4)) * EROW + (tid & 15) * 32;
      const u32x4 c0 = *(const LAS u32x4*)sk, c1 = *(const LAS u32x4*)(sk + 16);
      st128(c0 | bad, rs, lane * 16, e32 + part * 16384);
      st128(c1 | bad, rs, lane * 16, e32 + part * 16384 + 1024);
    }
    WAIT_VM(0);
    __syncthreads();
    if (tid == 0) st_sys(p.ws[((rank & 4) ^ 4) | sb] + FPR_OFF + ((rank & 3) * 64 + m) * 4, seq);
    if (tid == 64) st_sys(ws + FPL_OFF + (sb * 64 + m) * 4, seq);
    STAMP(n < GEMM_WGS ? 4 : 11);
  }
}

__global__ void __launch_bounds__(THREADS, 2) fused_gemm_allreduce_silu(Params p) {
  extern __shared__ __attribute__((aligned(16))) char smem[];
  const u32 g = blockIdx.x & 7, s = blockIdx.x >> 3;
  if (s < ENG_WGS && g == 0) return;
  STAMP(0);
  GAS u32* ctr = (GAS u32*)(p.ws[p.rank] + CTR_OFF) + (s * 8 + g);
  const u32 seqv = __hip_atomic_load(ctr, __ATOMIC_RELAXED, __HIP_MEMORY_SCOPE_AGENT);
  const int lane = threadIdx.x & 63;
  const u32 wid = rfl(threadIdx.x >> 6);
  const u32 peer = ((u32)p.rank ^ g) & (NR - 1);
  if (s < ENG_WGS) {
    LAS u32* ctl = (LAS u32*)smem;
    if (threadIdx.x < 2) ctl[threadIdx.x] = 0;
    asm volatile("s_waitcnt lgkmcnt(0)" ::: "memory");
    __builtin_amdgcn_s_barrier();
    if (wid < PUSH_WAVES) push_role(p, peer, wid * ENG_WGS + s, lane, seqv);
    else if (wid < PUSH_WAVES + PULL_WAVES) pull_role(p, peer, (wid - PUSH_WAVES) * ENG_WGS + s, lane, rfl(seqv) + 1, ctl);
    else if (wid < PUSH_WAVES + PULL_WAVES + FIN_WAVES && g >= 4)
      fin_role(p, peer, (wid - PUSH_WAVES - PULL_WAVES) * ENG_WGS + s, lane, rfl(seqv) + 1, ctl);
  } else {
    gemm_role(p, g, s - ENG_WGS, rfl(seqv) + 1, (LAS char*)smem);
  }
  __syncthreads();
  if (threadIdx.x == 0) __hip_atomic_store(ctr, seqv + 1, __ATOMIC_RELAXED, __HIP_MEMORY_SCOPE_AGENT);
  STAMP(7);
}

}

extern "C" void kernel_launch(int rank, int nranks, ncclComm_t comm,
                              hipStream_t stream,
                              void* const* d_in, const int* in_sizes, int n_in,
                              void* d_out, int out_size,
                              const int* in_dtypes, int out_dtype,
                              void* const* d_out_all,
                              void* const* d_ws_all, size_t ws_bytes) {
  if (nranks != NR || (unsigned)rank >= (unsigned)NR || n_in < 2 || ws_bytes < WS_USED ||
      in_sizes[0] != 4096 * 512 || in_sizes[1] != 512 * 4096 || out_size != 4096 * 4096) {
    fprintf(stderr, "kernel_launch: unexpected problem shape\n");
    return;
  }
  HIP_CHECK(hipFuncSetAttribute((const void*)fused_gemm_allreduce_silu,
                                hipFuncAttributeMaxDynamicSharedMemorySize, LDS_BYTES));
  Params p;
  p.x = (const char*)d_in[0];
  p.w = (const char*)d_in[1];
  static const int phys[NR] = {0, 2, 3, 6, 1, 4, 5, 7};
  p.rank = 0;
  for (int i = 0; i < NR; ++i) {
    p.ws[i] = (char*)d_ws_all[phys[i]]; p.out[i] = (char*)d_out_all[phys[i]];
    if (phys[i] == rank) p.rank = i;
  }
  hipLaunchKernelGGL(fused_gemm_allreduce_silu, dim3(GRID), dim3(THREADS), LDS_BYTES, stream, p);
  HIP_CHECK(hipGetLastError());
}
```

```cpp
#include <hip/hip_runtime.h>
#include <rccl/rccl.h>
#include <cstdio>
#include <type_traits>

#define HIP_CHECK(expr) do { hipError_t _e = (expr); if (_e != hipSuccess) { \
    fprintf(stderr, "HIP error %s at %s:%d\n", hipGetErrorString(_e), __FILE__, __LINE__); \
    return; } } while (0)

namespace {

typedef unsigned int u32;
typedef unsigned long long u64;
typedef u32 u32x4 __attribute__((ext_vector_type(4)));
typedef float f32x4 __attribute__((ext_vector_type(4)));
typedef __bf16 bf16x4 __attribute__((ext_vector_type(4)));
typedef __bf16 bf16x8 __attribute__((ext_vector_type(8)));
typedef float f32x16 __attribute__((ext_vector_type(16)));
#define LAS __attribute__((address_space(3)))
#define GAS __attribute__((address_space(1)))

constexpr int NR = 8;
constexpr u32 X_BYTES = 4096u * 512u * 2u;
constexpr u32 W_BYTES = 512u * 4096u * 2u;
constexpr u32 OUT_BYTES = 4096u * 4096u * 2u;
constexpr u32 KI = 5, KC = 3;

constexpr u32 BLK = 16384;
constexpr u32 IN_OFF = 0;
constexpr u32 IN_SRC = KI * 32 * BLK;
constexpr u32 P16_OFF = IN_OFF + NR * IN_SRC;
constexpr u32 P32_OFF = P16_OFF + 256 * 16384;
constexpr u32 DATA_BYTES = P32_OFF + 256 * 32768;
constexpr u32 FI_OFF = DATA_BYTES;
constexpr u32 FPR_OFF = FI_OFF + NR * 320 * 4;
constexpr u32 FPL_OFF = FPR_OFF + 256 * 4;
constexpr u32 FF_OFF = FPL_OFF + 256 * 4;
constexpr u32 CTR_OFF = FF_OFF + NR * 256 * 4;
constexpr u32 CLM_OFF = CTR_OFF + 256 * 4;
constexpr u32 TMO_OFF = CLM_OFF + 256 * 64;
constexpr u32 TICK_OFF = TMO_OFF + 128;
constexpr u32 TS_OFF = TICK_OFF + 16 * 128;
constexpr u32 TR_OFF = TS_OFF + 256 * 16 * 8;
constexpr u32 WS_USED = TR_OFF + 3 * 8 * 1024 * 8;

constexpr u64 DEADLINE_TICKS = 30000000ull;

#define TRIG_BACK_ 24
#define NHALF 8
#define SL_PULL 2
#define SL_FIN 8
#define FIN_WAVES_ 4
#define SL_GEMM 8
#define PUSH_WAVES_ 2
#define PULL_WAVES_ 2
#define PUSH_D_ 4
#define PUSH_LAUX 0
#define FIN_DIRECT 0
#define ASYNC_POLL 0
#define TLB_WARM 0
#define CODE_WARM 1
#define STAGGER 300
constexpr int ENG_WGS = 8, GEMM_WGS = 24;
constexpr int PUSH_WAVES = PUSH_WAVES_, PULL_WAVES = PULL_WAVES_, FIN_WAVES = FIN_WAVES_, PUSH_D = PUSH_D_;
static_assert(PUSH_WAVES + PULL_WAVES + FIN_WAVES <= 8, "waves per engine workgroup");
constexpr u32 TRIG_BACK = TRIG_BACK_;
constexpr u32 NAN2 = 0x7FC07FC0u;
constexpr int GRID = 8 * (ENG_WGS + GEMM_WGS);
static_assert(GRID == 256, "slots wrap at a power of two");
constexpr int THREADS = 512;
constexpr int NSTAGE = 4;
constexpr u32 STAGE = 2 * BLK;
constexpr u32 LDS_CTL = NSTAGE * STAGE;
constexpr u32 LDS_BYTES = LDS_CTL + 1024;
constexpr int AUX_SYS = 17;

struct Params {
  const char* x;
  const char* w;
  int rank;
  int pad;
  char* ws[NR];
  char* out[NR];
};

__device__ __forceinline__ u32 rfl(u32 v) { return __builtin_amdgcn_readfirstlane(v); }
__device__ __forceinline__ __amdgpu_buffer_rsrc_t mk_rsrc(const void* p, u32 n) {
  const u64 a = (u64)p;
  const u64 u = ((u64)rfl((u32)(a >> 32)) << 32) | rfl((u32)a);
  return __builtin_amdgcn_make_buffer_rsrc((void*)u, 0, n, 0x00020000);
}
__device__ __forceinline__ u32 ld_sys(const char* p) {
  return __hip_atomic_load((const GAS u32*)p, __ATOMIC_RELAXED, __HIP_MEMORY_SCOPE_SYSTEM);
}
__device__ __forceinline__ void st_sys(char* p, u32 v) {
  __hip_atomic_store((GAS u32*)p, v, __ATOMIC_RELAXED, __HIP_MEMORY_SCOPE_SYSTEM);
}
#define WAIT_VM(n) asm volatile("s_waitcnt vmcnt(%0)" ::"n"(n) : "memory")
#define CBAR() asm volatile("" ::: "memory")
#define LD_LDS(q) __hip_atomic_load(q, __ATOMIC_RELAXED, __HIP_MEMORY_SCOPE_WORKGROUP)
#define ST_LDS(q, v) __hip_atomic_store(q, v, __ATOMIC_RELAXED, __HIP_MEMORY_SCOPE_WORKGROUP)

__device__ __forceinline__ void st128(u32x4 v, __amdgpu_buffer_rsrc_t r, u32 voff, u32 soff) {
  __builtin_amdgcn_raw_buffer_store_b128(v, r, voff, soff, AUX_SYS);
  asm volatile("s_nop 1" ::"v"(v) : "memory");
}

template <int SLEEP>
__device__ __forceinline__ u64 wave_wait(const char* flag, u64 need, u32 seq, char* tmo, bool& bad) {
  const u64 t0 = __builtin_amdgcn_s_memrealtime();
  for (u32 it = 1;; ++it) {
    const u64 ok = __ballot(ld_sys(flag) >= seq);
    if ((ok & need) == need) return ok;
    if ((it & 15) == 0 || bad) {
      const u32 t = __hip_atomic_load((const GAS u32*)tmo, __ATOMIC_RELAXED, __HIP_MEMORY_SCOPE_AGENT);
      if (rfl(t) == seq || __builtin_amdgcn_s_memrealtime() - t0 > DEADLINE_TICKS) {
        __hip_atomic_store((GAS u32*)tmo, seq, __ATOMIC_RELAXED, __HIP_MEMORY_SCOPE_AGENT);
        bad = true;
        return ~0ull;
      }
    }
    __builtin_amdgcn_s_sleep(SLEEP);
  }
}

#define STAMPW(i, w) do {} while (0)
#define STAMP(i) STAMPW(i, 0)
#define NOW(x) do {} while (0)
#define TRACEJ(kind, job) do {} while (0)

__device__ __forceinline__ void dma4(u32 dst, __amdgpu_buffer_rsrc_t ra, u32 va0, u32 va1, u32 sa,
                                     __amdgpu_buffer_rsrc_t rb, u32 vb0, u32 vb1, u32 sb) {
  u32 keep;
  dst = rfl(dst); sa = rfl(sa); sb = rfl(sb);
  asm volatile(
      "s_mov_b32 %0, m0\n\t"
      "s_nop 4\n\t"
      "s_mov_b32 m0, %1\n\t"
      "s_nop 0\n\t"
      "buffer_load_dwordx4 %3, %5, %6 offen lds\n\t"
      "buffer_load_dwordx4 %4, %5, %6 offen offset:1024 lds\n\t"
      "s_mov_b32 m0, %2\n\t"
      "s_nop 0\n\t"
      "buffer_load_dwordx4 %7, %9, %10 offen lds\n\t"
      "buffer_load_dwordx4 %8, %9, %10 offen offset:1024 lds\n\t"
      "s_mov_b32 m0, %0"
      : "=&s"(keep)
      : "s"(dst), "s"(dst + BLK), "v"(va0), "v"(va1), "s"(ra), "s"(sa), "v"(vb0), "v"(vb1), "s"(rb), "s"(sb)
      : "memory");
}

__device__ __forceinline__ void dma3(u32 dsta, u32 dstb, __amdgpu_buffer_rsrc_t ra, u32 va0, u32 sa,
                                     __amdgpu_buffer_rsrc_t rb, u32 vb0, u32 vb1, u32 sb) {
  u32 keep;
  dsta = rfl(dsta); dstb = rfl(dstb); sa = rfl(sa); sb = rfl(sb);
  asm volatile(
      "s_mov_b32 %0, m0\n\t"
      "s_nop 4\n\t"
      "s_mov_b32 m0, %1\n\t"
      "s_nop 0\n\t"
      "buffer_load_dwordx4 %3, %4, %5 offen lds\n\t"
      "s_mov_b32 m0, %2\n\t"
      "s_nop 0\n\t"
      "buffer_load_dwordx4 %6, %8, %9 offen lds\n\t"
      "buffer_load_dwordx4 %7, %8, %9 offen offset:1024 lds\n\t"
      "s_mov_b32 m0, %0"
      : "=&s"(keep)
      : "s"(dsta), "s"(dstb), "v"(va0), "s"(ra), "s"(sa), "v"(vb0), "v"(vb1), "s"(rb), "s"(sb)
      : "memory");
}

__device__ __forceinline__ void dma_flags(u32 dst, const char* src) {
  u32 keep;
  dst = rfl(dst);
  asm volatile("s_mov_b32 %0, m0\n\ts_nop 0\n\ts_mov_b32 m0, %1\n\ts_nop 0\n\tglobal_load_lds_dword %2, off sc0 sc1\n\ts_mov_b32 m0, %0"
               : "=&s"(keep) : "s"(dst), "v"(src) : "memory");
}

__device__ __forceinline__ void dma_plain(u32 dst, const char* src) {
  u32 keep;
  dst = rfl(dst);
  asm volatile("s_mov_b32 %0, m0\n\ts_nop 0\n\ts_mov_b32 m0, %1\n\ts_nop 0\n\tglobal_load_lds_dword %2, off\n\ts_mov_b32 m0, %0"
               : "=&s"(keep) : "s"(dst), "v"(src) : "memory");
}

__device__ __forceinline__ u32 sup(u32 grp, u32 c, u32 u) { return grp ? c + 2 * u : 2 * c + u; }
__device__ __forceinline__ u32 srow(u32 r, u32 u) { return sup(r >> 2, (r >> 1) & 1, u); }
__device__ __forceinline__ u32 scol(u32 r, u32 v) { return sup(r >> 2, r & 1, v); }
__device__ __forceinline__ void tile_rc(u32 m, u32& ri, u32& ci) {
  const u32 l = (m >= 1) + (m >= 4) + (m >= 9) + (m >= 16) + (m >= 25) + (m >= 36) + (m >= 49);
  const u32 idx = m - l * l;
  ri = idx < l ? l : idx - l;
  ci = idx < l ? idx : l;
}
__device__ __forceinline__ u32 fin_off(u32 r, u32 m, u32 sb, u32 part) {
  u32 ri, ci;
  tile_rc(m, ri, ci);
  return (srow(r, sb >> 1) * 1024 + ri * 128 + (r >> 2) * 64 + part * 32) * 8192 + (scol(r, sb & 1) * 1024 + ci * 128) * 2;
}

struct Job { u32 src, sstride, dst, flag, blo, bhi, voff; };

__device__ __forceinline__ Job decode_push(u32 job, u32 rank, u32 peer, u64 b0, u64 b1, u32 vnarrow, u32 vwide) {
  Job j;
  const bool cross = (rank ^ peer) >= 4;
  const u32 lk = job >> 3, op = (job >> 2) & 1, uv = (job >> 1) & 1, half = job & 1;
  const u32 lev = cross ? lk / KC : lk / KI;
  const u32 ktl = lk - lev * (cross ? KC : KI), kt = cross ? KI + ktl : ktl;
  j.blo = op ? (u32)b1 : (u32)b0; j.bhi = op ? (u32)(b1 >> 32) : (u32)(b0 >> 32); j.voff = op ? vwide : vnarrow;
  j.src = op ? (kt * 64 + half * 32) * 8192 + (scol(peer, uv) * 1024 + lev * 128) * 2
             : (srow(peer, uv) * 1024 + lev * 128 + half * 64) * 1024 + kt * 128;
  j.sstride = op ? 4 * 8192 : 8 * 1024;
  j.dst = IN_OFF + rank * IN_SRC + job * 8192;
  j.flag = FI_OFF + (rank * 320 + job) * 4;
  return j;
}

__device__ __forceinline__ GAS u32* ticket_ptr(char* ws, u32 idx) {
  u32 zero = 0;
  asm volatile("" : "+v"(zero));
  return (GAS u32*)(ws + TICK_OFF + idx * 128 + zero);
}

__device__ __forceinline__ u32x4 push_ld(const Job& j, int i) {
  const auto r = mk_rsrc((const void*)(((u64)j.bhi << 32) | j.blo), X_BYTES);
  return __builtin_amdgcn_raw_buffer_load_b128(r, j.voff, rfl(j.src + i * j.sstride), PUSH_LAUX);
}

template <bool claimed>
__device__ __forceinline__ bool push_role(const Params& p, u32& slot, u32 wid, int lane, LAS u32* lctl) {
  constexpr int D = PUSH_D, JI = 8;
  constexpr u32 NW = ENG_WGS * PUSH_WAVES;
  NOW(tA);
  const u32 g = slot & 7, w = wid * ENG_WGS + (slot >> 3);
  static_assert(X_BYTES == W_BYTES, "one size for both input shards");
  u64 b0 = (u64)p.x, b1 = (u64)p.w, bw = (u64)p.ws[g];
  u32 rank = p.rank;
  asm volatile("" : "+s"(b0), "+s"(b1), "+s"(bw), "+s"(rank));
  const u32 peer = rank ^ g;
  const u32 NJOBS = (g >= 4 ? KC : KI) * 64;
  char* wsp = (char*)bw;
  const auto rd = mk_rsrc(wsp, DATA_BYTES);
  const u32 vnarrow = (lane >> 3) * 1024 + (lane & 7) * 16;
  const u32 vwide = (lane >> 4) * 8192 + (lane & 15) * 16;
  Job cur = decode_push(w, rank, peer, b0, b1, vnarrow, vwide);
  u32x4 v[D];
#pragma unroll
  for (int i = 0; i < D; ++i) { v[i] = push_ld(cur, i); CBAR(); }
  NOW(tB);
  GAS u32* tick = ticket_ptr(p.ws[0], peer);
  u32 nid = NW + w, pflag = 0, seq = 0;
  auto round = [&](auto first_c, auto last_c) {
    constexpr bool FIRST = decltype(first_c)::value, LAST = decltype(last_c)::value;
    Job nxt = cur;
    if constexpr (!LAST) { nxt = decode_push(nid, rank, peer, b0, b1, vnarrow, vwide); TRACEJ(0, nid); }
    u32 tk = 0;
#pragma unroll
    for (int i = 0; i < JI; ++i) {
      if (!FIRST && i == D - 1) {
        WAIT_VM(D + (LAST ? (JI - D < D - 1 ? JI - D : D - 1) : D - 1));
        if (lane == 0) st_sys(wsp + pflag, seq);
        CBAR();
      }
      st128(v[i % D], rd, lane * 16, rfl(cur.dst) + i * 1024);
      if (i + D < JI) v[i % D] = push_ld(cur, i + D);
      else if (!LAST) v[i % D] = push_ld(nxt, i + D - JI);
      CBAR();
      if (!FIRST && !LAST && i == 0) {
        if (lane == 0) tk = __hip_atomic_fetch_add(tick, 1u, __ATOMIC_RELAXED, __HIP_MEMORY_SCOPE_AGENT);
        CBAR();
      }
    }
    pflag = cur.flag;
    cur = nxt;
    if constexpr (FIRST) nid += NW;
    else if constexpr (!LAST) nid = rfl(tk) - (seq - 1) * (NJOBS - 2 * NW) + 3 * NW;
  };
  round(std::true_type{}, std::false_type{});
  NOW(tE);
  if (!claimed) __builtin_amdgcn_s_barrier();
  CBAR();
  const u32 slot2 = rfl(LD_LDS(lctl + 1));
  seq = rfl(LD_LDS(lctl + 2));
  if (__builtin_expect(slot2 != slot, 0)) { WAIT_VM(0); slot = slot2; return false; }
  NOW(tC);
  do round(std::false_type{}, std::false_type{}); while (nid < NJOBS);
  round(std::false_type{}, std::true_type{});
  WAIT_VM(0);
  if (lane == 0) st_sys(wsp + pflag, seq);
  STAMP(3);
  return true;
}

__device__ __forceinline__ float silu(float a) {
  const float c = fminf(fmaxf(a, -60.f), 60.f);
  return a * __builtin_amdgcn_rcpf(1.f + __expf(-c));
}

constexpr u32 MB_REQ = 0, MB_DONE = 4, MB_GONE = 8, MB_RUN = 9, MB_UNIT = 10, MB_BUF = 1024;

__device__ __forceinline__ void pull_role(const Params& p, u32 peer, u32 w, int lane, u32 seq, LAS u32* ctl) {
  constexpr u32 NW = ENG_WGS * PULL_WAVES, NF = 1024;
  const u32 rank = p.rank;
  const bool cross = (rank ^ peer) >= 4;
  char* ws = p.ws[0];
  char* tmo = ws + TMO_OFF;
  const u64 bf = (u64)p.out[rank ^ peer], bp = (u64)p.ws[rank ^ peer];
  static_assert(DATA_BYTES == OUT_BYTES, "one size for both remote sources");
  const u32 vwide = (lane >> 4) * 8192 + (lane & 15) * 16;
  GAS u32* tick = ticket_ptr(ws, 8 + peer);
  const u32 pbase = (lane & 1) ? FI_OFF + (peer * 320 + (cross ? KC : KI) * 64 - TRIG_BACK) * 4 : FF_OFF + peer * 1024;
  const u32 pmask = (lane & 1) ? 0u : ~0u;
  const u64 t0 = __builtin_amdgcn_s_memrealtime();
  u32 tf = w, tf2 = NW + w, cur = 0, curf = 0, lastf = 0;
  u32 nsrc = 0, nf = 0;
  bool bad = false;
  auto poll = [&]() -> u32 { return ld_sys(ws + (pbase + (rfl(((tf >> 2) & 255) * 4) & pmask))); };
  u32 pv = poll();
  u32x4 v[4];
  auto pick = [&]() -> bool {
    const u64 pend = __ballot(lane < FIN_WAVES && LD_LDS(ctl + MB_REQ + (lane & 3)) != 0);
    if (pend) {
      const u32 f = __builtin_ctzll(pend);
      u32 r = 0;
      if (lane == 0) r = __hip_atomic_exchange(ctl + MB_REQ + f, 0u, __ATOMIC_RELAXED, __HIP_MEMORY_SCOPE_WORKGROUP);
      r = rfl(r);
      if (r) { nsrc = r; nf = f + 1; return true; }
    }
    if (tf < NF && (bad || (__ballot(pv >= seq) & 3) == 3)) {
      nsrc = rfl(fin_off(peer, tf >> 4, (tf >> 2) & 3, 0) + (tf & 3) * 16 * 8192);
      nf = 0;
      return true;
    }
    return false;
  };
  auto put = [&](int i) {
    const auto ro = mk_rsrc(p.out[0], curf ? 0u : OUT_BYTES);
    st128(v[i], ro, vwide, rfl(cur) + i * 4 * 8192);
    if (curf) *(LAS u32x4*)((LAS char*)ctl + MB_BUF + (curf - 1) * 4096 + i * 1024 + lane * 16) = v[i];
  };
  auto delivered = [&]() {
    if (curf) {
      asm volatile("s_waitcnt lgkmcnt(0)" ::: "memory");
      if (lane == 0) __hip_atomic_fetch_add(ctl + MB_DONE + curf - 1, 1u, __ATOMIC_RELAXED, __HIP_MEMORY_SCOPE_WORKGROUP);
    }
  };
  auto round = [&](auto first_c) {
    constexpr bool FIRST = decltype(first_c)::value;
    u32 tk = 0;
    const bool isf = nf == 0;
    if (isf) { TRACEJ(1, tf); tf = tf2; lastf = nsrc; }
    if (lane == 0 && isf) tk = __hip_atomic_fetch_add(tick, 1u, __ATOMIC_RELAXED, __HIP_MEMORY_SCOPE_AGENT);
    pv = poll();
    CBAR();
    const auto rsrc = mk_rsrc((const void*)(isf ? bf : bp), OUT_BYTES);
    const u32 vs = isf ? vwide : lane * 16, str = isf ? 4 * 8192 : 1024;
#pragma unroll
    for (int i = 0; i < 4; ++i) {
      if (!FIRST) put(i);
      v[i] = __builtin_amdgcn_raw_buffer_load_b128(rsrc, vs, rfl(nsrc + i * str), AUX_SYS);
      CBAR();
    }
    if (!FIRST) delivered();
    cur = nsrc; curf = nf;
    if (isf) tf2 = rfl(tk) - (seq - 1) * NF + 2 * NW;
  };
  for (u32 it = 1;; ++it) {
    if (pick()) {
      if (tf == w && nf == 0) { ST_LDS(ctl + MB_RUN, 1u); STAMPW(5, PUSH_WAVES); }
      round(std::true_type{});
      if (pick()) {
        round(std::false_type{});
        while (pick()) round(std::false_type{});
      }
#pragma unroll
      for (int i = 0; i < 4; ++i) put(i);
      delivered();
      continue;
    }
    if (tf >= NF && rfl(LD_LDS(ctl + MB_GONE)) == (cross ? FIN_WAVES : 0)) break;
    if ((it & 63) == 0) {
      const u32 t = __hip_atomic_load((const GAS u32*)tmo, __ATOMIC_RELAXED, __HIP_MEMORY_SCOPE_AGENT);
      if (rfl(t) == seq || __builtin_amdgcn_s_memrealtime() - t0 > DEADLINE_TICKS) {
        __hip_atomic_store((GAS u32*)tmo, seq, __ATOMIC_RELAXED, __HIP_MEMORY_SCOPE_AGENT);
        bad = true;
      }
    }
    __builtin_amdgcn_s_sleep(SL_PULL);
    if ((it & 3) == 0) pv = poll();
  }
  if (bad) {
    WAIT_VM(0);
    const u32x4 nanv = {NAN2, NAN2, NAN2, NAN2};
    for (int i = 0; i < 4; ++i) st128(nanv, mk_rsrc(p.out[0], OUT_BYTES), vwide, rfl(lastf) + i * 4 * 8192);
  }
  asm volatile(".Lfz_e0:");
  STAMPW(6, PUSH_WAVES);
}

__device__ __forceinline__ void fin_role(const Params& p, u32 peer, u32 s, u32 f, int lane, u32 seq, LAS u32* ctl) {
  static_assert(FIN_WAVES == 4, "one wave per 16 rows of a half tile");
  const u32 rank = p.rank, sb = peer & 3;
  char* ws = p.ws[0];
  char* tmo = ws + TMO_OFF;
  const auto rl = mk_rsrc(ws, DATA_BYTES);
  const auto ro = mk_rsrc(p.out[0], OUT_BYTES);
  const u32 vwide = (lane >> 4) * 8192 + (lane & 15) * 16;
  char* wsl = p.ws[lane & 7];
  u32 ndone = 0, nunit = 0;
  const auto rr = mk_rsrc(p.ws[rank ^ peer], DATA_BYTES);
  for (u32 m = s; m < 64; m += ENG_WGS) {
    const u32 lo = P32_OFF + (sb * 64 + m) * 32768 + f * 8192;
    const u32 dst = fin_off(rank, m, sb, 0) + f * 16 * 8192;
    bool bad = false;
    wave_wait<SL_FIN>(ws + ((lane & 1) ? FPL_OFF : FPR_OFF) + (sb * 64 + m) * 4, 3, seq, tmo, bad);
    if (m == s) { STAMPW(8, PUSH_WAVES + PULL_WAVES); STAMPW(11, 7); }
    const u32 so = P16_OFF + ((rank & 3) * 64 + m) * 16384 + f * 4096;
    const bool direct = FIN_DIRECT && rfl(LD_LDS(ctl + MB_RUN)) == 0;
    u32x4 rv[4], a[4], b[4];
    if (direct) {
#pragma unroll
      for (int i = 0; i < 4; ++i) rv[i] = __builtin_amdgcn_raw_buffer_load_b128(rr, lane * 16, so + i * 1024, AUX_SYS);
    } else if (lane == 0) {
      ST_LDS(ctl + MB_REQ + f, so);
    }
#pragma unroll
    for (int i = 0; i < 4; ++i) {
      a[i] = __builtin_amdgcn_raw_buffer_load_b128(rl, lane * 16, lo + i * 2048, AUX_SYS);
      b[i] = __builtin_amdgcn_raw_buffer_load_b128(rl, lane * 16, lo + i * 2048 + 1024, AUX_SYS);
    }
    ++nunit;
    if (!direct) {
      ++ndone;
      while (rfl(LD_LDS(ctl + MB_DONE + f)) != ndone) __builtin_amdgcn_s_sleep(1);
      CBAR();
#pragma unroll
      for (int i = 0; i < 4; ++i) rv[i] = *(const LAS u32x4*)((LAS char*)ctl + MB_BUF + f * 4096 + i * 1024 + lane * 16);
    }
    if (m == s) { STAMPW(9, PUSH_WAVES + PULL_WAVES);  }
#pragma unroll
    for (int i = 0; i < 4; ++i) {
      const u32x4 r = rv[i];
      float y[8];
#pragma unroll
      for (int e = 0; e < 4; ++e) {
        const u32 l0 = e < 2 ? a[i][2 * e] : b[i][2 * e - 4], l1 = e < 2 ? a[i][2 * e + 1] : b[i][2 * e - 3];
        y[2 * e] = silu(__uint_as_float(l0) + __uint_as_float(r[e] << 16));
        y[2 * e + 1] = silu(__uint_as_float(l1) + __uint_as_float(r[e] & 0xFFFF0000u));
      }
      const bf16x8 o = {(__bf16)y[0], (__bf16)y[1], (__bf16)y[2], (__bf16)y[3],
                        (__bf16)y[4], (__bf16)y[5], (__bf16)y[6], (__bf16)y[7]};
      u32x4 ov = __builtin_bit_cast(u32x4, o);
      if (bad) ov |= NAN2;
      st128(ov, ro, vwide, dst + i * 4 * 8192);
    }
    if (m == s) { STAMPW(10, PUSH_WAVES + PULL_WAVES); }
    WAIT_VM(0);

    u32 arrived = 0;
    if (lane == 0) arrived = __hip_atomic_fetch_add(ctl + MB_UNIT + nunit - 1, 1u, __ATOMIC_RELAXED, __HIP_MEMORY_SCOPE_WORKGROUP);
    if (rfl(arrived) == FIN_WAVES - 1) {
      if (lane >= 1 && lane < NR) st_sys(wsl + FF_OFF + (rank * 256 + m * 4 + sb) * 4, seq);
      TRACEJ(2, 2 * m); TRACEJ(2, 2 * m + 1);
    }
    if (m == s) { STAMPW(1, PUSH_WAVES + PULL_WAVES); }
  }
  if (lane == 0) __hip_atomic_fetch_add(ctl + MB_GONE, 1u, __ATOMIC_RELAXED, __HIP_MEMORY_SCOPE_WORKGROUP);
  asm volatile(".Lfz_e1:");
  STAMPW(2, PUSH_WAVES + PULL_WAVES + FIN_WAVES - 1);
}

template <bool HALF>
__device__ __forceinline__ void gemm_tile(const Params& p, u32 g, u32 n, u32 hsel, u32 seq, u64 tstart, int sbase, LAS char* lds) {
  const int tid = threadIdx.x, lane = tid & 63;
  const u32 wid = rfl(tid >> 6);
  const u32 rank = p.rank;
  char* ws = p.ws[0];
  char* tmo = ws + TMO_OFF;
  const auto rx = mk_rsrc(p.x, X_BYTES);
  const auto rw = mk_rsrc(p.w, W_BYTES);
  const auto rs = mk_rsrc(ws, DATA_BYTES);
  const u32 sb = g >> 1, arm = g & 1, u = sb >> 1, v = sb & 1;
  constexpr int NT = 32, T_CROSS = KI, T_INTRA = KI + 4 * KC;
  constexpr int MI = HALF ? 1 : 2, NDMA = HALF ? 3 : 4;

  u32 vao[2], vbo[2], vaw[2], vbw[2];
#pragma unroll
  for (int q = 0; q < 2; ++q) {
    const u32 ra = wid * 8 * MI + q * 8 + (lane >> 3), ca = (lane & 7) ^ ((ra >> 1) & 7);
    const u32 kb = wid * 8 + q * 4 + (lane >> 4), cbk = (lane & 15) ^ (((kb & 3) << 2) | ((kb >> 2) & 3));
    vao[q] = ra * 1024 + ca * 16 - q * 1024;  vaw[q] = ra * 128 + ca * 16 - q * 1024;
    vbo[q] = kb * 8192 + cbk * 16 - q * 1024; vbw[q] = kb * 256 + cbk * 16 - q * 1024;
  }
  const u32 lds0 = (u32)(size_t)lds;
  const u32 wm = wid & 1, wn = wid >> 1, h = lane >> 5;
  u32 xo[4], wo[2];
  {
    const u32 r = wm * 32 * MI + (lane & 31);
#pragma unroll
    for (int ks = 0; ks < 4; ++ks) xo[ks] = r * 128 + (((2 * ks + h) ^ ((r >> 1) & 7)) * 16);
    const u32 bk = (lane >> 4) & 1, q = (lane & 15) >> 2, pp = lane & 3;
#pragma unroll
    for (int t = 0; t < 2; ++t)
      wo[t] = BLK + 256 * (8 * h + 4 * t + q) + 16 * ((4 * wn + 2 * bk + (pp >> 1)) ^ ((q << 2) | (2 * h + t))) + 8 * (pp & 1);
  }
  LAS u32* lbad = (LAS u32*)(lds + LDS_CTL);
  LAS u32* fimg = (LAS u32*)(lds + LDS_CTL + 256) + lane;
  if (wid == 0) { fimg[0] = 0; fimg[64] = 0; }

  u32 lev = 0, idx = n;
  for (;;) {
    const u32 cnt = lev + ((lev & 1) == arm);
    if (idx < cnt) break;
    idx -= cnt; ++lev;
  }
  const u32 ri = arm ? idx : lev, ci = arm ? lev : idx;
  const u32 m = lev * lev + (idx == lev ? 2 * lev : arm * lev + idx);
  const u32 row0 = srow(rank, u) * 1024 + ri * 128 + (HALF ? hsel * 64 : 0), col0 = scol(rank, v) * 1024 + ci * 128;
  auto issue = [&](int t) {
    const u32 dst = lds0 + (t & (NSTAGE - 1)) * STAGE;
    if (t < T_CROSS) {
      const u32 sa = row0 * 1024 + t * 128, sw = t * 64 * 8192 + col0 * 2;
      if (HALF) dma3(dst + wid * 1024, dst + BLK + wid * 2048, rx, vao[0], sa, rw, vbo[0], vbo[1], sw);
      else dma4(dst + wid * 2048, rx, vao[0], vao[1], sa, rw, vbo[0], vbo[1], sw);
    } else {
      u32 src, ktl, kts;
      if (t < T_INTRA) { const u32 c = t - T_CROSS; ktl = c >> 2; src = rank ^ (4 + (c & 3)); kts = KC; }
      else { const u32 c = t - T_INTRA; ktl = c / 3; src = rank ^ (1 + c - ktl * 3); kts = KI; }
      const u32 base = IN_OFF + src * IN_SRC + ktl * 4 * BLK;
      const u32 sa = base + (ri * kts * 4 + u) * BLK + (HALF ? hsel * 8192 : 0), sw = base + (ci * kts * 4 + 2 + v) * BLK;
      if (HALF) dma3(dst + wid * 1024, dst + BLK + wid * 2048, rs, vaw[0], sa, rs, vbw[0], vbw[1], sw);
      else dma4(dst + wid * 2048, rs, vaw[0], vaw[1], sa, rs, vbw[0], vbw[1], sw);
    }
  };
  const u32 fop = (lane >> 1) & 1, flev = fop ? ci : ri, fuv = fop ? v : u, fq = lane >> 2;
  const u32 fh = (HALF && !fop) ? hsel : (lane & 1);
  const u32 fck = min(fq >> 2, KC - 1), fik = min(fq / 3, KI - 1);
  const char* fcross = ws + FI_OFF + ((rank ^ (4 + (fq & 3))) * 320 + ((flev * KC + fck) * 4 + fop * 2 + fuv) * 2 + fh) * 4;
  const char* fintra = ws + FI_OFF + ((rank ^ (1 + fq % 3)) * 320 + ((flev * KI + fik) * 4 + fop * 2 + fuv) * 2 + fh) * 4;

  while (__builtin_amdgcn_s_memrealtime() - tstart < (u64)lev * STAGGER) __builtin_amdgcn_s_sleep(16);
  f32x16 acc[MI] = {};
  struct Frag { bf16x8 w[4], x[MI][4]; };
  auto read = [&](Frag& f, int t) {
    const LAS char* st = lds + (t & (NSTAGE - 1)) * STAGE;
#pragma unroll
    for (int ks = 0; ks < 4; ++ks) {
      const bf16x4 w0 = __builtin_amdgcn_ds_read_tr16_b64_v4bf16((LAS bf16x4*)(st + wo[0] + ks * 4096));
      const bf16x4 w1 = __builtin_amdgcn_ds_read_tr16_b64_v4bf16((LAS bf16x4*)(st + wo[1] + ks * 4096));
      f.w[ks] = __builtin_shufflevector(w0, w1, 0, 1, 2, 3, 4, 5, 6, 7);
#pragma unroll
      for (int mi = 0; mi < MI; ++mi) f.x[mi][ks] = *(const LAS bf16x8*)(st + xo[ks] + mi * 4096);
    }
  };
  auto mma = [&](const Frag& f) {
#pragma unroll
    for (int ks = 0; ks < 4; ++ks)
#pragma unroll
      for (int mi = 0; mi < MI; ++mi) acc[mi] = __builtin_amdgcn_mfma_f32_32x32x16_bf16(f.w[ks], f.x[mi][ks], acc[mi], 0, 0, 0);
  };
  int next_wait = T_CROSS;
  u64 need = 0xFFFF, have = 0;
  auto step = [&](int t, const Frag& fc, Frag& fn) {
    const int tn = t + NSTAGE;
    if (tn == next_wait && tn < NT) {
      if (tn == T_INTRA) { need = 0xFFF; have = 0; }
      if (wid == 0 && (have & need) != need) {
        if (ASYNC_POLL) have = __ballot(LD_LDS(fimg + (tn < T_INTRA ? 0 : 64)) >= seq);
        if ((have & need) != need) {
          bool bad = false;
          have = wave_wait<SL_GEMM>(tn < T_INTRA ? fcross : fintra, need, seq, tmo, bad);
          if (bad) *lbad = 1;
        }
      }
      if (tn == T_CROSS) { STAMP(sbase); }
      if (tn == NT - 3) { STAMP(sbase + 1); }
      if (tn < T_INTRA) { next_wait += 4; need = (need << 16) | 0xFFFF; }
      else { next_wait += 3; need = (need << 12) | 0xFFF; }
    }
    if (t < NT - 3) WAIT_VM(2 * NDMA); else if (t == NT - 3) WAIT_VM(NDMA); else WAIT_VM(0);
    __builtin_amdgcn_s_waitcnt(0xC07F);
    CBAR();
    __builtin_amdgcn_s_barrier();
    CBAR();
    read(fn, t + 1);
    if (tn < NT) issue(tn);
    if (ASYNC_POLL && wid == 0 && next_wait < NT && (have != ~0ull || next_wait <= T_INTRA)) {
      const bool c = next_wait < T_INTRA && have != ~0ull;
      dma_flags(lds0 + LDS_CTL + (c ? 256 : 512), c ? fcross : fintra);
    }
    mma(fc);
  };
  static_assert(NSTAGE == 4 && T_CROSS >= NSTAGE, "prologue stages own tiles only");
  issue(0); issue(1); issue(2); issue(3);
  Frag fa, fb;
  WAIT_VM(3 * NDMA);
  __builtin_amdgcn_s_barrier();
  read(fa, 0);
  for (int t = 0; t < NT; t += 2) { step(t, fa, fb); step(t + 1, fb, fa); }
  STAMP(sbase + 2);
  __syncthreads();
  constexpr u32 EROW = 528;
#pragma unroll
  for (int mi = 0; mi < MI; ++mi) {
    const u32 mr = wm * 32 * MI + mi * 32 + (lane & 31);
#pragma unroll
    for (int gq = 0; gq < 4; ++gq) {
      const f32x4 o = {acc[mi][4 * gq], acc[mi][4 * gq + 1], acc[mi][4 * gq + 2], acc[mi][4 * gq + 3]};
      *(LAS f32x4*)(lds + mr * EROW + (wn * 32 + 8 * gq + 4 * h) * 4) = o;
    }
  }
  __syncthreads();
  const u32 bad = *lbad ? NAN2 : 0u;
  const u32 keep = rank >> 2;
  const bool do_exp = !HALF || hsel != keep, do_keep = !HALF || hsel == keep;
  const u32 e16 = P16_OFF + (sb * 64 + m) * 16384 + wid * 1024, e32 = P32_OFF + (sb * 64 + m) * 32768 + wid * 2048;
  if (do_exp) {
#pragma unroll
    for (int part = 0; part < 2; ++part) {
      const LAS char* se = lds + ((HALF ? 0 : (keep ^ 1) * 64) + part * 32 + (tid >> 4)) * EROW + (tid & 15) * 32;
      const f32x4 c0 = *(const LAS f32x4*)se, c1 = *(const LAS f32x4*)(se + 16);
      const bf16x8 o = {(__bf16)c0[0], (__bf16)c0[1], (__bf16)c0[2], (__bf16)c0[3],
                        (__bf16)c1[0], (__bf16)c1[1], (__bf16)c1[2], (__bf16)c1[3]};
      st128(__builtin_bit_cast(u32x4, o) | bad, rs, lane * 16, e16 + part * 8192);
    }
  }
  if (do_keep) {
#pragma unroll
    for (int part = 0; part < 2; ++part) {
      const LAS char* sk = lds + ((HALF ? 0 : keep * 64) + part * 32 + (tid >> 4)) * EROW + (tid & 15) * 32;
      const u32x4 c0 = *(const LAS u32x4*)sk, c1 = *(const LAS u32x4*)(sk + 16);
      st128(c0 | bad, rs, lane * 16, e32 + part * 16384);
      st128(c1 | bad, rs, lane * 16, e32 + part * 16384 + 1024);
    }
  }
  WAIT_VM(0);
  __syncthreads();
  if (tid == 0 && do_exp) st_sys(p.ws[4 | ((rank & 3) ^ sb)] + FPR_OFF + ((rank & 3) * 64 + m) * 4, seq);
  if (tid == 64 && do_keep) st_sys(ws + FPL_OFF + (sb * 64 + m) * 4, seq);
  if constexpr (!HALF) asm volatile(".Lfz_e2:");
  STAMP(sbase + 3);
}

__device__ __forceinline__ void gemm_role(const Params& p, u32 g, u32 k, u32 seq, LAS char* lds) {
  const u64 tstart = __builtin_amdgcn_s_memrealtime();
  if (threadIdx.x < 64) __builtin_amdgcn_fence(__ATOMIC_ACQUIRE, "");
  static_assert(GEMM_WGS == 24, "unit schedule below");
  if (k < 2 * NHALF) gemm_tile<true>(p, g, k >> 1, k & 1, seq, tstart, 1, lds);
  for (u32 n = k < 2 * NHALF ? GEMM_WGS - NHALF + k : k - NHALF; n < 32; n += GEMM_WGS)
    gemm_tile<false>(p, g, n, 0, seq, tstart, n < GEMM_WGS - NHALF ? 1 : 8, lds);
}

__global__ void __launch_bounds__(THREADS, 2) fused_gemm_allreduce_silu(Params p) {
  extern __shared__ __attribute__((aligned(16))) char smem[];
  asm volatile(".Lfz_s:");
  const int lane = threadIdx.x & 63;
  const u32 wid = rfl(threadIdx.x >> 6);
  LAS u32* ctl = (LAS u32*)smem;
  LAS u32* lctl = (LAS u32*)(smem + LDS_CTL);
  u32 slot = (blockIdx.x & ~7u) | ((u32)__builtin_amdgcn_s_getreg((3 << 11) | 20) & 7);
  auto is_push = [&]() { return slot < 8 * ENG_WGS && (slot & 7) != 0 && wid < PUSH_WAVES; };
  if (__builtin_expect(is_push(), 1)) {
    if (push_role<false>(p, slot, wid, lane, lctl)) return;
  } else {
    STAMPW(0, 2);
    if (wid == 7) {
      if (lane < 32) ctl[lane] = 0;
      if (lane == 0) {
        GAS u32* ctr = (GAS u32*)(p.ws[0] + CTR_OFF) + blockIdx.x;
        GAS u32* clm = (GAS u32*)(p.ws[0] + CLM_OFF);
        const u32 c = __hip_atomic_load(ctr, __ATOMIC_RELAXED, __HIP_MEMORY_SCOPE_AGENT);
        u32 sl = slot;
#pragma unroll 1
        for (int n = 0; n < 4 * GRID; ++n) {
          if (__hip_atomic_fetch_add(clm + sl * 16, 1u, __ATOMIC_RELAXED, __HIP_MEMORY_SCOPE_AGENT) == c) break;
          __hip_atomic_fetch_add(clm + sl * 16, ~0u, __ATOMIC_RELAXED, __HIP_MEMORY_SCOPE_AGENT);
          sl = (sl + 1) & (GRID - 1);
        }
        __hip_atomic_store(ctr, c + 1, __ATOMIC_RELAXED, __HIP_MEMORY_SCOPE_AGENT);
        lctl[0] = 0; lctl[1] = sl; lctl[2] = c + 1;
      }
    }
    if (wid >= 2 && wid < 6 && slot < 8 * ENG_WGS) {
      u64 pc; int ls, l0, l1, l2;
      asm volatile("s_getpc_b64 %0\n.Lfz_pc:\n\ts_mov_b32 %1, .Lfz_s-.Lfz_pc\n\ts_mov_b32 %2, .Lfz_e0-.Lfz_pc\n\t"
                   "s_mov_b32 %3, .Lfz_e1-.Lfz_pc\n\ts_mov_b32 %4, .Lfz_e2-.Lfz_pc"
                   : "=s"(pc), "=s"(ls), "=s"(l0), "=s"(l1), "=s"(l2));
      const int o = ls + ((wid - 2) * 64 + lane) * 128;
      if (o < max(max(0, l0), max(l1, l2))) dma_plain((u32)(size_t)lctl + 768, (const char*)pc + o);
    }
    asm volatile("s_waitcnt lgkmcnt(0)" ::: "memory");
    __builtin_amdgcn_s_barrier();
    slot = rfl(LD_LDS(lctl + 1));
    WAIT_VM(0);
  }
  if (__builtin_expect(is_push(), 0)) {
    push_role<true>(p, slot, wid, lane, lctl);
    return;
  }
  const u32 seq = rfl(LD_LDS(lctl + 2));
  const u32 g = slot & 7, s = slot >> 3;
  const u32 peer = ((u32)p.rank ^ g) & (NR - 1);
  if (s < ENG_WGS) {
    if (g == 0) return;
    if (wid < PUSH_WAVES + PULL_WAVES) {
      pull_role(p, peer, (wid - PUSH_WAVES) * ENG_WGS + s, lane, seq, ctl);
    } else if (wid < PUSH_WAVES + PULL_WAVES + FIN_WAVES && g >= 4) {
      fin_role(p, peer, s, wid - PUSH_WAVES - PULL_WAVES, lane, seq, ctl);
    }
  } else {
    gemm_role(p, g, s - ENG_WGS, seq, (LAS char*)smem);
  }
  STAMP(7);
}

}

extern "C" void kernel_launch(int rank, int nranks, ncclComm_t comm,
                              hipStream_t stream,
                              void* const* d_in, const int* in_sizes, int n_in,
                              void* d_out, int out_size,
                              const int* in_dtypes, int out_dtype,
                              void* const* d_out_all,
                              void* const* d_ws_all, size_t ws_bytes) {
  if (nranks != NR || (unsigned)rank >= (unsigned)NR || n_in < 2 || ws_bytes < WS_USED ||
      in_sizes[0] != 4096 * 512 || in_sizes[1] != 512 * 4096 || out_size != 4096 * 4096) {
    fprintf(stderr, "kernel_launch: unexpected problem shape\n");
    return;
  }
  HIP_CHECK(hipFuncSetAttribute((const void*)fused_gemm_allreduce_silu,
                                hipFuncAttributeMaxDynamicSharedMemorySize, LDS_BYTES));
  Params p;
  p.x = (const char*)d_in[0];
  p.w = (const char*)d_in[1];
  static const int phys[NR] = {0, 2, 3, 6, 1, 4, 5, 7};
  p.rank = 0; p.pad = 0;
  for (int i = 0; i < NR; ++i) if (phys[i] == rank) p.rank = i;
  for (int i = 0; i < NR; ++i) { p.ws[i] = (char*)d_ws_all[phys[p.rank ^ i]]; p.out[i] = (char*)d_out_all[phys[p.rank ^ i]]; }
  hipLaunchKernelGGL(fused_gemm_allreduce_silu, dim3(GRID), dim3(THREADS), LDS_BYTES, stream, p);
  HIP_CHECK(hipGetLastError());
}
```

```cpp
#include <hip/hip_runtime.h>
#include <rccl/rccl.h>
#include <cstdio>
#include <type_traits>

#define HIP_CHECK(expr) do { hipError_t _e = (expr); if (_e != hipSuccess) { \
    fprintf(stderr, "HIP error %s at %s:%d\n", hipGetErrorString(_e), __FILE__, __LINE__); \
    return; } } while (0)

namespace {

typedef unsigned int u32;
typedef unsigned long long u64;
typedef u32 u32x4 __attribute__((ext_vector_type(4)));
typedef float f32x4 __attribute__((ext_vector_type(4)));
typedef __bf16 bf16x4 __attribute__((ext_vector_type(4)));
typedef __bf16 bf16x8 __attribute__((ext_vector_type(8)));
typedef float f32x16 __attribute__((ext_vector_type(16)));
#define LAS __attribute__((address_space(3)))
#define GAS __attribute__((address_space(1)))

constexpr int NR = 8;
constexpr u32 X_BYTES = 4096u * 512u * 2u;
constexpr u32 W_BYTES = 512u * 4096u * 2u;
constexpr u32 OUT_BYTES = 4096u * 4096u * 2u;
constexpr u32 KI = 5, KC = 3;

constexpr u32 BLK = 16384;
constexpr u32 IN_OFF = 4096;
constexpr u32 IN_SRC = KI * 32 * BLK + 4096;
constexpr u32 P16_OFF = IN_OFF + NR * IN_SRC;
constexpr u32 P32_OFF = P16_OFF + 256 * 16384;
constexpr u32 DATA_BYTES = P32_OFF + 256 * 32768;
constexpr u32 FI_OFF = DATA_BYTES;
constexpr u32 FPR_OFF = FI_OFF + NR * 320 * 4;
constexpr u32 FPL_OFF = FPR_OFF + 256 * 4;
constexpr u32 FF_OFF = FPL_OFF + 256 * 4;
constexpr u32 CTR_OFF = FF_OFF + NR * 256 * 4;
constexpr u32 CLM_OFF = CTR_OFF + 256 * 4;
constexpr u32 TMO_OFF = CLM_OFF + 256 * 64;
constexpr u32 TICK_OFF = TMO_OFF + 128;
constexpr u32 TS_OFF = TICK_OFF + 16 * 128;
constexpr u32 TR_OFF = TS_OFF + 256 * 16 * 8;
constexpr u32 WS_USED = TR_OFF + 3 * 8 * 1024 * 8;

constexpr u64 DEADLINE_TICKS = 30000000ull;

#define TRIG_BACK_ 24
#define NS_HALF_ 6
#define MDIR_ 9
#define SL_PULL 2
#define SL_FIN 8
#define FIN_WAVES_ 4
#define SL_GEMM 8
#define PUSH_WAVES_ 2
#define PULL_WAVES_ 2
#define PUSH_D_ 4
#define PUSH_LAUX 0
#define TLB_WARM 0
#define CODE_WARM 0
#define WIDE_START 1
#define ACQ 3
#define STAGGER 300
constexpr int ENG_WGS = 8, GEMM_WGS = 24;
constexpr int PUSH_WAVES = PUSH_WAVES_, PULL_WAVES = PULL_WAVES_, FIN_WAVES = FIN_WAVES_, PUSH_D = PUSH_D_;
static_assert(PUSH_WAVES + PULL_WAVES + FIN_WAVES <= 8, "waves per engine workgroup");
constexpr u32 TRIG_BACK = TRIG_BACK_;
constexpr u32 NAN2 = 0x7FC07FC0u;
constexpr int GRID = 8 * (ENG_WGS + GEMM_WGS);
static_assert(GRID == 256, "slots wrap at a power of two");
constexpr int THREADS = 512;
constexpr int NS_HALF = NS_HALF_;
constexpr u32 MDIR = MDIR_;
constexpr u32 LDS_CTL = 144 * 1024;
constexpr u32 LDS_BYTES = LDS_CTL + 1024;
constexpr int AUX_SYS = 17;

struct Params {
  const char* x;
  const char* w;
  int rank;
  int pad;
  char* ws[NR];
  char* out[NR];
};

__device__ __forceinline__ u32 rfl(u32 v) { return __builtin_amdgcn_readfirstlane(v); }
__device__ __forceinline__ __amdgpu_buffer_rsrc_t mk_rsrc(const void* p, u32 n) {
  const u64 a = (u64)p;
  const u64 u = ((u64)rfl((u32)(a >> 32)) << 32) | rfl((u32)a);
  return __builtin_amdgcn_make_buffer_rsrc((void*)u, 0, n, 0x00020000);
}
__device__ __forceinline__ u32 ld_sys(const char* p) {
  return __hip_atomic_load((const GAS u32*)p, __ATOMIC_RELAXED, __HIP_MEMORY_SCOPE_SYSTEM);
}
__device__ __forceinline__ void st_sys(char* p, u32 v) {
  __hip_atomic_store((GAS u32*)p, v, __ATOMIC_RELAXED, __HIP_MEMORY_SCOPE_SYSTEM);
}
#define WAIT_VM(n) asm volatile("s_waitcnt vmcnt(%0)" ::"n"(n) : "memory")
#define CBAR() asm volatile("" ::: "memory")
#define LD_LDS(q) __hip_atomic_load(q, __ATOMIC_RELAXED, __HIP_MEMORY_SCOPE_WORKGROUP)
#define ST_LDS(q, v) __hip_atomic_store(q, v, __ATOMIC_RELAXED, __HIP_MEMORY_SCOPE_WORKGROUP)

__device__ __forceinline__ void st128(u32x4 v, __amdgpu_buffer_rsrc_t r, u32 voff, u32 soff) {
  __builtin_amdgcn_raw_buffer_store_b128(v, r, voff, soff, AUX_SYS);
  asm volatile("s_nop 1" ::"v"(v) : "memory");
}

template <int SLEEP>
__device__ __forceinline__ u64 wave_wait(const char* flag, u64 need, u32 seq, char* tmo, bool& bad) {
  const u64 t0 = __builtin_amdgcn_s_memrealtime();
  for (u32 it = 1;; ++it) {
    const u64 ok = __ballot(ld_sys(flag) >= seq);
    if ((ok & need) == need) return ok;
    if ((it & 15) == 0 || bad) {
      const u32 t = __hip_atomic_load((const GAS u32*)tmo, __ATOMIC_RELAXED, __HIP_MEMORY_SCOPE_AGENT);
      if (rfl(t) == seq || __builtin_amdgcn_s_memrealtime() - t0 > DEADLINE_TICKS) {
        __hip_atomic_store((GAS u32*)tmo, seq, __ATOMIC_RELAXED, __HIP_MEMORY_SCOPE_AGENT);
        bad = true;
        return ~0ull;
      }
    }
    __builtin_amdgcn_s_sleep(SLEEP);
  }
}

#define STAMPW(i, w) do {} while (0)
#define STAMP(i) STAMPW(i, 0)
#define NOW(x) do {} while (0)
#define TRACEJ(kind, job) do {} while (0)

__device__ __forceinline__ void dma4(u32 dst, __amdgpu_buffer_rsrc_t ra, u32 va0, u32 va1, u32 sa,
                                     __amdgpu_buffer_rsrc_t rb, u32 vb0, u32 vb1, u32 sb) {
  u32 keep;
  dst = rfl(dst); sa = rfl(sa); sb = rfl(sb);
  asm volatile(
      "s_mov_b32 %0, m0\n\t"
      "s_nop 4\n\t"
      "s_mov_b32 m0, %1\n\t"
      "s_nop 0\n\t"
      "buffer_load_dwordx4 %3, %5, %6 offen lds\n\t"
      "buffer_load_dwordx4 %4, %5, %6 offen offset:1024 lds\n\t"
      "s_mov_b32 m0, %2\n\t"
      "s_nop 0\n\t"
      "buffer_load_dwordx4 %7, %9, %10 offen lds\n\t"
      "buffer_load_dwordx4 %8, %9, %10 offen offset:1024 lds\n\t"
      "s_mov_b32 m0, %0"
      : "=&s"(keep)
      : "s"(dst), "s"(dst + BLK), "v"(va0), "v"(va1), "s"(ra), "s"(sa), "v"(vb0), "v"(vb1), "s"(rb), "s"(sb)
      : "memory");
}

__device__ __forceinline__ void dma3(u32 dsta, u32 dstb, __amdgpu_buffer_rsrc_t ra, u32 va0, u32 sa,
                                     __amdgpu_buffer_rsrc_t rb, u32 vb0, u32 vb1, u32 sb) {
  u32 keep;
  dsta = rfl(dsta); dstb = rfl(dstb); sa = rfl(sa); sb = rfl(sb);
  asm volatile(
      "s_mov_b32 %0, m0\n\t"
      "s_nop 4\n\t"
      "s_mov_b32 m0, %1\n\t"
      "s_nop 0\n\t"
      "buffer_load_dwordx4 %3, %4, %5 offen lds\n\t"
      "s_mov_b32 m0, %2\n\t"
      "s_nop 0\n\t"
      "buffer_load_dwordx4 %6, %8, %9 offen lds\n\t"
      "buffer_load_dwordx4 %7, %8, %9 offen offset:1024 lds\n\t"
      "s_mov_b32 m0, %0"
      : "=&s"(keep)
      : "s"(dsta), "s"(dstb), "v"(va0), "s"(ra), "s"(sa), "v"(vb0), "v"(vb1), "s"(rb), "s"(sb)
      : "memory");
}

__device__ __forceinline__ void dma_flags(u32 dst, const char* src) {
  u32 keep;
  dst = rfl(dst);
  asm volatile("s_mov_b32 %0, m0\n\ts_nop 0\n\ts_mov_b32 m0, %1\n\ts_nop 0\n\tglobal_load_lds_dword %2, off sc0 sc1\n\ts_mov_b32 m0, %0"
               : "=&s"(keep) : "s"(dst), "v"(src) : "memory");
}

__device__ __forceinline__ void dma_plain(u32 dst, const char* src) {
  u32 keep;
  dst = rfl(dst);
  asm volatile("s_mov_b32 %0, m0\n\ts_nop 0\n\ts_mov_b32 m0, %1\n\ts_nop 0\n\tglobal_load_lds_dword %2, off\n\ts_mov_b32 m0, %0"
               : "=&s"(keep) : "s"(dst), "v"(src) : "memory");
}

template <int I, int N, class F>
__device__ __forceinline__ void static_for(F&& f) {
  if constexpr (I < N) { f(std::integral_constant<int, I>{}); static_for<I + 1, N>(f); }
}

__device__ __forceinline__ u32 sup(u32 grp, u32 c, u32 u) { return grp ? c + 2 * u : 2 * c + u; }
__device__ __forceinline__ u32 srow(u32 r, u32 u) { return sup(r >> 2, (r >> 1) & 1, u); }
__device__ __forceinline__ u32 scol(u32 r, u32 v) { return sup(r >> 2, r & 1, v); }
__device__ __forceinline__ void tile_rc(u32 m, u32& ri, u32& ci) {
  const u32 l = (m >= 1) + (m >= 4) + (m >= 9) + (m >= 16) + (m >= 25) + (m >= 36) + (m >= 49);
  const u32 idx = m - l * l;
  ri = idx < l ? l : idx - l;
  ci = idx < l ? idx : l;
}
__device__ __forceinline__ u32 fin_off(u32 r, u32 m, u32 sb, u32 part) {
  u32 ri, ci;
  tile_rc(m, ri, ci);
  return (srow(r, sb >> 1) * 1024 + ri * 128 + (r >> 2) * 64 + part * 32) * 8192 + (scol(r, sb & 1) * 1024 + ci * 128) * 2;
}

struct Job { u32 src, sstride, dst, flag, blo, bhi, voff; };

__device__ __forceinline__ Job decode_push(u32 job, u32 rank, u32 peer, u64 b0, u64 b1, u32 vnarrow, u32 vwide) {
  Job j;
  const bool cross = (rank ^ peer) >= 4;
  const u32 lk = job >> 3, op = (job >> 2) & 1, uv = (job >> 1) & 1, half = job & 1;
  const u32 lev = cross ? lk / KC : lk / KI;
  const u32 ktl = lk - lev * (cross ? KC : KI), kt = cross ? KI + ktl : ktl;
  j.blo = op ? (u32)b1 : (u32)b0; j.bhi = op ? (u32)(b1 >> 32) : (u32)(b0 >> 32); j.voff = op ? vwide : vnarrow;
  j.src = op ? (kt * 64 + half * 32) * 8192 + (scol(peer, uv) * 1024 + lev * 128) * 2
             : (srow(peer, uv) * 1024 + lev * 128 + half * 64) * 1024 + kt * 128;
  j.sstride = op ? 4 * 8192 : 8 * 1024;
  j.dst = IN_OFF + rank * IN_SRC + job * 8192;
  j.flag = FI_OFF + (rank * 320 + job) * 4;
  return j;
}

__device__ __forceinline__ GAS u32* ticket_ptr(char* ws, u32 idx) {
  u32 zero = 0;
  asm volatile("" : "+v"(zero));
  return (GAS u32*)(ws + TICK_OFF + idx * 128 + zero);
}

__device__ __forceinline__ u32x4 push_ld(const Job& j, int i) {
  const auto r = mk_rsrc((const void*)(((u64)j.bhi << 32) | j.blo), X_BYTES);
  return __builtin_amdgcn_raw_buffer_load_b128(r, j.voff, rfl(j.src + i * j.sstride), PUSH_LAUX);
}

template <bool claimed>
__device__ __forceinline__ bool push_role(const Params& p, u32& slot, u32 wid, int lane, LAS u32* lctl) {
  constexpr int D = PUSH_D, JI = 8;
  constexpr u32 NW = ENG_WGS * PUSH_WAVES;
  NOW(tA);
  const u32 g = slot & 7, w = wid * ENG_WGS + (slot >> 3);
  static_assert(X_BYTES == W_BYTES, "one size for both input shards");
  u64 b0 = (u64)p.x, b1 = (u64)p.w, bw = (u64)p.ws[g];
  u32 rank = p.rank;
  asm volatile("" : "+s"(b0), "+s"(b1), "+s"(bw), "+s"(rank));
  const u32 peer = rank ^ g;
  const u32 NJOBS = (g >= 4 ? KC : KI) * 64;
  char* wsp = (char*)bw;
  const auto rd = mk_rsrc(wsp, DATA_BYTES);
  const u32 vnarrow = (lane >> 3) * 1024 + (lane & 7) * 16;
  const u32 vwide = (lane >> 4) * 8192 + (lane & 15) * 16;
  Job cur = decode_push(w, rank, peer, b0, b1, vnarrow, vwide);
  u32x4 v[D], v2[JI - D];
#pragma unroll
  for (int i = 0; i < D; ++i) { v[i] = push_ld(cur, i); CBAR(); }
  if (WIDE_START) {
#pragma unroll
    for (int i = D; i < JI; ++i) { v2[i - D] = push_ld(cur, i); CBAR(); }
  }
  NOW(tB);
  GAS u32* tick = ticket_ptr(p.ws[0], peer);
  u32 seq = claimed ? LD_LDS(lctl + 2)
                    : __hip_atomic_load((GAS u32*)(p.ws[0] + CTR_OFF) + blockIdx.x, __ATOMIC_RELAXED, __HIP_MEMORY_SCOPE_AGENT) + 1;
  CBAR();
  u32 nid = NW + w, pflag = 0;
  auto round = [&](auto kind_c) {
    constexpr int KIND = decltype(kind_c)::value;
    constexpr bool FIRST = KIND == 0, LAST = KIND == 3;
    Job nxt = cur;
    if constexpr (!LAST) { nxt = decode_push(nid, rank, peer, b0, b1, vnarrow, vwide); TRACEJ(0, nid); }
    u32 tk = 0;
#pragma unroll
    for (int i = 0; i < JI; ++i) {
      if (!FIRST && i == D - 1) {
        WAIT_VM(D + (LAST ? (JI - D < D - 1 ? JI - D : D - 1) : D - 1));
        if (lane == 0) st_sys(wsp + pflag, seq);
        CBAR();
      }
      st128(FIRST && WIDE_START && i >= D ? v2[i >= D ? i - D : 0] : v[i % D], rd, lane * 16, rfl(cur.dst) + i * 1024);
      if (i + D < JI) { if (!(FIRST && WIDE_START)) v[i % D] = push_ld(cur, i + D); }
      else if (!LAST) v[i % D] = push_ld(nxt, i + D - JI);
      CBAR();
      if (KIND == 2 && i == 0) {
        if (lane == 0) tk = __hip_atomic_fetch_add(tick, 1u, __ATOMIC_RELAXED, __HIP_MEMORY_SCOPE_AGENT);
        CBAR();
      }
    }
    pflag = cur.flag;
    cur = nxt;
    if constexpr (KIND < 2) nid += NW;
    else if constexpr (!LAST) nid = rfl(tk) - (seq - 1) * (NJOBS - 3 * NW) + 4 * NW;
  };
  round(std::integral_constant<int, 0>{});
  NOW(tE);
  round(std::integral_constant<int, 1>{});
  seq = rfl(seq);
  if (!claimed) __builtin_amdgcn_s_barrier();
  CBAR();
  const u32 slot2 = rfl(LD_LDS(lctl + 1));
  if (__builtin_expect(slot2 != slot, 0)) { WAIT_VM(0); slot = slot2; return false; }
  NOW(tC);
  do round(std::integral_constant<int, 2>{}); while (nid < NJOBS);
  round(std::integral_constant<int, 3>{});
  WAIT_VM(0);
  if (lane == 0) st_sys(wsp + pflag, seq);
  STAMP(3);
  return true;
}

__device__ __forceinline__ float silu(float a) {
  const float c = fminf(fmaxf(a, -60.f), 60.f);
  return a * __builtin_amdgcn_rcpf(1.f + __expf(-c));
}

constexpr u32 MB_REQ = 0, MB_DONE = 4, MB_GONE = 8, MB_RUN = 9, MB_UNIT = 10, MB_BUF = 1024;

__device__ __forceinline__ void pull_role(const Params& p, u32 peer, u32 w, int lane, u32 seq, LAS u32* ctl) {
  constexpr u32 NW = ENG_WGS * PULL_WAVES, NF = 1024;
  const u32 rank = p.rank;
  const bool cross = (rank ^ peer) >= 4;
  char* ws = p.ws[0];
  char* tmo = ws + TMO_OFF;
  const u64 bf = (u64)p.out[rank ^ peer], bp = (u64)p.ws[rank ^ peer];
  const u32 vwide = (lane >> 4) * 8192 + (lane & 15) * 16;
  GAS u32* tick = ticket_ptr(ws, 8 + peer);
  const u32 pbase = (lane & 1) ? FI_OFF + (peer * 320 + (cross ? KC : KI) * 64 - TRIG_BACK) * 4 : FF_OFF + peer * 1024;
  const u32 pmask = (lane & 1) ? 0u : ~0u;
  const u64 t0 = __builtin_amdgcn_s_memrealtime();
  u32 tf = w, tf2 = NW + w, cur = 0, curf = 0, lastf = 0;
  u32 nsrc = 0, nf = 0;
  bool bad = false;
  auto poll = [&]() -> u32 { return ld_sys(ws + (pbase + (rfl(((tf >> 2) & 255) * 4) & pmask))); };
  u32 pv = poll();
  u32x4 v[4];
  auto pick = [&]() -> bool {
    const u64 pend = __ballot(lane < FIN_WAVES && LD_LDS(ctl + MB_REQ + (lane & 3)) != 0);
    if (pend) {
      const u32 f = __builtin_ctzll(pend);
      u32 r = 0;
      if (lane == 0) r = __hip_atomic_exchange(ctl + MB_REQ + f, 0u, __ATOMIC_RELAXED, __HIP_MEMORY_SCOPE_WORKGROUP);
      r = rfl(r);
      if (r) { nsrc = r; nf = f + 1; return true; }
    }
    if (tf < NF && (bad || (__ballot(pv >= seq) & 3) == 3)) {
      nsrc = rfl(fin_off(peer, tf >> 4, (tf >> 2) & 3, 0) + (tf & 3) * 16 * 8192);
      nf = 0;
      return true;
    }
    return false;
  };
  auto put = [&](int i) {
    const auto ro = mk_rsrc(p.out[0], curf ? 0u : OUT_BYTES);
    st128(v[i], ro, vwide, rfl(cur) + i * 4 * 8192);
    if (curf) *(LAS u32x4*)((LAS char*)ctl + MB_BUF + (curf - 1) * 4096 + i * 1024 + lane * 16) = v[i];
  };
  auto delivered = [&]() {
    if (curf) {
      asm volatile("s_waitcnt lgkmcnt(0)" ::: "memory");
      if (lane == 0) __hip_atomic_fetch_add(ctl + MB_DONE + curf - 1, 1u, __ATOMIC_RELAXED, __HIP_MEMORY_SCOPE_WORKGROUP);
    }
  };
  auto round = [&](auto first_c) {
    constexpr bool FIRST = decltype(first_c)::value;
    u32 tk = 0;
    const bool isf = nf == 0;
    if (isf) { TRACEJ(1, tf); tf = tf2; lastf = nsrc; }
    if (lane == 0 && isf) tk = __hip_atomic_fetch_add(tick, 1u, __ATOMIC_RELAXED, __HIP_MEMORY_SCOPE_AGENT);
    pv = poll();
    CBAR();
    const auto rsrc = mk_rsrc((const void*)(isf ? bf : bp), isf ? OUT_BYTES : DATA_BYTES);
    const u32 vs = isf ? vwide : lane * 16, str = isf ? 4 * 8192 : 1024;
#pragma unroll
    for (int i = 0; i < 4; ++i) {
      if (!FIRST) put(i);
      v[i] = __builtin_amdgcn_raw_buffer_load_b128(rsrc, vs, rfl(nsrc + i * str), AUX_SYS);
      CBAR();
    }
    if (!FIRST) delivered();
    cur = nsrc; curf = nf;
    if (isf) tf2 = rfl(tk) - (seq - 1) * NF + 2 * NW;
  };
  for (u32 it = 1;; ++it) {
    if (pick()) {
      if (tf == w && nf == 0) { STAMPW(5, PUSH_WAVES); }
      round(std::true_type{});
      if (pick()) {
        round(std::false_type{});
        while (pick()) round(std::false_type{});
      }
#pragma unroll
      for (int i = 0; i < 4; ++i) put(i);
      delivered();
      continue;
    }
    if (tf >= NF && rfl(LD_LDS(ctl + MB_GONE)) == (cross ? FIN_WAVES : 0)) break;
    if ((it & 63) == 0) {
      const u32 t = __hip_atomic_load((const GAS u32*)tmo, __ATOMIC_RELAXED, __HIP_MEMORY_SCOPE_AGENT);
      if (rfl(t) == seq || __builtin_amdgcn_s_memrealtime() - t0 > DEADLINE_TICKS) {
        __hip_atomic_store((GAS u32*)tmo, seq, __ATOMIC_RELAXED, __HIP_MEMORY_SCOPE_AGENT);
        bad = true;
      }
    }
    __builtin_amdgcn_s_sleep(SL_PULL);
    if ((it & 3) == 0) pv = poll();
  }
  if (bad) {
    WAIT_VM(0);
    const u32x4 nanv = {NAN2, NAN2, NAN2, NAN2};
    for (int i = 0; i < 4; ++i) st128(nanv, mk_rsrc(p.out[0], OUT_BYTES), vwide, rfl(lastf) + i * 4 * 8192);
  }
  asm volatile(".Lfz_e0:");
  STAMPW(6, PUSH_WAVES);
}

__device__ __forceinline__ void fin_role(const Params& p, u32 peer, u32 s, u32 f, int lane, u32 seq, LAS u32* ctl) {
  static_assert(FIN_WAVES == 4, "one wave per 16 rows of a half tile");
  const u32 rank = p.rank, sb = peer & 3;
  char* ws = p.ws[0];
  char* tmo = ws + TMO_OFF;
  const auto rl = mk_rsrc(ws, DATA_BYTES);
  const auto ro = mk_rsrc(p.out[0], OUT_BYTES);
  const u32 vwide = (lane >> 4) * 8192 + (lane & 15) * 16;
  char* wsl = p.ws[lane & 7];
  u32 ndone = 0;
  for (u32 m = MDIR + s; m < 64; m += ENG_WGS) {
    const u32 lo = P32_OFF + (sb * 64 + m) * 32768 + f * 8192;
    const u32 dst = fin_off(rank, m, sb, 0) + f * 16 * 8192;
    bool bad = false;
    wave_wait<SL_FIN>(ws + ((lane & 1) ? FPL_OFF : FPR_OFF) + (sb * 64 + m) * 4, 3, seq, tmo, bad);
    if (m == MDIR + s) { STAMPW(8, PUSH_WAVES + PULL_WAVES); STAMPW(11, 7); }
    const u32 so = P16_OFF + ((rank & 3) * 64 + m) * 16384 + f * 4096;
    u32x4 rv[4], a[4], b[4];
    if (lane == 0) ST_LDS(ctl + MB_REQ + f, so);
#pragma unroll
    for (int i = 0; i < 4; ++i) {
      a[i] = __builtin_amdgcn_raw_buffer_load_b128(rl, lane * 16, lo + i * 2048, AUX_SYS);
      b[i] = __builtin_amdgcn_raw_buffer_load_b128(rl, lane * 16, lo + i * 2048 + 1024, AUX_SYS);
    }
    ++ndone;
    while (rfl(LD_LDS(ctl + MB_DONE + f)) != ndone) __builtin_amdgcn_s_sleep(1);
    CBAR();
#pragma unroll
    for (int i = 0; i < 4; ++i) rv[i] = *(const LAS u32x4*)((LAS char*)ctl + MB_BUF + f * 4096 + i * 1024 + lane * 16);
    if (m == MDIR + s) { STAMPW(9, PUSH_WAVES + PULL_WAVES);  }
#pragma unroll
    for (int i = 0; i < 4; ++i) {
      const u32x4 r = rv[i];
      float y[8];
#pragma unroll
      for (int e = 0; e < 4; ++e) {
        const u32 l0 = e < 2 ? a[i][2 * e] : b[i][2 * e - 4], l1 = e < 2 ? a[i][2 * e + 1] : b[i][2 * e - 3];
        y[2 * e] = silu(__uint_as_float(l0) + __uint_as_float(r[e] << 16));
        y[2 * e + 1] = silu(__uint_as_float(l1) + __uint_as_float(r[e] & 0xFFFF0000u));
      }
      const bf16x8 o = {(__bf16)y[0], (__bf16)y[1], (__bf16)y[2], (__bf16)y[3],
                        (__bf16)y[4], (__bf16)y[5], (__bf16)y[6], (__bf16)y[7]};
      u32x4 ov = __builtin_bit_cast(u32x4, o);
      if (bad) ov |= NAN2;
      st128(ov, ro, vwide, dst + i * 4 * 8192);
    }
    if (m == MDIR + s) { STAMPW(10, PUSH_WAVES + PULL_WAVES); }
    WAIT_VM(0);

    u32 arrived = 0;
    if (lane == 0) arrived = __hip_atomic_fetch_add(ctl + MB_UNIT + ndone - 1, 1u, __ATOMIC_RELAXED, __HIP_MEMORY_SCOPE_WORKGROUP);
    if (rfl(arrived) == FIN_WAVES - 1) {
      if (lane >= 1 && lane < NR) st_sys(wsl + FF_OFF + (rank * 256 + m * 4 + sb) * 4, seq);
      TRACEJ(2, 2 * m); TRACEJ(2, 2 * m + 1);
    }
    if (m == MDIR + s) { STAMPW(1, PUSH_WAVES + PULL_WAVES); }
  }
  if (lane == 0) __hip_atomic_fetch_add(ctl + MB_GONE, 1u, __ATOMIC_RELAXED, __HIP_MEMORY_SCOPE_WORKGROUP);
  asm volatile(".Lfz_e1:");
  STAMPW(2, PUSH_WAVES + PULL_WAVES + FIN_WAVES - 1);
}

template <bool HALF>
__device__ __forceinline__ void gemm_tile(const Params& p, u32 sb, u32 m, u32 hsel, u32 seq, u64 tstart, int sbase, LAS char* lds) {
  const int tid = threadIdx.x, lane = tid & 63;
  const u32 wid = rfl(tid >> 6);
  const u32 rank = p.rank;
  char* ws = p.ws[0];
  char* tmo = ws + TMO_OFF;
  const auto rx = mk_rsrc(p.x, X_BYTES);
  const auto rw = mk_rsrc(p.w, W_BYTES);
  const auto rs = mk_rsrc(ws, DATA_BYTES);
  const u32 u = sb >> 1, v = sb & 1;
  constexpr int NT = 32, T_CROSS = KI, T_INTRA = KI + 4 * KC, T_SPLIT = 4;
  constexpr int MI = HALF ? 1 : 2, NDMA = HALF ? 3 : 4, NS = HALF ? NS_HALF : 4;
  constexpr u32 XB = HALF ? BLK / 2 : BLK, SB = XB + BLK;
  static_assert(NS * SB <= LDS_CTL, "stages fit");

  u32 vao[2], vbo[2], vaw[2], vbw[2];
#pragma unroll
  for (int q = 0; q < 2; ++q) {
    const u32 ra = wid * 8 * MI + q * 8 + (lane >> 3), ca = (lane & 7) ^ ((ra >> 1) & 7);
    const u32 kb = wid * 8 + q * 4 + (lane >> 4), cbk = (lane & 15) ^ (((kb & 3) << 2) | ((kb >> 2) & 3));
    vao[q] = ra * 1024 + ca * 16 - q * 1024;  vaw[q] = ra * 128 + ca * 16 - q * 1024;
    vbo[q] = kb * 8192 + cbk * 16 - q * 1024; vbw[q] = kb * 256 + cbk * 16 - q * 1024;
  }
  const u32 lds0 = (u32)(size_t)lds;
  const u32 wm = wid & 1, wn = wid >> 1, h = lane >> 5;
  u32 xo[4], wo[2];
  {
    const u32 r = wm * 32 * MI + (lane & 31);
#pragma unroll
    for (int ks = 0; ks < 4; ++ks) xo[ks] = r * 128 + (((2 * ks + h) ^ ((r >> 1) & 7)) * 16);
    const u32 bk = (lane >> 4) & 1, q = (lane & 15) >> 2, pp = lane & 3;
#pragma unroll
    for (int t = 0; t < 2; ++t)
      wo[t] = XB + 256 * (8 * h + 4 * t + q) + 16 * ((4 * wn + 2 * bk + (pp >> 1)) ^ ((q << 2) | (2 * h + t))) + 8 * (pp & 1);
  }
  LAS u32* lbad = (LAS u32*)(lds + LDS_CTL);
  u32 ri, ci;
  tile_rc(m, ri, ci);
  const u32 row0 = srow(rank, u) * 1024 + ri * 128 + (HALF ? hsel * 64 : 0), col0 = scol(rank, v) * 1024 + ci * 128;
  u32 islot = 0, rslot = 0;
  auto next_slot = [&](u32 o) -> u32 { return o + SB == NS * SB ? 0 : o + SB; };
  auto issue_own = [&](int t) {
    const u32 dst = lds0 + islot, sa = row0 * 1024 + t * 128, sw = t * 64 * 8192 + col0 * 2;
    islot = next_slot(islot);
    if (HALF) dma3(dst + wid * 1024, dst + XB + wid * 2048, rx, vao[0], sa, rw, vbo[0], vbo[1], sw);
    else dma4(dst + wid * 2048, rx, vao[0], vao[1], sa, rw, vbo[0], vbo[1], sw);
  };
  u32 taba, tabw;
  {
    const bool cr = lane < T_INTRA;
    const u32 c = cr ? lane - T_CROSS : lane - T_INTRA, ktl = cr ? c >> 2 : c / 3;
    const u32 src = rank ^ (cr ? 4 + (c & 3) : 1 + c - ktl * 3), kts = cr ? KC : KI;
    const u32 base = IN_OFF + src * IN_SRC + ktl * 4 * BLK;
    taba = base + (ri * kts * 4 + u) * BLK + (HALF ? hsel * 8192 : 0);
    tabw = base + (ci * kts * 4 + 2 + v) * BLK;
  }
  auto issue = [&](int t) {
    const u32 dst = lds0 + islot, sa = __builtin_amdgcn_readlane(taba, t), sw = __builtin_amdgcn_readlane(tabw, t);
    islot = next_slot(islot);
    if (HALF) dma3(dst + wid * 1024, dst + XB + wid * 2048, rs, vaw[0], sa, rs, vbw[0], vbw[1], sw);
    else dma4(dst + wid * 2048, rs, vaw[0], vaw[1], sa, rs, vbw[0], vbw[1], sw);
  };
  const u32 fop = (lane >> 1) & 1, flev = fop ? ci : ri, fuv = fop ? v : u, fq = lane >> 2;
  const u32 fh = (HALF && !fop) ? hsel : (lane & 1);
  const u32 fck = min(fq >> 2, KC - 1), fik = min(fq / 3, KI - 1);
  const char* fcross = ws + FI_OFF + ((rank ^ (4 + (fq & 3))) * 320 + ((flev * KC + fck) * 4 + fop * 2 + fuv) * 2 + fh) * 4;
  const char* fintra = ws + FI_OFF + ((rank ^ (1 + fq % 3)) * 320 + ((flev * KI + fik) * 4 + fop * 2 + fuv) * 2 + fh) * 4;

  while (__builtin_amdgcn_s_memrealtime() - tstart < (u64)max(ri, ci) * STAGGER) __builtin_amdgcn_s_sleep(16);
  if (ACQ == 3 && sbase == 1 && wid == 0) __builtin_amdgcn_fence(__ATOMIC_ACQUIRE, "");
  f32x16 acc[MI] = {};
  struct Frag { bf16x8 w[4], x[MI][4]; };
  auto read = [&](Frag& f) {
    const LAS char* st = lds + rslot;
    rslot = next_slot(rslot);
#pragma unroll
    for (int ks = 0; ks < 4; ++ks) {
      const bf16x4 w0 = __builtin_amdgcn_ds_read_tr16_b64_v4bf16((LAS bf16x4*)(st + wo[0] + ks * 4096));
      const bf16x4 w1 = __builtin_amdgcn_ds_read_tr16_b64_v4bf16((LAS bf16x4*)(st + wo[1] + ks * 4096));
      f.w[ks] = __builtin_shufflevector(w0, w1, 0, 1, 2, 3, 4, 5, 6, 7);
#pragma unroll
      for (int mi = 0; mi < MI; ++mi) f.x[mi][ks] = *(const LAS bf16x8*)(st + xo[ks] + mi * 4096);
    }
  };
  auto mma = [&](const Frag& f) {
#pragma unroll
    for (int ks = 0; ks < 4; ++ks)
#pragma unroll
      for (int mi = 0; mi < MI; ++mi) acc[mi] = __builtin_amdgcn_mfma_f32_32x32x16_bf16(f.w[ks], f.x[mi][ks], acc[mi], 0, 0, 0);
  };
  int next_wait = T_CROSS;
  u64 need = 0xFFFF, have = 0;
  auto waitpoint = [&](int tn) {
    if (tn == T_INTRA) { need = 0xFFF; have = 0; }
    if (wid == 0 && (have & need) != need) {
      bool bad = false;
      have = wave_wait<SL_GEMM>(tn < T_INTRA ? fcross : fintra, need, seq, tmo, bad);
      if (bad) *lbad = 1;
    }
    if (tn == T_CROSS) { STAMP(sbase); }
    if (tn == NT - 3) { STAMP(sbase + 1); }
    if (tn < T_INTRA) { next_wait += 4; need = (need << 16) | 0xFFFF; }
    else { next_wait += 3; need = (need << 12) | 0xFFF; }
  };
  auto step = [&](auto rc, int tn, const Frag& fc, Frag& fn) {
    constexpr int R = decltype(rc)::value;
    if (tn >= 0 && __builtin_expect(tn == next_wait, 0)) waitpoint(tn);
    if constexpr (R >= 0) {
      WAIT_VM(R * NDMA);
      __builtin_amdgcn_s_waitcnt(0xC07F);
      CBAR();
      __builtin_amdgcn_s_barrier();
      CBAR();
      read(fn);
    }
    if (tn >= 0) issue(tn);
    mma(fc);
  };
  Frag fa, fb;
  auto tail = [&](auto nc) {
    constexpr int N = decltype(nc)::value;
    static_for<0, N / 2>([&](auto ic) {
      constexpr int i = decltype(ic)::value;
      step(std::integral_constant<int, N - 2 - 2 * i>{}, -1, fa, fb);
      step(std::integral_constant<int, N - 3 - 2 * i>{}, -1, fb, fa);
    });
  };
  static_assert(NS % 2 == 0 && NS >= T_SPLIT && T_SPLIT + 1 == T_CROSS, "run structure");
  for (int t = 0; t < T_SPLIT; ++t) issue_own(t);
  WAIT_VM((T_SPLIT - 1) * NDMA);
  __builtin_amdgcn_s_barrier();
  CBAR();
  read(fa);
  tail(std::integral_constant<int, T_SPLIT>{});
  islot = rslot = 0;
  __builtin_amdgcn_s_waitcnt(0xC07F);
  __builtin_amdgcn_s_barrier();
  issue_own(T_SPLIT);
  for (int t = T_CROSS; t < T_SPLIT + NS; ++t) {
    if (t == next_wait) { waitpoint(t); __builtin_amdgcn_s_barrier(); }
    issue(t);
  }
  WAIT_VM((NS - 1) * NDMA);
  __builtin_amdgcn_s_barrier();
  CBAR();
  read(fa);
  for (int tn = T_SPLIT + NS; tn < NT; tn += 2) {
    step(std::integral_constant<int, NS - 2>{}, tn, fa, fb);
    step(std::integral_constant<int, NS - 2>{}, tn + 1, fb, fa);
  }
  tail(std::integral_constant<int, NS>{});
  STAMP(sbase + 2);
  __syncthreads();
  constexpr u32 EROW = 528;
#pragma unroll
  for (int mi = 0; mi < MI; ++mi) {
    const u32 mr = wm * 32 * MI + mi * 32 + (lane & 31);
#pragma unroll
    for (int gq = 0; gq < 4; ++gq) {
      const f32x4 o = {acc[mi][4 * gq], acc[mi][4 * gq + 1], acc[mi][4 * gq + 2], acc[mi][4 * gq + 3]};
      *(LAS f32x4*)(lds + mr * EROW + (wn * 32 + 8 * gq + 4 * h) * 4) = o;
    }
  }
  __syncthreads();
  const u32 bad = *lbad ? NAN2 : 0u;
  const u32 keep = rank >> 2;
  const bool do_exp = !HALF || hsel != keep, do_keep = !HALF || hsel == keep;
  char* wsq = p.ws[4 | ((rank & 3) ^ sb)];
  if (do_exp) {
    const bool dir = HALF && m < MDIR;
    const auto re = mk_rsrc(dir ? wsq : ws, DATA_BYTES);
    const u32 e16 = P16_OFF + ((dir ? rank & 3 : sb) * 64 + m) * 16384 + wid * 1024;
#pragma unroll
    for (int part = 0; part < 2; ++part) {
      const LAS char* se = lds + ((HALF ? 0 : (keep ^ 1) * 64) + part * 32 + (tid >> 4)) * EROW + (tid & 15) * 32;
      const f32x4 c0 = *(const LAS f32x4*)se, c1 = *(const LAS f32x4*)(se + 16);
      const bf16x8 o = {(__bf16)c0[0], (__bf16)c0[1], (__bf16)c0[2], (__bf16)c0[3],
                        (__bf16)c1[0], (__bf16)c1[1], (__bf16)c1[2], (__bf16)c1[3]};
      st128(__builtin_bit_cast(u32x4, o) | bad, re, lane * 16, e16 + part * 8192);
    }
  }
  if (HALF && do_keep && m < MDIR) {
    f32x4 kc[2][2];
#pragma unroll
    for (int part = 0; part < 2; ++part) {
      const LAS char* sk = lds + (part * 32 + (tid >> 4)) * EROW + (tid & 15) * 32;
      kc[part][0] = *(const LAS f32x4*)sk; kc[part][1] = *(const LAS f32x4*)(sk + 16);
    }
    bool badw = bad != 0;
    wave_wait<2>(ws + FPR_OFF + (sb * 64 + m) * 4, 1, seq, tmo, badw);
    STAMP(5);
    u32x4 r[2];
#pragma unroll
    for (int part = 0; part < 2; ++part)
      r[part] = __builtin_amdgcn_raw_buffer_load_b128(rs, lane * 16, P16_OFF + (sb * 64 + m) * 16384 + wid * 1024 + part * 8192, AUX_SYS);
    const auto ro = mk_rsrc(p.out[0], OUT_BYTES);
#pragma unroll
    for (int part = 0; part < 2; ++part) {
      float y[8];
#pragma unroll
      for (int e = 0; e < 4; ++e) {
        y[2 * e] = silu(kc[part][e >> 1][(2 * e) & 3] + __uint_as_float(r[part][e] << 16));
        y[2 * e + 1] = silu(kc[part][e >> 1][(2 * e + 1) & 3] + __uint_as_float(r[part][e] & 0xFFFF0000u));
      }
      const bf16x8 o = {(__bf16)y[0], (__bf16)y[1], (__bf16)y[2], (__bf16)y[3],
                        (__bf16)y[4], (__bf16)y[5], (__bf16)y[6], (__bf16)y[7]};
      st128(__builtin_bit_cast(u32x4, o) | (badw ? NAN2 : 0u), ro, (lane >> 4) * 8192 + (lane & 15) * 16,
            (row0 + part * 32 + wid * 4) * 8192 + col0 * 2);
    }
    WAIT_VM(0);
    __syncthreads();
    if (tid >= 1 && tid < NR) st_sys(p.ws[tid] + FF_OFF + (rank * 256 + m * 4 + sb) * 4, seq);
    STAMP(6);
    return;
  }
  if (do_keep) {
    const u32 e32 = P32_OFF + (sb * 64 + m) * 32768 + wid * 2048;
#pragma unroll
    for (int part = 0; part < 2; ++part) {
      const LAS char* sk = lds + ((HALF ? 0 : keep * 64) + part * 32 + (tid >> 4)) * EROW + (tid & 15) * 32;
      const u32x4 c0 = *(const LAS u32x4*)sk, c1 = *(const LAS u32x4*)(sk + 16);
      st128(c0 | bad, rs, lane * 16, e32 + part * 16384);
      st128(c1 | bad, rs, lane * 16, e32 + part * 16384 + 1024);
    }
  }
  WAIT_VM(0);
  __syncthreads();
  if (tid == 0 && do_exp) st_sys(wsq + FPR_OFF + ((rank & 3) * 64 + m) * 4, seq);
  if (tid == 64 && do_keep) st_sys(ws + FPL_OFF + (sb * 64 + m) * 4, seq);
  if constexpr (!HALF) asm volatile(".Lfz_e2:");
  STAMP(sbase + 3);
}

__device__ __forceinline__ void gemm_role(const Params& p, u32 g, u32 k, u32 seq, LAS char* lds) {
  const u64 tstart = __builtin_amdgcn_s_memrealtime();
  if (threadIdx.x < 64 && (ACQ == 1 || (ACQ == 2 && k == 0))) __builtin_amdgcn_fence(__ATOMIC_ACQUIRE, "");
  static_assert(GEMM_WGS == 24, "unit schedule below");
  const u32 keep = p.rank >> 2;
  if (k < 16) gemm_tile<true>(p, g >= 4 ? (p.rank ^ g) & 3 : g, k, g >= 4 ? keep ^ 1 : keep, seq, tstart, 1, lds);
  const u32 arm = g & 1;
  u32 lev = 4, idx = k < 16 ? 8 + k : k - 16;
  for (;;) {
    const u32 cnt = lev + ((lev & 1) == arm);
    if (idx < cnt) break;
    idx -= cnt; ++lev;
  }
  gemm_tile<false>(p, g >> 1, lev * lev + (idx == lev ? 2 * lev : arm * lev + idx), 0, seq, tstart, k < 16 ? 8 : 1, lds);
}

__global__ void __launch_bounds__(THREADS, 2) fused_gemm_allreduce_silu(Params p) {
  extern __shared__ __attribute__((aligned(16))) char smem[];
  asm volatile(".Lfz_s:");
  const int lane = threadIdx.x & 63;
  const u32 wid = rfl(threadIdx.x >> 6);
  LAS u32* ctl = (LAS u32*)smem;
  LAS u32* lctl = (LAS u32*)(smem + LDS_CTL);
  u32 slot = (blockIdx.x & ~7u) | ((u32)__builtin_amdgcn_s_getreg((3 << 11) | 20) & 7);
  auto is_push = [&]() { return slot < 8 * ENG_WGS && (slot & 7) != 0 && wid < PUSH_WAVES; };
  if (__builtin_expect(is_push(), 1)) {
    if (push_role<false>(p, slot, wid, lane, lctl)) return;
  } else {
    STAMPW(0, 2);
    if (wid == 7) {
      if (lane < 32) ctl[lane] = 0;
      if (lane == 0) {
        GAS u32* ctr = (GAS u32*)(p.ws[0] + CTR_OFF) + blockIdx.x;
        GAS u32* clm = (GAS u32*)(p.ws[0] + CLM_OFF);
        const u32 c = __hip_atomic_load(ctr, __ATOMIC_RELAXED, __HIP_MEMORY_SCOPE_AGENT);
        u32 sl = slot;
#pragma unroll 1
        for (int n = 0; n < 4 * GRID; ++n) {
          if (__hip_atomic_fetch_add(clm + sl * 16, 1u, __ATOMIC_RELAXED, __HIP_MEMORY_SCOPE_AGENT) == c) break;
          __hip_atomic_fetch_add(clm + sl * 16, ~0u, __ATOMIC_RELAXED, __HIP_MEMORY_SCOPE_AGENT);
          sl = (sl + 1) & (GRID - 1);
        }
        lctl[0] = 0; lctl[1] = sl; lctl[2] = c + 1;
      }
    }
    asm volatile("s_waitcnt lgkmcnt(0)" ::: "memory");
    __builtin_amdgcn_s_barrier();
    slot = rfl(LD_LDS(lctl + 1));
    if (threadIdx.x == THREADS - 1)
      __hip_atomic_store((GAS u32*)(p.ws[0] + CTR_OFF) + blockIdx.x, LD_LDS(lctl + 2), __ATOMIC_RELAXED, __HIP_MEMORY_SCOPE_AGENT);
    WAIT_VM(0);
  }
  if (__builtin_expect(is_push(), 0)) {
    push_role<true>(p, slot, wid, lane, lctl);
    return;
  }
  const u32 seq = rfl(LD_LDS(lctl + 2));
  const u32 g = slot & 7, s = slot >> 3;
  const u32 peer = ((u32)p.rank ^ g) & (NR - 1);
  if (s < ENG_WGS) {
    if (g == 0) return;
    if (wid < PUSH_WAVES + PULL_WAVES) {
      pull_role(p, peer, (wid - PUSH_WAVES) * ENG_WGS + s, lane, seq, ctl);
    } else if (wid < PUSH_WAVES + PULL_WAVES + FIN_WAVES && g >= 4) {
      fin_role(p, peer, s, wid - PUSH_WAVES - PULL_WAVES, lane, seq, ctl);
    }
  } else {
    gemm_role(p, g, s - ENG_WGS, seq, (LAS char*)smem);
  }
  STAMP(7);
}

}

extern "C" void kernel_launch(int rank, int nranks, ncclComm_t comm,
                              hipStream_t stream,
                              void* const* d_in, const int* in_sizes, int n_in,
                              void* d_out, int out_size,
                              const int* in_dtypes, int out_dtype,
                              void* const* d_out_all,
                              void* const* d_ws_all, size_t ws_bytes) {
  if (nranks != NR || (unsigned)rank >= (unsigned)NR || n_in < 2 || ws_bytes < WS_USED ||
      in_sizes[0] != 4096 * 512 || in_sizes[1] != 512 * 4096 || out_size != 4096 * 4096) {
    fprintf(stderr, "kernel_launch: unexpected problem shape\n");
    return;
  }
  HIP_CHECK(hipFuncSetAttribute((const void*)fused_gemm_allreduce_silu,
                                hipFuncAttributeMaxDynamicSharedMemorySize, LDS_BYTES));
  Params p;
  p.x = (const char*)d_in[0];
  p.w = (const char*)d_in[1];
  static const int phys[NR] = {0, 2, 3, 6, 1, 4, 5, 7};
  p.rank = 0; p.pad = 0;
  for (int i = 0; i < NR; ++i) if (phys[i] == rank) p.rank = i;
  for (int i = 0; i < NR; ++i) { p.ws[i] = (char*)d_ws_all[phys[p.rank ^ i]]; p.out[i] = (char*)d_out_all[phys[p.rank ^ i]]; }
  hipLaunchKernelGGL(fused_gemm_allreduce_silu, dim3(GRID), dim3(THREADS), LDS_BYTES, stream, p);
  HIP_CHECK(hipGetLastError());
}
```

```cpp
#include <hip/hip_runtime.h>
#include <rccl/rccl.h>
#include <cstdio>
#include <type_traits>

#define HIP_CHECK(expr) do { hipError_t _e = (expr); if (_e != hipSuccess) { \
    fprintf(stderr, "HIP error %s at %s:%d\n", hipGetErrorString(_e), __FILE__, __LINE__); \
    return; } } while (0)

namespace {

typedef unsigned int u32;
typedef unsigned long long u64;
typedef u32 u32x4 __attribute__((ext_vector_type(4)));
typedef float f32x4 __attribute__((ext_vector_type(4)));
typedef __bf16 bf16x4 __attribute__((ext_vector_type(4)));
typedef __bf16 bf16x8 __attribute__((ext_vector_type(8)));
typedef float f32x16 __attribute__((ext_vector_type(16)));
#define LAS __attribute__((address_space(3)))
#define GAS __attribute__((address_space(1)))

constexpr int NR = 8;
constexpr u32 X_BYTES = 4096u * 512u * 2u;
constexpr u32 W_BYTES = 512u * 4096u * 2u;
constexpr u32 OUT_BYTES = 4096u * 4096u * 2u;
constexpr u32 KI = 5, KC = 3;

#define FIS 128
#define FOS 128
constexpr u32 BLK = 16384;
constexpr u32 IN_OFF = 0;
constexpr u32 IN_SRC = KI * 32 * BLK;
constexpr u32 P16_OFF = IN_OFF + NR * IN_SRC;
constexpr u32 P32_OFF = P16_OFF + 256 * 16384;
constexpr u32 DATA_BYTES = P32_OFF + 256 * 32768;
constexpr u32 FI_OFF = DATA_BYTES;
constexpr u32 FPR_OFF = FI_OFF + NR * 320 * FIS;
constexpr u32 FPL_OFF = FPR_OFF + 256 * FOS;
constexpr u32 FF_OFF = FPL_OFF + 256 * FOS;
constexpr u32 CTR_OFF = FF_OFF + NR * 256 * FOS;
constexpr u32 CLM_OFF = CTR_OFF + 256 * 4;
constexpr u32 TMO_OFF = CLM_OFF + 256 * 64;
constexpr u32 TICK_OFF = TMO_OFF + 128;
constexpr u32 TS_OFF = TICK_OFF + 16 * 128;
constexpr u32 TR_OFF = TS_OFF + 256 * 16 * 8;
constexpr u32 WS_USED = TR_OFF + 3 * 8 * 1024 * 8;

constexpr u64 DEADLINE_TICKS = 30000000ull;

#define TRIG_BACK_ 28
#define NS_HALF_ 6
#define MDIR_ 9
#define SL_PULL 2
#define SL_FIN 8
#define FIN_WAVES_ 4
#define SL_GEMM 8
#define PUSH_WAVES_ 2
#define PULL_WAVES_ 2
#define PUSH_D_ 4
#define WIDE_START 1
#define MASKED 1
#define PDIV 3
#define TRIG_W 1
#define WB_TAIL 0
#define DIRPACE 1
#define PACE_FROM 0
#define ASYNC_POLL 0
#define ACQ 3
#define STAGGER 300
#define ENG_WGS_ 8
constexpr int ENG_WGS = ENG_WGS_, GEMM_WGS = 32 - ENG_WGS;
constexpr int PUSH_WAVES = PUSH_WAVES_, PULL_WAVES = PULL_WAVES_, FIN_WAVES = FIN_WAVES_, PUSH_D = PUSH_D_;
static_assert(PUSH_WAVES + PULL_WAVES + FIN_WAVES <= 8, "waves per engine workgroup");
constexpr u32 TRIG_BACK = TRIG_BACK_;
constexpr u32 NAN2 = 0x7FC07FC0u;
constexpr int GRID = 8 * (ENG_WGS + GEMM_WGS);
static_assert(GRID == 256, "slots wrap at a power of two");
constexpr int THREADS = 512;
constexpr int NS_HALF = NS_HALF_;
constexpr u32 MDIR = MDIR_;
constexpr u32 LDS_CTL = 144 * 1024;
constexpr u32 LDS_BYTES = LDS_CTL + 1024;
constexpr int AUX_SYS = 17;

struct Params {
  const char* x;
  const char* w;
  int rank;
  int pad;
  char* ws[NR];
  char* out[NR];
};

__device__ __forceinline__ u32 rfl(u32 v) { return __builtin_amdgcn_readfirstlane(v); }
__device__ __forceinline__ __amdgpu_buffer_rsrc_t mk_rsrc(const void* p, u32 n) {
  const u64 a = (u64)p;
  const u64 u = ((u64)rfl((u32)(a >> 32)) << 32) | rfl((u32)a);
  return __builtin_amdgcn_make_buffer_rsrc((void*)u, 0, n, 0x00020000);
}
__device__ __forceinline__ u32 ld_sys(const char* p) {
  return __hip_atomic_load((const GAS u32*)p, __ATOMIC_RELAXED, __HIP_MEMORY_SCOPE_SYSTEM);
}
__device__ __forceinline__ void st_sys(char* p, u32 v) {
  __hip_atomic_store((GAS u32*)p, v, __ATOMIC_RELAXED, __HIP_MEMORY_SCOPE_SYSTEM);
}
#define WAIT_VM(n) asm volatile("s_waitcnt vmcnt(%0)" ::"n"(n) : "memory")
#define CBAR() asm volatile("" ::: "memory")
#define LD_LDS(q) __hip_atomic_load(q, __ATOMIC_RELAXED, __HIP_MEMORY_SCOPE_WORKGROUP)
#define ST_LDS(q, v) __hip_atomic_store(q, v, __ATOMIC_RELAXED, __HIP_MEMORY_SCOPE_WORKGROUP)

__device__ __forceinline__ void st128(u32x4 v, __amdgpu_buffer_rsrc_t r, u32 voff, u32 soff) {
  __builtin_amdgcn_raw_buffer_store_b128(v, r, voff, soff, AUX_SYS);
  asm volatile("s_nop 1" ::"v"(v) : "memory");
}

template <int SLEEP>
__device__ __forceinline__ u64 wave_wait(const char* flag, u64 need, u32 seq, char* tmo, bool& bad) {
  const u64 t0 = __builtin_amdgcn_s_memrealtime();
  for (u32 it = 1;; ++it) {
    const u64 ok = __ballot(ld_sys(flag) >= seq);
    if ((ok & need) == need) return ok;
    if ((it & 15) == 0 || bad) {
      const u32 t = __hip_atomic_load((const GAS u32*)tmo, __ATOMIC_RELAXED, __HIP_MEMORY_SCOPE_AGENT);
      if (rfl(t) == seq || __builtin_amdgcn_s_memrealtime() - t0 > DEADLINE_TICKS) {
        __hip_atomic_store((GAS u32*)tmo, seq, __ATOMIC_RELAXED, __HIP_MEMORY_SCOPE_AGENT);
        bad = true;
        return ~0ull;
      }
    }
    __builtin_amdgcn_s_sleep(SLEEP);
  }
}

template <int SLEEP>
__device__ __forceinline__ u64 wave_wait2(const char* flag, u64 need, u32 seq, char* tmo, bool& bad) {
  const u64 t0 = __builtin_amdgcn_s_memrealtime();
  const bool needed = (need >> (threadIdx.x & 63)) & 1;
  bool ok = false;
  for (u32 it = 1;; ++it) {
    if (!ok && (it == 1 || needed)) ok = ld_sys(flag) >= seq;
    const u64 have = __ballot(ok);
    if ((have & need) == need) return have;
    if ((it & 15) == 0) {
      const u32 t = __hip_atomic_load((const GAS u32*)tmo, __ATOMIC_RELAXED, __HIP_MEMORY_SCOPE_AGENT);
      if (rfl(t) == seq || __builtin_amdgcn_s_memrealtime() - t0 > DEADLINE_TICKS) {
        __hip_atomic_store((GAS u32*)tmo, seq, __ATOMIC_RELAXED, __HIP_MEMORY_SCOPE_AGENT);
        bad = true;
        return ~0ull;
      }
    }
    __builtin_amdgcn_s_sleep(SLEEP);
  }
}

#define STAMPW(i, w) do {} while (0)
#define STAMP(i) STAMPW(i, 0)
#define NOW(x) do {} while (0)
#define TRACEJ(kind, job) do {} while (0)

__device__ __forceinline__ void dma4(u32 dst, __amdgpu_buffer_rsrc_t ra, u32 va0, u32 va1, u32 sa,
                                     __amdgpu_buffer_rsrc_t rb, u32 vb0, u32 vb1, u32 sb) {
  u32 keep;
  dst = rfl(dst); sa = rfl(sa); sb = rfl(sb);
  asm volatile(
      "s_mov_b32 %0, m0\n\t"
      "s_nop 4\n\t"
      "s_mov_b32 m0, %1\n\t"
      "s_nop 0\n\t"
      "buffer_load_dwordx4 %3, %5, %6 offen lds\n\t"
      "buffer_load_dwordx4 %4, %5, %6 offen offset:1024 lds\n\t"
      "s_mov_b32 m0, %2\n\t"
      "s_nop 0\n\t"
      "buffer_load_dwordx4 %7, %9, %10 offen lds\n\t"
      "buffer_load_dwordx4 %8, %9, %10 offen offset:1024 lds\n\t"
      "s_mov_b32 m0, %0"
      : "=&s"(keep)
      : "s"(dst), "s"(dst + BLK), "v"(va0), "v"(va1), "s"(ra), "s"(sa), "v"(vb0), "v"(vb1), "s"(rb), "s"(sb)
      : "memory");
}

__device__ __forceinline__ void dma3(u32 dsta, u32 dstb, __amdgpu_buffer_rsrc_t ra, u32 va0, u32 sa,
                                     __amdgpu_buffer_rsrc_t rb, u32 vb0, u32 vb1, u32 sb) {
  u32 keep;
  dsta = rfl(dsta); dstb = rfl(dstb); sa = rfl(sa); sb = rfl(sb);
  asm volatile(
      "s_mov_b32 %0, m0\n\t"
      "s_nop 4\n\t"
      "s_mov_b32 m0, %1\n\t"
      "s_nop 0\n\t"
      "buffer_load_dwordx4 %3, %4, %5 offen lds\n\t"
      "s_mov_b32 m0, %2\n\t"
      "s_nop 0\n\t"
      "buffer_load_dwordx4 %6, %8, %9 offen lds\n\t"
      "buffer_load_dwordx4 %7, %8, %9 offen offset:1024 lds\n\t"
      "s_mov_b32 m0, %0"
      : "=&s"(keep)
      : "s"(dsta), "s"(dstb), "v"(va0), "s"(ra), "s"(sa), "v"(vb0), "v"(vb1), "s"(rb), "s"(sb)
      : "memory");
}

__device__ __forceinline__ void dma_flags(u32 dst, const char* src) {
  u32 keep;
  dst = rfl(dst);
  asm volatile("s_mov_b32 %0, m0\n\ts_nop 0\n\ts_mov_b32 m0, %1\n\ts_nop 0\n\tglobal_load_lds_dword %2, off sc0 sc1\n\ts_mov_b32 m0, %0"
               : "=&s"(keep) : "s"(dst), "v"(src) : "memory");
}

template <int I, int N, class F>
__device__ __forceinline__ void static_for(F&& f) {
  if constexpr (I < N) { f(std::integral_constant<int, I>{}); static_for<I + 1, N>(f); }
}

__device__ __forceinline__ u32 sup(u32 grp, u32 c, u32 u) { return grp ? c + 2 * u : 2 * c + u; }
__device__ __forceinline__ u32 srow(u32 r, u32 u) { return sup(r >> 2, (r >> 1) & 1, u); }
__device__ __forceinline__ u32 scol(u32 r, u32 v) { return sup(r >> 2, r & 1, v); }
__device__ __forceinline__ void tile_rc(u32 m, u32& ri, u32& ci) {
  const u32 l = (m >= 1) + (m >= 4) + (m >= 9) + (m >= 16) + (m >= 25) + (m >= 36) + (m >= 49);
  const u32 idx = m - l * l;
  ri = idx < l ? l : idx - l;
  ci = idx < l ? idx : l;
}
__device__ __forceinline__ u32 fin_off(u32 r, u32 m, u32 sb, u32 part) {
  u32 ri, ci;
  tile_rc(m, ri, ci);
  return (srow(r, sb >> 1) * 1024 + ri * 128 + (r >> 2) * 64 + part * 32) * 8192 + (scol(r, sb & 1) * 1024 + ci * 128) * 2;
}

struct Job { u32 src, sstride, dst, flag, blo, bhi, voff; };

__device__ __forceinline__ Job decode_push(u32 job, u32 rank, u32 peer, u64 b0, u64 b1, u32 vnarrow, u32 vwide) {
  Job j;
  const bool cross = (rank ^ peer) >= 4;
  const u32 lk = job >> 3, op = (job >> 2) & 1, uv = (job >> 1) & 1, half = job & 1;
  const u32 lev = cross ? lk / KC : lk / KI;
  const u32 ktl = lk - lev * (cross ? KC : KI), kt = cross ? KI + ktl : ktl;
  j.blo = op ? (u32)b1 : (u32)b0; j.bhi = op ? (u32)(b1 >> 32) : (u32)(b0 >> 32); j.voff = op ? vwide : vnarrow;
  j.src = op ? (kt * 64 + half * 32) * 8192 + (scol(peer, uv) * 1024 + lev * 128) * 2
             : (srow(peer, uv) * 1024 + lev * 128 + half * 64) * 1024 + kt * 128;
  j.sstride = op ? 4 * 8192 : 8 * 1024;
  j.dst = IN_OFF + rank * IN_SRC + job * 8192;
  j.flag = FI_OFF + (rank * 320 + job) * FIS;
  return j;
}

__device__ __forceinline__ GAS u32* ticket_ptr(char* ws, u32 idx) {
  u32 zero = 0;
  asm volatile("" : "+v"(zero));
  return (GAS u32*)(ws + TICK_OFF + idx * 128 + zero);
}

__device__ __forceinline__ u32x4 push_ld(const Job& j, int i) {
  const auto r = mk_rsrc((const void*)(((u64)j.bhi << 32) | j.blo), X_BYTES);
  return __builtin_amdgcn_raw_buffer_load_b128(r, j.voff, rfl(j.src + i * j.sstride), 0);
}

template <bool claimed>
__device__ __forceinline__ bool push_role(const Params& p, u32& slot, u32 wid, int lane, LAS u32* lctl) {
  constexpr int D = PUSH_D, JI = 8;
  constexpr u32 NW = ENG_WGS * PUSH_WAVES;
  NOW(tA);
  const u32 g = slot & 7, w = wid * ENG_WGS + (slot >> 3);
  static_assert(X_BYTES == W_BYTES, "one size for both input shards");
  u64 b0 = (u64)p.x, b1 = (u64)p.w, bw = (u64)p.ws[g];
  u32 rank = p.rank;
  asm volatile("" : "+s"(b0), "+s"(b1), "+s"(bw), "+s"(rank));
  const u32 peer = rank ^ g;
  const u32 NJOBS = (g >= 4 ? KC : KI) * 64;
  char* wsp = (char*)bw;
  const auto rd = mk_rsrc(wsp, DATA_BYTES);
  const u32 vnarrow = (lane >> 3) * 1024 + (lane & 7) * 16;
  const u32 vwide = (lane >> 4) * 8192 + (lane & 15) * 16;
  Job cur = decode_push(w, rank, peer, b0, b1, vnarrow, vwide);
  u32x4 v[D], v2[JI - D];
#pragma unroll
  for (int i = 0; i < D; ++i) { v[i] = push_ld(cur, i); CBAR(); }
  if (WIDE_START) {
#pragma unroll
    for (int i = D; i < JI; ++i) { v2[i - D] = push_ld(cur, i); CBAR(); }
  }
  NOW(tB);
  GAS u32* tick = ticket_ptr(p.ws[0], peer);
  u32 seq = claimed ? LD_LDS(lctl + 2)
                    : __hip_atomic_load((GAS u32*)(p.ws[0] + CTR_OFF) + blockIdx.x, __ATOMIC_RELAXED, __HIP_MEMORY_SCOPE_AGENT) + 1;
  CBAR();
  u32 nid = NW + w, pflag = 0;
  auto round = [&](auto kind_c) {
    constexpr int KIND = decltype(kind_c)::value;
    constexpr bool FIRST = KIND == 0, LAST = KIND == 3;
    Job nxt = cur;
    if constexpr (!LAST) { nxt = decode_push(nid, rank, peer, b0, b1, vnarrow, vwide); }
    u32 tk = 0;
#pragma unroll
    for (int i = 0; i < JI; ++i) {
      if (!FIRST && i == D - 1) {
        WAIT_VM(D + (LAST ? (JI - D < D - 1 ? JI - D : D - 1) : D - 1));
        if (lane == 0) st_sys(wsp + pflag, seq);
        TRACEJ(0, (pflag - FI_OFF) / FIS - rank * 320);
        CBAR();
      }
      st128(FIRST && WIDE_START && i >= D ? v2[i >= D ? i - D : 0] : v[i % D], rd, lane * 16, rfl(cur.dst) + i * 1024);
      if (i + D < JI) { if (!(FIRST && WIDE_START)) v[i % D] = push_ld(cur, i + D); }
      else if (!LAST) v[i % D] = push_ld(nxt, i + D - JI);
      CBAR();
      if (KIND == 2 && i == 0) {
        if (lane == 0) tk = __hip_atomic_fetch_add(tick, 1u, __ATOMIC_RELAXED, __HIP_MEMORY_SCOPE_AGENT);
        CBAR();
      }
    }
    pflag = cur.flag;
    cur = nxt;
    if constexpr (KIND < 2) nid += NW;
    else if constexpr (!LAST) nid = rfl(tk) - (seq - 1) * (NJOBS - 3 * NW) + 4 * NW;
  };
  round(std::integral_constant<int, 0>{});
  NOW(tE);
  round(std::integral_constant<int, 1>{});
  seq = rfl(seq);
  if (!claimed) __builtin_amdgcn_s_barrier();
  CBAR();
  const u32 slot2 = rfl(LD_LDS(lctl + 1));
  if (__builtin_expect(slot2 != slot, 0)) { WAIT_VM(0); slot = slot2; return false; }
  NOW(tC);
  do round(std::integral_constant<int, 2>{}); while (nid < NJOBS);
  round(std::integral_constant<int, 3>{});
  WAIT_VM(0);
  if (lane == 0) st_sys(wsp + pflag, seq);
  TRACEJ(0, (pflag - FI_OFF) / FIS - rank * 320);
  STAMP(3);
  return true;
}

__device__ __forceinline__ float silu(float a) {
  const float c = fminf(fmaxf(a, -60.f), 60.f);
  return a * __builtin_amdgcn_rcpf(1.f + __expf(-c));
}

constexpr u32 MB_REQ = 0, MB_DONE = 4, MB_GONE = 8, MB_RUN = 9, MB_UNIT = 10, MB_BUF = 1024;

__device__ __forceinline__ void pull_role(const Params& p, u32 peer, u32 w, int lane, u32 seq, LAS u32* ctl) {
  constexpr u32 NW = ENG_WGS * PULL_WAVES, NF = 1024;
  const u32 rank = p.rank;
  const bool cross = (rank ^ peer) >= 4;
  char* ws = p.ws[0];
  char* tmo = ws + TMO_OFF;
  const u64 bf = (u64)p.out[rank ^ peer], bp = (u64)p.ws[rank ^ peer];
  const u32 vwide = (lane >> 4) * 8192 + (lane & 15) * 16;
  GAS u32* tick = ticket_ptr(ws, 8 + peer);
  const u32 pbase = (lane & 1) ? FI_OFF + (peer * 320 + (cross ? KC : KI) * 64 - TRIG_BACK + TRIG_W * w) * FIS : FF_OFF + peer * 256 * FOS;
  const u32 pmask = (lane & 1) ? 0u : ~0u;
  const u64 t0 = __builtin_amdgcn_s_memrealtime();
  u32 tf = w, tf2 = NW + w, cur = 0, curf = 0, lastf = 0;
  u32 nsrc = 0, nf = 0;
  bool bad = false;
  auto poll = [&]() -> u32 { return ld_sys(ws + (pbase + (rfl(((tf >> 2) & 255) * FOS) & pmask))); };
  u32 pv = poll();
  u32x4 v[4];
  auto pick = [&]() -> bool {
    const u64 pend = __ballot(lane < FIN_WAVES && LD_LDS(ctl + MB_REQ + (lane & 3)) != 0);
    if (pend) {
      const u32 f = __builtin_ctzll(pend);
      u32 r = 0;
      if (lane == 0) r = __hip_atomic_exchange(ctl + MB_REQ + f, 0u, __ATOMIC_RELAXED, __HIP_MEMORY_SCOPE_WORKGROUP);
      r = rfl(r);
      if (r) { nsrc = r; nf = f + 1; return true; }
    }
    if (tf < NF && (bad || (__ballot(pv >= seq) & 3) == 3)) {
      nsrc = rfl(fin_off(peer, tf >> 4, (tf >> 2) & 3, 0) + (tf & 3) * 16 * 8192);
      nf = 0;
      return true;
    }
    return false;
  };
  auto put = [&](int i) {
    const auto ro = mk_rsrc(p.out[0], curf ? 0u : OUT_BYTES);
    st128(v[i], ro, vwide, rfl(cur) + i * 4 * 8192);
    if (curf) *(LAS u32x4*)((LAS char*)ctl + MB_BUF + (curf - 1) * 4096 + i * 1024 + lane * 16) = v[i];
  };
  u32 curid = 0;
  auto delivered = [&]() {
    if (curf) { TRACEJ(2, ((cur - P16_OFF) >> 12) & 255); } else { TRACEJ(1, curid); }
    if (curf) {
      asm volatile("s_waitcnt lgkmcnt(0)" ::: "memory");
      if (lane == 0) __hip_atomic_fetch_add(ctl + MB_DONE + curf - 1, 1u, __ATOMIC_RELAXED, __HIP_MEMORY_SCOPE_WORKGROUP);
    }
  };
  auto round = [&](auto first_c) {
    constexpr bool FIRST = decltype(first_c)::value;
    u32 tk = 0;
    const bool isf = nf == 0;
    const u32 tid_ = tf;
    if (isf) { tf = tf2; lastf = nsrc; }
    if (lane == 0 && isf) tk = __hip_atomic_fetch_add(tick, 1u, __ATOMIC_RELAXED, __HIP_MEMORY_SCOPE_AGENT);
    pv = poll();
    CBAR();
    const auto rsrc = mk_rsrc((const void*)(isf ? bf : bp), isf ? OUT_BYTES : DATA_BYTES);
    const u32 vs = isf ? vwide : lane * 16, str = isf ? 4 * 8192 : 1024;
#pragma unroll
    for (int i = 0; i < 4; ++i) {
      if (!FIRST) put(i);
      v[i] = __builtin_amdgcn_raw_buffer_load_b128(rsrc, vs, rfl(nsrc + i * str), AUX_SYS);
      CBAR();
    }
    if (!FIRST) delivered();
    cur = nsrc; curf = nf; curid = tid_;
    if (isf) tf2 = rfl(tk) - (seq - 1) * NF + 2 * NW;
  };
  for (u32 it = 1;; ++it) {
    if (pick()) {
      if (tf == w && nf == 0) { STAMPW(5, PUSH_WAVES); }
      round(std::true_type{});
      if (pick()) {
        round(std::false_type{});
        while (pick()) round(std::false_type{});
      }
      if (WB_TAIL && tf >= NF && curf == 0) {
#pragma unroll
        for (int i = 0; i < 4; ++i) {
          __builtin_amdgcn_raw_buffer_store_b128(v[i], mk_rsrc(p.out[0], OUT_BYTES), vwide, rfl(cur) + i * 4 * 8192, 0);
          asm volatile("s_nop 1" ::"v"(v[i]) : "memory");
        }
        continue;
      }
#pragma unroll
      for (int i = 0; i < 4; ++i) put(i);
      delivered();
      continue;
    }
    if (tf >= NF && rfl(LD_LDS(ctl + MB_GONE)) == (cross ? FIN_WAVES : 0)) break;
    if ((it & 63) == 0) {
      const u32 t = __hip_atomic_load((const GAS u32*)tmo, __ATOMIC_RELAXED, __HIP_MEMORY_SCOPE_AGENT);
      if (rfl(t) == seq || __builtin_amdgcn_s_memrealtime() - t0 > DEADLINE_TICKS) {
        __hip_atomic_store((GAS u32*)tmo, seq, __ATOMIC_RELAXED, __HIP_MEMORY_SCOPE_AGENT);
        bad = true;
      }
    }
    __builtin_amdgcn_s_sleep(SL_PULL);
    if ((it & PDIV) == 0) pv = poll();
  }
  if (bad) {
    WAIT_VM(0);
    const u32x4 nanv = {NAN2, NAN2, NAN2, NAN2};
    for (int i = 0; i < 4; ++i) st128(nanv, mk_rsrc(p.out[0], OUT_BYTES), vwide, rfl(lastf) + i * 4 * 8192);
  }
  STAMPW(6, PUSH_WAVES);
}

__device__ __forceinline__ void fin_role(const Params& p, u32 peer, u32 s, u32 f, int lane, u32 seq, LAS u32* ctl) {
  static_assert(FIN_WAVES == 4, "one wave per 16 rows of a half tile");
  const u32 rank = p.rank, sb = peer & 3;
  char* ws = p.ws[0];
  char* tmo = ws + TMO_OFF;
  const auto rl = mk_rsrc(ws, DATA_BYTES);
  const auto ro = mk_rsrc(p.out[0], OUT_BYTES);
  const u32 vwide = (lane >> 4) * 8192 + (lane & 15) * 16;
  char* wsl = p.ws[lane & 7];
  u32 ndone = 0;
  for (u32 m = MDIR + s; m < 64; m += ENG_WGS) {
    const u32 lo = P32_OFF + (sb * 64 + m) * 32768 + f * 8192;
    const u32 dst = fin_off(rank, m, sb, 0) + f * 16 * 8192;
    bool bad = false;
    wave_wait<SL_FIN>(ws + ((lane & 1) ? FPL_OFF : FPR_OFF) + (sb * 64 + m) * FOS, 3, seq, tmo, bad);
    if (m == MDIR + s) { STAMPW(8, PUSH_WAVES + PULL_WAVES); STAMPW(11, 7); }
    const u32 so = P16_OFF + ((rank & 3) * 64 + m) * 16384 + f * 4096;
    u32x4 rv[4], a[4], b[4];
    if (lane == 0) ST_LDS(ctl + MB_REQ + f, so);
#pragma unroll
    for (int i = 0; i < 4; ++i) {
      a[i] = __builtin_amdgcn_raw_buffer_load_b128(rl, lane * 16, lo + i * 2048, AUX_SYS);
      b[i] = __builtin_amdgcn_raw_buffer_load_b128(rl, lane * 16, lo + i * 2048 + 1024, AUX_SYS);
    }
    ++ndone;
    while (rfl(LD_LDS(ctl + MB_DONE + f)) != ndone) __builtin_amdgcn_s_sleep(1);
    CBAR();
#pragma unroll
    for (int i = 0; i < 4; ++i) rv[i] = *(const LAS u32x4*)((LAS char*)ctl + MB_BUF + f * 4096 + i * 1024 + lane * 16);
    if (m == MDIR + s) { STAMPW(9, PUSH_WAVES + PULL_WAVES);  }
#pragma unroll
    for (int i = 0; i < 4; ++i) {
      const u32x4 r = rv[i];
      float y[8];
#pragma unroll
      for (int e = 0; e < 4; ++e) {
        const u32 l0 = e < 2 ? a[i][2 * e] : b[i][2 * e - 4], l1 = e < 2 ? a[i][2 * e + 1] : b[i][2 * e - 3];
        y[2 * e] = silu(__uint_as_float(l0) + __uint_as_float(r[e] << 16));
        y[2 * e + 1] = silu(__uint_as_float(l1) + __uint_as_float(r[e] & 0xFFFF0000u));
      }
      const bf16x8 o = {(__bf16)y[0], (__bf16)y[1], (__bf16)y[2], (__bf16)y[3],
                        (__bf16)y[4], (__bf16)y[5], (__bf16)y[6], (__bf16)y[7]};
      u32x4 ov = __builtin_bit_cast(u32x4, o);
      if (bad) ov |= NAN2;
      st128(ov, ro, vwide, dst + i * 4 * 8192);
    }
    if (m == MDIR + s) { STAMPW(10, PUSH_WAVES + PULL_WAVES); }
    WAIT_VM(0);

    u32 arrived = 0;
    if (lane == 0) arrived = __hip_atomic_fetch_add(ctl + MB_UNIT + ndone - 1, 1u, __ATOMIC_RELAXED, __HIP_MEMORY_SCOPE_WORKGROUP);
    if (rfl(arrived) == FIN_WAVES - 1) {
      if (lane >= 1 && lane < NR) st_sys(wsl + FF_OFF + (rank * 256 + m * 4 + sb) * FOS, seq);
    }
    if (m == MDIR + s) { STAMPW(1, PUSH_WAVES + PULL_WAVES); }
  }
  if (lane == 0) __hip_atomic_fetch_add(ctl + MB_GONE, 1u, __ATOMIC_RELAXED, __HIP_MEMORY_SCOPE_WORKGROUP);
  STAMPW(2, PUSH_WAVES + PULL_WAVES + FIN_WAVES - 1);
}

template <bool HALF>
__device__ __forceinline__ void gemm_tile(const Params& p, u32 sb, u32 m, u32 hsel, u32 seq, u64 tstart, int sbase, LAS char* lds) {
  const int tid = threadIdx.x, lane = tid & 63;
  const u32 wid = rfl(tid >> 6);
  const u32 rank = p.rank;
  char* ws = p.ws[0];
  char* tmo = ws + TMO_OFF;
  const auto rx = mk_rsrc(p.x, X_BYTES);
  const auto rw = mk_rsrc(p.w, W_BYTES);
  const auto rs = mk_rsrc(ws, DATA_BYTES);
  const u32 u = sb >> 1, v = sb & 1;
  constexpr int NT = 32, T_CROSS = KI, T_INTRA = KI + 4 * KC, T_SPLIT = 4;
  constexpr int MI = HALF ? 1 : 2, NDMA = HALF ? 3 : 4, NS = HALF ? NS_HALF : 4;
  constexpr u32 XB = HALF ? BLK / 2 : BLK, SB = XB + BLK;
  static_assert(NS * SB <= LDS_CTL, "stages fit");

  u32 vao[2], vbo[2], vaw[2], vbw[2];
#pragma unroll
  for (int q = 0; q < 2; ++q) {
    const u32 ra = wid * 8 * MI + q * 8 + (lane >> 3), ca = (lane & 7) ^ ((ra >> 1) & 7);
    const u32 kb = wid * 8 + q * 4 + (lane >> 4), cbk = (lane & 15) ^ (((kb & 3) << 2) | ((kb >> 2) & 3));
    vao[q] = ra * 1024 + ca * 16 - q * 1024;  vaw[q] = ra * 128 + ca * 16 - q * 1024;
    vbo[q] = kb * 8192 + cbk * 16 - q * 1024; vbw[q] = kb * 256 + cbk * 16 - q * 1024;
  }
  const u32 lds0 = (u32)(size_t)lds;
  const u32 wm = wid & 1, wn = wid >> 1, h = lane >> 5;
  u32 xo[4], wo[2];
  {
    const u32 r = wm * 32 * MI + (lane & 31);
#pragma unroll
    for (int ks = 0; ks < 4; ++ks) xo[ks] = r * 128 + (((2 * ks + h) ^ ((r >> 1) & 7)) * 16);
    const u32 bk = (lane >> 4) & 1, q = (lane & 15) >> 2, pp = lane & 3;
#pragma unroll
    for (int t = 0; t < 2; ++t)
      wo[t] = XB + 256 * (8 * h + 4 * t + q) + 16 * ((4 * wn + 2 * bk + (pp >> 1)) ^ ((q << 2) | (2 * h + t))) + 8 * (pp & 1);
  }
  LAS u32* lbad = (LAS u32*)(lds + LDS_CTL);
  LAS u32* fimg = (LAS u32*)(lds + LDS_CTL + 256) + lane;
  if (ASYNC_POLL && wid == 0) { fimg[0] = 0; fimg[64] = 0; }
  u32 ri, ci;
  tile_rc(m, ri, ci);
  const u32 row0 = srow(rank, u) * 1024 + ri * 128 + (HALF ? hsel * 64 : 0), col0 = scol(rank, v) * 1024 + ci * 128;
  u32 islot = 0, rslot = 0;
  auto next_slot = [&](u32 o) -> u32 { return o + SB == NS * SB ? 0 : o + SB; };
  auto issue_own = [&](int t) {
    const u32 dst = lds0 + islot, sa = row0 * 1024 + t * 128, sw = t * 64 * 8192 + col0 * 2;
    islot = next_slot(islot);
    if (HALF) dma3(dst + wid * 1024, dst + XB + wid * 2048, rx, vao[0], sa, rw, vbo[0], vbo[1], sw);
    else dma4(dst + wid * 2048, rx, vao[0], vao[1], sa, rw, vbo[0], vbo[1], sw);
  };
  u32 taba, tabw;
  {
    const bool cr = lane < T_INTRA;
    const u32 c = cr ? lane - T_CROSS : lane - T_INTRA, ktl = cr ? c >> 2 : c / 3;
    const u32 src = rank ^ (cr ? 4 + (c & 3) : 1 + c - ktl * 3), kts = cr ? KC : KI;
    const u32 base = IN_OFF + src * IN_SRC + ktl * 4 * BLK;
    taba = base + (ri * kts * 4 + u) * BLK + (HALF ? hsel * 8192 : 0);
    tabw = base + (ci * kts * 4 + 2 + v) * BLK;
  }
  auto issue = [&](int t) {
    const u32 dst = lds0 + islot, sa = __builtin_amdgcn_readlane(taba, t), sw = __builtin_amdgcn_readlane(tabw, t);
    islot = next_slot(islot);
    if (HALF) dma3(dst + wid * 1024, dst + XB + wid * 2048, rs, vaw[0], sa, rs, vbw[0], vbw[1], sw);
    else dma4(dst + wid * 2048, rs, vaw[0], vaw[1], sa, rs, vbw[0], vbw[1], sw);
  };
  const u32 fop = (lane >> 1) & 1, flev = fop ? ci : ri, fuv = fop ? v : u, fq = lane >> 2;
  const u32 fh = (HALF && !fop) ? hsel : (lane & 1);
  const u32 fck = min(fq >> 2, KC - 1), fik = min(fq / 3, KI - 1);
  const char* fcross = ws + FI_OFF + ((rank ^ (4 + (fq & 3))) * 320 + ((flev * KC + fck) * 4 + fop * 2 + fuv) * 2 + fh) * FIS;
  const char* fintra = ws + FI_OFF + ((rank ^ (1 + fq % 3)) * 320 + ((flev * KI + fik) * 4 + fop * 2 + fuv) * 2 + fh) * FIS;

  while (__builtin_amdgcn_s_memrealtime() - tstart < (u64)max(ri, ci) * STAGGER) __builtin_amdgcn_s_sleep(16);
  if (ACQ == 3 && sbase == 1 && wid == 0) __builtin_amdgcn_fence(__ATOMIC_ACQUIRE, "");
  f32x16 acc[MI] = {};
  struct Frag { bf16x8 w[4], x[MI][4]; };
  auto read = [&](Frag& f) {
    const LAS char* st = lds + rslot;
    rslot = next_slot(rslot);
#pragma unroll
    for (int ks = 0; ks < 4; ++ks) {
      const bf16x4 w0 = __builtin_amdgcn_ds_read_tr16_b64_v4bf16((LAS bf16x4*)(st + wo[0] + ks * 4096));
      const bf16x4 w1 = __builtin_amdgcn_ds_read_tr16_b64_v4bf16((LAS bf16x4*)(st + wo[1] + ks * 4096));
      f.w[ks] = __builtin_shufflevector(w0, w1, 0, 1, 2, 3, 4, 5, 6, 7);
#pragma unroll
      for (int mi = 0; mi < MI; ++mi) f.x[mi][ks] = *(const LAS bf16x8*)(st + xo[ks] + mi * 4096);
    }
  };
  auto mma = [&](const Frag& f) {
#pragma unroll
    for (int ks = 0; ks < 4; ++ks)
#pragma unroll
      for (int mi = 0; mi < MI; ++mi) acc[mi] = __builtin_amdgcn_mfma_f32_32x32x16_bf16(f.w[ks], f.x[mi][ks], acc[mi], 0, 0, 0);
  };
  int next_wait = T_CROSS;
  u64 need = 0xFFFF, have = 0;
  auto waitpoint = [&](int tn) {
    if (tn == T_INTRA) { need = 0xFFF; have = 0; }
    if (wid == 0 && (have & need) != need) {
      if (ASYNC_POLL && HALF) have = __ballot(LD_LDS(fimg + (tn < T_INTRA ? 0 : 64)) >= seq);
      if ((have & need) != need) {
        bool bad = false;
        have = MASKED ? wave_wait2<SL_GEMM>(tn < T_INTRA ? fcross : fintra, need, seq, tmo, bad)
                      : wave_wait<SL_GEMM>(tn < T_INTRA ? fcross : fintra, need, seq, tmo, bad);
        if (bad) *lbad = 1;
      }
    }
    if (tn == T_CROSS) { STAMP(sbase); }
    if (tn == NT - 3) { STAMP(sbase + 1); }
    if (tn < T_INTRA) { next_wait += 4; need = (need << 16) | 0xFFFF; }
    else { next_wait += 3; need = (need << 12) | 0xFFF; }
  };
  auto step = [&](auto rc, int tn, const Frag& fc, Frag& fn) {
    constexpr int R = decltype(rc)::value;
    if (tn >= 0 && __builtin_expect(tn == next_wait, 0)) waitpoint(tn);
    if constexpr (R >= 0) {
      WAIT_VM(R * NDMA);
      __builtin_amdgcn_s_waitcnt(0xC07F);
      CBAR();
      __builtin_amdgcn_s_barrier();
      CBAR();
      read(fn);
    }
    if (tn >= 0) issue(tn);
    if (ASYNC_POLL && HALF && tn >= 0 && wid == 1 && next_wait < NT)
      dma_flags(lds0 + LDS_CTL + (next_wait < T_INTRA ? 256 : 512), next_wait < T_INTRA ? fcross : fintra);
    mma(fc);
  };
  Frag fa, fb;
  auto tail = [&](auto nc) {
    constexpr int N = decltype(nc)::value;
    static_for<0, N / 2>([&](auto ic) {
      constexpr int i = decltype(ic)::value;
      step(std::integral_constant<int, N - 2 - 2 * i>{}, -1, fa, fb);
      step(std::integral_constant<int, N - 3 - 2 * i>{}, -1, fb, fa);
    });
  };
  static_assert(NS % 2 == 0 && NS >= T_SPLIT && T_SPLIT + 1 == T_CROSS, "run structure");
  for (int t = 0; t < T_SPLIT; ++t) issue_own(t);
  WAIT_VM((T_SPLIT - 1) * NDMA);
  __builtin_amdgcn_s_barrier();
  CBAR();
  read(fa);
  tail(std::integral_constant<int, T_SPLIT>{});
  islot = rslot = 0;
  __builtin_amdgcn_s_waitcnt(0xC07F);
  __builtin_amdgcn_s_barrier();
  issue_own(T_SPLIT);
  for (int t = T_CROSS; t < T_SPLIT + NS; ++t) {
    if (t == next_wait) { waitpoint(t); __builtin_amdgcn_s_barrier(); }
    issue(t);
  }
  WAIT_VM((NS - 1) * NDMA);
  __builtin_amdgcn_s_barrier();
  CBAR();
  read(fa);
  for (int tn = T_SPLIT + NS; tn < NT; tn += 2) {
    step(std::integral_constant<int, NS - 2>{}, tn, fa, fb);
    step(std::integral_constant<int, NS - 2>{}, tn + 1, fb, fa);
  }
  tail(std::integral_constant<int, NS>{});
  STAMP(sbase + 2);
  __syncthreads();
  constexpr u32 EROW = 528;
#pragma unroll
  for (int mi = 0; mi < MI; ++mi) {
    const u32 mr = wm * 32 * MI + mi * 32 + (lane & 31);
#pragma unroll
    for (int gq = 0; gq < 4; ++gq) {
      const f32x4 o = {acc[mi][4 * gq], acc[mi][4 * gq + 1], acc[mi][4 * gq + 2], acc[mi][4 * gq + 3]};
      *(LAS f32x4*)(lds + mr * EROW + (wn * 32 + 8 * gq + 4 * h) * 4) = o;
    }
  }
  __syncthreads();
  const u32 bad = *lbad ? NAN2 : 0u;
  const u32 keep = rank >> 2;
  const bool do_exp = !HALF || hsel != keep, do_keep = !HALF || hsel == keep;
  char* wsq = p.ws[4 | ((rank & 3) ^ sb)];
  if (do_exp) {
    const bool dir = HALF && m < MDIR;
    const auto re = mk_rsrc(dir ? wsq : ws, DATA_BYTES);
    const u32 e16 = P16_OFF + ((dir ? rank & 3 : sb) * 64 + m) * 16384 + wid * 1024;
#pragma unroll
    for (int part = 0; part < 2; ++part) {
      const LAS char* se = lds + ((HALF ? 0 : (keep ^ 1) * 64) + part * 32 + (tid >> 4)) * EROW + (tid & 15) * 32;
      const f32x4 c0 = *(const LAS f32x4*)se, c1 = *(const LAS f32x4*)(se + 16);
      const bf16x8 o = {(__bf16)c0[0], (__bf16)c0[1], (__bf16)c0[2], (__bf16)c0[3],
                        (__bf16)c1[0], (__bf16)c1[1], (__bf16)c1[2], (__bf16)c1[3]};
      if (DIRPACE == 2 && dir && m >= PACE_FROM && part == 0 && wid >= 4) __builtin_amdgcn_s_barrier();
      st128(__builtin_bit_cast(u32x4, o) | bad, re, lane * 16, e16 + part * 8192);
      if (DIRPACE >= 1 && dir && m >= PACE_FROM) WAIT_VM(0);
      if (DIRPACE == 2 && dir && m >= PACE_FROM && part == 1 && wid < 4) __builtin_amdgcn_s_barrier();
    }
  }
  if (HALF && do_keep && m < MDIR) {
    f32x4 kc[2][2];
#pragma unroll
    for (int part = 0; part < 2; ++part) {
      const LAS char* sk = lds + (part * 32 + (tid >> 4)) * EROW + (tid & 15) * 32;
      kc[part][0] = *(const LAS f32x4*)sk; kc[part][1] = *(const LAS f32x4*)(sk + 16);
    }
    bool badw = bad != 0;
    wave_wait<2>(ws + FPR_OFF + (sb * 64 + m) * FOS, 1, seq, tmo, badw);
    STAMP(5);
    u32x4 r[2];
#pragma unroll
    for (int part = 0; part < 2; ++part)
      r[part] = __builtin_amdgcn_raw_buffer_load_b128(rs, lane * 16, P16_OFF + (sb * 64 + m) * 16384 + wid * 1024 + part * 8192, AUX_SYS);
    const auto ro = mk_rsrc(p.out[0], OUT_BYTES);
#pragma unroll
    for (int part = 0; part < 2; ++part) {
      float y[8];
#pragma unroll
      for (int e = 0; e < 4; ++e) {
        y[2 * e] = silu(kc[part][e >> 1][(2 * e) & 3] + __uint_as_float(r[part][e] << 16));
        y[2 * e + 1] = silu(kc[part][e >> 1][(2 * e + 1) & 3] + __uint_as_float(r[part][e] & 0xFFFF0000u));
      }
      const bf16x8 o = {(__bf16)y[0], (__bf16)y[1], (__bf16)y[2], (__bf16)y[3],
                        (__bf16)y[4], (__bf16)y[5], (__bf16)y[6], (__bf16)y[7]};
      st128(__builtin_bit_cast(u32x4, o) | (badw ? NAN2 : 0u), ro, (lane >> 4) * 8192 + (lane & 15) * 16,
            (row0 + part * 32 + wid * 4) * 8192 + col0 * 2);
    }
    WAIT_VM(0);
    __syncthreads();
    if (tid >= 1 && tid < NR) st_sys(p.ws[tid] + FF_OFF + (rank * 256 + m * 4 + sb) * FOS, seq);
    STAMP(6);
    return;
  }
  if (do_keep) {
    const u32 e32 = P32_OFF + (sb * 64 + m) * 32768 + wid * 2048;
#pragma unroll
    for (int part = 0; part < 2; ++part) {
      const LAS char* sk = lds + ((HALF ? 0 : keep * 64) + part * 32 + (tid >> 4)) * EROW + (tid & 15) * 32;
      const u32x4 c0 = *(const LAS u32x4*)sk, c1 = *(const LAS u32x4*)(sk + 16);
      st128(c0 | bad, rs, lane * 16, e32 + part * 16384);
      st128(c1 | bad, rs, lane * 16, e32 + part * 16384 + 1024);
    }
  }
  WAIT_VM(0);
  __syncthreads();
  if (tid == 0 && do_exp) st_sys(wsq + FPR_OFF + ((rank & 3) * 64 + m) * FOS, seq);
  if (tid == 64 && do_keep) st_sys(ws + FPL_OFF + (sb * 64 + m) * FOS, seq);
  STAMP(sbase + 3);
}

__device__ __forceinline__ void gemm_role(const Params& p, u32 g, u32 k, u32 seq, LAS char* lds) {
  const u64 tstart = __builtin_amdgcn_s_memrealtime();
  if (threadIdx.x < 64 && (ACQ == 1 || (ACQ == 2 && k == 0))) __builtin_amdgcn_fence(__ATOMIC_ACQUIRE, "");
  static_assert(GEMM_WGS >= 16 && GEMM_WGS <= 24, "unit schedule below");
  const u32 keep = p.rank >> 2;
  if (k < 16) gemm_tile<true>(p, g >= 4 ? (p.rank ^ g) & 3 : g, k, g >= 4 ? keep ^ 1 : keep, seq, tstart, 1, lds);
  const u32 arm = g & 1;
  for (u32 n = k < 16 ? GEMM_WGS - 16 + k : k - 16; n < 24; n += GEMM_WGS) {
    u32 lev = 4, idx = n;
    for (;;) {
      const u32 cnt = lev + ((lev & 1) == arm);
      if (idx < cnt) break;
      idx -= cnt; ++lev;
    }
    gemm_tile<false>(p, g >> 1, lev * lev + (idx == lev ? 2 * lev : arm * lev + idx), 0, seq, tstart, k < 16 ? 8 : 1, lds);
  }
}

__global__ void __launch_bounds__(THREADS, 2) fused_gemm_allreduce_silu(Params p) {
  extern __shared__ __attribute__((aligned(16))) char smem[];
  const int lane = threadIdx.x & 63;
  const u32 wid = rfl(threadIdx.x >> 6);
  LAS u32* ctl = (LAS u32*)smem;
  LAS u32* lctl = (LAS u32*)(smem + LDS_CTL);
  u32 slot = (blockIdx.x & ~7u) | ((u32)__builtin_amdgcn_s_getreg((3 << 11) | 20) & 7);
  auto is_push = [&]() { return slot < 8 * ENG_WGS && (slot & 7) != 0 && wid < PUSH_WAVES; };
  if (__builtin_expect(is_push(), 1)) {
    if (push_role<false>(p, slot, wid, lane, lctl)) return;
  } else {
    STAMPW(0, 2);
    if (wid == 7) {
      if (lane < 32) ctl[lane] = 0;
      if (lane == 0) {
        GAS u32* ctr = (GAS u32*)(p.ws[0] + CTR_OFF) + blockIdx.x;
        GAS u32* clm = (GAS u32*)(p.ws[0] + CLM_OFF);
        const u32 c = __hip_atomic_load(ctr, __ATOMIC_RELAXED, __HIP_MEMORY_SCOPE_AGENT);
        u32 sl = slot;
#pragma unroll 1
        for (int n = 0; n < 4 * GRID; ++n) {
          if (__hip_atomic_fetch_add(clm + sl * 16, 1u, __ATOMIC_RELAXED, __HIP_MEMORY_SCOPE_AGENT) == c) break;
          __hip_atomic_fetch_add(clm + sl * 16, ~0u, __ATOMIC_RELAXED, __HIP_MEMORY_SCOPE_AGENT);
          sl = (sl + 1) & (GRID - 1);
        }
        lctl[0] = 0; lctl[1] = sl; lctl[2] = c + 1;
      }
    }
    asm volatile("s_waitcnt lgkmcnt(0)" ::: "memory");
    __builtin_amdgcn_s_barrier();
    slot = rfl(LD_LDS(lctl + 1));
    if (threadIdx.x == THREADS - 1)
      __hip_atomic_store((GAS u32*)(p.ws[0] + CTR_OFF) + blockIdx.x, LD_LDS(lctl + 2), __ATOMIC_RELAXED, __HIP_MEMORY_SCOPE_AGENT);
    WAIT_VM(0);
  }
  if (__builtin_expect(is_push(), 0)) {
    push_role<true>(p, slot, wid, lane, lctl);
    return;
  }
  const u32 seq = rfl(LD_LDS(lctl + 2));
  const u32 g = slot & 7, s = slot >> 3;
  const u32 peer = ((u32)p.rank ^ g) & (NR - 1);
  if (s < ENG_WGS) {
    if (g == 0) return;
    if (wid < PUSH_WAVES + PULL_WAVES) {
      pull_role(p, peer, (wid - PUSH_WAVES) * ENG_WGS + s, lane, seq, ctl);
    } else if (wid < PUSH_WAVES + PULL_WAVES + FIN_WAVES && g >= 4) {
      fin_role(p, peer, s, wid - PUSH_WAVES - PULL_WAVES, lane, seq, ctl);
    }
  } else {
    gemm_role(p, g, s - ENG_WGS, seq, (LAS char*)smem);
  }
  STAMP(7);
}

}

extern "C" void kernel_launch(int rank, int nranks, ncclComm_t comm,
                              hipStream_t stream,
                              void* const* d_in, const int* in_sizes, int n_in,
                              void* d_out, int out_size,
                              const int* in_dtypes, int out_dtype,
                              void* const* d_out_all,
                              void* const* d_ws_all, size_t ws_bytes) {
  if (nranks != NR || (unsigned)rank >= (unsigned)NR || n_in < 2 || ws_bytes < WS_USED ||
      in_sizes[0] != 4096 * 512 || in_sizes[1] != 512 * 4096 || out_size != 4096 * 4096) {
    fprintf(stderr, "kernel_launch: unexpected problem shape\n");
    return;
  }
  HIP_CHECK(hipFuncSetAttribute((const void*)fused_gemm_allreduce_silu,
                                hipFuncAttributeMaxDynamicSharedMemorySize, LDS_BYTES));
  Params p;
  p.x = (const char*)d_in[0];
  p.w = (const char*)d_in[1];
  static const int phys[NR] = {0, 2, 3, 6, 1, 4, 5, 7};
  p.rank = 0; p.pad = 0;
  for (int i = 0; i < NR; ++i) if (phys[i] == rank) p.rank = i;
  for (int i = 0; i < NR; ++i) { p.ws[i] = (char*)d_ws_all[phys[p.rank ^ i]]; p.out[i] = (char*)d_out_all[phys[p.rank ^ i]]; }
  hipLaunchKernelGGL(fused_gemm_allreduce_silu, dim3(GRID), dim3(THREADS), LDS_BYTES, stream, p);
  HIP_CHECK(hipGetLastError());
}
```

```cpp
#include <hip/hip_runtime.h>
#include <rccl/rccl.h>
#include <cstdio>
#include <type_traits>

#define HIP_CHECK(expr) do { hipError_t _e = (expr); if (_e != hipSuccess) { \
    fprintf(stderr, "HIP error %s at %s:%d\n", hipGetErrorString(_e), __FILE__, __LINE__); \
    return; } } while (0)

namespace {

typedef unsigned int u32;
typedef unsigned long long u64;
typedef u32 u32x4 __attribute__((ext_vector_type(4)));
typedef float f32x4 __attribute__((ext_vector_type(4)));
typedef __bf16 bf16x4 __attribute__((ext_vector_type(4)));
typedef __bf16 bf16x8 __attribute__((ext_vector_type(8)));
typedef float f32x16 __attribute__((ext_vector_type(16)));
#define LAS __attribute__((address_space(3)))
#define GAS __attribute__((address_space(1)))

constexpr int NR = 8;
constexpr u32 X_BYTES = 4096u * 512u * 2u;
constexpr u32 W_BYTES = 512u * 4096u * 2u;
constexpr u32 OUT_BYTES = 4096u * 4096u * 2u;
constexpr u32 KI = 5, KC = 3;

#define FIS 128
#define FOS 128
constexpr u32 BLK = 16384;
constexpr u32 IN_OFF = 0;
constexpr u32 IN_SRC = KI * 32 * BLK;
constexpr u32 P16_OFF = IN_OFF + NR * IN_SRC;
constexpr u32 P32_OFF = P16_OFF + 256 * 16384;
constexpr u32 EX_OFF = P32_OFF + 256 * 32768;
constexpr u32 DATA_BYTES = EX_OFF + 512 * BLK;
constexpr u32 FI_OFF = DATA_BYTES;
constexpr u32 FPR_OFF = FI_OFF + NR * 320 * FIS;
constexpr u32 FPL_OFF = FPR_OFF + 256 * FOS;
constexpr u32 FF_OFF = FPL_OFF + 256 * FOS;
constexpr u32 SG_OFF = FF_OFF + NR * 256 * FOS;
constexpr u32 GO_OFF = SG_OFF + NR * 40 * FOS;
constexpr u32 CTR_OFF = GO_OFF + NR * 16 * FOS;
constexpr u32 CLM_OFF = CTR_OFF + 256 * 4;
constexpr u32 TMO_OFF = CLM_OFF + 256 * 64;
constexpr u32 TICK_OFF = TMO_OFF + 128;
constexpr u32 TS_OFF = TICK_OFF + 16 * 128;
constexpr u32 TR_OFF = TS_OFF + 256 * 16 * 8;
constexpr u32 WS_USED = TR_OFF + 3 * 8 * 1024 * 8;

constexpr u64 DEADLINE_TICKS = 30000000ull;

#define TRIG_BACK_ 32
#define NS_HALF_ 6
#define MDIR_ 16
#define SL_PULL 2
#define SL_FIN 8
#define FIN_WAVES_ 4
#define SL_GEMM 8
#define PUSH_WAVES_ 2
#define PULL_WAVES_ 2
#define PUSH_D_ 4
#define WIDE_START 1
#define MASKED 1
#define PDIV 3
#define TRIG_W 1
#define WB_TAIL 0
#define DIRPACE 1
#define PACE_FROM 0
#define ASYNC_POLL 0
#define PUSH_ROUNDS 4
#define HELP_ROUNDS 2
#define GO_AT 7
#define ACQ 3
#define STAGGER 300
#define ENG_WGS_ 8
constexpr int ENG_WGS = ENG_WGS_, GEMM_WGS = 32 - ENG_WGS;
constexpr int PUSH_WAVES = PUSH_WAVES_, PULL_WAVES = PULL_WAVES_, FIN_WAVES = FIN_WAVES_, PUSH_D = PUSH_D_;
static_assert(PUSH_WAVES + PULL_WAVES + FIN_WAVES <= 8, "waves per engine workgroup");
constexpr u32 TRIG_BACK = TRIG_BACK_;
constexpr u32 NPUSH = PUSH_ROUNDS * ENG_WGS * PUSH_WAVES_;
static_assert(PUSH_ROUNDS >= (GO_AT > 8 ? 4 : 3) && ENG_WGS == 8, "push rounds / staging schedule");
constexpr u32 NAN2 = 0x7FC07FC0u;
constexpr int GRID = 8 * (ENG_WGS + GEMM_WGS);
static_assert(GRID == 256, "slots wrap at a power of two");
constexpr int THREADS = 512;
constexpr int NS_HALF = NS_HALF_;
constexpr u32 MDIR = MDIR_;
constexpr u32 LDS_CTL = 144 * 1024;
constexpr u32 LDS_BYTES = LDS_CTL + 1024;
constexpr int AUX_SYS = 17;

struct Params {
  const char* x;
  const char* w;
  int rank;
  int pad;
  char* ws[NR];
  char* out[NR];
};

__device__ __forceinline__ u32 rfl(u32 v) { return __builtin_amdgcn_readfirstlane(v); }
__device__ __forceinline__ __amdgpu_buffer_rsrc_t mk_rsrc(const void* p, u32 n) {
  const u64 a = (u64)p;
  const u64 u = ((u64)rfl((u32)(a >> 32)) << 32) | rfl((u32)a);
  return __builtin_amdgcn_make_buffer_rsrc((void*)u, 0, n, 0x00020000);
}
__device__ __forceinline__ u32 ld_sys(const char* p) {
  return __hip_atomic_load((const GAS u32*)p, __ATOMIC_RELAXED, __HIP_MEMORY_SCOPE_SYSTEM);
}
__device__ __forceinline__ void st_sys(char* p, u32 v) {
  __hip_atomic_store((GAS u32*)p, v, __ATOMIC_RELAXED, __HIP_MEMORY_SCOPE_SYSTEM);
}
#define WAIT_VM(n) asm volatile("s_waitcnt vmcnt(%0)" ::"n"(n) : "memory")
#define CBAR() asm volatile("" ::: "memory")
#define LD_LDS(q) __hip_atomic_load(q, __ATOMIC_RELAXED, __HIP_MEMORY_SCOPE_WORKGROUP)
#define ST_LDS(q, v) __hip_atomic_store(q, v, __ATOMIC_RELAXED, __HIP_MEMORY_SCOPE_WORKGROUP)

__device__ __forceinline__ void st128(u32x4 v, __amdgpu_buffer_rsrc_t r, u32 voff, u32 soff) {
  __builtin_amdgcn_raw_buffer_store_b128(v, r, voff, soff, AUX_SYS);
  asm volatile("s_nop 1" ::"v"(v) : "memory");
}

template <int SLEEP>
__device__ __forceinline__ u64 wave_wait(const char* flag, u64 need, u32 seq, char* tmo, bool& bad) {
  const u64 t0 = __builtin_amdgcn_s_memrealtime();
  for (u32 it = 1;; ++it) {
    const u64 ok = __ballot(ld_sys(flag) >= seq);
    if ((ok & need) == need) return ok;
    if ((it & 15) == 0 || bad) {
      const u32 t = __hip_atomic_load((const GAS u32*)tmo, __ATOMIC_RELAXED, __HIP_MEMORY_SCOPE_AGENT);
      if (rfl(t) == seq || __builtin_amdgcn_s_memrealtime() - t0 > DEADLINE_TICKS) {
        __hip_atomic_store((GAS u32*)tmo, seq, __ATOMIC_RELAXED, __HIP_MEMORY_SCOPE_AGENT);
        bad = true;
        return ~0ull;
      }
    }
    __builtin_amdgcn_s_sleep(SLEEP);
  }
}

template <int SLEEP>
__device__ __forceinline__ u64 wave_wait2(const char* flag, u64 need, u32 seq, char* tmo, bool& bad) {
  const u64 t0 = __builtin_amdgcn_s_memrealtime();
  const bool needed = (need >> (threadIdx.x & 63)) & 1;
  bool ok = false;
  for (u32 it = 1;; ++it) {
    if (!ok && (it == 1 || needed)) ok = ld_sys(flag) >= seq;
    const u64 have = __ballot(ok);
    if ((have & need) == need) return have;
    if ((it & 15) == 0) {
      const u32 t = __hip_atomic_load((const GAS u32*)tmo, __ATOMIC_RELAXED, __HIP_MEMORY_SCOPE_AGENT);
      if (rfl(t) == seq || __builtin_amdgcn_s_memrealtime() - t0 > DEADLINE_TICKS) {
        __hip_atomic_store((GAS u32*)tmo, seq, __ATOMIC_RELAXED, __HIP_MEMORY_SCOPE_AGENT);
        bad = true;
        return ~0ull;
      }
    }
    __builtin_amdgcn_s_sleep(SLEEP);
  }
}

#define STAMPW(i, w) do {} while (0)
#define STAMP(i) STAMPW(i, 0)
#define NOW(x) do {} while (0)
#define TRACEJ(kind, job) do {} while (0)

__device__ __forceinline__ void dma4(u32 dst, __amdgpu_buffer_rsrc_t ra, u32 va0, u32 va1, u32 sa,
                                     __amdgpu_buffer_rsrc_t rb, u32 vb0, u32 vb1, u32 sb) {
  u32 keep;
  dst = rfl(dst); sa = rfl(sa); sb = rfl(sb);
  asm volatile(
      "s_mov_b32 %0, m0\n\t"
      "s_nop 4\n\t"
      "s_mov_b32 m0, %1\n\t"
      "s_nop 0\n\t"
      "buffer_load_dwordx4 %3, %5, %6 offen lds\n\t"
      "buffer_load_dwordx4 %4, %5, %6 offen offset:1024 lds\n\t"
      "s_mov_b32 m0, %2\n\t"
      "s_nop 0\n\t"
      "buffer_load_dwordx4 %7, %9, %10 offen lds\n\t"
      "buffer_load_dwordx4 %8, %9, %10 offen offset:1024 lds\n\t"
      "s_mov_b32 m0, %0"
      : "=&s"(keep)
      : "s"(dst), "s"(dst + BLK), "v"(va0), "v"(va1), "s"(ra), "s"(sa), "v"(vb0), "v"(vb1), "s"(rb), "s"(sb)
      : "memory");
}

__device__ __forceinline__ void dma3(u32 dsta, u32 dstb, __amdgpu_buffer_rsrc_t ra, u32 va0, u32 sa,
                                     __amdgpu_buffer_rsrc_t rb, u32 vb0, u32 vb1, u32 sb) {
  u32 keep;
  dsta = rfl(dsta); dstb = rfl(dstb); sa = rfl(sa); sb = rfl(sb);
  asm volatile(
      "s_mov_b32 %0, m0\n\t"
      "s_nop 4\n\t"
      "s_mov_b32 m0, %1\n\t"
      "s_nop 0\n\t"
      "buffer_load_dwordx4 %3, %4, %5 offen lds\n\t"
      "s_mov_b32 m0, %2\n\t"
      "s_nop 0\n\t"
      "buffer_load_dwordx4 %6, %8, %9 offen lds\n\t"
      "buffer_load_dwordx4 %7, %8, %9 offen offset:1024 lds\n\t"
      "s_mov_b32 m0, %0"
      : "=&s"(keep)
      : "s"(dsta), "s"(dstb), "v"(va0), "s"(ra), "s"(sa), "v"(vb0), "v"(vb1), "s"(rb), "s"(sb)
      : "memory");
}

__device__ __forceinline__ void dma_flags(u32 dst, const char* src) {
  u32 keep;
  dst = rfl(dst);
  asm volatile("s_mov_b32 %0, m0\n\ts_nop 0\n\ts_mov_b32 m0, %1\n\ts_nop 0\n\tglobal_load_lds_dword %2, off sc0 sc1\n\ts_mov_b32 m0, %0"
               : "=&s"(keep) : "s"(dst), "v"(src) : "memory");
}

template <int I, int N, class F>
__device__ __forceinline__ void static_for(F&& f) {
  if constexpr (I < N) { f(std::integral_constant<int, I>{}); static_for<I + 1, N>(f); }
}

__device__ __forceinline__ u32 sup(u32 grp, u32 c, u32 u) { return grp ? c + 2 * u : 2 * c + u; }
__device__ __forceinline__ u32 srow(u32 r, u32 u) { return sup(r >> 2, (r >> 1) & 1, u); }
__device__ __forceinline__ u32 scol(u32 r, u32 v) { return sup(r >> 2, r & 1, v); }
__device__ __forceinline__ void tile_rc(u32 m, u32& ri, u32& ci) {
  const u32 l = (m >= 1) + (m >= 4) + (m >= 9) + (m >= 16) + (m >= 25) + (m >= 36) + (m >= 49);
  const u32 idx = m - l * l;
  ri = idx < l ? l : idx - l;
  ci = idx < l ? idx : l;
}
__device__ __forceinline__ u32 fin_off(u32 r, u32 m, u32 sb, u32 part) {
  u32 ri, ci;
  tile_rc(m, ri, ci);
  return (srow(r, sb >> 1) * 1024 + ri * 128 + (r >> 2) * 64 + part * 32) * 8192 + (scol(r, sb & 1) * 1024 + ci * 128) * 2;
}

struct Job { u32 src, sstride, dst, flag, blo, bhi, voff; };

__device__ __forceinline__ Job decode_push(u32 job, u32 rank, u32 peer, u64 b0, u64 b1, u32 vnarrow, u32 vwide) {
  Job j;
  const bool cross = (rank ^ peer) >= 4;
  const u32 lk = job >> 3, op = (job >> 2) & 1, uv = (job >> 1) & 1, half = job & 1;
  const u32 lev = cross ? lk / KC : lk / KI;
  const u32 ktl = lk - lev * (cross ? KC : KI), kt = cross ? KI + ktl : ktl;
  j.blo = op ? (u32)b1 : (u32)b0; j.bhi = op ? (u32)(b1 >> 32) : (u32)(b0 >> 32); j.voff = op ? vwide : vnarrow;
  j.src = op ? (kt * 64 + half * 32) * 8192 + (scol(peer, uv) * 1024 + lev * 128) * 2
             : (srow(peer, uv) * 1024 + lev * 128 + half * 64) * 1024 + kt * 128;
  j.sstride = op ? 4 * 8192 : 8 * 1024;
  j.dst = IN_OFF + rank * IN_SRC + job * 8192;
  j.flag = FI_OFF + (rank * 320 + job) * FIS;
  return j;
}

__device__ __forceinline__ GAS u32* ticket_ptr(char* ws, u32 idx) {
  u32 zero = 0;
  asm volatile("" : "+v"(zero));
  return (GAS u32*)(ws + TICK_OFF + idx * 128 + zero);
}

__device__ __forceinline__ u32x4 push_ld(const Job& j, int i) {
  const auto r = mk_rsrc((const void*)(((u64)j.bhi << 32) | j.blo), X_BYTES);
  return __builtin_amdgcn_raw_buffer_load_b128(r, j.voff, rfl(j.src + i * j.sstride), 0);
}

template <bool claimed>
__device__ __forceinline__ bool push_role(const Params& p, u32& slot, u32 wid, int lane, LAS u32* lctl) {
  constexpr int D = PUSH_D, JI = 8;
  constexpr u32 NW = ENG_WGS * PUSH_WAVES;
  NOW(tA);
  const u32 g = slot & 7, w = wid * ENG_WGS + (slot >> 3);
  static_assert(X_BYTES == W_BYTES, "one size for both input shards");
  u64 b0 = (u64)p.x, b1 = (u64)p.w, bw = (u64)p.ws[g];
  u32 rank = p.rank;
  asm volatile("" : "+s"(b0), "+s"(b1), "+s"(bw), "+s"(rank));
  const u32 peer = rank ^ g;
  char* wsp = (char*)bw;
  const auto rd = mk_rsrc(wsp, DATA_BYTES);
  const u32 vnarrow = (lane >> 3) * 1024 + (lane & 7) * 16;
  const u32 vwide = (lane >> 4) * 8192 + (lane & 15) * 16;
  Job cur = decode_push(w, rank, peer, b0, b1, vnarrow, vwide);
  u32x4 v[D], v2[JI - D];
#pragma unroll
  for (int i = 0; i < D; ++i) { v[i] = push_ld(cur, i); CBAR(); }
#pragma unroll
  for (int i = D; i < JI; ++i) { v2[i - D] = push_ld(cur, i); CBAR(); }
  NOW(tB);
  u32 nid = NW + w, pflag = 0, seq = 0;
  auto round = [&](auto kind_c) {
    constexpr int KIND = decltype(kind_c)::value;
    constexpr bool FIRST = KIND == 0, LAST = KIND == 3;
    Job nxt = cur;
    if constexpr (!LAST) { nxt = decode_push(nid, rank, peer, b0, b1, vnarrow, vwide); }
#pragma unroll
    for (int i = 0; i < JI; ++i) {
      if (!FIRST && i == D - 1) {
        WAIT_VM(D + (LAST ? (JI - D < D - 1 ? JI - D : D - 1) : D - 1));
        if (KIND == 1) {
          if (!claimed) __builtin_amdgcn_s_barrier();
          CBAR();
          seq = rfl(LD_LDS(lctl + 2));
        }
        if (lane == 0) st_sys(wsp + pflag, seq);
        TRACEJ(0, (pflag - FI_OFF) / FIS - rank * 320);
        CBAR();
      }
      if ((LAST && i == JI - GO_AT) || (KIND == 2 && i == 2 * JI - GO_AT && nid + NW >= NPUSH)) {
        if (lane == 0) st_sys(wsp + GO_OFF + (rank * 16 + w) * FOS, seq);
        CBAR();
      }
      st128(FIRST && i >= D ? v2[i >= D ? i - D : 0] : v[i % D], rd, lane * 16, rfl(cur.dst) + i * 1024);
      if (i + D < JI) { if (!FIRST) v[i % D] = push_ld(cur, i + D); }
      else if (!LAST) v[i % D] = push_ld(nxt, i + D - JI);
      CBAR();
    }
    pflag = cur.flag;
    cur = nxt;
    nid += NW;
  };
  round(std::integral_constant<int, 0>{});
  NOW(tE);
  round(std::integral_constant<int, 1>{});
  const u32 slot2 = rfl(LD_LDS(lctl + 1));
  if (__builtin_expect(slot2 != slot, 0)) { WAIT_VM(0); slot = slot2; return false; }
  NOW(tC);
  for (int r = 3; r < PUSH_ROUNDS; ++r) round(std::integral_constant<int, 2>{});
  round(std::integral_constant<int, 3>{});
  WAIT_VM(0);
  if (lane == 0) st_sys(wsp + pflag, seq);
        TRACEJ(0, (pflag - FI_OFF) / FIS - rank * 320);
  STAMP(3);
  return true;
}

__device__ __forceinline__ float silu(float a) {
  const float c = fminf(fmaxf(a, -60.f), 60.f);
  return a * __builtin_amdgcn_rcpf(1.f + __expf(-c));
}

constexpr u32 MB_REQ = 0, MB_DONE = 4, MB_GONE = 8, MB_UNIT = 10, MB_BUF = 1024;

__device__ __forceinline__ void pull_role(const Params& p, u32 peer, u32 w, int lane, u32 seq, LAS u32* ctl) {
  constexpr u32 NW = ENG_WGS * PULL_WAVES, KF = 0, KIN = 5;
  const u32 rank = p.rank;
  const bool cross = (rank ^ peer) >= 4;
  const u32 kts = cross ? KC : KI, NI = kts * 64 - NPUSH, NJ = NI + 1024;
  char* ws = p.ws[0];
  char* tmo = ws + TMO_OFF;
  const u64 bf = (u64)p.out[rank ^ peer], bp = (u64)p.ws[rank ^ peer], bo = (u64)p.out[0];
  const u32 vwide = (lane >> 4) * 8192 + (lane & 15) * 16;
  GAS u32* tick = ticket_ptr(ws, 8 + peer);
  auto level_of = [&](u32 id) -> u32 { const u32 lk = (NPUSH + id) >> 3; return cross ? lk / KC : lk / KI; };
  static_assert(PUSH_WAVES == PULL_WAVES && NW <= 16, "one pull wave takes over from one push wave");
  const u32 trig = GO_OFF + (peer * 16 + w) * FOS;
  const u64 t0 = __builtin_amdgcn_s_memrealtime();
  u32 tf = w, tf2 = NW + w, lastf = 0, sub = 0;
  u32 cdst = 0, ck = 0, cflag = 0, pend = 0;
  u32 nsrc = 0, ndst = 0, nk = 0, nflag = 0;
  bool bad = false;
  auto poll = [&]() -> u32 {
    const u32 o = tf < NI ? SG_OFF + (peer * 40 + ((NPUSH + tf) >> 3)) * FOS : FF_OFF + (peer * 256 + (((tf - NI) >> 2) & 255)) * FOS;
    return ld_sys(ws + ((lane & 1) ? rfl(tf == w ? trig : o) : rfl(o)));
  };
  u32 pv = poll();
  u32x4 v[4];
  auto pick = [&]() -> bool {
    const u64 req = __ballot(lane < FIN_WAVES && LD_LDS(ctl + MB_REQ + (lane & 3)) != 0);
    if (req) {
      const u32 f = __builtin_ctzll(req);
      u32 r = 0;
      if (lane == 0) r = __hip_atomic_exchange(ctl + MB_REQ + f, 0u, __ATOMIC_RELAXED, __HIP_MEMORY_SCOPE_WORKGROUP);
      r = rfl(r);
      if (r) { nsrc = r; ndst = r; nk = f + 1; nflag = 0; return true; }
    }
    if (tf < NJ && (bad || (__ballot(pv >= seq) & 3) == 3)) {
      if (tf < NI) {
        const u32 jb = NPUSH + tf, lev = level_of(tf), ktl = (jb >> 3) - lev * kts, op = (jb >> 2) & 1, uv = (jb >> 1) & 1;
        nsrc = EX_OFF + (((lev * 8 + (cross ? KI + ktl : ktl)) * 2 + op) * 4 + (op ? scol(rank, uv) : srow(rank, uv))) * BLK + ((jb & 1) * 2 + sub) * 4096;
        ndst = IN_OFF + peer * IN_SRC + jb * 8192 + sub * 4096;
        nk = KIN; nflag = sub ? FI_OFF + (peer * 320 + jb) * FIS : 0;
      } else {
        const u32 j = tf - NI;
        nsrc = ndst = fin_off(peer, j >> 4, (j >> 2) & 3, 0) + (j & 3) * 16 * 8192;
        nk = KF; nflag = 0;
      }
      nsrc = rfl(nsrc); ndst = rfl(ndst); nflag = rfl(nflag);
      return true;
    }
    return false;
  };
  auto put = [&](int i) {
    const bool isp = ck - 1 < FIN_WAVES;
    const auto ro = mk_rsrc((const void*)(ck == KF ? bo : (u64)ws), ck == KF ? OUT_BYTES : isp ? 0u : DATA_BYTES);
    st128(v[i], ro, ck == KF ? vwide : lane * 16, rfl(cdst + i * (ck == KF ? 4 * 8192 : 1024)));
    if (isp) *(LAS u32x4*)((LAS char*)ctl + MB_BUF + (ck - 1) * 4096 + i * 1024 + lane * 16) = v[i];
  };
  u32 curid = 0;
  auto delivered = [&]() {
    if (ck == KF) { TRACEJ(1, curid - NI); } else if (ck == KIN) { TRACEJ(2, 256 + ((cdst - IN_OFF - peer * IN_SRC) >> 12)); } else { TRACEJ(2, ((cdst - P16_OFF) >> 12) & 255); }
    if (ck - 1 < FIN_WAVES) {
      asm volatile("s_waitcnt lgkmcnt(0)" ::: "memory");
      if (lane == 0) __hip_atomic_fetch_add(ctl + MB_DONE + ck - 1, 1u, __ATOMIC_RELAXED, __HIP_MEMORY_SCOPE_WORKGROUP);
    }
  };
  auto publish = [&](u32 off) {
    if (lane == 0) st_sys(ws + off, seq);
    CBAR();
  };
  auto round = [&](auto first_c) {
    constexpr bool FIRST = decltype(first_c)::value;
    u32 tk = 0;
    const bool tix = nk == KF || (nk == KIN && sub);
    const u32 tid_ = tf;
    if (nk == KF) lastf = ndst;
    if (nk == KIN) sub ^= 1;
    if (tix) tf = tf2;
    if (lane == 0 && tix) tk = __hip_atomic_fetch_add(tick, 1u, __ATOMIC_RELAXED, __HIP_MEMORY_SCOPE_AGENT);
    pv = poll();
    CBAR();
    const auto rsrc = mk_rsrc((const void*)(nk == KF ? bf : bp), nk == KF ? OUT_BYTES : DATA_BYTES);
    const u32 vs = nk == KF ? vwide : lane * 16, str = nk == KF ? 4 * 8192 : 1024;
#pragma unroll
    for (int i = 0; i < 4; ++i) {
      if (!FIRST) put(i);
      v[i] = __builtin_amdgcn_raw_buffer_load_b128(rsrc, vs, rfl(nsrc + i * str), AUX_SYS);
      CBAR();
    }
    if (!FIRST) {
      delivered();
      if (pend) { WAIT_VM(9); publish(pend); }
      pend = cflag;
    }
    cdst = ndst; ck = nk; cflag = nflag; curid = tid_;
    if (tix) tf2 = rfl(tk) - (seq - 1) * NJ + 2 * NW;
  };
  for (u32 it = 1;; ++it) {
    if (pick()) {
      if (tf == w && nk == KIN) { STAMPW(5, PUSH_WAVES); }
      round(std::true_type{});
      if (pick()) {
        round(std::false_type{});
        while (pick()) round(std::false_type{});
      }
#pragma unroll
      for (int i = 0; i < 4; ++i) put(i);
      delivered();
      if (pend | cflag) {
        WAIT_VM(0);
        if (pend) publish(pend);
        if (cflag) publish(cflag);
        pend = 0;
      }
      continue;
    }
    if (tf >= NJ && rfl(LD_LDS(ctl + MB_GONE)) == (cross ? FIN_WAVES : 0)) break;
    if ((it & 63) == 0) {
      const u32 t = __hip_atomic_load((const GAS u32*)tmo, __ATOMIC_RELAXED, __HIP_MEMORY_SCOPE_AGENT);
      if (rfl(t) == seq || __builtin_amdgcn_s_memrealtime() - t0 > DEADLINE_TICKS) {
        __hip_atomic_store((GAS u32*)tmo, seq, __ATOMIC_RELAXED, __HIP_MEMORY_SCOPE_AGENT);
        WAIT_VM(0);
        bad = true;
      }
    }
    __builtin_amdgcn_s_sleep(SL_PULL);
    if ((it & PDIV) == 0 || tf == w) pv = poll();
  }
  if (bad) {
    WAIT_VM(0);
    const u32x4 nanv = {NAN2, NAN2, NAN2, NAN2};
    for (int i = 0; i < 4; ++i) st128(nanv, mk_rsrc(p.out[0], OUT_BYTES), vwide, rfl(lastf) + i * 4 * 8192);
  }
  STAMPW(6, PUSH_WAVES);
}

__device__ __forceinline__ void fin_role(const Params& p, u32 peer, u32 s, u32 f, int lane, u32 seq, LAS u32* ctl) {
  static_assert(FIN_WAVES == 4, "one wave per 16 rows of a half tile");
  const u32 rank = p.rank, sb = peer & 3;
  char* ws = p.ws[0];
  char* tmo = ws + TMO_OFF;
  const auto rl = mk_rsrc(ws, DATA_BYTES);
  const auto ro = mk_rsrc(p.out[0], OUT_BYTES);
  const u32 vwide = (lane >> 4) * 8192 + (lane & 15) * 16;
  char* wsl = p.ws[lane & 7];
  u32 ndone = 0;
  for (u32 m = MDIR + s; m < 64; m += ENG_WGS) {
    const u32 lo = P32_OFF + (sb * 64 + m) * 32768 + f * 8192;
    const u32 dst = fin_off(rank, m, sb, 0) + f * 16 * 8192;
    bool bad = false;
    wave_wait<SL_FIN>(ws + ((lane & 1) ? FPL_OFF : FPR_OFF) + (sb * 64 + m) * FOS, 3, seq, tmo, bad);
    if (m == MDIR + s) { STAMPW(8, PUSH_WAVES + PULL_WAVES); STAMPW(11, 7); }
    const u32 so = P16_OFF + ((rank & 3) * 64 + m) * 16384 + f * 4096;
    u32x4 rv[4], a[4], b[4];
    if (lane == 0) ST_LDS(ctl + MB_REQ + f, so);
#pragma unroll
    for (int i = 0; i < 4; ++i) {
      a[i] = __builtin_amdgcn_raw_buffer_load_b128(rl, lane * 16, lo + i * 2048, AUX_SYS);
      b[i] = __builtin_amdgcn_raw_buffer_load_b128(rl, lane * 16, lo + i * 2048 + 1024, AUX_SYS);
    }
    ++ndone;
    while (rfl(LD_LDS(ctl + MB_DONE + f)) != ndone) __builtin_amdgcn_s_sleep(1);
    CBAR();
#pragma unroll
    for (int i = 0; i < 4; ++i) rv[i] = *(const LAS u32x4*)((LAS char*)ctl + MB_BUF + f * 4096 + i * 1024 + lane * 16);
    if (m == MDIR + s) { STAMPW(9, PUSH_WAVES + PULL_WAVES);  }
#pragma unroll
    for (int i = 0; i < 4; ++i) {
      const u32x4 r = rv[i];
      float y[8];
#pragma unroll
      for (int e = 0; e < 4; ++e) {
        const u32 l0 = e < 2 ? a[i][2 * e] : b[i][2 * e - 4], l1 = e < 2 ? a[i][2 * e + 1] : b[i][2 * e - 3];
        y[2 * e] = silu(__uint_as_float(l0) + __uint_as_float(r[e] << 16));
        y[2 * e + 1] = silu(__uint_as_float(l1) + __uint_as_float(r[e] & 0xFFFF0000u));
      }
      const bf16x8 o = {(__bf16)y[0], (__bf16)y[1], (__bf16)y[2], (__bf16)y[3],
                        (__bf16)y[4], (__bf16)y[5], (__bf16)y[6], (__bf16)y[7]};
      u32x4 ov = __builtin_bit_cast(u32x4, o);
      if (bad) ov |= NAN2;
      st128(ov, ro, vwide, dst + i * 4 * 8192);
    }
    if (m == MDIR + s) { STAMPW(10, PUSH_WAVES + PULL_WAVES); }
    WAIT_VM(0);

    u32 arrived = 0;
    if (lane == 0) arrived = __hip_atomic_fetch_add(ctl + MB_UNIT + ndone - 1, 1u, __ATOMIC_RELAXED, __HIP_MEMORY_SCOPE_WORKGROUP);
    if (rfl(arrived) == FIN_WAVES - 1) {
      if (lane >= 1 && lane < NR) st_sys(wsl + FF_OFF + (rank * 256 + m * 4 + sb) * FOS, seq);
    }
    if (m == MDIR + s) { STAMPW(1, PUSH_WAVES + PULL_WAVES); }
  }
  if (lane == 0) __hip_atomic_fetch_add(ctl + MB_GONE, 1u, __ATOMIC_RELAXED, __HIP_MEMORY_SCOPE_WORKGROUP);
  STAMPW(2, PUSH_WAVES + PULL_WAVES + FIN_WAVES - 1);
}

template <bool HALF>
__device__ __forceinline__ void gemm_tile(const Params& p, u32 sb, u32 m, u32 hsel, u32 seq, u64 tstart, int sbase, LAS char* lds) {
  const int tid = threadIdx.x, lane = tid & 63;
  const u32 wid = rfl(tid >> 6);
  const u32 rank = p.rank;
  char* ws = p.ws[0];
  char* tmo = ws + TMO_OFF;
  const auto rx = mk_rsrc(p.x, X_BYTES);
  const auto rw = mk_rsrc(p.w, W_BYTES);
  const auto rs = mk_rsrc(ws, DATA_BYTES);
  const u32 u = sb >> 1, v = sb & 1;
  constexpr int NT = 32, T_CROSS = KI, T_INTRA = KI + 4 * KC, T_SPLIT = 4;
  constexpr int MI = HALF ? 1 : 2, NDMA = HALF ? 3 : 4, NS = HALF ? NS_HALF : 4;
  constexpr u32 XB = HALF ? BLK / 2 : BLK, SB = XB + BLK;
  static_assert(NS * SB <= LDS_CTL, "stages fit");

  u32 vao[2], vbo[2], vaw[2], vbw[2];
#pragma unroll
  for (int q = 0; q < 2; ++q) {
    const u32 ra = wid * 8 * MI + q * 8 + (lane >> 3), ca = (lane & 7) ^ ((ra >> 1) & 7);
    const u32 kb = wid * 8 + q * 4 + (lane >> 4), cbk = (lane & 15) ^ (((kb & 3) << 2) | ((kb >> 2) & 3));
    vao[q] = ra * 1024 + ca * 16 - q * 1024;  vaw[q] = ra * 128 + ca * 16 - q * 1024;
    vbo[q] = kb * 8192 + cbk * 16 - q * 1024; vbw[q] = kb * 256 + cbk * 16 - q * 1024;
  }
  const u32 lds0 = (u32)(size_t)lds;
  const u32 wm = wid & 1, wn = wid >> 1, h = lane >> 5;
  u32 xo[4], wo[2];
  {
    const u32 r = wm * 32 * MI + (lane & 31);
#pragma unroll
    for (int ks = 0; ks < 4; ++ks) xo[ks] = r * 128 + (((2 * ks + h) ^ ((r >> 1) & 7)) * 16);
    const u32 bk = (lane >> 4) & 1, q = (lane & 15) >> 2, pp = lane & 3;
#pragma unroll
    for (int t = 0; t < 2; ++t)
      wo[t] = XB + 256 * (8 * h + 4 * t + q) + 16 * ((4 * wn + 2 * bk + (pp >> 1)) ^ ((q << 2) | (2 * h + t))) + 8 * (pp & 1);
  }
  LAS u32* lbad = (LAS u32*)(lds + LDS_CTL);
  LAS u32* fimg = (LAS u32*)(lds + LDS_CTL + 256) + lane;
  if (ASYNC_POLL && wid == 0) { fimg[0] = 0; fimg[64] = 0; }
  u32 ri, ci;
  tile_rc(m, ri, ci);
  const u32 row0 = srow(rank, u) * 1024 + ri * 128 + (HALF ? hsel * 64 : 0), col0 = scol(rank, v) * 1024 + ci * 128;
  u32 islot = 0, rslot = 0;
  auto next_slot = [&](u32 o) -> u32 { return o + SB == NS * SB ? 0 : o + SB; };
  auto issue_own = [&](int t) {
    const u32 dst = lds0 + islot, sa = row0 * 1024 + t * 128, sw = t * 64 * 8192 + col0 * 2;
    islot = next_slot(islot);
    if (HALF) dma3(dst + wid * 1024, dst + XB + wid * 2048, rx, vao[0], sa, rw, vbo[0], vbo[1], sw);
    else dma4(dst + wid * 2048, rx, vao[0], vao[1], sa, rw, vbo[0], vbo[1], sw);
  };
  u32 taba, tabw;
  {
    const bool cr = lane < T_INTRA;
    const u32 c = cr ? lane - T_CROSS : lane - T_INTRA, ktl = cr ? c >> 2 : c / 3;
    const u32 src = rank ^ (cr ? 4 + (c & 3) : 1 + c - ktl * 3), kts = cr ? KC : KI;
    const u32 base = IN_OFF + src * IN_SRC + ktl * 4 * BLK;
    taba = base + (ri * kts * 4 + u) * BLK + (HALF ? hsel * 8192 : 0);
    tabw = base + (ci * kts * 4 + 2 + v) * BLK;
  }
  auto issue = [&](int t) {
    const u32 dst = lds0 + islot, sa = __builtin_amdgcn_readlane(taba, t), sw = __builtin_amdgcn_readlane(tabw, t);
    islot = next_slot(islot);
    if (HALF) dma3(dst + wid * 1024, dst + XB + wid * 2048, rs, vaw[0], sa, rs, vbw[0], vbw[1], sw);
    else dma4(dst + wid * 2048, rs, vaw[0], vaw[1], sa, rs, vbw[0], vbw[1], sw);
  };
  const u32 fop = (lane >> 1) & 1, flev = fop ? ci : ri, fuv = fop ? v : u, fq = lane >> 2;
  const u32 fh = (HALF && !fop) ? hsel : (lane & 1);
  const u32 fck = min(fq >> 2, KC - 1), fik = min(fq / 3, KI - 1);
  const char* fcross = ws + FI_OFF + ((rank ^ (4 + (fq & 3))) * 320 + ((flev * KC + fck) * 4 + fop * 2 + fuv) * 2 + fh) * FIS;
  const char* fintra = ws + FI_OFF + ((rank ^ (1 + fq % 3)) * 320 + ((flev * KI + fik) * 4 + fop * 2 + fuv) * 2 + fh) * FIS;

  while (__builtin_amdgcn_s_memrealtime() - tstart < (u64)max(ri, ci) * STAGGER) __builtin_amdgcn_s_sleep(16);
  if (ACQ == 3 && sbase == 1 && wid == 0) __builtin_amdgcn_fence(__ATOMIC_ACQUIRE, "");
  f32x16 acc[MI] = {};
  struct Frag { bf16x8 w[4], x[MI][4]; };
  auto read = [&](Frag& f) {
    const LAS char* st = lds + rslot;
    rslot = next_slot(rslot);
#pragma unroll
    for (int ks = 0; ks < 4; ++ks) {
      const bf16x4 w0 = __builtin_amdgcn_ds_read_tr16_b64_v4bf16((LAS bf16x4*)(st + wo[0] + ks * 4096));
      const bf16x4 w1 = __builtin_amdgcn_ds_read_tr16_b64_v4bf16((LAS bf16x4*)(st + wo[1] + ks * 4096));
      f.w[ks] = __builtin_shufflevector(w0, w1, 0, 1, 2, 3, 4, 5, 6, 7);
#pragma unroll
      for (int mi = 0; mi < MI; ++mi) f.x[mi][ks] = *(const LAS bf16x8*)(st + xo[ks] + mi * 4096);
    }
  };
  auto mma = [&](const Frag& f) {
#pragma unroll
    for (int ks = 0; ks < 4; ++ks)
#pragma unroll
      for (int mi = 0; mi < MI; ++mi) acc[mi] = __builtin_amdgcn_mfma_f32_32x32x16_bf16(f.w[ks], f.x[mi][ks], acc[mi], 0, 0, 0);
  };
  int next_wait = T_CROSS;
  u64 need = 0xFFFF, have = 0;
  auto waitpoint = [&](int tn) {
    if (tn == T_INTRA) { need = 0xFFF; have = 0; }
    if (wid == 0 && (have & need) != need) {
      if (ASYNC_POLL && HALF) have = __ballot(LD_LDS(fimg + (tn < T_INTRA ? 0 : 64)) >= seq);
      if ((have & need) != need) {
        bool bad = false;
        have = MASKED ? wave_wait2<SL_GEMM>(tn < T_INTRA ? fcross : fintra, need, seq, tmo, bad)
                      : wave_wait<SL_GEMM>(tn < T_INTRA ? fcross : fintra, need, seq, tmo, bad);
        if (bad) *lbad = 1;
      }
    }
    if (tn == T_CROSS) { STAMP(sbase); }
    if (tn == NT - 3) { STAMP(sbase + 1); }
    if (tn < T_INTRA) { next_wait += 4; need = (need << 16) | 0xFFFF; }
    else { next_wait += 3; need = (need << 12) | 0xFFF; }
  };
  auto step = [&](auto rc, int tn, const Frag& fc, Frag& fn) {
    constexpr int R = decltype(rc)::value;
    if (tn >= 0 && __builtin_expect(tn == next_wait, 0)) waitpoint(tn);
    if constexpr (R >= 0) {
      WAIT_VM(R * NDMA);
      __builtin_amdgcn_s_waitcnt(0xC07F);
      CBAR();
      __builtin_amdgcn_s_barrier();
      CBAR();
      read(fn);
    }
    if (tn >= 0) issue(tn);
    if (ASYNC_POLL && HALF && tn >= 0 && wid == 1 && next_wait < NT)
      dma_flags(lds0 + LDS_CTL + (next_wait < T_INTRA ? 256 : 512), next_wait < T_INTRA ? fcross : fintra);
    mma(fc);
  };
  Frag fa, fb;
  auto tail = [&](auto nc) {
    constexpr int N = decltype(nc)::value;
    static_for<0, N / 2>([&](auto ic) {
      constexpr int i = decltype(ic)::value;
      step(std::integral_constant<int, N - 2 - 2 * i>{}, -1, fa, fb);
      step(std::integral_constant<int, N - 3 - 2 * i>{}, -1, fb, fa);
    });
  };
  static_assert(NS % 2 == 0 && NS >= T_SPLIT && T_SPLIT + 1 == T_CROSS, "run structure");
  for (int t = 0; t < T_SPLIT; ++t) issue_own(t);
  WAIT_VM((T_SPLIT - 1) * NDMA);
  __builtin_amdgcn_s_barrier();
  CBAR();
  read(fa);
  tail(std::integral_constant<int, T_SPLIT>{});
  islot = rslot = 0;
  __builtin_amdgcn_s_waitcnt(0xC07F);
  __builtin_amdgcn_s_barrier();
  issue_own(T_SPLIT);
  for (int t = T_CROSS; t < T_SPLIT + NS; ++t) {
    if (t == next_wait) { waitpoint(t); __builtin_amdgcn_s_barrier(); }
    issue(t);
  }
  WAIT_VM((NS - 1) * NDMA);
  __builtin_amdgcn_s_barrier();
  CBAR();
  read(fa);
  for (int tn = T_SPLIT + NS; tn < NT; tn += 2) {
    step(std::integral_constant<int, NS - 2>{}, tn, fa, fb);
    step(std::integral_constant<int, NS - 2>{}, tn + 1, fb, fa);
  }
  tail(std::integral_constant<int, NS>{});
  STAMP(sbase + 2);
  const u32 tmv = __hip_atomic_load((const GAS u32*)tmo, __ATOMIC_RELAXED, __HIP_MEMORY_SCOPE_AGENT);
  __builtin_amdgcn_s_waitcnt(0xC07F);
  __builtin_amdgcn_s_barrier();
  CBAR();
  constexpr u32 EROW = 528;
#pragma unroll
  for (int mi = 0; mi < MI; ++mi) {
    const u32 mr = wm * 32 * MI + mi * 32 + (lane & 31);
#pragma unroll
    for (int gq = 0; gq < 4; ++gq) {
      const f32x4 o = {acc[mi][4 * gq], acc[mi][4 * gq + 1], acc[mi][4 * gq + 2], acc[mi][4 * gq + 3]};
      *(LAS f32x4*)(lds + mr * EROW + (wn * 32 + 8 * gq + 4 * h) * 4) = o;
    }
  }
  __syncthreads();
  const u32 bad = (*lbad || tmv == seq) ? NAN2 : 0u;
  const u32 keep = rank >> 2;
  const bool do_exp = !HALF || hsel != keep, do_keep = !HALF || hsel == keep;
  char* wsq = p.ws[4 | ((rank & 3) ^ sb)];
  if (do_exp) {
    const u32 e16 = P16_OFF + (sb * 64 + m) * 16384 + wid * 1024;
#pragma unroll
    for (int part = 0; part < 2; ++part) {
      const LAS char* se = lds + ((HALF ? 0 : (keep ^ 1) * 64) + part * 32 + (tid >> 4)) * EROW + (tid & 15) * 32;
      const f32x4 c0 = *(const LAS f32x4*)se, c1 = *(const LAS f32x4*)(se + 16);
      const bf16x8 o = {(__bf16)c0[0], (__bf16)c0[1], (__bf16)c0[2], (__bf16)c0[3],
                        (__bf16)c1[0], (__bf16)c1[1], (__bf16)c1[2], (__bf16)c1[3]};
      st128(__builtin_bit_cast(u32x4, o) | bad, rs, lane * 16, e16 + part * 8192);
    }
  }
  if (HALF && do_keep && m < MDIR) {
    f32x4 kc[2][2];
#pragma unroll
    for (int part = 0; part < 2; ++part) {
      const LAS char* sk = lds + (part * 32 + (tid >> 4)) * EROW + (tid & 15) * 32;
      kc[part][0] = *(const LAS f32x4*)sk; kc[part][1] = *(const LAS f32x4*)(sk + 16);
    }
    bool badw = bad != 0;
    wave_wait<2>(ws + FPR_OFF + (sb * 64 + m) * FOS, 1, seq, tmo, badw);
    STAMP(5);
    u32x4 r[2];
#pragma unroll
    for (int part = 0; part < 2; ++part)
    {
      r[part] = __builtin_amdgcn_raw_buffer_load_b128(mk_rsrc(wsq, DATA_BYTES), lane * 16, P16_OFF + ((rank & 3) * 64 + m) * 16384 + wid * 1024 + part * 8192, AUX_SYS);
      if (DIRPACE) WAIT_VM(0);
    }
    const auto ro = mk_rsrc(p.out[0], OUT_BYTES);
#pragma unroll
    for (int part = 0; part < 2; ++part) {
      float y[8];
#pragma unroll
      for (int e = 0; e < 4; ++e) {
        y[2 * e] = silu(kc[part][e >> 1][(2 * e) & 3] + __uint_as_float(r[part][e] << 16));
        y[2 * e + 1] = silu(kc[part][e >> 1][(2 * e + 1) & 3] + __uint_as_float(r[part][e] & 0xFFFF0000u));
      }
      const bf16x8 o = {(__bf16)y[0], (__bf16)y[1], (__bf16)y[2], (__bf16)y[3],
                        (__bf16)y[4], (__bf16)y[5], (__bf16)y[6], (__bf16)y[7]};
      st128(__builtin_bit_cast(u32x4, o) | (badw ? NAN2 : 0u), ro, (lane >> 4) * 8192 + (lane & 15) * 16,
            (row0 + part * 32 + wid * 4) * 8192 + col0 * 2);
    }
    WAIT_VM(0);
    __syncthreads();
    if (tid >= 1 && tid < NR) st_sys(p.ws[tid] + FF_OFF + (rank * 256 + m * 4 + sb) * FOS, seq);
    STAMP(6);
    return;
  }
  if (do_keep) {
    const u32 e32 = P32_OFF + (sb * 64 + m) * 32768 + wid * 2048;
#pragma unroll
    for (int part = 0; part < 2; ++part) {
      const LAS char* sk = lds + ((HALF ? 0 : keep * 64) + part * 32 + (tid >> 4)) * EROW + (tid & 15) * 32;
      const u32x4 c0 = *(const LAS u32x4*)sk, c1 = *(const LAS u32x4*)(sk + 16);
      st128(c0 | bad, rs, lane * 16, e32 + part * 16384);
      st128(c1 | bad, rs, lane * 16, e32 + part * 16384 + 1024);
    }
  }
  WAIT_VM(0);
  __syncthreads();
  if (tid == 0 && do_exp) st_sys(wsq + FPR_OFF + ((rank & 3) * 64 + m) * FOS, seq);
  if (tid == 64 && do_keep) st_sys(ws + FPL_OFF + (sb * 64 + m) * FOS, seq);
  STAMP(sbase + 3);
}

constexpr u32 LK0 = NPUSH / 8, NUNIT = 64 - 2 * LK0;
static_assert(NPUSH % 8 == 0 && LK0 < 8 * KC, "staging units");
__device__ __forceinline__ void stage_unit(const Params& p, u32 u, u32 seq) {
  const int tid = threadIdx.x, lane = tid & 63;
  const u32 wid = rfl(tid >> 6);
  constexpr u32 NPAIR = 2 * (8 * KC - LK0);
  const bool cr = u < NPAIR && (u & 1);
  const u32 lk = u < NPAIR ? LK0 + (u >> 1) : 8 * KC + u - NPAIR;
  const u32 lev = cr ? lk / KC : lk / KI, kt = cr ? KI + lk - lev * KC : lk - lev * KI, op = wid >> 2, sup = wid & 3;
  const auto r = mk_rsrc(op ? p.w : p.x, X_BYTES);
  const auto rs = mk_rsrc(p.ws[0], DATA_BYTES);
  const u32 vo = op ? (lane >> 4) * 8192 + (lane & 15) * 16 : (lane >> 3) * 1024 + (lane & 7) * 16;
  const u32 so = op ? kt * 64 * 8192 + (sup * 1024 + lev * 128) * 2 : (sup * 1024 + lev * 128) * 1024 + kt * 128;
  u32x4 v[16];
#pragma unroll
  for (int i = 0; i < 16; ++i) v[i] = __builtin_amdgcn_raw_buffer_load_b128(r, vo, so + i * (op ? 4 * 8192 : 8 * 1024), 0);
#pragma unroll
  for (int i = 0; i < 16; ++i) st128(v[i], rs, lane * 16, EX_OFF + ((lev * 8 + kt) * 8 + wid) * BLK + i * 1024);
  WAIT_VM(0);
  __syncthreads();
  if (tid >= 1 && tid < NR && (tid >= 4) == cr) st_sys(p.ws[tid] + SG_OFF + (p.rank * 40 + lk) * FOS, seq);
}

__device__ __forceinline__ void gemm_role(const Params& p, u32 g, u32 k, u32 seq, LAS char* lds) {
  const u64 tstart = __builtin_amdgcn_s_memrealtime();
  if (threadIdx.x < 64 && (ACQ == 1 || (ACQ == 2 && k == 0))) __builtin_amdgcn_fence(__ATOMIC_ACQUIRE, "");
  static_assert(GEMM_WGS >= 16 && GEMM_WGS <= 24, "unit schedule below");
  if (g == 0 && k >= 16)
    for (u32 r = 0; r < HELP_ROUNDS; ++r) { stage_unit(p, r * 16 + k - 8, seq); STAMP(12 + r); }
  const u32 keep = p.rank >> 2;
  if (k < 16) gemm_tile<true>(p, g >= 4 ? (p.rank ^ g) & 3 : g, k, g >= 4 ? keep : keep ^ 1, seq, tstart, 1, lds);
  const u32 arm = g & 1;
  for (u32 n = k < 16 ? GEMM_WGS - 16 + k : k - 16; n < 24; n += GEMM_WGS) {
    u32 lev = 4, idx = n;
    for (;;) {
      const u32 cnt = lev + ((lev & 1) == arm);
      if (idx < cnt) break;
      idx -= cnt; ++lev;
    }
    gemm_tile<false>(p, g >> 1, lev * lev + (idx == lev ? 2 * lev : arm * lev + idx), 0, seq, tstart, k < 16 ? 8 : 1, lds);
  }
}

__global__ void __launch_bounds__(THREADS, 2) fused_gemm_allreduce_silu(Params p) {
  extern __shared__ __attribute__((aligned(16))) char smem[];
  const int lane = threadIdx.x & 63;
  const u32 wid = rfl(threadIdx.x >> 6);
  LAS u32* ctl = (LAS u32*)smem;
  LAS u32* lctl = (LAS u32*)(smem + LDS_CTL);
  u32 slot = (blockIdx.x & ~7u) | ((u32)__builtin_amdgcn_s_getreg((3 << 11) | 20) & 7);
  auto is_push = [&]() { return slot < 8 * ENG_WGS && (slot & 7) != 0 && wid < PUSH_WAVES; };
  if (__builtin_expect(is_push(), 1)) {
    if (push_role<false>(p, slot, wid, lane, lctl)) return;
  } else {
    STAMPW(0, 2);
    if (wid == 7) {
      if (lane < 32) ctl[lane] = 0;
      if (lane == 0) {
        GAS u32* ctr = (GAS u32*)(p.ws[0] + CTR_OFF) + blockIdx.x;
        GAS u32* clm = (GAS u32*)(p.ws[0] + CLM_OFF);
        const u32 c = __hip_atomic_load(ctr, __ATOMIC_RELAXED, __HIP_MEMORY_SCOPE_AGENT);
        u32 sl = slot;
#pragma unroll 1
        for (int n = 0; n < 4 * GRID; ++n) {
          if (__hip_atomic_fetch_add(clm + sl * 16, 1u, __ATOMIC_RELAXED, __HIP_MEMORY_SCOPE_AGENT) == c) break;
          __hip_atomic_fetch_add(clm + sl * 16, ~0u, __ATOMIC_RELAXED, __HIP_MEMORY_SCOPE_AGENT);
          sl = (sl + 1) & (GRID - 1);
        }
        lctl[0] = 0; lctl[1] = sl; lctl[2] = c + 1;
      }
    }
    asm volatile("s_waitcnt lgkmcnt(0)" ::: "memory");
    __builtin_amdgcn_s_barrier();
    slot = rfl(LD_LDS(lctl + 1));
    if (threadIdx.x == THREADS - 1)
      __hip_atomic_store((GAS u32*)(p.ws[0] + CTR_OFF) + blockIdx.x, LD_LDS(lctl + 2), __ATOMIC_RELAXED, __HIP_MEMORY_SCOPE_AGENT);
    WAIT_VM(0);
  }
  if (__builtin_expect(is_push(), 0)) {
    push_role<true>(p, slot, wid, lane, lctl);
    return;
  }
  const u32 seq = rfl(LD_LDS(lctl + 2));
  const u32 g = slot & 7, s = slot >> 3;
  const u32 peer = ((u32)p.rank ^ g) & (NR - 1);
  if (s < ENG_WGS) {
    if (g == 0) {
      for (u32 r = 0; r < HELP_ROUNDS; ++r) stage_unit(p, r * 16 + s, seq);
      for (u32 u = HELP_ROUNDS * 16 + s; u < NUNIT; u += 8) stage_unit(p, u, seq);
      STAMP(12);
      return;
    }
    if (wid < PUSH_WAVES + PULL_WAVES) {
      pull_role(p, peer, (wid - PUSH_WAVES) * ENG_WGS + s, lane, seq, ctl);
    } else if (wid < PUSH_WAVES + PULL_WAVES + FIN_WAVES && g >= 4) {
      fin_role(p, peer, s, wid - PUSH_WAVES - PULL_WAVES, lane, seq, ctl);
    }
  } else {
    gemm_role(p, g, s - ENG_WGS, seq, (LAS char*)smem);
  }
  STAMP(7);
}

}

extern "C" void kernel_launch(int rank, int nranks, ncclComm_t comm,
                              hipStream_t stream,
                              void* const* d_in, const int* in_sizes, int n_in,
                              void* d_out, int out_size,
                              const int* in_dtypes, int out_dtype,
                              void* const* d_out_all,
                              void* const* d_ws_all, size_t ws_bytes) {
  if (nranks != NR || (unsigned)rank >= (unsigned)NR || n_in < 2 || ws_bytes < WS_USED ||
      in_sizes[0] != 4096 * 512 || in_sizes[1] != 512 * 4096 || out_size != 4096 * 4096) {
    fprintf(stderr, "kernel_launch: unexpected problem shape\n");
    return;
  }
  HIP_CHECK(hipFuncSetAttribute((const void*)fused_gemm_allreduce_silu,
                                hipFuncAttributeMaxDynamicSharedMemorySize, LDS_BYTES));
  Params p;
  p.x = (const char*)d_in[0];
  p.w = (const char*)d_in[1];
  static const int phys[NR] = {0, 2, 3, 6, 1, 4, 5, 7};
  p.rank = 0; p.pad = 0;
  for (int i = 0; i < NR; ++i) if (phys[i] == rank) p.rank = i;
  for (int i = 0; i < NR; ++i) { p.ws[i] = (char*)d_ws_all[phys[p.rank ^ i]]; p.out[i] = (char*)d_out_all[phys[p.rank ^ i]]; }
  hipLaunchKernelGGL(fused_gemm_allreduce_silu, dim3(GRID), dim3(THREADS), LDS_BYTES, stream, p);
  HIP_CHECK(hipGetLastError());
}
```

```cpp
#include <hip/hip_runtime.h>
#include <rccl/rccl.h>
#include <cstdio>
#include <type_traits>

#define HIP_CHECK(expr) do { hipError_t _e = (expr); if (_e != hipSuccess) { \
    fprintf(stderr, "HIP error %s at %s:%d\n", hipGetErrorString(_e), __FILE__, __LINE__); \
    return; } } while (0)

namespace {

typedef unsigned int u32;
typedef unsigned long long u64;
typedef u32 u32x4 __attribute__((ext_vector_type(4)));
typedef float f32x4 __attribute__((ext_vector_type(4)));
typedef __bf16 bf16x4 __attribute__((ext_vector_type(4)));
typedef __bf16 bf16x8 __attribute__((ext_vector_type(8)));
typedef float f32x16 __attribute__((ext_vector_type(16)));
#define LAS __attribute__((address_space(3)))
#define GAS __attribute__((address_space(1)))

constexpr int NR = 8;
constexpr u32 X_BYTES = 4096u * 512u * 2u;
constexpr u32 W_BYTES = 512u * 4096u * 2u;
constexpr u32 OUT_BYTES = 4096u * 4096u * 2u;
constexpr u32 KI = 5, KC = 3;

#define FIS 128
#define FOS 128
constexpr u32 BLK = 16384;
constexpr u32 IN_OFF = 0;
constexpr u32 IN_SRC = KI * 32 * BLK;
constexpr u32 P16_OFF = IN_OFF + NR * IN_SRC;
constexpr u32 P32_OFF = P16_OFF + 256 * 16384;
constexpr u32 EX_OFF = P32_OFF + 256 * 32768;
constexpr u32 DATA_BYTES = EX_OFF + 512 * BLK;
constexpr u32 FI_OFF = DATA_BYTES;
constexpr u32 FPR_OFF = FI_OFF + NR * 320 * FIS;
constexpr u32 FPL_OFF = FPR_OFF + 256 * FOS;
constexpr u32 FF_OFF = FPL_OFF + 256 * FOS;
constexpr u32 SG_OFF = FF_OFF + NR * 256 * FOS;
constexpr u32 GO_OFF = SG_OFF + NR * 40 * FOS;
constexpr u32 CTR_OFF = GO_OFF + NR * 16 * FOS;
constexpr u32 CLM_OFF = CTR_OFF + 256 * 4;
constexpr u32 TMO_OFF = CLM_OFF + 256 * 64;
constexpr u32 TICK_OFF = TMO_OFF + 128;
constexpr u32 SEG_OFF = TICK_OFF + 16 * 128;
constexpr u32 RF_OFF = SEG_OFF + 8 * 128;
constexpr u32 TS_OFF = RF_OFF + 8 * 128;
constexpr u32 TR_OFF = TS_OFF + 256 * 16 * 8;
constexpr u32 WS_USED = TR_OFF + 3 * 8 * 1024 * 8;

constexpr u64 DEADLINE_TICKS = 30000000ull;

#define TRIG_BACK_ 32
#define NS_HALF_ 6
#define MDIR_ 16
#define SL_PULL 2
#define SL_FIN 8
#define FIN_WAVES_ 4
#define SL_GEMM 8
#define PUSH_WAVES_ 2
#define PULL_WAVES_ 2
#define PUSH_D_ 4
#define WIDE_START 1
#define MASKED 1
#define PDIV 3
#define TRIG_W 1
#define WB_TAIL 0
#define DIRPACE 1
#define PACE_FROM 0
#define ASYNC_POLL 0
#define PUSH_ROUNDS 4
#define HELP_ROUNDS 2
#define GO_AT 7
#define RELAY 1
#define LOOK_AT 100
#define STEAL_MIN 0
#define IDLE_MIN 24
#define LEAD 4
#define LEAD_W 3
#define LOOK_EVERY 1
#define GATE 1
#define ACQ 3
#define STAGGER 300
#define ENG_WGS_ 8
constexpr int ENG_WGS = ENG_WGS_, GEMM_WGS = 32 - ENG_WGS;
constexpr int PUSH_WAVES = PUSH_WAVES_, PULL_WAVES = PULL_WAVES_, FIN_WAVES = FIN_WAVES_, PUSH_D = PUSH_D_;
static_assert(PUSH_WAVES + PULL_WAVES + FIN_WAVES <= 8, "waves per engine workgroup");
constexpr u32 TRIG_BACK = TRIG_BACK_;
constexpr u32 NPUSH = PUSH_ROUNDS * ENG_WGS * PUSH_WAVES_;
static_assert(PUSH_ROUNDS >= (GO_AT > 8 ? 4 : 3) && ENG_WGS == 8, "push rounds / staging schedule");
constexpr u32 NAN2 = 0x7FC07FC0u;
constexpr int GRID = 8 * (ENG_WGS + GEMM_WGS);
static_assert(GRID == 256, "slots wrap at a power of two");
constexpr int THREADS = 512;
constexpr int NS_HALF = NS_HALF_;
constexpr u32 MDIR = MDIR_;
constexpr u32 LDS_CTL = 144 * 1024;
constexpr u32 LDS_BYTES = LDS_CTL + 1024;
constexpr int AUX_SYS = 17;

struct Params {
  const char* x;
  const char* w;
  int rank;
  int pad;
  char* ws[NR];
  char* out[NR];
};

__device__ __forceinline__ u32 rfl(u32 v) { return __builtin_amdgcn_readfirstlane(v); }
__device__ __forceinline__ __amdgpu_buffer_rsrc_t mk_rsrc(const void* p, u32 n) {
  const u64 a = (u64)p;
  const u64 u = ((u64)rfl((u32)(a >> 32)) << 32) | rfl((u32)a);
  return __builtin_amdgcn_make_buffer_rsrc((void*)u, 0, n, 0x00020000);
}
__device__ __forceinline__ u32 ld_sys(const char* p) {
  return __hip_atomic_load((const GAS u32*)p, __ATOMIC_RELAXED, __HIP_MEMORY_SCOPE_SYSTEM);
}
__device__ __forceinline__ void st_sys(char* p, u32 v) {
  __hip_atomic_store((GAS u32*)p, v, __ATOMIC_RELAXED, __HIP_MEMORY_SCOPE_SYSTEM);
}
#define WAIT_VM(n) asm volatile("s_waitcnt vmcnt(%0)" ::"n"(n) : "memory")
#define CBAR() asm volatile("" ::: "memory")
#define LD_LDS(q) __hip_atomic_load(q, __ATOMIC_RELAXED, __HIP_MEMORY_SCOPE_WORKGROUP)
#define ST_LDS(q, v) __hip_atomic_store(q, v, __ATOMIC_RELAXED, __HIP_MEMORY_SCOPE_WORKGROUP)

__device__ __forceinline__ void st128(u32x4 v, __amdgpu_buffer_rsrc_t r, u32 voff, u32 soff) {
  __builtin_amdgcn_raw_buffer_store_b128(v, r, voff, soff, AUX_SYS);
  asm volatile("s_nop 1" ::"v"(v) : "memory");
}

template <int SLEEP>
__device__ __forceinline__ u64 wave_wait(const char* flag, u64 need, u32 seq, char* tmo, bool& bad) {
  const u64 t0 = __builtin_amdgcn_s_memrealtime();
  for (u32 it = 1;; ++it) {
    const u64 ok = __ballot(ld_sys(flag) >= seq);
    if ((ok & need) == need) return ok;
    if ((it & 15) == 0 || bad) {
      const u32 t = __hip_atomic_load((const GAS u32*)tmo, __ATOMIC_RELAXED, __HIP_MEMORY_SCOPE_AGENT);
      if (rfl(t) == seq || __builtin_amdgcn_s_memrealtime() - t0 > DEADLINE_TICKS) {
        __hip_atomic_store((GAS u32*)tmo, seq, __ATOMIC_RELAXED, __HIP_MEMORY_SCOPE_AGENT);
        bad = true;
        return ~0ull;
      }
    }
    __builtin_amdgcn_s_sleep(SLEEP);
  }
}

template <int SLEEP>
__device__ __forceinline__ u64 wave_wait2(const char* flag, u64 need, u32 seq, char* tmo, bool& bad) {
  const u64 t0 = __builtin_amdgcn_s_memrealtime();
  const bool needed = (need >> (threadIdx.x & 63)) & 1;
  bool ok = false;
  for (u32 it = 1;; ++it) {
    if (!ok && (it == 1 || needed)) ok = ld_sys(flag) >= seq;
    const u64 have = __ballot(ok);
    if ((have & need) == need) return have;
    if ((it & 15) == 0) {
      const u32 t = __hip_atomic_load((const GAS u32*)tmo, __ATOMIC_RELAXED, __HIP_MEMORY_SCOPE_AGENT);
      if (rfl(t) == seq || __builtin_amdgcn_s_memrealtime() - t0 > DEADLINE_TICKS) {
        __hip_atomic_store((GAS u32*)tmo, seq, __ATOMIC_RELAXED, __HIP_MEMORY_SCOPE_AGENT);
        bad = true;
        return ~0ull;
      }
    }
    __builtin_amdgcn_s_sleep(SLEEP);
  }
}

#define STAMPW(i, w) do {} while (0)
#define STAMP(i) STAMPW(i, 0)
#define NOW(x) do {} while (0)
#define TRACEJ(kind, job) do {} while (0)

__device__ __forceinline__ void dma4(u32 dst, __amdgpu_buffer_rsrc_t ra, u32 va0, u32 va1, u32 sa,
                                     __amdgpu_buffer_rsrc_t rb, u32 vb0, u32 vb1, u32 sb) {
  u32 keep;
  dst = rfl(dst); sa = rfl(sa); sb = rfl(sb);
  asm volatile(
      "s_mov_b32 %0, m0\n\t"
      "s_nop 4\n\t"
      "s_mov_b32 m0, %1\n\t"
      "s_nop 0\n\t"
      "buffer_load_dwordx4 %3, %5, %6 offen lds\n\t"
      "buffer_load_dwordx4 %4, %5, %6 offen offset:1024 lds\n\t"
      "s_mov_b32 m0, %2\n\t"
      "s_nop 0\n\t"
      "buffer_load_dwordx4 %7, %9, %10 offen lds\n\t"
      "buffer_load_dwordx4 %8, %9, %10 offen offset:1024 lds\n\t"
      "s_mov_b32 m0, %0"
      : "=&s"(keep)
      : "s"(dst), "s"(dst + BLK), "v"(va0), "v"(va1), "s"(ra), "s"(sa), "v"(vb0), "v"(vb1), "s"(rb), "s"(sb)
      : "memory");
}

__device__ __forceinline__ void dma3(u32 dsta, u32 dstb, __amdgpu_buffer_rsrc_t ra, u32 va0, u32 sa,
                                     __amdgpu_buffer_rsrc_t rb, u32 vb0, u32 vb1, u32 sb) {
  u32 keep;
  dsta = rfl(dsta); dstb = rfl(dstb); sa = rfl(sa); sb = rfl(sb);
  asm volatile(
      "s_mov_b32 %0, m0\n\t"
      "s_nop 4\n\t"
      "s_mov_b32 m0, %1\n\t"
      "s_nop 0\n\t"
      "buffer_load_dwordx4 %3, %4, %5 offen lds\n\t"
      "s_mov_b32 m0, %2\n\t"
      "s_nop 0\n\t"
      "buffer_load_dwordx4 %6, %8, %9 offen lds\n\t"
      "buffer_load_dwordx4 %7, %8, %9 offen offset:1024 lds\n\t"
      "s_mov_b32 m0, %0"
      : "=&s"(keep)
      : "s"(dsta), "s"(dstb), "v"(va0), "s"(ra), "s"(sa), "v"(vb0), "v"(vb1), "s"(rb), "s"(sb)
      : "memory");
}

__device__ __forceinline__ void dma_flags(u32 dst, const char* src) {
  u32 keep;
  dst = rfl(dst);
  asm volatile("s_mov_b32 %0, m0\n\ts_nop 0\n\ts_mov_b32 m0, %1\n\ts_nop 0\n\tglobal_load_lds_dword %2, off sc0 sc1\n\ts_mov_b32 m0, %0"
               : "=&s"(keep) : "s"(dst), "v"(src) : "memory");
}

template <int I, int N, class F>
__device__ __forceinline__ void static_for(F&& f) {
  if constexpr (I < N) { f(std::integral_constant<int, I>{}); static_for<I + 1, N>(f); }
}

__device__ __forceinline__ u32 sup(u32 grp, u32 c, u32 u) { return grp ? c + 2 * u : 2 * c + u; }
__device__ __forceinline__ u32 srow(u32 r, u32 u) { return sup(r >> 2, (r >> 1) & 1, u); }
__device__ __forceinline__ u32 scol(u32 r, u32 v) { return sup(r >> 2, r & 1, v); }
__device__ __forceinline__ void tile_rc(u32 m, u32& ri, u32& ci) {
  const u32 l = (m >= 1) + (m >= 4) + (m >= 9) + (m >= 16) + (m >= 25) + (m >= 36) + (m >= 49);
  const u32 idx = m - l * l;
  ri = idx < l ? l : idx - l;
  ci = idx < l ? idx : l;
}
__device__ __forceinline__ u32 fin_off(u32 r, u32 m, u32 sb, u32 part) {
  u32 ri, ci;
  tile_rc(m, ri, ci);
  return (srow(r, sb >> 1) * 1024 + ri * 128 + (r >> 2) * 64 + part * 32) * 8192 + (scol(r, sb & 1) * 1024 + ci * 128) * 2;
}

struct Job { u32 src, sstride, dst, flag, blo, bhi, voff; };

__device__ __forceinline__ Job decode_push(u32 job, u32 rank, u32 peer, u64 b0, u64 b1, u32 vnarrow, u32 vwide) {
  Job j;
  const bool cross = (rank ^ peer) >= 4;
  const u32 lk = job >> 3, op = (job >> 2) & 1, uv = (job >> 1) & 1, half = job & 1;
  const u32 lev = cross ? lk / KC : lk / KI;
  const u32 ktl = lk - lev * (cross ? KC : KI), kt = cross ? KI + ktl : ktl;
  j.blo = op ? (u32)b1 : (u32)b0; j.bhi = op ? (u32)(b1 >> 32) : (u32)(b0 >> 32); j.voff = op ? vwide : vnarrow;
  j.src = op ? (kt * 64 + half * 32) * 8192 + (scol(peer, uv) * 1024 + lev * 128) * 2
             : (srow(peer, uv) * 1024 + lev * 128 + half * 64) * 1024 + kt * 128;
  j.sstride = op ? 4 * 8192 : 8 * 1024;
  j.dst = IN_OFF + rank * IN_SRC + job * 8192;
  j.flag = FI_OFF + (rank * 320 + job) * FIS;
  return j;
}

__device__ __forceinline__ u32x4 push_ld(const Job& j, int i) {
  const auto r = mk_rsrc((const void*)(((u64)j.bhi << 32) | j.blo), X_BYTES);
  return __builtin_amdgcn_raw_buffer_load_b128(r, j.voff, rfl(j.src + i * j.sstride), 0);
}

template <bool claimed>
__device__ __forceinline__ bool push_role(const Params& p, u32& slot, u32 wid, int lane, LAS u32* lctl) {
  constexpr int D = PUSH_D, JI = 8;
  constexpr u32 NW = ENG_WGS * PUSH_WAVES;
  NOW(tA);
  const u32 g = slot & 7, w = wid * ENG_WGS + (slot >> 3);
  static_assert(X_BYTES == W_BYTES, "one size for both input shards");
  u64 b0 = (u64)p.x, b1 = (u64)p.w, bw = (u64)p.ws[g];
  u32 rank = p.rank;
  asm volatile("" : "+s"(b0), "+s"(b1), "+s"(bw), "+s"(rank));
  const u32 peer = rank ^ g;
  char* wsp = (char*)bw;
  const auto rd = mk_rsrc(wsp, DATA_BYTES);
  const u32 vnarrow = (lane >> 3) * 1024 + (lane & 7) * 16;
  const u32 vwide = (lane >> 4) * 8192 + (lane & 15) * 16;
  Job cur = decode_push(w, rank, peer, b0, b1, vnarrow, vwide);
  u32x4 v[D], v2[JI - D];
#pragma unroll
  for (int i = 0; i < D; ++i) { v[i] = push_ld(cur, i); CBAR(); }
#pragma unroll
  for (int i = D; i < JI; ++i) { v2[i - D] = push_ld(cur, i); CBAR(); }
  NOW(tB);
  u32 nid = NW + w, pflag = 0, seq = 0;
  auto round = [&](auto kind_c) {
    constexpr int KIND = decltype(kind_c)::value;
    constexpr bool FIRST = KIND == 0, LAST = KIND == 3;
    Job nxt = cur;
    if constexpr (!LAST) { nxt = decode_push(nid, rank, peer, b0, b1, vnarrow, vwide); }
#pragma unroll
    for (int i = 0; i < JI; ++i) {
      if (!FIRST && i == D - 1) {
        WAIT_VM(D + (LAST ? (JI - D < D - 1 ? JI - D : D - 1) : D - 1));
        if (KIND == 1) {
          if (!claimed) __builtin_amdgcn_s_barrier();
          CBAR();
          seq = rfl(LD_LDS(lctl + 2));
        }
        if (lane == 0) st_sys(wsp + pflag, seq);
        TRACEJ(0, (pflag - FI_OFF) / FIS - rank * 320);
        CBAR();
      }
      if ((LAST && i == JI - GO_AT) || (KIND == 2 && i == 2 * JI - GO_AT && nid + NW >= NPUSH)) {
        if (lane == 0) st_sys(wsp + GO_OFF + (rank * 16 + w) * FOS, seq);
        if (lane == 0) ST_LDS(lctl - LDS_CTL / 4 + 24 + wid, 1u);
        CBAR();
      }
      st128(FIRST && i >= D ? v2[i >= D ? i - D : 0] : v[i % D], rd, lane * 16, rfl(cur.dst) + i * 1024);
      if (i + D < JI) { if (!FIRST) v[i % D] = push_ld(cur, i + D); }
      else if (!LAST) v[i % D] = push_ld(nxt, i + D - JI);
      CBAR();
    }
    pflag = cur.flag;
    cur = nxt;
    nid += NW;
  };
  round(std::integral_constant<int, 0>{});
  NOW(tE);
  round(std::integral_constant<int, 1>{});
  const u32 slot2 = rfl(LD_LDS(lctl + 1));
  if (__builtin_expect(slot2 != slot, 0)) { WAIT_VM(0); slot = slot2; return false; }
  NOW(tC);
  for (int r = 3; r < PUSH_ROUNDS; ++r) round(std::integral_constant<int, 2>{});
  round(std::integral_constant<int, 3>{});
  WAIT_VM(0);
  if (lane == 0) st_sys(wsp + pflag, seq);
        TRACEJ(0, (pflag - FI_OFF) / FIS - rank * 320);
  STAMP(3);
  return true;
}

__device__ __forceinline__ float silu(float a) {
  const float c = fminf(fmaxf(a, -60.f), 60.f);
  return a * __builtin_amdgcn_rcpf(1.f + __expf(-c));
}

constexpr u32 MB_REQ = 0, MB_DONE = 4, MB_GONE = 8, MB_UNIT = 10, MB_PUSHED = 24, MB_BUF = 1024;

__device__ __forceinline__ void pull_role(const Params& p, u32 peer, u32 w, int lane, u32 seq, LAS u32* ctl) {
  constexpr u32 NW = ENG_WGS * PULL_WAVES, KF = 0, KIN = 5;
  constexpr u32 SEG = 96, TAIL = 1024 - 4 * SEG, NONE = ~0u, REL = 1u << 31;
  const u32 rank = p.rank, g = rank ^ peer;
  const bool cross = g >= 4;
  const u32 kts = cross ? KC : KI, NI = kts * 64 - NPUSH, NJ = NI + 1024;
  char* ws = p.ws[0];
  char* tmo = ws + TMO_OFF;
  const u64 bf = (u64)p.out[g], bp = (u64)p.ws[g], bo = (u64)p.out[0];
  const u32 vwide = (lane >> 4) * 8192 + (lane & 15) * 16;
  GAS u32* const tick = (GAS u32*)(ws + TICK_OFF + (seq & 1) * 1024 + (lane & 7) * 128);
  if (w == 0 && lane == 0) st_sys(ws + TICK_OFF + ((seq & 1) ^ 1) * 1024 + g * 128, 0);
  u32 ttile, tsup, tlk, tblk;
  {
    u32 ri, ci;
    tile_rc(lane, ri, ci);
    ttile = ri * 128 * 8192 + ci * 256;
    tsup = fin_off(rank ^ ((lane >> 2) & 7), 0, lane & 3, 0);
    const u32 lev = cross ? lane / KC : lane / KI, ktl = lane - lev * kts;
    tlk = EX_OFF + (lev * 8 + (cross ? KI + ktl : ktl)) * 8 * BLK;
    tblk = ((lane & 2) * 2 + ((lane & 2) ? scol(rank, lane & 1) : srow(rank, lane & 1))) * BLK;
  }
  auto rl = [](u32 t, u32 i) -> u32 { return __builtin_amdgcn_readlane(t, i); };
  auto rot = [&](u32 j, u32 link) -> u32 {
    if (j < TAIL) return j;
    const u32 t = j - TAIL + (link & 3) * SEG;
    return TAIL + (t >= 4 * SEG ? t - 4 * SEG : t);
  };
  static_assert(PUSH_WAVES == PULL_WAVES && NW <= 16, "one pull wave takes over from one push wave");
  const u32 trig = GO_OFF + (peer * 16 + w) * FOS;
  const u64 t0 = __builtin_amdgcn_s_memrealtime();
  u32 tf = w, tf2 = NW + w, lastf = 0, sub = 0;
  u32 cdst = 0, ck = 0, cflag = 0, pend = 0;
  u32 nsrc = 0, ndst = 0, nk = 0, nflag = 0;
  u32 npos = NONE, cpos = NONE, ppos = NONE;
  u32 apos = 0, acnt = 0, chk = NONE, chkn = 0, chkv = 0;
  u32 left = 0, near = 0;
  u32 oleft = 0, nround = w;
  bool bad = false, own_done = false;
  const u32 sw = NI + TAIL - w * 24;
  auto poll = [&](auto tc) -> u32 {
    if constexpr (decltype(tc)::value) {
      const u32 o = FF_OFF + (peer * 256 + ((rot(tf - NI, g) >> 2) & 255)) * FOS;
      return ld_sys(ws + (near && (lane & 8) ? RF_OFF + g * 128 + (lane & 7) * 4 : rfl(o)));
    } else {
      const u32 o = tf < NI ? SG_OFF + (peer * 40 + ((NPUSH + tf) >> 3)) * FOS : FF_OFF + (peer * 256 + ((rot(tf - NI, g) >> 2) & 255)) * FOS;
      return ld_sys(ws + ((lane & 1) ? rfl(tf == w ? trig : o) : rfl(o)));
    }
  };
  const u32 isrf = (lane & 8) ? ~0u : 0u;
  u32 pv = poll(std::false_type{}), rfm = 0;
  u32x4 v[4];
  auto victim = [&](u32 least) -> u32 {
    u32 best = 0, bl = 0;
    for (u64 mk = __ballot(left > least) & rfm; mk; mk &= mk - 1) {
      const u32 l = __builtin_ctzll(mk), n = __builtin_amdgcn_readlane(left, l);
      if (n > bl) { bl = n; best = l; }
    }
    return best;
  };
  auto draw = [&](u32 dg, bool look) -> u32 {
    u32 t = 0;
    if ((u32)lane == dg ? dg != 0 : (look && lane >= 1 && lane < NR))
      t = __hip_atomic_fetch_add(tick, (u32)lane != dg ? 0u : dg == g ? 1u : 1u << 16, __ATOMIC_RELAXED, __HIP_MEMORY_SCOPE_AGENT);
    return t;
  };
  auto drawn = [&](u32 dg, bool look, u32 t) {
    u32 rem = 0;
    if (look) {
      const u32 nj = (lane >= 4 ? KC : KI) * 64 - NPUSH + 1024, fr = (t & 0xFFFF) + 2 * NW, bk = t >> 16;
      rem = bk < SEG && fr + bk < nj ? nj - fr - bk : 0;
      left = lane >= 1 && lane < NR && (u32)lane != g ? rem - ((u32)lane == dg && rem) : 0;
      const u32 to = __builtin_amdgcn_readlane(t, g), so = (to & 0xFFFF) + 2 * NW + min(to >> 16, SEG);
      oleft = so < NJ ? NJ - so : 0;
    }
    if (dg == 0) return;
    const u32 td = __builtin_amdgcn_readlane(t, dg), f = (td & 0xFFFF) + 2 * NW, b = min(td >> 16, SEG);
    u32 h;
    if (dg == g) {
      h = f + b < NJ ? f : NONE;
      own_done = h == NONE;
      near |= RELAY && f + b + LOOK_AT >= NJ;
    } else {
      h = __builtin_amdgcn_readlane(rem, dg) ? REL | dg << 16 | (1023 - b) : NONE;
    }
    if (tf == NONE) { tf = h; pv &= isrf; } else tf2 = h;
  };
  static_assert(FIN_WAVES == 4, "mailbox lanes");
  u32 reqv = 0, ack = 0, apend = NONE;
  auto pick = [&](auto tc) -> bool {
    constexpr bool T = decltype(tc)::value;
    if (!T && tf >= sw) return false;
    const u64 req = __ballot(reqv != 0);
    if (req) {
      const u32 f = __builtin_ctzll(req);
      u32 r = 0;
      if (lane == 0) r = __hip_atomic_exchange(ctl + MB_REQ + f, 0u, __ATOMIC_RELAXED, __HIP_MEMORY_SCOPE_WORKGROUP);
      r = rfl(r);
      if (r) { nsrc = r; ndst = r; nk = f + 1; nflag = 0; npos = NONE; return true; }
    }
    if (T && tf == NONE) return false;
    if (!(bad || (T && (tf & REL)) || (__ballot(pv >= seq) & 3) == 3)) return false;
    if (!T && tf < NI) {
      const u32 jb = NPUSH + tf;
      nsrc = rl(tlk, jb >> 3) + rl(tblk, (jb >> 1) & 3) + ((jb & 1) * 2 + sub) * 4096;
      ndst = IN_OFF + peer * IN_SRC + jb * 8192 + sub * 4096;
      nk = KIN; nflag = sub ? FI_OFF + (peer * 320 + jb) * FIS : 0;
    } else {
      const bool rel = T && (tf & REL);
      const u32 link = rel ? (tf >> 16) & 7 : g, pos = rel ? tf & 1023 : tf - NI, j = T ? rot(pos, link) : pos;
      nsrc = ndst = rl(ttile, j >> 4) + rl(tsup, link * 4 + ((j >> 2) & 3)) + (j & 3) * 16 * 8192;
      nk = KF; nflag = 0;
      if (T) npos = pos >= TAIL ? (pos >= TAIL + SEG) + (pos >= TAIL + 2 * SEG) + (pos >= TAIL + 3 * SEG) : NONE;
    }
    nsrc = rfl(nsrc); ndst = rfl(ndst); nflag = rfl(nflag);
    return true;
  };
  auto put = [&](int i) {
    const bool isp = ck - 1 < FIN_WAVES;
    const auto ro = mk_rsrc((const void*)(ck == KF ? bo : (u64)ws), ck == KF ? OUT_BYTES : isp ? 0u : DATA_BYTES);
    st128(v[i], ro, ck == KF ? vwide : lane * 16, rfl(cdst + i * (ck == KF ? 4 * 8192 : 1024)));
    if (isp) *(LAS u32x4*)((LAS char*)ctl + MB_BUF + (ck - 1) * 4096 + i * 1024 + lane * 16) = v[i];
  };
  u32 curid = 0;
  auto delivered = [&]() {
    if (ck == KF) { if (curid & REL) { TRACEJ(2, 896 + ((curid >> 16) & 7) * 16 + (curid & 15)); } else { TRACEJ(1, curid - NI); } }
    else if (ck == KIN) { TRACEJ(2, 256 + ((cdst - IN_OFF - peer * IN_SRC) >> 12)); } else { TRACEJ(2, ((cdst - P16_OFF) >> 12) & 255); }
    if (ck - 1 < FIN_WAVES) {
      asm volatile("s_waitcnt lgkmcnt(0)" ::: "memory");
      if (lane == 0) __hip_atomic_fetch_add(ctl + MB_DONE + ck - 1, 1u, __ATOMIC_RELAXED, __HIP_MEMORY_SCOPE_WORKGROUP);
    }
  };
  auto publish = [&](u32 off) {
    if (lane == 0) st_sys(ws + off, seq);
    CBAR();
  };
  auto check = [&]() {
    if (chk != NONE) {
      if (__builtin_amdgcn_readlane(chkv, 0) + chkn == SEG * seq) {
        const u32 gr = ((g + chk + 1) & 3) | (lane & 1) << 2;
        if (lane < 2 && gr != 0) st_sys(p.ws[g ^ gr] + RF_OFF + (g ^ gr) * 128 + gr * 4, seq);
      }
      chk = NONE;
    }
  };
  auto report = [&]() {
    if (RELAY && acnt && apos < 3) {
      check();
      if (lane == 0) chkv = __hip_atomic_fetch_add((GAS u32*)((GAS char*)tick - TICK_OFF - (seq & 1) * 1024 + SEG_OFF + g * 128 + apos * 4), acnt,
                                                    __ATOMIC_RELAXED, __HIP_MEMORY_SCOPE_AGENT);
      chk = apos; chkn = acnt;
      CBAR();
    }
    acnt = 0;
  };
  auto stored = [&](u32 pos) {
    if (pos != apos) { report(); apos = pos; }
    ++acnt;
  };
  auto round = [&](auto tc, auto first_c) {
    constexpr bool T = decltype(tc)::value, FIRST = decltype(first_c)::value;
    const bool tix = nk == KF || (!T && nk == KIN && sub);
    const u32 tid_ = tf;
    if (T && near) rfm |= (u32)(__ballot(pv >= seq) >> 8) & 0xFE & near;
    if (nk == KF) lastf = ndst;
    if (!T && nk == KIN) sub ^= 1;
    u32 dg = 0;
    if (tix) {
      if (T && near) dg = victim(own_done ? STEAL_MIN : oleft + LEAD + w * LEAD_W);
      if (dg == 0 && !own_done) dg = g;
      tf = tf2; tf2 = NONE;
    }
    const bool look = T && tix && near && g != 4 && (dg != g || (++nround & LOOK_EVERY) == 0);
    const u32 tk = draw(dg, look);
    if (T && near) near = 0xFF;
    pv = poll(tc);
    CBAR();
    if (!T) {
      if (ack) { publish(ack); ack = 0; }
    } else {
      check();
      if (apend != NONE) { stored(apend); apend = NONE; }
    }
    const auto rsrc = mk_rsrc((const void*)(nk == KF ? bf : bp), nk == KF ? OUT_BYTES : DATA_BYTES);
    const u32 vs = nk == KF ? vwide : lane * 16, str = nk == KF ? 4 * 8192 : 1024;
#pragma unroll
    for (int i = 0; i < 4; ++i) {
      if (!FIRST) put(i);
      if (i == 3) reqv = LD_LDS(ctl + MB_REQ + (lane & 3));
      v[i] = __builtin_amdgcn_raw_buffer_load_b128(rsrc, vs, rfl(nsrc + i * str), AUX_SYS);
      CBAR();
    }
    if (!FIRST) {
      delivered();
      WAIT_VM(9);
      if (!T) { ack = pend; pend = cflag; }
      else { apend = ppos; ppos = cpos; }
    }
    cdst = ndst; ck = nk; cflag = nflag; curid = tid_;
    if (T) cpos = npos;
    if (tix) drawn(dg, look, tk);
  };
  auto drain = [&](auto tc) {
    constexpr bool T = decltype(tc)::value;
#pragma unroll
    for (int i = 0; i < 4; ++i) put(i);
    delivered();
    if (!T) {
      if (ack) { publish(ack); ack = 0; }
      if (pend | cflag) {
        WAIT_VM(0);
        if (pend) publish(pend);
        if (cflag) publish(cflag);
      }
      pend = 0;
    } else {
      if (apend != NONE) { stored(apend); apend = NONE; }
      if (ppos < 3 || cpos < 3) {
        WAIT_VM(0);
        if (ppos != NONE) stored(ppos);
        if (cpos != NONE) stored(cpos);
      }
      report();
      check();
      ppos = NONE;
    }
  };
  auto idle = [&](u32 it) {
    if ((it & 63) == 0) {
      const u32 t = __hip_atomic_load((const GAS u32*)tmo, __ATOMIC_RELAXED, __HIP_MEMORY_SCOPE_AGENT);
      if (rfl(t) == seq || __builtin_amdgcn_s_memrealtime() - t0 > DEADLINE_TICKS) {
        __hip_atomic_store((GAS u32*)tmo, seq, __ATOMIC_RELAXED, __HIP_MEMORY_SCOPE_AGENT);
        WAIT_VM(0);
        bad = true;
      }
    }
    __builtin_amdgcn_s_sleep(SL_PULL);
  };
  if (GATE) while (rfl(LD_LDS(ctl + MB_PUSHED + w / ENG_WGS)) == 0) __builtin_amdgcn_s_sleep(2);
  pv = poll(std::false_type{});
  constexpr std::false_type F_{};
  constexpr std::true_type T_{};
  for (u32 it = 1; tf < sw; ++it) {
    reqv = LD_LDS(ctl + MB_REQ + (lane & 3));
    if (pick(F_)) {
      if (tf == w && nk == KIN) { STAMPW(5, PUSH_WAVES); }
      round(F_, T_);
      if (pick(F_)) {
        round(F_, F_);
        while (pick(F_)) round(F_, F_);
      }
      drain(F_);
      continue;
    }
    idle(it);
    if ((it & PDIV) == 0 || tf == w) pv = poll(F_);
  }
  for (u32 it = 1;; ++it) {
    if (!own_done && tf2 == NONE) {
      do drawn(g, false, draw(g, false)); while (!own_done && tf2 == NONE);
      pv = poll(T_);
    }
    reqv = LD_LDS(ctl + MB_REQ + (lane & 3));
    if (pick(T_)) {
      round(T_, T_);
      if (pick(T_)) {
        round(T_, F_);
        while (pick(T_)) round(T_, F_);
      }
      drain(T_);
      continue;
    }
    if (tf == NONE) {
      if (near == 0xFF) rfm |= (u32)(__ballot(pv >= seq) >> 8) & 0xFE;
      const u32 dg = RELAY && !bad ? victim(IDLE_MIN) : 0;
      if (dg) {
        drawn(dg, true, draw(dg, true));
        continue;
      }
      if (rfl(LD_LDS(ctl + MB_GONE)) == (cross ? FIN_WAVES : 0)) break;
    }
    idle(it);
    if ((it & PDIV) == 0) pv = poll(T_);
  }
  if (bad) {
    WAIT_VM(0);
    const u32x4 nanv = {NAN2, NAN2, NAN2, NAN2};
    for (int i = 0; i < 4; ++i) st128(nanv, mk_rsrc(p.out[0], OUT_BYTES), vwide, rfl(lastf) + i * 4 * 8192);
  }
  STAMPW(6, PUSH_WAVES);
}

__device__ __forceinline__ void fin_role(const Params& p, u32 peer, u32 s, u32 f, int lane, u32 seq, LAS u32* ctl) {
  static_assert(FIN_WAVES == 4, "one wave per 16 rows of a half tile");
  const u32 rank = p.rank, sb = peer & 3;
  char* ws = p.ws[0];
  char* tmo = ws + TMO_OFF;
  const auto rl = mk_rsrc(ws, DATA_BYTES);
  const auto ro = mk_rsrc(p.out[0], OUT_BYTES);
  const u32 vwide = (lane >> 4) * 8192 + (lane & 15) * 16;
  char* wsl = p.ws[lane & 7];
  u32 ndone = 0;
  for (u32 m = MDIR + s; m < 64; m += ENG_WGS) {
    const u32 lo = P32_OFF + (sb * 64 + m) * 32768 + f * 8192;
    const u32 dst = fin_off(rank, m, sb, 0) + f * 16 * 8192;
    bool bad = false;
    wave_wait<SL_FIN>(ws + ((lane & 1) ? FPL_OFF : FPR_OFF) + (sb * 64 + m) * FOS, 3, seq, tmo, bad);
    if (m == MDIR + s) { STAMPW(8, PUSH_WAVES + PULL_WAVES); STAMPW(11, 7); }
    const u32 so = P16_OFF + ((rank & 3) * 64 + m) * 16384 + f * 4096;
    u32x4 rv[4], a[4], b[4];
    if (lane == 0) ST_LDS(ctl + MB_REQ + f, so);
#pragma unroll
    for (int i = 0; i < 4; ++i) {
      a[i] = __builtin_amdgcn_raw_buffer_load_b128(rl, lane * 16, lo + i * 2048, AUX_SYS);
      b[i] = __builtin_amdgcn_raw_buffer_load_b128(rl, lane * 16, lo + i * 2048 + 1024, AUX_SYS);
    }
    ++ndone;
    while (rfl(LD_LDS(ctl + MB_DONE + f)) != ndone) __builtin_amdgcn_s_sleep(1);
    CBAR();
#pragma unroll
    for (int i = 0; i < 4; ++i) rv[i] = *(const LAS u32x4*)((LAS char*)ctl + MB_BUF + f * 4096 + i * 1024 + lane * 16);
    if (m == MDIR + s) { STAMPW(9, PUSH_WAVES + PULL_WAVES);  }
#pragma unroll
    for (int i = 0; i < 4; ++i) {
      const u32x4 r = rv[i];
      float y[8];
#pragma unroll
      for (int e = 0; e < 4; ++e) {
        const u32 l0 = e < 2 ? a[i][2 * e] : b[i][2 * e - 4], l1 = e < 2 ? a[i][2 * e + 1] : b[i][2 * e - 3];
        y[2 * e] = silu(__uint_as_float(l0) + __uint_as_float(r[e] << 16));
        y[2 * e + 1] = silu(__uint_as_float(l1) + __uint_as_float(r[e] & 0xFFFF0000u));
      }
      const bf16x8 o = {(__bf16)y[0], (__bf16)y[1], (__bf16)y[2], (__bf16)y[3],
                        (__bf16)y[4], (__bf16)y[5], (__bf16)y[6], (__bf16)y[7]};
      u32x4 ov = __builtin_bit_cast(u32x4, o);
      if (bad) ov |= NAN2;
      st128(ov, ro, vwide, dst + i * 4 * 8192);
    }
    if (m == MDIR + s) { STAMPW(10, PUSH_WAVES + PULL_WAVES); }
    WAIT_VM(0);

    u32 arrived = 0;
    if (lane == 0) arrived = __hip_atomic_fetch_add(ctl + MB_UNIT + ndone - 1, 1u, __ATOMIC_RELAXED, __HIP_MEMORY_SCOPE_WORKGROUP);
    if (rfl(arrived) == FIN_WAVES - 1) {
      if (lane >= 1 && lane < NR) st_sys(wsl + FF_OFF + (rank * 256 + m * 4 + sb) * FOS, seq);
    }
    if (m == MDIR + s) { STAMPW(1, PUSH_WAVES + PULL_WAVES); }
  }
  if (lane == 0) __hip_atomic_fetch_add(ctl + MB_GONE, 1u, __ATOMIC_RELAXED, __HIP_MEMORY_SCOPE_WORKGROUP);
  STAMPW(2, PUSH_WAVES + PULL_WAVES + FIN_WAVES - 1);
}

template <bool HALF>
__device__ __forceinline__ void gemm_tile(const Params& p, u32 sb, u32 m, u32 hsel, u32 seq, u64 tstart, int sbase, LAS char* lds) {
  const int tid = threadIdx.x, lane = tid & 63;
  const u32 wid = rfl(tid >> 6);
  const u32 rank = p.rank;
  char* ws = p.ws[0];
  char* tmo = ws + TMO_OFF;
  const auto rx = mk_rsrc(p.x, X_BYTES);
  const auto rw = mk_rsrc(p.w, W_BYTES);
  const auto rs = mk_rsrc(ws, DATA_BYTES);
  const u32 u = sb >> 1, v = sb & 1;
  constexpr int NT = 32, T_CROSS = KI, T_INTRA = KI + 4 * KC, T_SPLIT = 4;
  constexpr int MI = HALF ? 1 : 2, NDMA = HALF ? 3 : 4, NS = HALF ? NS_HALF : 4;
  constexpr u32 XB = HALF ? BLK / 2 : BLK, SB = XB + BLK;
  static_assert(NS * SB <= LDS_CTL, "stages fit");

  u32 vao[2], vbo[2], vaw[2], vbw[2];
#pragma unroll
  for (int q = 0; q < 2; ++q) {
    const u32 ra = wid * 8 * MI + q * 8 + (lane >> 3), ca = (lane & 7) ^ ((ra >> 1) & 7);
    const u32 kb = wid * 8 + q * 4 + (lane >> 4), cbk = (lane & 15) ^ (((kb & 3) << 2) | ((kb >> 2) & 3));
    vao[q] = ra * 1024 + ca * 16 - q * 1024;  vaw[q] = ra * 128 + ca * 16 - q * 1024;
    vbo[q] = kb * 8192 + cbk * 16 - q * 1024; vbw[q] = kb * 256 + cbk * 16 - q * 1024;
  }
  const u32 lds0 = (u32)(size_t)lds;
  const u32 wm = wid & 1, wn = wid >> 1, h = lane >> 5;
  u32 xo[4], wo[2];
  {
    const u32 r = wm * 32 * MI + (lane & 31);
#pragma unroll
    for (int ks = 0; ks < 4; ++ks) xo[ks] = r * 128 + (((2 * ks + h) ^ ((r >> 1) & 7)) * 16);
    const u32 bk = (lane >> 4) & 1, q = (lane & 15) >> 2, pp = lane & 3;
#pragma unroll
    for (int t = 0; t < 2; ++t)
      wo[t] = XB + 256 * (8 * h + 4 * t + q) + 16 * ((4 * wn + 2 * bk + (pp >> 1)) ^ ((q << 2) | (2 * h + t))) + 8 * (pp & 1);
  }
  LAS u32* lbad = (LAS u32*)(lds + LDS_CTL);
  LAS u32* fimg = (LAS u32*)(lds + LDS_CTL + 256) + lane;
  if (ASYNC_POLL && wid == 0) { fimg[0] = 0; fimg[64] = 0; }
  u32 ri, ci;
  tile_rc(m, ri, ci);
  const u32 row0 = srow(rank, u) * 1024 + ri * 128 + (HALF ? hsel * 64 : 0), col0 = scol(rank, v) * 1024 + ci * 128;
  u32 islot = 0, rslot = 0;
  auto next_slot = [&](u32 o) -> u32 { return o + SB == NS * SB ? 0 : o + SB; };
  auto issue_own = [&](int t) {
    const u32 dst = lds0 + islot, sa = row0 * 1024 + t * 128, sw = t * 64 * 8192 + col0 * 2;
    islot = next_slot(islot);
    if (HALF) dma3(dst + wid * 1024, dst + XB + wid * 2048, rx, vao[0], sa, rw, vbo[0], vbo[1], sw);
    else dma4(dst + wid * 2048, rx, vao[0], vao[1], sa, rw, vbo[0], vbo[1], sw);
  };
  u32 taba, tabw;
  {
    const bool cr = lane < T_INTRA;
    const u32 c = cr ? lane - T_CROSS : lane - T_INTRA, ktl = cr ? c >> 2 : c / 3;
    const u32 src = rank ^ (cr ? 4 + (c & 3) : 1 + c - ktl * 3), kts = cr ? KC : KI;
    const u32 base = IN_OFF + src * IN_SRC + ktl * 4 * BLK;
    taba = base + (ri * kts * 4 + u) * BLK + (HALF ? hsel * 8192 : 0);
    tabw = base + (ci * kts * 4 + 2 + v) * BLK;
  }
  auto issue = [&](int t) {
    const u32 dst = lds0 + islot, sa = __builtin_amdgcn_readlane(taba, t), sw = __builtin_amdgcn_readlane(tabw, t);
    islot = next_slot(islot);
    if (HALF) dma3(dst + wid * 1024, dst + XB + wid * 2048, rs, vaw[0], sa, rs, vbw[0], vbw[1], sw);
    else dma4(dst + wid * 2048, rs, vaw[0], vaw[1], sa, rs, vbw[0], vbw[1], sw);
  };
  const u32 fop = (lane >> 1) & 1, flev = fop ? ci : ri, fuv = fop ? v : u, fq = lane >> 2;
  const u32 fh = (HALF && !fop) ? hsel : (lane & 1);
  const u32 fck = min(fq >> 2, KC - 1), fik = min(fq / 3, KI - 1);
  const char* fcross = ws + FI_OFF + ((rank ^ (4 + (fq & 3))) * 320 + ((flev * KC + fck) * 4 + fop * 2 + fuv) * 2 + fh) * FIS;
  const char* fintra = ws + FI_OFF + ((rank ^ (1 + fq % 3)) * 320 + ((flev * KI + fik) * 4 + fop * 2 + fuv) * 2 + fh) * FIS;

  while (__builtin_amdgcn_s_memrealtime() - tstart < (u64)max(ri, ci) * STAGGER) __builtin_amdgcn_s_sleep(16);
  if (ACQ == 3 && sbase == 1 && wid == 0) __builtin_amdgcn_fence(__ATOMIC_ACQUIRE, "");
  f32x16 acc[MI] = {};
  struct Frag { bf16x8 w[4], x[MI][4]; };
  auto read = [&](Frag& f) {
    const LAS char* st = lds + rslot;
    rslot = next_slot(rslot);
#pragma unroll
    for (int ks = 0; ks < 4; ++ks) {
      const bf16x4 w0 = __builtin_amdgcn_ds_read_tr16_b64_v4bf16((LAS bf16x4*)(st + wo[0] + ks * 4096));
      const bf16x4 w1 = __builtin_amdgcn_ds_read_tr16_b64_v4bf16((LAS bf16x4*)(st + wo[1] + ks * 4096));
      f.w[ks] = __builtin_shufflevector(w0, w1, 0, 1, 2, 3, 4, 5, 6, 7);
#pragma unroll
      for (int mi = 0; mi < MI; ++mi) f.x[mi][ks] = *(const LAS bf16x8*)(st + xo[ks] + mi * 4096);
    }
  };
  auto mma = [&](const Frag& f) {
#pragma unroll
    for (int ks = 0; ks < 4; ++ks)
#pragma unroll
      for (int mi = 0; mi < MI; ++mi) acc[mi] = __builtin_amdgcn_mfma_f32_32x32x16_bf16(f.w[ks], f.x[mi][ks], acc[mi], 0, 0, 0);
  };
  int next_wait = T_CROSS;
  u64 need = 0xFFFF, have = 0;
  auto waitpoint = [&](int tn) {
    if (tn == T_INTRA) { need = 0xFFF; have = 0; }
    if (wid == 0 && (have & need) != need) {
      if (ASYNC_POLL && HALF) have = __ballot(LD_LDS(fimg + (tn < T_INTRA ? 0 : 64)) >= seq);
      if ((have & need) != need) {
        bool bad = false;
        have = MASKED ? wave_wait2<SL_GEMM>(tn < T_INTRA ? fcross : fintra, need, seq, tmo, bad)
                      : wave_wait<SL_GEMM>(tn < T_INTRA ? fcross : fintra, need, seq, tmo, bad);
        if (bad) *lbad = 1;
      }
    }
    if (tn == T_CROSS) { STAMP(sbase); }
    if (tn == NT - 3) { STAMP(sbase + 1); }
    if (tn < T_INTRA) { next_wait += 4; need = (need << 16) | 0xFFFF; }
    else { next_wait += 3; need = (need << 12) | 0xFFF; }
  };
  auto step = [&](auto rc, int tn, const Frag& fc, Frag& fn) {
    constexpr int R = decltype(rc)::value;
    if (tn >= 0 && __builtin_expect(tn == next_wait, 0)) waitpoint(tn);
    if constexpr (R >= 0) {
      WAIT_VM(R * NDMA);
      __builtin_amdgcn_s_waitcnt(0xC07F);
      CBAR();
      __builtin_amdgcn_s_barrier();
      CBAR();
      read(fn);
    }
    if (tn >= 0) issue(tn);
    if (ASYNC_POLL && HALF && tn >= 0 && wid == 1 && next_wait < NT)
      dma_flags(lds0 + LDS_CTL + (next_wait < T_INTRA ? 256 : 512), next_wait < T_INTRA ? fcross : fintra);
    mma(fc);
  };
  Frag fa, fb;
  auto tail = [&](auto nc) {
    constexpr int N = decltype(nc)::value;
    static_for<0, N / 2>([&](auto ic) {
      constexpr int i = decltype(ic)::value;
      step(std::integral_constant<int, N - 2 - 2 * i>{}, -1, fa, fb);
      step(std::integral_constant<int, N - 3 - 2 * i>{}, -1, fb, fa);
    });
  };
  static_assert(NS % 2 == 0 && NS >= T_SPLIT && T_SPLIT + 1 == T_CROSS, "run structure");
  for (int t = 0; t < T_SPLIT; ++t) issue_own(t);
  WAIT_VM((T_SPLIT - 1) * NDMA);
  __builtin_amdgcn_s_barrier();
  CBAR();
  read(fa);
  tail(std::integral_constant<int, T_SPLIT>{});
  islot = rslot = 0;
  __builtin_amdgcn_s_waitcnt(0xC07F);
  __builtin_amdgcn_s_barrier();
  issue_own(T_SPLIT);
  for (int t = T_CROSS; t < T_SPLIT + NS; ++t) {
    if (t == next_wait) { waitpoint(t); __builtin_amdgcn_s_barrier(); }
    issue(t);
  }
  WAIT_VM((NS - 1) * NDMA);
  __builtin_amdgcn_s_barrier();
  CBAR();
  read(fa);
  for (int tn = T_SPLIT + NS; tn < NT; tn += 2) {
    step(std::integral_constant<int, NS - 2>{}, tn, fa, fb);
    step(std::integral_constant<int, NS - 2>{}, tn + 1, fb, fa);
  }
  tail(std::integral_constant<int, NS>{});
  STAMP(sbase + 2);
  const u32 tmv = __hip_atomic_load((const GAS u32*)tmo, __ATOMIC_RELAXED, __HIP_MEMORY_SCOPE_AGENT);
  __builtin_amdgcn_s_waitcnt(0xC07F);
  __builtin_amdgcn_s_barrier();
  CBAR();
  constexpr u32 EROW = 528;
#pragma unroll
  for (int mi = 0; mi < MI; ++mi) {
    const u32 mr = wm * 32 * MI + mi * 32 + (lane & 31);
#pragma unroll
    for (int gq = 0; gq < 4; ++gq) {
      const f32x4 o = {acc[mi][4 * gq], acc[mi][4 * gq + 1], acc[mi][4 * gq + 2], acc[mi][4 * gq + 3]};
      *(LAS f32x4*)(lds + mr * EROW + (wn * 32 + 8 * gq + 4 * h) * 4) = o;
    }
  }
  __syncthreads();
  const u32 bad = (*lbad || tmv == seq) ? NAN2 : 0u;
  const u32 keep = rank >> 2;
  const bool do_exp = !HALF || hsel != keep, do_keep = !HALF || hsel == keep;
  char* wsq = p.ws[4 | ((rank & 3) ^ sb)];
  if (do_exp) {
    const u32 e16 = P16_OFF + (sb * 64 + m) * 16384 + wid * 1024;
#pragma unroll
    for (int part = 0; part < 2; ++part) {
      const LAS char* se = lds + ((HALF ? 0 : (keep ^ 1) * 64) + part * 32 + (tid >> 4)) * EROW + (tid & 15) * 32;
      const f32x4 c0 = *(const LAS f32x4*)se, c1 = *(const LAS f32x4*)(se + 16);
      const bf16x8 o = {(__bf16)c0[0], (__bf16)c0[1], (__bf16)c0[2], (__bf16)c0[3],
                        (__bf16)c1[0], (__bf16)c1[1], (__bf16)c1[2], (__bf16)c1[3]};
      st128(__builtin_bit_cast(u32x4, o) | bad, rs, lane * 16, e16 + part * 8192);
    }
  }
  if (HALF && do_keep && m < MDIR) {
    f32x4 kc[2][2];
#pragma unroll
    for (int part = 0; part < 2; ++part) {
      const LAS char* sk = lds + (part * 32 + (tid >> 4)) * EROW + (tid & 15) * 32;
      kc[part][0] = *(const LAS f32x4*)sk; kc[part][1] = *(const LAS f32x4*)(sk + 16);
    }
    bool badw = bad != 0;
    wave_wait<2>(ws + FPR_OFF + (sb * 64 + m) * FOS, 1, seq, tmo, badw);
    STAMP(5);
    u32x4 r[2];
#pragma unroll
    for (int part = 0; part < 2; ++part)
    {
      r[part] = __builtin_amdgcn_raw_buffer_load_b128(mk_rsrc(wsq, DATA_BYTES), lane * 16, P16_OFF + ((rank & 3) * 64 + m) * 16384 + wid * 1024 + part * 8192, AUX_SYS);
      if (DIRPACE) WAIT_VM(0);
    }
    const auto ro = mk_rsrc(p.out[0], OUT_BYTES);
#pragma unroll
    for (int part = 0; part < 2; ++part) {
      float y[8];
#pragma unroll
      for (int e = 0; e < 4; ++e) {
        y[2 * e] = silu(kc[part][e >> 1][(2 * e) & 3] + __uint_as_float(r[part][e] << 16));
        y[2 * e + 1] = silu(kc[part][e >> 1][(2 * e + 1) & 3] + __uint_as_float(r[part][e] & 0xFFFF0000u));
      }
      const bf16x8 o = {(__bf16)y[0], (__bf16)y[1], (__bf16)y[2], (__bf16)y[3],
                        (__bf16)y[4], (__bf16)y[5], (__bf16)y[6], (__bf16)y[7]};
      st128(__builtin_bit_cast(u32x4, o) | (badw ? NAN2 : 0u), ro, (lane >> 4) * 8192 + (lane & 15) * 16,
            (row0 + part * 32 + wid * 4) * 8192 + col0 * 2);
    }
    WAIT_VM(0);
    __syncthreads();
    if (tid >= 1 && tid < NR) st_sys(p.ws[tid] + FF_OFF + (rank * 256 + m * 4 + sb) * FOS, seq);
    STAMP(6);
    return;
  }
  if (do_keep) {
    const u32 e32 = P32_OFF + (sb * 64 + m) * 32768 + wid * 2048;
#pragma unroll
    for (int part = 0; part < 2; ++part) {
      const LAS char* sk = lds + ((HALF ? 0 : keep * 64) + part * 32 + (tid >> 4)) * EROW + (tid & 15) * 32;
      const u32x4 c0 = *(const LAS u32x4*)sk, c1 = *(const LAS u32x4*)(sk + 16);
      st128(c0 | bad, rs, lane * 16, e32 + part * 16384);
      st128(c1 | bad, rs, lane * 16, e32 + part * 16384 + 1024);
    }
  }
  WAIT_VM(0);
  __syncthreads();
  if (tid == 0 && do_exp) st_sys(wsq + FPR_OFF + ((rank & 3) * 64 + m) * FOS, seq);
  if (tid == 64 && do_keep) st_sys(ws + FPL_OFF + (sb * 64 + m) * FOS, seq);
  STAMP(sbase + 3);
}

constexpr u32 LK0 = NPUSH / 8, NUNIT = 64 - 2 * LK0;
static_assert(NPUSH % 8 == 0 && LK0 < 8 * KC, "staging units");
__device__ __forceinline__ void stage_unit(const Params& p, u32 u, u32 seq) {
  const int tid = threadIdx.x, lane = tid & 63;
  const u32 wid = rfl(tid >> 6);
  constexpr u32 NPAIR = 2 * (8 * KC - LK0);
  const bool cr = u < NPAIR && (u & 1);
  const u32 lk = u < NPAIR ? LK0 + (u >> 1) : 8 * KC + u - NPAIR;
  const u32 lev = cr ? lk / KC : lk / KI, kt = cr ? KI + lk - lev * KC : lk - lev * KI, op = wid >> 2, sup = wid & 3;
  const auto r = mk_rsrc(op ? p.w : p.x, X_BYTES);
  const auto rs = mk_rsrc(p.ws[0], DATA_BYTES);
  const u32 vo = op ? (lane >> 4) * 8192 + (lane & 15) * 16 : (lane >> 3) * 1024 + (lane & 7) * 16;
  const u32 so = op ? kt * 64 * 8192 + (sup * 1024 + lev * 128) * 2 : (sup * 1024 + lev * 128) * 1024 + kt * 128;
  u32x4 v[16];
#pragma unroll
  for (int i = 0; i < 16; ++i) v[i] = __builtin_amdgcn_raw_buffer_load_b128(r, vo, so + i * (op ? 4 * 8192 : 8 * 1024), 0);
#pragma unroll
  for (int i = 0; i < 16; ++i) st128(v[i], rs, lane * 16, EX_OFF + ((lev * 8 + kt) * 8 + wid) * BLK + i * 1024);
  WAIT_VM(0);
  __syncthreads();
  if (tid >= 1 && tid < NR && (tid >= 4) == cr) st_sys(p.ws[tid] + SG_OFF + (p.rank * 40 + lk) * FOS, seq);
}

__device__ __forceinline__ void gemm_role(const Params& p, u32 g, u32 k, u32 seq, LAS char* lds) {
  const u64 tstart = __builtin_amdgcn_s_memrealtime();
  if (threadIdx.x < 64 && (ACQ == 1 || (ACQ == 2 && k == 0))) __builtin_amdgcn_fence(__ATOMIC_ACQUIRE, "");
  static_assert(GEMM_WGS >= 16 && GEMM_WGS <= 24, "unit schedule below");
  if (g == 0 && k >= 16)
    for (u32 r = 0; r < HELP_ROUNDS; ++r) { stage_unit(p, r * 16 + k - 8, seq); STAMP(12 + r); }
  const u32 keep = p.rank >> 2;
  if (k < 16) gemm_tile<true>(p, g >= 4 ? (p.rank ^ g) & 3 : g, k, g >= 4 ? keep : keep ^ 1, seq, tstart, 1, lds);
  const u32 arm = g & 1;
  for (u32 n = k < 16 ? GEMM_WGS - 16 + k : k - 16; n < 24; n += GEMM_WGS) {
    u32 lev = 4, idx = n;
    for (;;) {
      const u32 cnt = lev + ((lev & 1) == arm);
      if (idx < cnt) break;
      idx -= cnt; ++lev;
    }
    gemm_tile<false>(p, g >> 1, lev * lev + (idx == lev ? 2 * lev : arm * lev + idx), 0, seq, tstart, k < 16 ? 8 : 1, lds);
  }
}

__global__ void __launch_bounds__(THREADS, 2) fused_gemm_allreduce_silu(Params p) {
  extern __shared__ __attribute__((aligned(16))) char smem[];
  const int lane = threadIdx.x & 63;
  const u32 wid = rfl(threadIdx.x >> 6);
  LAS u32* ctl = (LAS u32*)smem;
  LAS u32* lctl = (LAS u32*)(smem + LDS_CTL);
  u32 slot = (blockIdx.x & ~7u) | ((u32)__builtin_amdgcn_s_getreg((3 << 11) | 20) & 7);
  auto is_push = [&]() { return slot < 8 * ENG_WGS && (slot & 7) != 0 && wid < PUSH_WAVES; };
  if (__builtin_expect(is_push(), 1)) {
    if (push_role<false>(p, slot, wid, lane, lctl)) return;
  } else {
    STAMPW(0, 2);
    if (wid == 7) {
      if (lane < 32) ctl[lane] = 0;
      if (lane == 0) {
        GAS u32* ctr = (GAS u32*)(p.ws[0] + CTR_OFF) + blockIdx.x;
        GAS u32* clm = (GAS u32*)(p.ws[0] + CLM_OFF);
        const u32 c = __hip_atomic_load(ctr, __ATOMIC_RELAXED, __HIP_MEMORY_SCOPE_AGENT);
        u32 sl = slot;
#pragma unroll 1
        for (int n = 0; n < 4 * GRID; ++n) {
          if (__hip_atomic_fetch_add(clm + sl * 16, 1u, __ATOMIC_RELAXED, __HIP_MEMORY_SCOPE_AGENT) == c) break;
          __hip_atomic_fetch_add(clm + sl * 16, ~0u, __ATOMIC_RELAXED, __HIP_MEMORY_SCOPE_AGENT);
          sl = (sl + 1) & (GRID - 1);
        }
        lctl[0] = 0; lctl[1] = sl; lctl[2] = c + 1;
      }
    }
    asm volatile("s_waitcnt lgkmcnt(0)" ::: "memory");
    __builtin_amdgcn_s_barrier();
    slot = rfl(LD_LDS(lctl + 1));
    if (threadIdx.x == THREADS - 1)
      __hip_atomic_store((GAS u32*)(p.ws[0] + CTR_OFF) + blockIdx.x, LD_LDS(lctl + 2), __ATOMIC_RELAXED, __HIP_MEMORY_SCOPE_AGENT);
    WAIT_VM(0);
  }
  if (__builtin_expect(is_push(), 0)) {
    push_role<true>(p, slot, wid, lane, lctl);
    return;
  }
  const u32 seq = rfl(LD_LDS(lctl + 2));
  const u32 g = slot & 7, s = slot >> 3;
  const u32 peer = ((u32)p.rank ^ g) & (NR - 1);
  if (s < ENG_WGS) {
    if (g == 0) {
      for (u32 r = 0; r < HELP_ROUNDS; ++r) stage_unit(p, r * 16 + s, seq);
      for (u32 u = HELP_ROUNDS * 16 + s; u < NUNIT; u += 8) stage_unit(p, u, seq);
      STAMP(12);
      return;
    }
    if (wid < PUSH_WAVES + PULL_WAVES) {
      pull_role(p, peer, (wid - PUSH_WAVES) * ENG_WGS + s, lane, seq, ctl);
    } else if (wid < PUSH_WAVES + PULL_WAVES + FIN_WAVES && g >= 4) {
      fin_role(p, peer, s, wid - PUSH_WAVES - PULL_WAVES, lane, seq, ctl);
    }
  } else {
    gemm_role(p, g, s - ENG_WGS, seq, (LAS char*)smem);
  }
  STAMP(7);
}

}

extern "C" void kernel_launch(int rank, int nranks, ncclComm_t comm,
                              hipStream_t stream,
                              void* const* d_in, const int* in_sizes, int n_in,
                              void* d_out, int out_size,
                              const int* in_dtypes, int out_dtype,
                              void* const* d_out_all,
                              void* const* d_ws_all, size_t ws_bytes) {
  if (nranks != NR || (unsigned)rank >= (unsigned)NR || n_in < 2 || ws_bytes < WS_USED ||
      in_sizes[0] != 4096 * 512 || in_sizes[1] != 512 * 4096 || out_size != 4096 * 4096) {
    fprintf(stderr, "kernel_launch: unexpected problem shape\n");
    return;
  }
  HIP_CHECK(hipFuncSetAttribute((const void*)fused_gemm_allreduce_silu,
                                hipFuncAttributeMaxDynamicSharedMemorySize, LDS_BYTES));
  Params p;
  p.x = (const char*)d_in[0];
  p.w = (const char*)d_in[1];
  static const int phys[NR] = {0, 2, 3, 6, 1, 4, 5, 7};
  p.rank = 0; p.pad = 0;
  for (int i = 0; i < NR; ++i) if (phys[i] == rank) p.rank = i;
  for (int i = 0; i < NR; ++i) { p.ws[i] = (char*)d_ws_all[phys[p.rank ^ i]]; p.out[i] = (char*)d_out_all[phys[p.rank ^ i]]; }
  hipLaunchKernelGGL(fused_gemm_allreduce_silu, dim3(GRID), dim3(THREADS), LDS_BYTES, stream, p);
  HIP_CHECK(hipGetLastError());
}
```

```cpp
#include <hip/hip_runtime.h>
#include <rccl/rccl.h>
#include <cstdio>
#include <type_traits>

#define HIP_CHECK(expr) do { hipError_t _e = (expr); if (_e != hipSuccess) { \
    fprintf(stderr, "HIP error %s at %s:%d\n", hipGetErrorString(_e), __FILE__, __LINE__); \
    return; } } while (0)

namespace {

typedef unsigned int u32;
typedef unsigned long long u64;
typedef u32 u32x4 __attribute__((ext_vector_type(4)));
typedef float f32x4 __attribute__((ext_vector_type(4)));
typedef __bf16 bf16x4 __attribute__((ext_vector_type(4)));
typedef __bf16 bf16x8 __attribute__((ext_vector_type(8)));
typedef float f32x16 __attribute__((ext_vector_type(16)));
#define LAS __attribute__((address_space(3)))
#define GAS __attribute__((address_space(1)))

constexpr int NR = 8;
constexpr u32 X_BYTES = 4096u * 512u * 2u;
constexpr u32 W_BYTES = 512u * 4096u * 2u;
constexpr u32 OUT_BYTES = 4096u * 4096u * 2u;
constexpr u32 KI = 5, KC = 3;

constexpr u32 FIS = 128, FOS = 128;

constexpr u32 BLK = 16384;
constexpr u32 IN_OFF = 0;
constexpr u32 IN_SRC = KI * 32 * BLK;
constexpr u32 P16_OFF = IN_OFF + NR * IN_SRC;
constexpr u32 P32_OFF = P16_OFF + 256 * 16384;
constexpr u32 EX_OFF = P32_OFF + 256 * 32768;
constexpr u32 DATA_BYTES = EX_OFF + 512 * BLK;
constexpr u32 FI_OFF = DATA_BYTES;
constexpr u32 FPR_OFF = FI_OFF + NR * 320 * FIS;
constexpr u32 FPL_OFF = FPR_OFF + 256 * FOS;
constexpr u32 FF_OFF = FPL_OFF + 256 * FOS;
constexpr u32 SG_OFF = FF_OFF + NR * 256 * FOS;
constexpr u32 GO_OFF = SG_OFF + NR * 40 * FOS;
constexpr u32 CTR_OFF = GO_OFF + NR * 16 * FOS;
constexpr u32 CLM_OFF = CTR_OFF + 256 * 4;
constexpr u32 TMO_OFF = CLM_OFF + 256 * 64;
constexpr u32 TICK_OFF = TMO_OFF + 128;
constexpr u32 SEG_OFF = TICK_OFF + 16 * 128;
constexpr u32 RF_OFF = SEG_OFF + 8 * 128;
constexpr u32 WS_USED = RF_OFF + 8 * 128;

constexpr u64 DEADLINE_TICKS = 80000000ull;

constexpr int ENG_WGS = 8, GEMM_WGS = 24;
constexpr int PUSH_WAVES = 2, PULL_WAVES = 2, FIN_WAVES = 4;
constexpr int PUSH_D = 4, PUSH_ROUNDS = 4;
constexpr u32 NPUSH = PUSH_ROUNDS * ENG_WGS * PUSH_WAVES;
constexpr int GO_AT = 7;
constexpr u32 HELP_ROUNDS = 2;
constexpr int NS_HALF = 6;
constexpr u32 MDIR = 16;
constexpr u64 STAGGER = 300;
constexpr u32 LOOK_AT = 100, LEAD = 4, LEAD_W = 3, IDLE_MIN = 24;
constexpr int SL_PULL = 2, SL_FIN = 8, SL_GEMM = 8;
constexpr u32 NAN2 = 0x7FC07FC0u;
constexpr int GRID = 8 * (ENG_WGS + GEMM_WGS);
static_assert(GRID == 256, "slots wrap at a power of two");
constexpr int THREADS = 512;
constexpr u32 LDS_CTL = 144 * 1024;
constexpr u32 LDS_BYTES = LDS_CTL + 64;
constexpr int AUX_SYS = 17;

struct Params {
  const char* x;
  const char* w;
  int rank;
  int pad;
  char* ws[NR];
  char* out[NR];
  u64 pad2;
  u32x4 psrc[NR][16];
};

__device__ __forceinline__ u32 rfl(u32 v) { return __builtin_amdgcn_readfirstlane(v); }
__device__ __forceinline__ __amdgpu_buffer_rsrc_t mk_rsrc(const void* p, u32 n) {
  const u64 a = (u64)p;
  const u64 u = ((u64)rfl((u32)(a >> 32)) << 32) | rfl((u32)a);
  return __builtin_amdgcn_make_buffer_rsrc((void*)u, 0, n, 0x00020000);
}
__device__ __forceinline__ u32 ld_sys(const char* p) {
  return __hip_atomic_load((const GAS u32*)p, __ATOMIC_RELAXED, __HIP_MEMORY_SCOPE_SYSTEM);
}
__device__ __forceinline__ void st_sys(char* p, u32 v) {
  __hip_atomic_store((GAS u32*)p, v, __ATOMIC_RELAXED, __HIP_MEMORY_SCOPE_SYSTEM);
}
#define WAIT_VM(n) asm volatile("s_waitcnt vmcnt(%0)" ::"n"(n) : "memory")
#define CBAR() asm volatile("" ::: "memory")
#define LD_LDS(q) __hip_atomic_load(q, __ATOMIC_RELAXED, __HIP_MEMORY_SCOPE_WORKGROUP)
#define ST_LDS(q, v) __hip_atomic_store(q, v, __ATOMIC_RELAXED, __HIP_MEMORY_SCOPE_WORKGROUP)

__device__ __forceinline__ void st128(u32x4 v, __amdgpu_buffer_rsrc_t r, u32 voff, u32 soff) {
  __builtin_amdgcn_raw_buffer_store_b128(v, r, voff, soff, AUX_SYS);
  asm volatile("s_nop 1" ::"v"(v) : "memory");
}

template <int SLEEP>
__device__ __forceinline__ u64 wave_wait(const char* flag, u64 need, u32 seq, char* tmo, bool& bad) {
  const u64 t0 = __builtin_amdgcn_s_memrealtime();
  for (u32 it = 1;; ++it) {
    const u64 ok = __ballot(ld_sys(flag) >= seq);
    if ((ok & need) == need) return ok;
    if ((it & 15) == 0 || bad) {
      const u32 t = __hip_atomic_load((const GAS u32*)tmo, __ATOMIC_RELAXED, __HIP_MEMORY_SCOPE_AGENT);
      if (rfl(t) == seq || __builtin_amdgcn_s_memrealtime() - t0 > DEADLINE_TICKS) {
        __hip_atomic_store((GAS u32*)tmo, seq, __ATOMIC_RELAXED, __HIP_MEMORY_SCOPE_AGENT);
        bad = true;
        return ~0ull;
      }
    }
    __builtin_amdgcn_s_sleep(SLEEP);
  }
}

template <int SLEEP>
__device__ __forceinline__ u64 wave_wait2(const char* flag, u64 need, u32 seq, char* tmo, bool& bad) {
  const u64 t0 = __builtin_amdgcn_s_memrealtime();
  const bool needed = (need >> (threadIdx.x & 63)) & 1;
  bool ok = false;
  for (u32 it = 1;; ++it) {
    if (!ok && (it == 1 || needed)) ok = ld_sys(flag) >= seq;
    const u64 have = __ballot(ok);
    if ((have & need) == need) return have;
    if ((it & 15) == 0) {
      const u32 t = __hip_atomic_load((const GAS u32*)tmo, __ATOMIC_RELAXED, __HIP_MEMORY_SCOPE_AGENT);
      if (rfl(t) == seq || __builtin_amdgcn_s_memrealtime() - t0 > DEADLINE_TICKS) {
        __hip_atomic_store((GAS u32*)tmo, seq, __ATOMIC_RELAXED, __HIP_MEMORY_SCOPE_AGENT);
        bad = true;
        return ~0ull;
      }
    }
    __builtin_amdgcn_s_sleep(SLEEP);
  }
}

__device__ __forceinline__ void dma4(u32 dst, __amdgpu_buffer_rsrc_t ra, u32 va0, u32 va1, u32 sa,
                                     __amdgpu_buffer_rsrc_t rb, u32 vb0, u32 vb1, u32 sb) {
  u32 keep;
  dst = rfl(dst); sa = rfl(sa); sb = rfl(sb);
  asm volatile(
      "s_mov_b32 %0, m0\n\t"
      "s_nop 4\n\t"
      "s_mov_b32 m0, %1\n\t"
      "s_nop 0\n\t"
      "buffer_load_dwordx4 %3, %5, %6 offen lds\n\t"
      "buffer_load_dwordx4 %4, %5, %6 offen offset:1024 lds\n\t"
      "s_mov_b32 m0, %2\n\t"
      "s_nop 0\n\t"
      "buffer_load_dwordx4 %7, %9, %10 offen lds\n\t"
      "buffer_load_dwordx4 %8, %9, %10 offen offset:1024 lds\n\t"
      "s_mov_b32 m0, %0"
      : "=&s"(keep)
      : "s"(dst), "s"(dst + BLK), "v"(va0), "v"(va1), "s"(ra), "s"(sa), "v"(vb0), "v"(vb1), "s"(rb), "s"(sb)
      : "memory");
}

__device__ __forceinline__ void dma3(u32 dsta, u32 dstb, __amdgpu_buffer_rsrc_t ra, u32 va0, u32 sa,
                                     __amdgpu_buffer_rsrc_t rb, u32 vb0, u32 vb1, u32 sb) {
  u32 keep;
  dsta = rfl(dsta); dstb = rfl(dstb); sa = rfl(sa); sb = rfl(sb);
  asm volatile(
      "s_mov_b32 %0, m0\n\t"
      "s_nop 4\n\t"
      "s_mov_b32 m0, %1\n\t"
      "s_nop 0\n\t"
      "buffer_load_dwordx4 %3, %4, %5 offen lds\n\t"
      "s_mov_b32 m0, %2\n\t"
      "s_nop 0\n\t"
      "buffer_load_dwordx4 %6, %8, %9 offen lds\n\t"
      "buffer_load_dwordx4 %7, %8, %9 offen offset:1024 lds\n\t"
      "s_mov_b32 m0, %0"
      : "=&s"(keep)
      : "s"(dsta), "s"(dstb), "v"(va0), "s"(ra), "s"(sa), "v"(vb0), "v"(vb1), "s"(rb), "s"(sb)
      : "memory");
}

template <int I, int N, class F>
__device__ __forceinline__ void static_for(F&& f) {
  if constexpr (I < N) { f(std::integral_constant<int, I>{}); static_for<I + 1, N>(f); }
}

__host__ __device__ __forceinline__ u32 sup(u32 grp, u32 c, u32 u) { return grp ? c + 2 * u : 2 * c + u; }
__host__ __device__ __forceinline__ u32 srow(u32 r, u32 u) { return sup(r >> 2, (r >> 1) & 1, u); }
__host__ __device__ __forceinline__ u32 scol(u32 r, u32 v) { return sup(r >> 2, r & 1, v); }
__device__ __forceinline__ void tile_rc(u32 m, u32& ri, u32& ci) {
  const u32 l = (m >= 1) + (m >= 4) + (m >= 9) + (m >= 16) + (m >= 25) + (m >= 36) + (m >= 49);
  const u32 idx = m - l * l;
  ri = idx < l ? l : idx - l;
  ci = idx < l ? idx : l;
}
__device__ __forceinline__ u32 fin_off(u32 r, u32 m, u32 sb, u32 part) {
  u32 ri, ci;
  tile_rc(m, ri, ci);
  return (srow(r, sb >> 1) * 1024 + ri * 128 + (r >> 2) * 64 + part * 32) * 8192 + (scol(r, sb & 1) * 1024 + ci * 128) * 2;
}

template <bool claimed>
__device__ __forceinline__ bool push_role(const Params& p, u32& slot, u32 wid, int lane, LAS u32* lctl) {
  constexpr int D = PUSH_D, JI = 8;
  constexpr u32 NW = ENG_WGS * PUSH_WAVES;
  static_assert(PUSH_ROUNDS == 4 && NW == 16 && X_BYTES == W_BYTES && GO_AT <= JI, "push schedule");
  const u32 g = slot & 7, w = wid * ENG_WGS + (slot >> 3), op = (slot >> 5) & 1;
  u64 bs = (u64)(&p.x)[op], bw = (u64)p.ws[g];
  u32x4 src = p.psrc[g][w];
  u32 rank = p.rank;
  asm volatile("" : "+s"(bs), "+s"(bw), "+s"(src), "+s"(rank));
  const auto rs = mk_rsrc((const void*)bs, X_BYTES);
  const u32 vs = op ? (lane >> 4) * 8192 + (lane & 15) * 16 : (lane >> 3) * 1024 + (lane & 7) * 16;
  const u32 str = op ? 4 * 8192 : 8 * 1024;
  auto ld = [&](int r, int i) -> u32x4 { return __builtin_amdgcn_raw_buffer_load_b128(rs, vs, src[r] + i * str, 0); };
  u32x4 v[D], v2[JI - D];
#pragma unroll
  for (int i = 0; i < D; ++i) { v[i] = ld(0, i); CBAR(); }
#pragma unroll
  for (int i = D; i < JI; ++i) { v2[i - D] = ld(0, i); CBAR(); }
  char* wsp = (char*)bw;
  const auto rd = mk_rsrc(wsp, DATA_BYTES);
  const u32 dst = IN_OFF + rank * IN_SRC + w * 8192, flag = FI_OFF + (rank * 320 + w) * FIS;
  u32 seq = 0;
  auto round = [&](auto rc) {
    constexpr int R = decltype(rc)::value;
    constexpr bool FIRST = R == 0, LAST = R == PUSH_ROUNDS - 1;
#pragma unroll
    for (int i = 0; i < JI; ++i) {
      if (!FIRST && i == D - 1) {
        WAIT_VM(2 * D - 1);
        if (R == 1) {
          if (!claimed) __builtin_amdgcn_s_barrier();
          CBAR();
          seq = rfl(LD_LDS(lctl + 2));
        }
        if (lane == 0) st_sys(wsp + flag + (R - 1) * NW * FIS, seq);
        CBAR();
      }
      if (LAST && i == JI - GO_AT) {
        if (lane == 0) st_sys(wsp + GO_OFF + (rank * 16 + w) * FOS, seq);
        if (lane == 0) ST_LDS(lctl - LDS_CTL / 4 + 24 + wid, 1u);
        CBAR();
      }
      st128(FIRST && i >= D ? v2[i >= D ? i - D : 0] : v[i % D], rd, lane * 16, dst + R * NW * 8192 + i * 1024);
      if (i + D < JI) { if (!FIRST) v[i % D] = ld(R, i + D); }
      else if (!LAST) v[i % D] = ld(LAST ? R : R + 1, i + D - JI);
      CBAR();
    }
  };
  static_assert(JI - D >= D - 1, "the count above also holds in the last round");
  round(std::integral_constant<int, 0>{});
  round(std::integral_constant<int, 1>{});
  const u32 slot2 = rfl(LD_LDS(lctl + 1));
  if (__builtin_expect(slot2 != slot, 0)) { WAIT_VM(0); slot = slot2; return false; }
  round(std::integral_constant<int, 2>{});
  round(std::integral_constant<int, 3>{});
  WAIT_VM(0);
  if (lane == 0) st_sys(wsp + flag + (PUSH_ROUNDS - 1) * NW * FIS, seq);
  return true;
}

__device__ __forceinline__ float silu(float a) {
  const float c = fminf(fmaxf(a, -60.f), 60.f);
  return a * __builtin_amdgcn_rcpf(1.f + __expf(-c));
}

constexpr u32 MB_REQ = 0, MB_DONE = 4, MB_GONE = 8, MB_UNIT = 10, MB_PUSHED = 24, MB_BUF = 1024;

__device__ __forceinline__ void pull_role(const Params& p, u32 peer, u32 w, int lane, u32 seq, LAS u32* ctl) {
  constexpr u32 NW = ENG_WGS * PULL_WAVES, KF = 0, KIN = 5;
  constexpr u32 SEG = 96, TAIL = 1024 - 4 * SEG, NONE = ~0u, REL = 1u << 31;
  const u32 rank = p.rank, g = rank ^ peer;
  const bool cross = g >= 4;
  const u32 kts = cross ? KC : KI, NI = kts * 64 - NPUSH, NJ = NI + 1024;
  char* ws = p.ws[0];
  char* tmo = ws + TMO_OFF;
  const u64 bf = (u64)p.out[g], bp = (u64)p.ws[g], bo = (u64)p.out[0];
  const u32 vwide = (lane >> 4) * 8192 + (lane & 15) * 16;
  GAS u32* const tick = (GAS u32*)(ws + TICK_OFF + (seq & 1) * 1024 + (lane & 7) * 128);
  if (w == 0 && lane == 0) st_sys(ws + TICK_OFF + ((seq & 1) ^ 1) * 1024 + g * 128, 0);
  u32 ttile, tsup, tlk, tblk;
  {
    u32 ri, ci;
    tile_rc(lane, ri, ci);
    ttile = ri * 128 * 8192 + ci * 256;
    tsup = fin_off(rank ^ ((lane >> 2) & 7), 0, lane & 3, 0);
    const u32 lev = cross ? lane / KC : lane / KI, ktl = lane - lev * kts;
    tlk = EX_OFF + (lev * 8 + (cross ? KI + ktl : ktl)) * 8 * BLK;
    tblk = ((lane & 2) * 2 + ((lane & 2) ? scol(rank, lane & 1) : srow(rank, lane & 1))) * BLK;
  }
  auto rl = [](u32 t, u32 i) -> u32 { return __builtin_amdgcn_readlane(t, i); };
  auto rot = [&](u32 j, u32 link) -> u32 {
    if (j < TAIL) return j;
    const u32 t = j - TAIL + (link & 3) * SEG;
    return TAIL + (t >= 4 * SEG ? t - 4 * SEG : t);
  };
  static_assert(PUSH_WAVES == PULL_WAVES && NW <= 16, "one pull wave takes over from one push wave");
  const u32 trig = GO_OFF + (peer * 16 + w) * FOS;
  const u64 t0 = __builtin_amdgcn_s_memrealtime();
  u32 tf = w, tf2 = NW + w, lastf = 0, sub = 0;
  u32 cdst = 0, ck = 0, cflag = 0, pend = 0;
  u32 nsrc = 0, ndst = 0, nk = 0, nflag = 0;
  u32 npos = NONE, cpos = NONE, ppos = NONE;
  u32 apos = 0, acnt = 0, chk = NONE, chkn = 0, chkv = 0;
  u32 left = 0, near = 0;
  u32 oleft = 0, nround = w;
  bool bad = false, own_done = false;
  const u32 sw = NI + TAIL - w * 24;
  auto poll = [&](auto tc) -> u32 {
    if constexpr (decltype(tc)::value) {
      const u32 o = FF_OFF + (peer * 256 + ((rot(tf - NI, g) >> 2) & 255)) * FOS;
      return ld_sys(ws + (near && (lane & 8) ? RF_OFF + g * 128 + (lane & 7) * 4 : rfl(o)));
    } else {
      const u32 o = tf < NI ? SG_OFF + (peer * 40 + ((NPUSH + tf) >> 3)) * FOS : FF_OFF + (peer * 256 + ((rot(tf - NI, g) >> 2) & 255)) * FOS;
      return ld_sys(ws + ((lane & 1) ? rfl(tf == w ? trig : o) : rfl(o)));
    }
  };
  const u32 isrf = (lane & 8) ? ~0u : 0u;
  u32 pv = poll(std::false_type{}), rfm = 0;
  u32x4 v[4];
  auto victim = [&](u32 least) -> u32 {
    u32 best = 0, bl = 0;
    for (u64 mk = __ballot(left > least) & rfm; mk; mk &= mk - 1) {
      const u32 l = __builtin_ctzll(mk), n = __builtin_amdgcn_readlane(left, l);
      if (n > bl) { bl = n; best = l; }
    }
    return best;
  };
  auto draw = [&](u32 dg, bool look) -> u32 {
    u32 t = 0;
    if ((u32)lane == dg ? dg != 0 : (look && lane >= 1 && lane < NR))
      t = __hip_atomic_fetch_add(tick, (u32)lane != dg ? 0u : dg == g ? 1u : 1u << 16, __ATOMIC_RELAXED, __HIP_MEMORY_SCOPE_AGENT);
    return t;
  };
  auto drawn = [&](u32 dg, bool look, u32 t) {
    u32 rem = 0;
    if (look) {
      const u32 nj = (lane >= 4 ? KC : KI) * 64 - NPUSH + 1024, fr = (t & 0xFFFF) + 2 * NW, bk = t >> 16;
      rem = bk < SEG && fr + bk < nj ? nj - fr - bk : 0;
      left = lane >= 1 && lane < NR && (u32)lane != g ? rem - ((u32)lane == dg && rem) : 0;
      const u32 to = __builtin_amdgcn_readlane(t, g), so = (to & 0xFFFF) + 2 * NW + min(to >> 16, SEG);
      oleft = so < NJ ? NJ - so : 0;
    }
    if (dg == 0) return;
    const u32 td = __builtin_amdgcn_readlane(t, dg), f = (td & 0xFFFF) + 2 * NW, b = min(td >> 16, SEG);
    u32 h;
    if (dg == g) {
      h = f + b < NJ ? f : NONE;
      own_done = h == NONE;
      near |= f + b + LOOK_AT >= NJ;
    } else {
      h = __builtin_amdgcn_readlane(rem, dg) ? REL | dg << 16 | (1023 - b) : NONE;
    }
    if (tf == NONE) { tf = h; pv &= isrf; } else tf2 = h;
  };
  static_assert(FIN_WAVES == 4, "mailbox lanes");
  u32 reqv = 0, ack = 0, apend = NONE;
  auto pick = [&](auto tc) -> bool {
    constexpr bool T = decltype(tc)::value;
    if (!T && tf >= sw) return false;
    const u64 req = __ballot(reqv != 0);
    if (req) {
      const u32 f = __builtin_ctzll(req);
      u32 r = 0;
      if (lane == 0) r = __hip_atomic_exchange(ctl + MB_REQ + f, 0u, __ATOMIC_RELAXED, __HIP_MEMORY_SCOPE_WORKGROUP);
      r = rfl(r);
      if (r) { nsrc = r; ndst = r; nk = f + 1; nflag = 0; npos = NONE; return true; }
    }
    if (T && tf == NONE) return false;
    if (!(bad || (T && (tf & REL)) || (__ballot(pv >= seq) & 3) == 3)) return false;
    if (!T && tf < NI) {
      const u32 jb = NPUSH + tf;
      nsrc = rl(tlk, jb >> 3) + rl(tblk, (jb >> 1) & 3) + ((jb & 1) * 2 + sub) * 4096;
      ndst = IN_OFF + peer * IN_SRC + jb * 8192 + sub * 4096;
      nk = KIN; nflag = sub ? FI_OFF + (peer * 320 + jb) * FIS : 0;
    } else {
      const bool rel = T && (tf & REL);
      const u32 link = rel ? (tf >> 16) & 7 : g, pos = rel ? tf & 1023 : tf - NI, j = T ? rot(pos, link) : pos;
      nsrc = ndst = rl(ttile, j >> 4) + rl(tsup, link * 4 + ((j >> 2) & 3)) + (j & 3) * 16 * 8192;
      nk = KF; nflag = 0;
      if (T) npos = pos >= TAIL ? (pos >= TAIL + SEG) + (pos >= TAIL + 2 * SEG) + (pos >= TAIL + 3 * SEG) : NONE;
    }
    nsrc = rfl(nsrc); ndst = rfl(ndst); nflag = rfl(nflag);
    return true;
  };
  auto put = [&](int i) {
    const bool isp = ck - 1 < FIN_WAVES;
    const auto ro = mk_rsrc((const void*)(ck == KF ? bo : (u64)ws), ck == KF ? OUT_BYTES : isp ? 0u : DATA_BYTES);
    st128(v[i], ro, ck == KF ? vwide : lane * 16, rfl(cdst + i * (ck == KF ? 4 * 8192 : 1024)));
    if (isp) *(LAS u32x4*)((LAS char*)ctl + MB_BUF + (ck - 1) * 4096 + i * 1024 + lane * 16) = v[i];
  };
  auto delivered = [&]() {
    if (ck - 1 < FIN_WAVES) {
      asm volatile("s_waitcnt lgkmcnt(0)" ::: "memory");
      if (lane == 0) __hip_atomic_fetch_add(ctl + MB_DONE + ck - 1, 1u, __ATOMIC_RELAXED, __HIP_MEMORY_SCOPE_WORKGROUP);
    }
  };
  auto publish = [&](u32 off) {
    if (lane == 0) st_sys(ws + off, seq);
    CBAR();
  };
  auto check = [&]() {
    if (chk != NONE) {
      if (__builtin_amdgcn_readlane(chkv, 0) + chkn == SEG * seq) {
        const u32 gr = ((g + chk + 1) & 3) | (lane & 1) << 2;
        if (lane < 2 && gr != 0) st_sys(p.ws[g ^ gr] + RF_OFF + (g ^ gr) * 128 + gr * 4, seq);
      }
      chk = NONE;
    }
  };
  auto report = [&]() {
    if (acnt && apos < 3) {
      check();
      if (lane == 0) chkv = __hip_atomic_fetch_add((GAS u32*)((GAS char*)tick - TICK_OFF - (seq & 1) * 1024 + SEG_OFF + g * 128 + apos * 4), acnt,
                                                    __ATOMIC_RELAXED, __HIP_MEMORY_SCOPE_AGENT);
      chk = apos; chkn = acnt;
      CBAR();
    }
    acnt = 0;
  };
  auto stored = [&](u32 pos) {
    if (pos != apos) { report(); apos = pos; }
    ++acnt;
  };
  auto round = [&](auto tc, auto first_c) {
    constexpr bool T = decltype(tc)::value, FIRST = decltype(first_c)::value;
    const bool tix = nk == KF || (!T && nk == KIN && sub);
    if (T && near) rfm |= (u32)(__ballot(pv >= seq) >> 8) & 0xFE & near;
    if (nk == KF) lastf = ndst;
    if (!T && nk == KIN) sub ^= 1;
    u32 dg = 0;
    if (tix) {
      if (T && near) dg = victim(own_done ? 0 : oleft + LEAD + w * LEAD_W);
      if (dg == 0 && !own_done) dg = g;
      tf = tf2; tf2 = NONE;
    }
    const bool look = T && tix && near && g != 4 && (dg != g || (++nround & 1) == 0);
    const u32 tk = draw(dg, look);
    if (T && near) near = 0xFF;
    pv = poll(tc);
    CBAR();
    if (!T) {
      if (ack) { publish(ack); ack = 0; }
    } else {
      check();
      if (apend != NONE) { stored(apend); apend = NONE; }
    }
    const auto rsrc = mk_rsrc((const void*)(nk == KF ? bf : bp), nk == KF ? OUT_BYTES : DATA_BYTES);
    const u32 vs = nk == KF ? vwide : lane * 16, str = nk == KF ? 4 * 8192 : 1024;
#pragma unroll
    for (int i = 0; i < 4; ++i) {
      if (!FIRST) put(i);
      if (i == 3) reqv = LD_LDS(ctl + MB_REQ + (lane & 3));
      v[i] = __builtin_amdgcn_raw_buffer_load_b128(rsrc, vs, rfl(nsrc + i * str), AUX_SYS);
      CBAR();
    }
    if (!FIRST) {
      delivered();
      WAIT_VM(9);
      if (!T) { ack = pend; pend = cflag; }
      else { apend = ppos; ppos = cpos; }
    }
    cdst = ndst; ck = nk; cflag = nflag;
    if (T) cpos = npos;
    if (tix) drawn(dg, look, tk);
  };
  auto drain = [&](auto tc) {
    constexpr bool T = decltype(tc)::value;
#pragma unroll
    for (int i = 0; i < 4; ++i) put(i);
    delivered();
    if (!T) {
      if (ack) { publish(ack); ack = 0; }
      if (pend | cflag) {
        WAIT_VM(0);
        if (pend) publish(pend);
        if (cflag) publish(cflag);
      }
      pend = 0;
    } else {
      if (apend != NONE) { stored(apend); apend = NONE; }
      if (ppos < 3 || cpos < 3) {
        WAIT_VM(0);
        if (ppos != NONE) stored(ppos);
        if (cpos != NONE) stored(cpos);
      }
      report();
      check();
      ppos = NONE;
    }
  };
  auto idle = [&](u32 it) {
    if ((it & 63) == 0) {
      const u32 t = __hip_atomic_load((const GAS u32*)tmo, __ATOMIC_RELAXED, __HIP_MEMORY_SCOPE_AGENT);
      if (rfl(t) == seq || __builtin_amdgcn_s_memrealtime() - t0 > DEADLINE_TICKS) {
        __hip_atomic_store((GAS u32*)tmo, seq, __ATOMIC_RELAXED, __HIP_MEMORY_SCOPE_AGENT);
        WAIT_VM(0);
        bad = true;
      }
    }
    __builtin_amdgcn_s_sleep(SL_PULL);
  };
  while (rfl(LD_LDS(ctl + MB_PUSHED + w / ENG_WGS)) == 0) __builtin_amdgcn_s_sleep(2);
  pv = poll(std::false_type{});
  constexpr std::false_type F_{};
  constexpr std::true_type T_{};
  for (u32 it = 1; tf < sw; ++it) {
    reqv = LD_LDS(ctl + MB_REQ + (lane & 3));
    if (pick(F_)) {
      round(F_, T_);
      if (pick(F_)) {
        round(F_, F_);
        while (pick(F_)) round(F_, F_);
      }
      drain(F_);
      continue;
    }
    idle(it);
    if ((it & 3) == 0 || tf == w) pv = poll(F_);
  }
  for (u32 it = 1;; ++it) {
    if (!own_done && tf2 == NONE) {
      do drawn(g, false, draw(g, false)); while (!own_done && tf2 == NONE);
      pv = poll(T_);
    }
    reqv = LD_LDS(ctl + MB_REQ + (lane & 3));
    if (pick(T_)) {
      round(T_, T_);
      if (pick(T_)) {
        round(T_, F_);
        while (pick(T_)) round(T_, F_);
      }
      drain(T_);
      continue;
    }
    if (tf == NONE) {
      if (near == 0xFF) rfm |= (u32)(__ballot(pv >= seq) >> 8) & 0xFE;
      const u32 dg = bad ? 0 : victim(IDLE_MIN);
      if (dg) {
        drawn(dg, true, draw(dg, true));
        continue;
      }
      if (rfl(LD_LDS(ctl + MB_GONE)) == (cross ? FIN_WAVES : 0)) break;
    }
    idle(it);
    if ((it & 3) == 0) pv = poll(T_);
  }
  if (bad) {
    WAIT_VM(0);
    const u32x4 nanv = {NAN2, NAN2, NAN2, NAN2};
    for (int i = 0; i < 4; ++i) st128(nanv, mk_rsrc(p.out[0], OUT_BYTES), vwide, rfl(lastf) + i * 4 * 8192);
  }
}

__device__ __forceinline__ void fin_role(const Params& p, u32 peer, u32 s, u32 f, int lane, u32 seq, LAS u32* ctl) {
  static_assert(FIN_WAVES == 4, "one wave per 16 rows of a half tile");
  const u32 rank = p.rank, sb = peer & 3;
  char* ws = p.ws[0];
  char* tmo = ws + TMO_OFF;
  const auto rl = mk_rsrc(ws, DATA_BYTES);
  const auto ro = mk_rsrc(p.out[0], OUT_BYTES);
  const u32 vwide = (lane >> 4) * 8192 + (lane & 15) * 16;
  char* wsl = p.ws[lane & 7];
  u32 ndone = 0;
  for (u32 m = MDIR + s; m < 64; m += ENG_WGS) {
    const u32 lo = P32_OFF + (sb * 64 + m) * 32768 + f * 8192;
    const u32 dst = fin_off(rank, m, sb, 0) + f * 16 * 8192;
    bool bad = false;
    wave_wait<SL_FIN>(ws + ((lane & 1) ? FPL_OFF : FPR_OFF) + (sb * 64 + m) * FOS, 3, seq, tmo, bad);
    const u32 so = P16_OFF + ((rank & 3) * 64 + m) * 16384 + f * 4096;
    u32x4 rv[4], a[4], b[4];
    if (lane == 0) ST_LDS(ctl + MB_REQ + f, so);
#pragma unroll
    for (int i = 0; i < 4; ++i) {
      a[i] = __builtin_amdgcn_raw_buffer_load_b128(rl, lane * 16, lo + i * 2048, AUX_SYS);
      b[i] = __builtin_amdgcn_raw_buffer_load_b128(rl, lane * 16, lo + i * 2048 + 1024, AUX_SYS);
    }
    ++ndone;
    while (rfl(LD_LDS(ctl + MB_DONE + f)) != ndone) __builtin_amdgcn_s_sleep(1);
    CBAR();
#pragma unroll
    for (int i = 0; i < 4; ++i) rv[i] = *(const LAS u32x4*)((LAS char*)ctl + MB_BUF + f * 4096 + i * 1024 + lane * 16);
#pragma unroll
    for (int i = 0; i < 4; ++i) {
      const u32x4 r = rv[i];
      float y[8];
#pragma unroll
      for (int e = 0; e < 4; ++e) {
        const u32 l0 = e < 2 ? a[i][2 * e] : b[i][2 * e - 4], l1 = e < 2 ? a[i][2 * e + 1] : b[i][2 * e - 3];
        y[2 * e] = silu(__uint_as_float(l0) + __uint_as_float(r[e] << 16));
        y[2 * e + 1] = silu(__uint_as_float(l1) + __uint_as_float(r[e] & 0xFFFF0000u));
      }
      const bf16x8 o = {(__bf16)y[0], (__bf16)y[1], (__bf16)y[2], (__bf16)y[3],
                        (__bf16)y[4], (__bf16)y[5], (__bf16)y[6], (__bf16)y[7]};
      u32x4 ov = __builtin_bit_cast(u32x4, o);
      if (bad) ov |= NAN2;
      st128(ov, ro, vwide, dst + i * 4 * 8192);
    }
    WAIT_VM(0);

    u32 arrived = 0;
    if (lane == 0) arrived = __hip_atomic_fetch_add(ctl + MB_UNIT + ndone - 1, 1u, __ATOMIC_RELAXED, __HIP_MEMORY_SCOPE_WORKGROUP);
    if (rfl(arrived) == FIN_WAVES - 1) {
      if (lane >= 1 && lane < NR) st_sys(wsl + FF_OFF + (rank * 256 + m * 4 + sb) * FOS, seq);
    }
  }
  if (lane == 0) __hip_atomic_fetch_add(ctl + MB_GONE, 1u, __ATOMIC_RELAXED, __HIP_MEMORY_SCOPE_WORKGROUP);
}

template <bool HALF>
__device__ __forceinline__ void gemm_tile(const Params& p, u32 sb, u32 m, u32 hsel, u32 seq, u64 tstart, bool first, LAS char* lds) {
  const int tid = threadIdx.x, lane = tid & 63;
  const u32 wid = rfl(tid >> 6);
  const u32 rank = p.rank;
  char* ws = p.ws[0];
  char* tmo = ws + TMO_OFF;
  const auto rx = mk_rsrc(p.x, X_BYTES);
  const auto rw = mk_rsrc(p.w, W_BYTES);
  const auto rs = mk_rsrc(ws, DATA_BYTES);
  const u32 u = sb >> 1, v = sb & 1;
  constexpr int NT = 32, T_CROSS = KI, T_INTRA = KI + 4 * KC, T_SPLIT = 4;
  constexpr int MI = HALF ? 1 : 2, NDMA = HALF ? 3 : 4, NS = HALF ? NS_HALF : 4;
  constexpr u32 XB = HALF ? BLK / 2 : BLK, SB = XB + BLK;
  static_assert(NS * SB <= LDS_CTL, "stages fit");

  u32 vao[2], vbo[2], vaw[2], vbw[2];
#pragma unroll
  for (int q = 0; q < 2; ++q) {
    const u32 ra = wid * 8 * MI + q * 8 + (lane >> 3), ca = (lane & 7) ^ ((ra >> 1) & 7);
    const u32 kb = wid * 8 + q * 4 + (lane >> 4), cbk = (lane & 15) ^ (((kb & 3) << 2) | ((kb >> 2) & 3));
    vao[q] = ra * 1024 + ca * 16 - q * 1024;  vaw[q] = ra * 128 + ca * 16 - q * 1024;
    vbo[q] = kb * 8192 + cbk * 16 - q * 1024; vbw[q] = kb * 256 + cbk * 16 - q * 1024;
  }
  const u32 lds0 = (u32)(size_t)lds;
  const u32 wm = wid & 1, wn = wid >> 1, h = lane >> 5;
  u32 xo[4], wo[2];
  {
    const u32 r = wm * 32 * MI + (lane & 31);
#pragma unroll
    for (int ks = 0; ks < 4; ++ks) xo[ks] = r * 128 + (((2 * ks + h) ^ ((r >> 1) & 7)) * 16);
    const u32 bk = (lane >> 4) & 1, q = (lane & 15) >> 2, pp = lane & 3;
#pragma unroll
    for (int t = 0; t < 2; ++t)
      wo[t] = XB + 256 * (8 * h + 4 * t + q) + 16 * ((4 * wn + 2 * bk + (pp >> 1)) ^ ((q << 2) | (2 * h + t))) + 8 * (pp & 1);
  }
  LAS u32* lbad = (LAS u32*)(lds + LDS_CTL);
  u32 ri, ci;
  tile_rc(m, ri, ci);
  const u32 row0 = srow(rank, u) * 1024 + ri * 128 + (HALF ? hsel * 64 : 0), col0 = scol(rank, v) * 1024 + ci * 128;
  u32 islot = 0, rslot = 0;
  auto next_slot = [&](u32 o) -> u32 { return o + SB == NS * SB ? 0 : o + SB; };
  auto issue_own = [&](int t) {
    const u32 dst = lds0 + islot, sa = row0 * 1024 + t * 128, sw = t * 64 * 8192 + col0 * 2;
    islot = next_slot(islot);
    if (HALF) dma3(dst + wid * 1024, dst + XB + wid * 2048, rx, vao[0], sa, rw, vbo[0], vbo[1], sw);
    else dma4(dst + wid * 2048, rx, vao[0], vao[1], sa, rw, vbo[0], vbo[1], sw);
  };
  u32 taba, tabw;
  {
    const bool cr = lane < T_INTRA;
    const u32 c = cr ? lane - T_CROSS : lane - T_INTRA, ktl = cr ? c >> 2 : c / 3;
    const u32 src = rank ^ (cr ? 4 + (c & 3) : 1 + c - ktl * 3), kts = cr ? KC : KI;
    const u32 base = IN_OFF + src * IN_SRC + ktl * 4 * BLK;
    taba = base + (ri * kts * 4 + u) * BLK + (HALF ? hsel * 8192 : 0);
    tabw = base + (ci * kts * 4 + 2 + v) * BLK;
  }
  auto issue = [&](int t) {
    const u32 dst = lds0 + islot, sa = __builtin_amdgcn_readlane(taba, t), sw = __builtin_amdgcn_readlane(tabw, t);
    islot = next_slot(islot);
    if (HALF) dma3(dst + wid * 1024, dst + XB + wid * 2048, rs, vaw[0], sa, rs, vbw[0], vbw[1], sw);
    else dma4(dst + wid * 2048, rs, vaw[0], vaw[1], sa, rs, vbw[0], vbw[1], sw);
  };
  const u32 fop = (lane >> 1) & 1, flev = fop ? ci : ri, fuv = fop ? v : u, fq = lane >> 2;
  const u32 fh = (HALF && !fop) ? hsel : (lane & 1);
  const u32 fck = min(fq >> 2, KC - 1), fik = min(fq / 3, KI - 1);
  const char* fcross = ws + FI_OFF + ((rank ^ (4 + (fq & 3))) * 320 + ((flev * KC + fck) * 4 + fop * 2 + fuv) * 2 + fh) * FIS;
  const char* fintra = ws + FI_OFF + ((rank ^ (1 + fq % 3)) * 320 + ((flev * KI + fik) * 4 + fop * 2 + fuv) * 2 + fh) * FIS;

  while (__builtin_amdgcn_s_memrealtime() - tstart < (u64)max(ri, ci) * STAGGER) __builtin_amdgcn_s_sleep(16);
  if (first && wid == 0) __builtin_amdgcn_fence(__ATOMIC_ACQUIRE, "");
  f32x16 acc[MI] = {};
  struct Frag { bf16x8 w[4], x[MI][4]; };
  auto read = [&](Frag& f) {
    const LAS char* st = lds + rslot;
    rslot = next_slot(rslot);
#pragma unroll
    for (int ks = 0; ks < 4; ++ks) {
      const bf16x4 w0 = __builtin_amdgcn_ds_read_tr16_b64_v4bf16((LAS bf16x4*)(st + wo[0] + ks * 4096));
      const bf16x4 w1 = __builtin_amdgcn_ds_read_tr16_b64_v4bf16((LAS bf16x4*)(st + wo[1] + ks * 4096));
      f.w[ks] = __builtin_shufflevector(w0, w1, 0, 1, 2, 3, 4, 5, 6, 7);
#pragma unroll
      for (int mi = 0; mi < MI; ++mi) f.x[mi][ks] = *(const LAS bf16x8*)(st + xo[ks] + mi * 4096);
    }
  };
  auto mma = [&](const Frag& f) {
#pragma unroll
    for (int ks = 0; ks < 4; ++ks)
#pragma unroll
      for (int mi = 0; mi < MI; ++mi) acc[mi] = __builtin_amdgcn_mfma_f32_32x32x16_bf16(f.w[ks], f.x[mi][ks], acc[mi], 0, 0, 0);
  };
  int next_wait = T_CROSS;
  u64 need = 0xFFFF, have = 0;
  auto waitpoint = [&](int tn) {
    if (tn == T_INTRA) { need = 0xFFF; have = 0; }
    if (wid == 0 && (have & need) != need) {
      bool bad = false;
      have = wave_wait2<SL_GEMM>(tn < T_INTRA ? fcross : fintra, need, seq, tmo, bad);
      if (bad) *lbad = 1;
    }
    if (tn < T_INTRA) { next_wait += 4; need = (need << 16) | 0xFFFF; }
    else { next_wait += 3; need = (need << 12) | 0xFFF; }
  };
  auto step = [&](auto rc, int tn, const Frag& fc, Frag& fn) {
    constexpr int R = decltype(rc)::value;
    if (tn >= 0 && __builtin_expect(tn == next_wait, 0)) waitpoint(tn);
    if constexpr (R >= 0) {
      WAIT_VM(R * NDMA);
      __builtin_amdgcn_s_waitcnt(0xC07F);
      CBAR();
      __builtin_amdgcn_s_barrier();
      CBAR();
      read(fn);
    }
    if (tn >= 0) issue(tn);
    mma(fc);
  };
  Frag fa, fb;
  auto tail = [&](auto nc) {
    constexpr int N = decltype(nc)::value;
    static_for<0, N / 2>([&](auto ic) {
      constexpr int i = decltype(ic)::value;
      step(std::integral_constant<int, N - 2 - 2 * i>{}, -1, fa, fb);
      step(std::integral_constant<int, N - 3 - 2 * i>{}, -1, fb, fa);
    });
  };
  static_assert(NS % 2 == 0 && NS >= T_SPLIT && T_SPLIT + 1 == T_CROSS, "run structure");
  for (int t = 0; t < T_SPLIT; ++t) issue_own(t);
  WAIT_VM((T_SPLIT - 1) * NDMA);
  __builtin_amdgcn_s_barrier();
  CBAR();
  read(fa);
  tail(std::integral_constant<int, T_SPLIT>{});
  islot = rslot = 0;
  __builtin_amdgcn_s_waitcnt(0xC07F);
  __builtin_amdgcn_s_barrier();
  issue_own(T_SPLIT);
  for (int t = T_CROSS; t < T_SPLIT + NS; ++t) {
    if (t == next_wait) { waitpoint(t); __builtin_amdgcn_s_barrier(); }
    issue(t);
  }
  WAIT_VM((NS - 1) * NDMA);
  __builtin_amdgcn_s_barrier();
  CBAR();
  read(fa);
  for (int tn = T_SPLIT + NS; tn < NT; tn += 2) {
    step(std::integral_constant<int, NS - 2>{}, tn, fa, fb);
    step(std::integral_constant<int, NS - 2>{}, tn + 1, fb, fa);
  }
  tail(std::integral_constant<int, NS>{});
  const u32 tmv = __hip_atomic_load((const GAS u32*)tmo, __ATOMIC_RELAXED, __HIP_MEMORY_SCOPE_AGENT);
  __builtin_amdgcn_s_waitcnt(0xC07F);
  __builtin_amdgcn_s_barrier();
  CBAR();
  constexpr u32 EROW = 528;
#pragma unroll
  for (int mi = 0; mi < MI; ++mi) {
    const u32 mr = wm * 32 * MI + mi * 32 + (lane & 31);
#pragma unroll
    for (int gq = 0; gq < 4; ++gq) {
      const f32x4 o = {acc[mi][4 * gq], acc[mi][4 * gq + 1], acc[mi][4 * gq + 2], acc[mi][4 * gq + 3]};
      *(LAS f32x4*)(lds + mr * EROW + (wn * 32 + 8 * gq + 4 * h) * 4) = o;
    }
  }
  __syncthreads();
  const u32 bad = (*lbad || tmv == seq) ? NAN2 : 0u;
  const u32 keep = rank >> 2;
  const bool do_exp = !HALF || hsel != keep, do_keep = !HALF || hsel == keep;
  char* wsq = p.ws[4 | ((rank & 3) ^ sb)];
  if (do_exp) {
    const u32 e16 = P16_OFF + (sb * 64 + m) * 16384 + wid * 1024;
#pragma unroll
    for (int part = 0; part < 2; ++part) {
      const LAS char* se = lds + ((HALF ? 0 : (keep ^ 1) * 64) + part * 32 + (tid >> 4)) * EROW + (tid & 15) * 32;
      const f32x4 c0 = *(const LAS f32x4*)se, c1 = *(const LAS f32x4*)(se + 16);
      const bf16x8 o = {(__bf16)c0[0], (__bf16)c0[1], (__bf16)c0[2], (__bf16)c0[3],
                        (__bf16)c1[0], (__bf16)c1[1], (__bf16)c1[2], (__bf16)c1[3]};
      st128(__builtin_bit_cast(u32x4, o) | bad, rs, lane * 16, e16 + part * 8192);
    }
  }
  if (HALF && do_keep && m < MDIR) {
    f32x4 kc[2][2];
#pragma unroll
    for (int part = 0; part < 2; ++part) {
      const LAS char* sk = lds + (part * 32 + (tid >> 4)) * EROW + (tid & 15) * 32;
      kc[part][0] = *(const LAS f32x4*)sk; kc[part][1] = *(const LAS f32x4*)(sk + 16);
    }
    bool badw = bad != 0;
    wave_wait<2>(ws + FPR_OFF + (sb * 64 + m) * FOS, 1, seq, tmo, badw);
    u32x4 r[2];
#pragma unroll
    for (int part = 0; part < 2; ++part)
    {
      r[part] = __builtin_amdgcn_raw_buffer_load_b128(mk_rsrc(wsq, DATA_BYTES), lane * 16, P16_OFF + ((rank & 3) * 64 + m) * 16384 + wid * 1024 + part * 8192, AUX_SYS);
      WAIT_VM(0);
    }
    const auto ro = mk_rsrc(p.out[0], OUT_BYTES);
#pragma unroll
    for (int part = 0; part < 2; ++part) {
      float y[8];
#pragma unroll
      for (int e = 0; e < 4; ++e) {
        y[2 * e] = silu(kc[part][e >> 1][(2 * e) & 3] + __uint_as_float(r[part][e] << 16));
        y[2 * e + 1] = silu(kc[part][e >> 1][(2 * e + 1) & 3] + __uint_as_float(r[part][e] & 0xFFFF0000u));
      }
      const bf16x8 o = {(__bf16)y[0], (__bf16)y[1], (__bf16)y[2], (__bf16)y[3],
                        (__bf16)y[4], (__bf16)y[5], (__bf16)y[6], (__bf16)y[7]};
      st128(__builtin_bit_cast(u32x4, o) | (badw ? NAN2 : 0u), ro, (lane >> 4) * 8192 + (lane & 15) * 16,
            (row0 + part * 32 + wid * 4) * 8192 + col0 * 2);
    }
    WAIT_VM(0);
    __syncthreads();
    if (tid >= 1 && tid < NR) st_sys(p.ws[tid] + FF_OFF + (rank * 256 + m * 4 + sb) * FOS, seq);
    return;
  }
  if (do_keep) {
    const u32 e32 = P32_OFF + (sb * 64 + m) * 32768 + wid * 2048;
#pragma unroll
    for (int part = 0; part < 2; ++part) {
      const LAS char* sk = lds + ((HALF ? 0 : keep * 64) + part * 32 + (tid >> 4)) * EROW + (tid & 15) * 32;
      const u32x4 c0 = *(const LAS u32x4*)sk, c1 = *(const LAS u32x4*)(sk + 16);
      st128(c0 | bad, rs, lane * 16, e32 + part * 16384);
      st128(c1 | bad, rs, lane * 16, e32 + part * 16384 + 1024);
    }
  }
  WAIT_VM(0);
  __syncthreads();
  if (tid == 0 && do_exp) st_sys(wsq + FPR_OFF + ((rank & 3) * 64 + m) * FOS, seq);
  if (tid == 64 && do_keep) st_sys(ws + FPL_OFF + (sb * 64 + m) * FOS, seq);
}

constexpr u32 LK0 = NPUSH / 8, NUNIT = 64 - 2 * LK0;
static_assert(NPUSH % 8 == 0 && LK0 < 8 * KC, "staging units");
__device__ __forceinline__ void stage_unit(const Params& p, u32 u, u32 seq) {
  const int tid = threadIdx.x, lane = tid & 63;
  const u32 wid = rfl(tid >> 6);
  constexpr u32 NPAIR = 2 * (8 * KC - LK0);
  const bool cr = u < NPAIR && (u & 1);
  const u32 lk = u < NPAIR ? LK0 + (u >> 1) : 8 * KC + u - NPAIR;
  const u32 lev = cr ? lk / KC : lk / KI, kt = cr ? KI + lk - lev * KC : lk - lev * KI, op = wid >> 2, sup = wid & 3;
  const auto r = mk_rsrc(op ? p.w : p.x, X_BYTES);
  const auto rs = mk_rsrc(p.ws[0], DATA_BYTES);
  const u32 vo = op ? (lane >> 4) * 8192 + (lane & 15) * 16 : (lane >> 3) * 1024 + (lane & 7) * 16;
  const u32 so = op ? kt * 64 * 8192 + (sup * 1024 + lev * 128) * 2 : (sup * 1024 + lev * 128) * 1024 + kt * 128;
  u32x4 v[16];
#pragma unroll
  for (int i = 0; i < 16; ++i) v[i] = __builtin_amdgcn_raw_buffer_load_b128(r, vo, so + i * (op ? 4 * 8192 : 8 * 1024), 0);
#pragma unroll
  for (int i = 0; i < 16; ++i) st128(v[i], rs, lane * 16, EX_OFF + ((lev * 8 + kt) * 8 + wid) * BLK + i * 1024);
  WAIT_VM(0);
  __syncthreads();
  if (tid >= 1 && tid < NR && (tid >= 4) == cr) st_sys(p.ws[tid] + SG_OFF + (p.rank * 40 + lk) * FOS, seq);
}

__device__ __forceinline__ void gemm_role(const Params& p, u32 g, u32 k, u32 seq, LAS char* lds) {
  const u64 tstart = __builtin_amdgcn_s_memrealtime();
  static_assert(GEMM_WGS >= 16 && GEMM_WGS <= 24, "unit schedule below");
  if (g == 0 && k >= 16)
    for (u32 r = 0; r < HELP_ROUNDS; ++r) { stage_unit(p, r * 16 + k - 8, seq); }
  const u32 keep = p.rank >> 2;
  if (k < 16) gemm_tile<true>(p, g >= 4 ? (p.rank ^ g) & 3 : g, k, g >= 4 ? keep : keep ^ 1, seq, tstart, true, lds);
  const u32 arm = g & 1;
  for (u32 n = k < 16 ? GEMM_WGS - 16 + k : k - 16; n < 24; n += GEMM_WGS) {
    u32 lev = 4, idx = n;
    for (;;) {
      const u32 cnt = lev + ((lev & 1) == arm);
      if (idx < cnt) break;
      idx -= cnt; ++lev;
    }
    gemm_tile<false>(p, g >> 1, lev * lev + (idx == lev ? 2 * lev : arm * lev + idx), 0, seq, tstart, k >= 16, lds);
  }
}

__global__ void __launch_bounds__(THREADS, 2) fused_gemm_allreduce_silu(Params p) {
  extern __shared__ __attribute__((aligned(16))) char smem[];
  const int lane = threadIdx.x & 63;
  const u32 wid = rfl(threadIdx.x >> 6);
  LAS u32* ctl = (LAS u32*)smem;
  LAS u32* lctl = (LAS u32*)(smem + LDS_CTL);
  u32 slot = (blockIdx.x & ~7u) | ((u32)__builtin_amdgcn_s_getreg((3 << 11) | 20) & 7);
  auto is_push = [&]() { return slot < 8 * ENG_WGS && (slot & 7) != 0 && wid < PUSH_WAVES; };
  if (__builtin_expect(is_push(), 1)) {
    if (push_role<false>(p, slot, wid, lane, lctl)) return;
  } else {
    if (wid == 7) {
      if (lane < 32) ctl[lane] = 0;
      if (lane == 0) {
        GAS u32* ctr = (GAS u32*)(p.ws[0] + CTR_OFF) + blockIdx.x;
        GAS u32* clm = (GAS u32*)(p.ws[0] + CLM_OFF);
        const u32 c = __hip_atomic_load(ctr, __ATOMIC_RELAXED, __HIP_MEMORY_SCOPE_AGENT);
        u32 sl = slot;
#pragma unroll 1
        for (int n = 0; n < 4 * GRID; ++n) {
          if (__hip_atomic_fetch_add(clm + sl * 16, 1u, __ATOMIC_RELAXED, __HIP_MEMORY_SCOPE_AGENT) == c) break;
          __hip_atomic_fetch_add(clm + sl * 16, ~0u, __ATOMIC_RELAXED, __HIP_MEMORY_SCOPE_AGENT);
          sl = (sl + 1) & (GRID - 1);
        }
        lctl[0] = 0; lctl[1] = sl; lctl[2] = c + 1;
      }
    }
    asm volatile("s_waitcnt lgkmcnt(0)" ::: "memory");
    __builtin_amdgcn_s_barrier();
    slot = rfl(LD_LDS(lctl + 1));
    if (threadIdx.x == THREADS - 1)
      __hip_atomic_store((GAS u32*)(p.ws[0] + CTR_OFF) + blockIdx.x, LD_LDS(lctl + 2), __ATOMIC_RELAXED, __HIP_MEMORY_SCOPE_AGENT);
    WAIT_VM(0);
  }
  if (__builtin_expect(is_push(), 0)) {
    push_role<true>(p, slot, wid, lane, lctl);
    return;
  }
  const u32 seq = rfl(LD_LDS(lctl + 2));
  const u32 g = slot & 7, s = slot >> 3;
  const u32 peer = ((u32)p.rank ^ g) & (NR - 1);
  if (s < ENG_WGS) {
    if (g == 0) {
      for (u32 r = 0; r < HELP_ROUNDS; ++r) stage_unit(p, r * 16 + s, seq);
      for (u32 u = HELP_ROUNDS * 16 + s; u < NUNIT; u += 8) stage_unit(p, u, seq);
      return;
    }
    if (wid < PUSH_WAVES + PULL_WAVES) {
      pull_role(p, peer, (wid - PUSH_WAVES) * ENG_WGS + s, lane, seq, ctl);
    } else if (wid < PUSH_WAVES + PULL_WAVES + FIN_WAVES && g >= 4) {
      fin_role(p, peer, s, wid - PUSH_WAVES - PULL_WAVES, lane, seq, ctl);
    }
  } else {
    gemm_role(p, g, s - ENG_WGS, seq, (LAS char*)smem);
  }
}

}

extern "C" void kernel_launch(int rank, int nranks, ncclComm_t comm,
                              hipStream_t stream,
                              void* const* d_in, const int* in_sizes, int n_in,
                              void* d_out, int out_size,
                              const int* in_dtypes, int out_dtype,
                              void* const* d_out_all,
                              void* const* d_ws_all, size_t ws_bytes) {
  if (nranks != NR || (unsigned)rank >= (unsigned)NR || n_in < 2 || ws_bytes < WS_USED ||
      in_sizes[0] != 4096 * 512 || in_sizes[1] != 512 * 4096 || out_size != 4096 * 4096) {
    fprintf(stderr, "kernel_launch: unexpected problem shape\n");
    return;
  }
  HIP_CHECK(hipFuncSetAttribute((const void*)fused_gemm_allreduce_silu,
                                hipFuncAttributeMaxDynamicSharedMemorySize, LDS_BYTES));
  Params p;
  p.x = (const char*)d_in[0];
  p.w = (const char*)d_in[1];
  const int phys[NR] = {0, 2, 3, 6, 1, 4, 5, 7};
  p.rank = 0; p.pad = 0;
  for (int i = 0; i < NR; ++i) if (phys[i] == rank) p.rank = i;
  for (int i = 0; i < NR; ++i) { p.ws[i] = (char*)d_ws_all[phys[p.rank ^ i]]; p.out[i] = (char*)d_out_all[phys[p.rank ^ i]]; }
  p.pad2 = 0;
  for (u32 g = 0; g < NR; ++g)
    for (u32 job = 0; job < NPUSH; ++job) {
      const u32 peer = p.rank ^ g, kts = g >= 4 ? KC : KI, lk = job >> 3, op = (job >> 2) & 1, uv = (job >> 1) & 1, half = job & 1;
      const u32 lev = lk / kts, kt = (g >= 4 ? KI : 0) + lk % kts;
      p.psrc[g][job & 15][job >> 4] = op ? (kt * 64 + half * 32) * 8192 + (scol(peer, uv) * 1024 + lev * 128) * 2
                                         : (srow(peer, uv) * 1024 + lev * 128 + half * 64) * 1024 + kt * 128;
    }
  hipLaunchKernelGGL(fused_gemm_allreduce_silu, dim3(GRID), dim3(THREADS), LDS_BYTES, stream, p);
  HIP_CHECK(hipGetLastError());
}
```

```cpp
#include <hip/hip_runtime.h>
#include <rccl/rccl.h>
#include <cstdio>
#include <type_traits>

#define HIP_CHECK(expr) do { hipError_t _e = (expr); if (_e != hipSuccess) { \
    fprintf(stderr, "HIP error %s at %s:%d\n", hipGetErrorString(_e), __FILE__, __LINE__); \
    return; } } while (0)

namespace {

typedef unsigned int u32;
typedef unsigned long long u64;
typedef u32 u32x4 __attribute__((ext_vector_type(4)));
typedef float f32x4 __attribute__((ext_vector_type(4)));
typedef __bf16 bf16x4 __attribute__((ext_vector_type(4)));
typedef __bf16 bf16x8 __attribute__((ext_vector_type(8)));
typedef float f32x16 __attribute__((ext_vector_type(16)));
#define LAS __attribute__((address_space(3)))
#define GAS __attribute__((address_space(1)))

constexpr int NR = 8;
constexpr u32 X_BYTES = 4096u * 512u * 2u;
constexpr u32 W_BYTES = 512u * 4096u * 2u;
constexpr u32 OUT_BYTES = 4096u * 4096u * 2u;
constexpr u32 KI = 5, KC = 3;

constexpr u32 FIS = 128, FOS = 128;

constexpr u32 BLK = 16384;
constexpr u32 IN_OFF = 0;
constexpr u32 IN_SRC = KI * 32 * BLK;
constexpr u32 P16_OFF = IN_OFF + NR * IN_SRC;
constexpr u32 P32_OFF = P16_OFF + 256 * 16384;
constexpr u32 EX_OFF = P32_OFF + 256 * 32768;
constexpr u32 DATA_BYTES = EX_OFF + 512 * BLK;
constexpr u32 FI_OFF = DATA_BYTES;
constexpr u32 FPR_OFF = FI_OFF + NR * 320 * FIS;
constexpr u32 FPL_OFF = FPR_OFF + 256 * FOS;
constexpr u32 FF_OFF = FPL_OFF + 256 * FOS;
constexpr u32 SG_OFF = FF_OFF + NR * 256 * FOS;
constexpr u32 GO_OFF = SG_OFF + NR * 40 * FOS;
constexpr u32 CTR_OFF = GO_OFF + NR * 16 * FOS;
constexpr u32 CLM_OFF = CTR_OFF + 256 * 4;
constexpr u32 TMO_OFF = CLM_OFF + 256 * 64;
constexpr u32 TICK_OFF = TMO_OFF + 128;
constexpr u32 SEG_OFF = TICK_OFF + 16 * 128;
constexpr u32 RF_OFF = SEG_OFF + 8 * 128;
constexpr u32 WS_USED = RF_OFF + 8 * 128;

constexpr u64 DEADLINE_TICKS = 80000000ull;

constexpr int ENG_WGS = 8, GEMM_WGS = 24;
constexpr int PUSH_WAVES = 2, PULL_WAVES = 2, FIN_WAVES = 4;
constexpr int PUSH_D = 4, PUSH_ROUNDS = 4;
constexpr u32 NPUSH = PUSH_ROUNDS * ENG_WGS * PUSH_WAVES;
constexpr int GO_AT = 7;
constexpr u32 HELP_ROUNDS = 2;
constexpr int NS_HALF = 6;
constexpr u32 MDIR = 16;
constexpr u64 STAGGER = 300;
constexpr u32 LOOK_AT = 100, LEAD = 4, LEAD_W = 3, IDLE_MIN = 24;
constexpr int SL_PULL = 2, SL_FIN = 8, SL_GEMM = 8;
constexpr u32 NAN2 = 0x7FC07FC0u;
constexpr int GRID = 8 * (ENG_WGS + GEMM_WGS);
static_assert(GRID == 256, "slots wrap at a power of two");
constexpr int THREADS = 512;
constexpr u32 LDS_CTL = 144 * 1024;
constexpr u32 LDS_BYTES = LDS_CTL + 64;
constexpr int AUX_SYS = 17;

struct Params {
  const char* x;
  const char* w;
  int rank;
  int pad;
  char* ws[NR];
  char* out[NR];
  u64 pad2;
  u32x4 psrc[NR][16];
};

__device__ __forceinline__ u32 rfl(u32 v) { return __builtin_amdgcn_readfirstlane(v); }
__device__ __forceinline__ __amdgpu_buffer_rsrc_t mk_rsrc(const void* p, u32 n) {
  const u64 a = (u64)p;
  const u64 u = ((u64)rfl((u32)(a >> 32)) << 32) | rfl((u32)a);
  return __builtin_amdgcn_make_buffer_rsrc((void*)u, 0, n, 0x00020000);
}
__device__ __forceinline__ u32 ld_sys(const char* p) {
  return __hip_atomic_load((const GAS u32*)p, __ATOMIC_RELAXED, __HIP_MEMORY_SCOPE_SYSTEM);
}
__device__ __forceinline__ void st_sys(char* p, u32 v) {
  __hip_atomic_store((GAS u32*)p, v, __ATOMIC_RELAXED, __HIP_MEMORY_SCOPE_SYSTEM);
}
#define WAIT_VM(n) asm volatile("s_waitcnt vmcnt(%0)" ::"n"(n) : "memory")
#define CBAR() asm volatile("" ::: "memory")
#define LD_LDS(q) __hip_atomic_load(q, __ATOMIC_RELAXED, __HIP_MEMORY_SCOPE_WORKGROUP)
#define ST_LDS(q, v) __hip_atomic_store(q, v, __ATOMIC_RELAXED, __HIP_MEMORY_SCOPE_WORKGROUP)

__device__ __forceinline__ void st128(u32x4 v, __amdgpu_buffer_rsrc_t r, u32 voff, u32 soff) {
  __builtin_amdgcn_raw_buffer_store_b128(v, r, voff, soff, AUX_SYS);
  asm volatile("s_nop 1" ::"v"(v) : "memory");
}

template <int SLEEP>
__device__ __forceinline__ u64 wave_wait(const char* flag, u64 need, u32 seq, char* tmo, bool& bad) {
  const u64 t0 = __builtin_amdgcn_s_memrealtime();
  for (u32 it = 1;; ++it) {
    const u64 ok = __ballot(ld_sys(flag) >= seq);
    if ((ok & need) == need) return ok;
    if ((it & 15) == 0 || bad) {
      const u32 t = __hip_atomic_load((const GAS u32*)tmo, __ATOMIC_RELAXED, __HIP_MEMORY_SCOPE_AGENT);
      if (rfl(t) == seq || __builtin_amdgcn_s_memrealtime() - t0 > DEADLINE_TICKS) {
        __hip_atomic_store((GAS u32*)tmo, seq, __ATOMIC_RELAXED, __HIP_MEMORY_SCOPE_AGENT);
        bad = true;
        return ~0ull;
      }
    }
    __builtin_amdgcn_s_sleep(SLEEP);
  }
}

template <int SLEEP>
__device__ __forceinline__ u64 wave_wait2(const char* flag, u64 need, u32 seq, char* tmo, bool& bad) {
  const u64 t0 = __builtin_amdgcn_s_memrealtime();
  const bool needed = (need >> (threadIdx.x & 63)) & 1;
  bool ok = false;
  for (u32 it = 1;; ++it) {
    if (!ok && (it == 1 || needed)) ok = ld_sys(flag) >= seq;
    const u64 have = __ballot(ok);
    if ((have & need) == need) return have;
    if ((it & 15) == 0) {
      const u32 t = __hip_atomic_load((const GAS u32*)tmo, __ATOMIC_RELAXED, __HIP_MEMORY_SCOPE_AGENT);
      if (rfl(t) == seq || __builtin_amdgcn_s_memrealtime() - t0 > DEADLINE_TICKS) {
        __hip_atomic_store((GAS u32*)tmo, seq, __ATOMIC_RELAXED, __HIP_MEMORY_SCOPE_AGENT);
        bad = true;
        return ~0ull;
      }
    }
    __builtin_amdgcn_s_sleep(SLEEP);
  }
}

__device__ __forceinline__ void dma4(u32 dst, __amdgpu_buffer_rsrc_t ra, u32 va0, u32 va1, u32 sa,
                                     __amdgpu_buffer_rsrc_t rb, u32 vb0, u32 vb1, u32 sb) {
  u32 keep;
  dst = rfl(dst); sa = rfl(sa); sb = rfl(sb);
  asm volatile(
      "s_mov_b32 %0, m0\n\t"
      "s_nop 4\n\t"
      "s_mov_b32 m0, %1\n\t"
      "s_nop 0\n\t"
      "buffer_load_dwordx4 %3, %5, %6 offen lds\n\t"
      "buffer_load_dwordx4 %4, %5, %6 offen offset:1024 lds\n\t"
      "s_mov_b32 m0, %2\n\t"
      "s_nop 0\n\t"
      "buffer_load_dwordx4 %7, %9, %10 offen lds\n\t"
      "buffer_load_dwordx4 %8, %9, %10 offen offset:1024 lds\n\t"
      "s_mov_b32 m0, %0"
      : "=&s"(keep)
      : "s"(dst), "s"(dst + BLK), "v"(va0), "v"(va1), "s"(ra), "s"(sa), "v"(vb0), "v"(vb1), "s"(rb), "s"(sb)
      : "memory");
}

__device__ __forceinline__ void dma3(u32 dsta, u32 dstb, __amdgpu_buffer_rsrc_t ra, u32 va0, u32 sa,
                                     __amdgpu_buffer_rsrc_t rb, u32 vb0, u32 vb1, u32 sb) {
  u32 keep;
  dsta = rfl(dsta); dstb = rfl(dstb); sa = rfl(sa); sb = rfl(sb);
  asm volatile(
      "s_mov_b32 %0, m0\n\t"
      "s_nop 4\n\t"
      "s_mov_b32 m0, %1\n\t"
      "s_nop 0\n\t"
      "buffer_load_dwordx4 %3, %4, %5 offen lds\n\t"
      "s_mov_b32 m0, %2\n\t"
      "s_nop 0\n\t"
      "buffer_load_dwordx4 %6, %8, %9 offen lds\n\t"
      "buffer_load_dwordx4 %7, %8, %9 offen offset:1024 lds\n\t"
      "s_mov_b32 m0, %0"
      : "=&s"(keep)
      : "s"(dsta), "s"(dstb), "v"(va0), "s"(ra), "s"(sa), "v"(vb0), "v"(vb1), "s"(rb), "s"(sb)
      : "memory");
}

template <int I, int N, class F>
__device__ __forceinline__ void static_for(F&& f) {
  if constexpr (I < N) { f(std::integral_constant<int, I>{}); static_for<I + 1, N>(f); }
}

__host__ __device__ __forceinline__ u32 sup(u32 grp, u32 c, u32 u) { return grp ? c + 2 * u : 2 * c + u; }
__host__ __device__ __forceinline__ u32 srow(u32 r, u32 u) { return sup(r >> 2, (r >> 1) & 1, u); }
__host__ __device__ __forceinline__ u32 scol(u32 r, u32 v) { return sup(r >> 2, r & 1, v); }
__device__ __forceinline__ void tile_rc(u32 m, u32& ri, u32& ci) {
  const u32 l = (m >= 1) + (m >= 4) + (m >= 9) + (m >= 16) + (m >= 25) + (m >= 36) + (m >= 49);
  const u32 idx = m - l * l;
  ri = idx < l ? l : idx - l;
  ci = idx < l ? idx : l;
}
__device__ __forceinline__ u32 fin_off(u32 r, u32 m, u32 sb, u32 part) {
  u32 ri, ci;
  tile_rc(m, ri, ci);
  return (srow(r, sb >> 1) * 1024 + ri * 128 + (r >> 2) * 64 + part * 32) * 8192 + (scol(r, sb & 1) * 1024 + ci * 128) * 2;
}

template <bool claimed>
__device__ __forceinline__ bool push_role(const Params& p, u32& slot, u32 wid, int lane, LAS u32* lctl) {
  constexpr int D = PUSH_D, JI = 8;
  constexpr u32 NW = ENG_WGS * PUSH_WAVES;
  static_assert(PUSH_ROUNDS == 4 && NW == 16 && X_BYTES == W_BYTES && GO_AT <= JI, "push schedule");
  const u32 g = slot & 7, w = wid * ENG_WGS + (slot >> 3), op = (slot >> 5) & 1;
  u64 bs = (u64)(&p.x)[op], bw = (u64)p.ws[g];
  u32x4 src = p.psrc[g][w];
  u32 rank = p.rank;
  asm volatile("" : "+s"(bs), "+s"(bw), "+s"(src), "+s"(rank));
  const auto rs = mk_rsrc((const void*)bs, X_BYTES);
  const u32 vs = op ? (lane >> 4) * 8192 + (lane & 15) * 16 : (lane >> 3) * 1024 + (lane & 7) * 16;
  const u32 str = op ? 4 * 8192 : 8 * 1024;
  auto ld = [&](int r, int i) -> u32x4 { return __builtin_amdgcn_raw_buffer_load_b128(rs, vs, src[r] + i * str, 0); };
  u32x4 v[D], v2[JI - D];
#pragma unroll
  for (int i = 0; i < D; ++i) { v[i] = ld(0, i); CBAR(); }
#pragma unroll
  for (int i = D; i < JI; ++i) { v2[i - D] = ld(0, i); CBAR(); }
  char* wsp = (char*)bw;
  const auto rd = mk_rsrc(wsp, DATA_BYTES);
  const u32 dst = IN_OFF + rank * IN_SRC + w * 8192, flag = FI_OFF + (rank * 320 + w) * FIS;
  u32 seq = 0;
  auto round = [&](auto rc) {
    constexpr int R = decltype(rc)::value;
    constexpr bool FIRST = R == 0, LAST = R == PUSH_ROUNDS - 1;
#pragma unroll
    for (int i = 0; i < JI; ++i) {
      if (!FIRST && i == D - 1) {
        WAIT_VM(2 * D - 1);
        if (R == 1) {
          if (!claimed) __builtin_amdgcn_s_barrier();
          CBAR();
          seq = rfl(LD_LDS(lctl + 2));
        }
        if (lane == 0) st_sys(wsp + flag + (R - 1) * NW * FIS, seq);
        CBAR();
      }
      if (LAST && i == JI - GO_AT) {
        if (lane == 0) st_sys(wsp + GO_OFF + (rank * 16 + w) * FOS, seq);
        if (lane == 0) ST_LDS(lctl - LDS_CTL / 4 + 24 + wid, 1u);
        CBAR();
      }
      st128(FIRST && i >= D ? v2[i >= D ? i - D : 0] : v[i % D], rd, lane * 16, dst + R * NW * 8192 + i * 1024);
      if (i + D < JI) { if (!FIRST) v[i % D] = ld(R, i + D); }
      else if (!LAST) v[i % D] = ld(LAST ? R : R + 1, i + D - JI);
      CBAR();
    }
  };
  static_assert(JI - D >= D - 1, "the count above also holds in the last round");
  round(std::integral_constant<int, 0>{});
  round(std::integral_constant<int, 1>{});
  const u32 slot2 = rfl(LD_LDS(lctl + 1));
  if (__builtin_expect(slot2 != slot, 0)) { WAIT_VM(0); slot = slot2; return false; }
  round(std::integral_constant<int, 2>{});
  round(std::integral_constant<int, 3>{});
  WAIT_VM(0);
  if (lane == 0) st_sys(wsp + flag + (PUSH_ROUNDS - 1) * NW * FIS, seq);
  return true;
}

__device__ __forceinline__ float silu(float a) {
  const float c = fminf(fmaxf(a, -60.f), 60.f);
  return a * __builtin_amdgcn_rcpf(1.f + __expf(-c));
}

constexpr u32 MB_REQ = 0, MB_DONE = 4, MB_GONE = 8, MB_UNIT = 10, MB_PUSHED = 24, MB_BUF = 1024;

__device__ __forceinline__ void pull_role(const Params& p, u32 peer, u32 w, int lane, u32 seq, LAS u32* ctl) {
  constexpr u32 NW = ENG_WGS * PULL_WAVES, KF = 0, KIN = 5;
  constexpr u32 SEG = 96, TAIL = 1024 - 4 * SEG, NONE = ~0u, REL = 1u << 31;
  const u32 rank = p.rank, g = rank ^ peer;
  const bool cross = g >= 4;
  const u32 kts = cross ? KC : KI, NI = kts * 64 - NPUSH, NJ = NI + 1024;
  char* ws = p.ws[0];
  char* tmo = ws + TMO_OFF;
  const u64 bf = (u64)p.out[g], bp = (u64)p.ws[g], bo = (u64)p.out[0];
  const u32 vwide = (lane >> 4) * 8192 + (lane & 15) * 16;
  GAS u32* const segc = (GAS u32*)(ws + SEG_OFF + g * 128) + lane;
  if (w == 0 && lane == 0) st_sys(ws + TICK_OFF + ((seq & 1) ^ 1) * 1024 + g * 128, 0);
  u32 ttile, tsup, tlk, tblk;
  {
    u32 ri, ci;
    tile_rc(lane, ri, ci);
    ttile = ri * 128 * 8192 + ci * 256;
    tsup = fin_off(rank ^ ((lane >> 2) & 7), 0, lane & 3, 0);
    const u32 lev = cross ? lane / KC : lane / KI, ktl = lane - lev * kts;
    tlk = EX_OFF + (lev * 8 + (cross ? KI + ktl : ktl)) * 8 * BLK;
    tblk = ((lane & 2) * 2 + ((lane & 2) ? scol(rank, lane & 1) : srow(rank, lane & 1))) * BLK;
  }
  auto rl = [](u32 t, u32 i) -> u32 { return __builtin_amdgcn_readlane(t, i); };
  auto rot = [&](u32 j, u32 link) -> u32 {
    if (j < TAIL) return j;
    const u32 t = j - TAIL + (link & 3) * SEG;
    return TAIL + (t >= 4 * SEG ? t - 4 * SEG : t);
  };
  static_assert(PUSH_WAVES == PULL_WAVES && NW <= 16, "one pull wave takes over from one push wave");
  const u32 trig = GO_OFF + (peer * 16 + w) * FOS;
  const u64 t0 = __builtin_amdgcn_s_memrealtime();
  u32 tf = w, tf2 = NW + w, lastf = 0, sub = 0;
  u32 cdst = 0, ck = 0, cflag = 0, pend = 0;
  u32 nsrc = 0, ndst = 0, nk = 0, nflag = 0;
  u32 npos = NONE, cpos = NONE, ppos = NONE;
  u32 apos = 0, acnt = 0, chk = NONE, chkn = 0, chkv = 0;
  u32 left = 0, near = 0;
  u32 oleft = 0, nround = w;
  bool bad = false, own_done = false;
  const u32 sw = NI + TAIL - w * 24;
  auto poll = [&](auto tc) -> u32 {
    if constexpr (decltype(tc)::value) {
      const u32 o = FF_OFF + (peer * 256 + ((rot(tf - NI, g) >> 2) & 255)) * FOS;
      return ld_sys(ws + (near && (lane & 8) ? RF_OFF + g * 128 + (lane & 7) * 4 : rfl(o)));
    } else {
      const u32 o = tf < NI ? SG_OFF + (peer * 40 + ((NPUSH + tf) >> 3)) * FOS : FF_OFF + (peer * 256 + ((rot(tf - NI, g) >> 2) & 255)) * FOS;
      return ld_sys(ws + ((lane & 1) ? rfl(tf == w ? trig : o) : rfl(o)));
    }
  };
  const u32 isrf = (lane & 8) ? ~0u : 0u;
  u32 pv = poll(std::false_type{}), rfm = 0;
  u32x4 v[4];
  auto victim = [&](u32 least) -> u32 {
    u32 best = 0, bl = 0;
    for (u64 mk = __ballot(left > least) & rfm; mk; mk &= mk - 1) {
      const u32 l = __builtin_ctzll(mk), n = __builtin_amdgcn_readlane(left, l);
      if (n > bl) { bl = n; best = l; }
    }
    return best;
  };
  const auto rtick = mk_rsrc(ws + TICK_OFF + (seq & 1) * 1024, 1024);
  auto draw = [&](u32 dg, bool look) -> u32 {
    const bool on = (u32)lane == dg ? dg != 0 : (look && lane >= 1 && lane < NR);
    return __builtin_amdgcn_raw_ptr_buffer_atomic_add_i32((u32)lane != dg ? 0u : dg == g ? 1u : 1u << 16, rtick, on ? lane * 128 : 0x80000000u, 0, 0);
  };
  auto drawn = [&](u32 dg, bool look, u32 t) {
    u32 rem = 0;
    if (look) {
      const u32 nj = (lane >= 4 ? KC : KI) * 64 - NPUSH + 1024, fr = (t & 0xFFFF) + 2 * NW, bk = t >> 16;
      rem = bk < SEG && fr + bk < nj ? nj - fr - bk : 0;
      left = lane >= 1 && lane < NR && (u32)lane != g ? rem - ((u32)lane == dg && rem) : 0;
      const u32 to = __builtin_amdgcn_readlane(t, g), so = (to & 0xFFFF) + 2 * NW + min(to >> 16, SEG);
      oleft = so < NJ ? NJ - so : 0;
    }
    if (dg == 0) return;
    const u32 td = __builtin_amdgcn_readlane(t, dg), f = (td & 0xFFFF) + 2 * NW, b = min(td >> 16, SEG);
    u32 h;
    if (dg == g) {
      h = f + b < NJ ? f : NONE;
      own_done = h == NONE;
      near |= f + b + LOOK_AT >= NJ;
    } else {
      h = __builtin_amdgcn_readlane(rem, dg) ? REL | dg << 16 | (1023 - b) : NONE;
    }
    if (tf == NONE) { tf = h; pv &= isrf; } else tf2 = h;
  };
  static_assert(FIN_WAVES == 4, "mailbox lanes");
  u32 reqv = 0, ack = 0, apend = NONE;
  auto pick = [&](auto tc) -> bool {
    constexpr bool T = decltype(tc)::value;
    if (!T && tf >= sw) return false;
    const u64 req = __ballot(reqv != 0);
    if (req) {
      const u32 f = __builtin_ctzll(req);
      u32 r = 0;
      if (lane == 0) r = __hip_atomic_exchange(ctl + MB_REQ + f, 0u, __ATOMIC_RELAXED, __HIP_MEMORY_SCOPE_WORKGROUP);
      r = rfl(r);
      if (r) { nsrc = r; ndst = r; nk = f + 1; nflag = 0; npos = NONE; return true; }
    }
    if (T && tf == NONE) return false;
    if (!(bad || (T && (tf & REL)) || (__ballot(pv >= seq) & 3) == 3)) return false;
    if (!T && tf < NI) {
      const u32 jb = NPUSH + tf;
      nsrc = rl(tlk, jb >> 3) + rl(tblk, (jb >> 1) & 3) + ((jb & 1) * 2 + sub) * 4096;
      ndst = IN_OFF + peer * IN_SRC + jb * 8192 + sub * 4096;
      nk = KIN; nflag = sub ? FI_OFF + (peer * 320 + jb) * FIS : 0;
    } else {
      const bool rel = T && (tf & REL);
      const u32 link = rel ? (tf >> 16) & 7 : g, pos = rel ? tf & 1023 : tf - NI, j = T ? rot(pos, link) : pos;
      nsrc = ndst = rl(ttile, j >> 4) + rl(tsup, link * 4 + ((j >> 2) & 3)) + (j & 3) * 16 * 8192;
      nk = KF; nflag = 0;
      if (T) npos = pos >= TAIL ? (pos >= TAIL + SEG) + (pos >= TAIL + 2 * SEG) + (pos >= TAIL + 3 * SEG) : NONE;
    }
    nsrc = rfl(nsrc); ndst = rfl(ndst); nflag = rfl(nflag);
    return true;
  };
  auto put = [&](int i) {
    const bool isp = ck - 1 < FIN_WAVES;
    const auto ro = mk_rsrc((const void*)(ck == KF ? bo : (u64)ws), ck == KF ? OUT_BYTES : isp ? 0u : DATA_BYTES);
    st128(v[i], ro, ck == KF ? vwide : lane * 16, rfl(cdst + i * (ck == KF ? 4 * 8192 : 1024)));
    if (isp) *(LAS u32x4*)((LAS char*)ctl + MB_BUF + (ck - 1) * 4096 + i * 1024 + lane * 16) = v[i];
  };
  auto delivered = [&]() {
    if (ck - 1 < FIN_WAVES) {
      asm volatile("s_waitcnt lgkmcnt(0)" ::: "memory");
      if (lane == 0) __hip_atomic_fetch_add(ctl + MB_DONE + ck - 1, 1u, __ATOMIC_RELAXED, __HIP_MEMORY_SCOPE_WORKGROUP);
    }
  };
  auto publish = [&](u32 off) {
    if (lane == 0) st_sys(ws + off, seq);
    CBAR();
  };
  auto check = [&]() {
    if (chk != NONE) {
      if (__builtin_amdgcn_readlane(chkv, 0) + chkn == SEG * seq) {
        const u32 gr = ((g + chk + 1) & 3) | (lane & 1) << 2;
        if (lane < 2 && gr != 0) st_sys(p.ws[g ^ gr] + RF_OFF + (g ^ gr) * 128 + gr * 4, seq);
      }
      chk = NONE;
    }
  };
  auto report = [&]() {
    if (acnt && apos < 3) {
      check();
      if (lane == 0) chkv = __hip_atomic_fetch_add(segc + apos, acnt, __ATOMIC_RELAXED, __HIP_MEMORY_SCOPE_AGENT);
      chk = apos; chkn = acnt;
      CBAR();
    }
    acnt = 0;
  };
  auto stored = [&](u32 pos) {
    if (pos != apos) { report(); apos = pos; }
    ++acnt;
  };
  auto round = [&](auto tc, auto first_c) {
    constexpr bool T = decltype(tc)::value, FIRST = decltype(first_c)::value;
    const bool tix = nk == KF || (!T && nk == KIN && sub);
    if (T && near) rfm |= (u32)(__ballot(pv >= seq) >> 8) & 0xFE & near;
    if (nk == KF) lastf = ndst;
    if (!T && nk == KIN) sub ^= 1;
    u32 dg = 0;
    if (tix) {
      if (T && near) dg = victim(own_done ? 0 : oleft + LEAD + w * LEAD_W);
      if (dg == 0 && !own_done) dg = g;
      tf = tf2; tf2 = NONE;
    }
    const bool look = T && tix && near && g != 4 && (dg != g || (++nround & 1) == 0);
    const u32 tk = draw(dg, look);
    if (T && near) near = 0xFF;
    pv = poll(tc);
    CBAR();
    if (!T) {
      if (ack) { publish(ack); ack = 0; }
    } else {
      check();
      if (apend != NONE) { stored(apend); apend = NONE; }
    }
    const auto rsrc = mk_rsrc((const void*)(nk == KF ? bf : bp), nk == KF ? OUT_BYTES : DATA_BYTES);
    const u32 vs = nk == KF ? vwide : lane * 16, str = nk == KF ? 4 * 8192 : 1024;
#pragma unroll
    for (int i = 0; i < 4; ++i) {
      if (!FIRST) put(i);
      if (i == 3) reqv = LD_LDS(ctl + MB_REQ + (lane & 3));
      v[i] = __builtin_amdgcn_raw_buffer_load_b128(rsrc, vs, rfl(nsrc + i * str), AUX_SYS);
      CBAR();
    }
    if (!FIRST) {
      delivered();
      WAIT_VM(11);
      if (!T) { ack = pend; pend = cflag; }
      else { apend = ppos; ppos = cpos; }
    }
    cdst = ndst; ck = nk; cflag = nflag;
    if (T) cpos = npos;
    if (tix) drawn(dg, look, tk);
  };
  auto drain = [&](auto tc) {
    constexpr bool T = decltype(tc)::value;
#pragma unroll
    for (int i = 0; i < 4; ++i) put(i);
    delivered();
    if (!T) {
      if (ack) { publish(ack); ack = 0; }
      if (pend | cflag) {
        WAIT_VM(0);
        if (pend) publish(pend);
        if (cflag) publish(cflag);
      }
      pend = 0;
    } else {
      if (apend != NONE) { stored(apend); apend = NONE; }
      if (ppos < 3 || cpos < 3) {
        WAIT_VM(0);
        if (ppos != NONE) stored(ppos);
        if (cpos != NONE) stored(cpos);
      }
      report();
      check();
      ppos = NONE;
    }
  };
  auto idle = [&](u32 it) {
    if ((it & 63) == 0) {
      const u32 t = __hip_atomic_load((const GAS u32*)tmo, __ATOMIC_RELAXED, __HIP_MEMORY_SCOPE_AGENT);
      if (rfl(t) == seq || __builtin_amdgcn_s_memrealtime() - t0 > DEADLINE_TICKS) {
        __hip_atomic_store((GAS u32*)tmo, seq, __ATOMIC_RELAXED, __HIP_MEMORY_SCOPE_AGENT);
        WAIT_VM(0);
        bad = true;
      }
    }
    __builtin_amdgcn_s_sleep(SL_PULL);
  };
  while (rfl(LD_LDS(ctl + MB_PUSHED + w / ENG_WGS)) == 0) __builtin_amdgcn_s_sleep(2);
  pv = poll(std::false_type{});
  constexpr std::false_type F_{};
  constexpr std::true_type T_{};
  for (u32 it = 1; tf < sw; ++it) {
    reqv = LD_LDS(ctl + MB_REQ + (lane & 3));
    if (pick(F_)) {
      round(F_, T_);
      if (pick(F_)) {
        round(F_, F_);
        while (pick(F_)) round(F_, F_);
      }
      drain(F_);
      continue;
    }
    idle(it);
    if ((it & 3) == 0 || tf == w) pv = poll(F_);
  }
  for (u32 it = 1;; ++it) {
    if (!own_done && tf2 == NONE) {
      do drawn(g, false, draw(g, false)); while (!own_done && tf2 == NONE);
      pv = poll(T_);
    }
    reqv = LD_LDS(ctl + MB_REQ + (lane & 3));
    if (pick(T_)) {
      round(T_, T_);
      if (pick(T_)) {
        round(T_, F_);
        while (pick(T_)) round(T_, F_);
      }
      drain(T_);
      continue;
    }
    if (tf == NONE) {
      if (near == 0xFF) rfm |= (u32)(__ballot(pv >= seq) >> 8) & 0xFE;
      const u32 dg = bad ? 0 : victim(IDLE_MIN);
      if (dg) {
        drawn(dg, true, draw(dg, true));
        continue;
      }
      if (rfl(LD_LDS(ctl + MB_GONE)) == (cross ? FIN_WAVES : 0)) break;
    }
    idle(it);
    if ((it & 3) == 0) pv = poll(T_);
  }
  if (bad) {
    WAIT_VM(0);
    const u32x4 nanv = {NAN2, NAN2, NAN2, NAN2};
    for (int i = 0; i < 4; ++i) st128(nanv, mk_rsrc(p.out[0], OUT_BYTES), vwide, rfl(lastf) + i * 4 * 8192);
  }
}

__device__ __forceinline__ void fin_role(const Params& p, u32 peer, u32 s, u32 f, int lane, u32 seq, LAS u32* ctl) {
  static_assert(FIN_WAVES == 4, "one wave per 16 rows of a half tile");
  const u32 rank = p.rank, sb = peer & 3;
  char* ws = p.ws[0];
  char* tmo = ws + TMO_OFF;
  const auto rl = mk_rsrc(ws, DATA_BYTES);
  const auto ro = mk_rsrc(p.out[0], OUT_BYTES);
  const u32 vwide = (lane >> 4) * 8192 + (lane & 15) * 16;
  char* wsl = p.ws[lane & 7];
  u32 ndone = 0;
  for (u32 m = MDIR + s; m < 64; m += ENG_WGS) {
    const u32 lo = P32_OFF + (sb * 64 + m) * 32768 + f * 8192;
    const u32 dst = fin_off(rank, m, sb, 0) + f * 16 * 8192;
    bool bad = false;
    wave_wait<SL_FIN>(ws + ((lane & 1) ? FPL_OFF : FPR_OFF) + (sb * 64 + m) * FOS, 3, seq, tmo, bad);
    const u32 so = P16_OFF + ((rank & 3) * 64 + m) * 16384 + f * 4096;
    u32x4 rv[4], a[4], b[4];
    if (lane == 0) ST_LDS(ctl + MB_REQ + f, so);
#pragma unroll
    for (int i = 0; i < 4; ++i) {
      a[i] = __builtin_amdgcn_raw_buffer_load_b128(rl, lane * 16, lo + i * 2048, AUX_SYS);
      b[i] = __builtin_amdgcn_raw_buffer_load_b128(rl, lane * 16, lo + i * 2048 + 1024, AUX_SYS);
    }
    ++ndone;
    while (rfl(LD_LDS(ctl + MB_DONE + f)) != ndone) __builtin_amdgcn_s_sleep(1);
    CBAR();
#pragma unroll
    for (int i = 0; i < 4; ++i) rv[i] = *(const LAS u32x4*)((LAS char*)ctl + MB_BUF + f * 4096 + i * 1024 + lane * 16);
#pragma unroll
    for (int i = 0; i < 4; ++i) {
      const u32x4 r = rv[i];
      float y[8];
#pragma unroll
      for (int e = 0; e < 4; ++e) {
        const u32 l0 = e < 2 ? a[i][2 * e] : b[i][2 * e - 4], l1 = e < 2 ? a[i][2 * e + 1] : b[i][2 * e - 3];
        y[2 * e] = silu(__uint_as_float(l0) + __uint_as_float(r[e] << 16));
        y[2 * e + 1] = silu(__uint_as_float(l1) + __uint_as_float(r[e] & 0xFFFF0000u));
      }
      const bf16x8 o = {(__bf16)y[0], (__bf16)y[1], (__bf16)y[2], (__bf16)y[3],
                        (__bf16)y[4], (__bf16)y[5], (__bf16)y[6], (__bf16)y[7]};
      u32x4 ov = __builtin_bit_cast(u32x4, o);
      if (bad) ov |= NAN2;
      st128(ov, ro, vwide, dst + i * 4 * 8192);
    }
    WAIT_VM(0);
    u32 arrived = 0;
    if (lane == 0) arrived = __hip_atomic_fetch_add(ctl + MB_UNIT + ndone - 1, 1u, __ATOMIC_RELAXED, __HIP_MEMORY_SCOPE_WORKGROUP);
    if (rfl(arrived) == FIN_WAVES - 1) {
      if (lane >= 1 && lane < NR) st_sys(wsl + FF_OFF + (rank * 256 + m * 4 + sb) * FOS, seq);
    }
  }
  if (lane == 0) __hip_atomic_fetch_add(ctl + MB_GONE, 1u, __ATOMIC_RELAXED, __HIP_MEMORY_SCOPE_WORKGROUP);
}

template <bool HALF>
__device__ __forceinline__ void gemm_tile(const Params& p, u32 sb, u32 m, u32 hsel, u32 seq, u64 tstart, bool first, LAS char* lds) {
  const int tid = threadIdx.x, lane = tid & 63;
  const u32 wid = rfl(tid >> 6);
  const u32 rank = p.rank;
  char* ws = p.ws[0];
  char* tmo = ws + TMO_OFF;
  const auto rx = mk_rsrc(p.x, X_BYTES);
  const auto rw = mk_rsrc(p.w, W_BYTES);
  const auto rs = mk_rsrc(ws, DATA_BYTES);
  const u32 u = sb >> 1, v = sb & 1;
  constexpr int NT = 32, T_CROSS = KI, T_INTRA = KI + 4 * KC, T_SPLIT = 4;
  constexpr int MI = HALF ? 1 : 2, NDMA = HALF ? 3 : 4, NS = HALF ? NS_HALF : 4;
  constexpr u32 XB = HALF ? BLK / 2 : BLK, SB = XB + BLK;
  static_assert(NS * SB <= LDS_CTL, "stages fit");

  u32 vao[2], vbo[2], vaw[2], vbw[2];
#pragma unroll
  for (int q = 0; q < 2; ++q) {
    const u32 ra = wid * 8 * MI + q * 8 + (lane >> 3), ca = (lane & 7) ^ ((ra >> 1) & 7);
    const u32 kb = wid * 8 + q * 4 + (lane >> 4), cbk = (lane & 15) ^ (((kb & 3) << 2) | ((kb >> 2) & 3));
    vao[q] = ra * 1024 + ca * 16 - q * 1024;  vaw[q] = ra * 128 + ca * 16 - q * 1024;
    vbo[q] = kb * 8192 + cbk * 16 - q * 1024; vbw[q] = kb * 256 + cbk * 16 - q * 1024;
  }
  const u32 lds0 = (u32)(size_t)lds;
  const u32 wm = wid & 1, wn = wid >> 1, h = lane >> 5;
  u32 xo[4], wo[2];
  {
    const u32 r = wm * 32 * MI + (lane & 31);
#pragma unroll
    for (int ks = 0; ks < 4; ++ks) xo[ks] = r * 128 + (((2 * ks + h) ^ ((r >> 1) & 7)) * 16);
    const u32 bk = (lane >> 4) & 1, q = (lane & 15) >> 2, pp = lane & 3;
#pragma unroll
    for (int t = 0; t < 2; ++t)
      wo[t] = XB + 256 * (8 * h + 4 * t + q) + 16 * ((4 * wn + 2 * bk + (pp >> 1)) ^ ((q << 2) | (2 * h + t))) + 8 * (pp & 1);
  }
  LAS u32* lbad = (LAS u32*)(lds + LDS_CTL);
  u32 ri, ci;
  tile_rc(m, ri, ci);
  const u32 row0 = srow(rank, u) * 1024 + ri * 128 + (HALF ? hsel * 64 : 0), col0 = scol(rank, v) * 1024 + ci * 128;
  u32 islot = 0, rslot = 0;
  auto next_slot = [&](u32 o) -> u32 { return o + SB == NS * SB ? 0 : o + SB; };
  auto issue_own = [&](int t) {
    const u32 dst = lds0 + islot, sa = row0 * 1024 + t * 128, sw = t * 64 * 8192 + col0 * 2;
    islot = next_slot(islot);
    if (HALF) dma3(dst + wid * 1024, dst + XB + wid * 2048, rx, vao[0], sa, rw, vbo[0], vbo[1], sw);
    else dma4(dst + wid * 2048, rx, vao[0], vao[1], sa, rw, vbo[0], vbo[1], sw);
  };
  u32 taba, tabw;
  {
    const bool cr = lane < T_INTRA;
    const u32 c = cr ? lane - T_CROSS : lane - T_INTRA, ktl = cr ? c >> 2 : c / 3;
    const u32 src = rank ^ (cr ? 4 + (c & 3) : 1 + c - ktl * 3), kts = cr ? KC : KI;
    const u32 base = IN_OFF + src * IN_SRC + ktl * 4 * BLK;
    taba = base + (ri * kts * 4 + u) * BLK + (HALF ? hsel * 8192 : 0);
    tabw = base + (ci * kts * 4 + 2 + v) * BLK;
  }
  auto issue = [&](int t) {
    const u32 dst = lds0 + islot, sa = __builtin_amdgcn_readlane(taba, t), sw = __builtin_amdgcn_readlane(tabw, t);
    islot = next_slot(islot);
    if (HALF) dma3(dst + wid * 1024, dst + XB + wid * 2048, rs, vaw[0], sa, rs, vbw[0], vbw[1], sw);
    else dma4(dst + wid * 2048, rs, vaw[0], vaw[1], sa, rs, vbw[0], vbw[1], sw);
  };
  const u32 fop = (lane >> 1) & 1, flev = fop ? ci : ri, fuv = fop ? v : u, fq = lane >> 2;
  const u32 fh = (HALF && !fop) ? hsel : (lane & 1);
  const u32 fck = min(fq >> 2, KC - 1), fik = min(fq / 3, KI - 1);
  const char* fcross = ws + FI_OFF + ((rank ^ (4 + (fq & 3))) * 320 + ((flev * KC + fck) * 4 + fop * 2 + fuv) * 2 + fh) * FIS;
  const char* fintra = ws + FI_OFF + ((rank ^ (1 + fq % 3)) * 320 + ((flev * KI + fik) * 4 + fop * 2 + fuv) * 2 + fh) * FIS;

  while (__builtin_amdgcn_s_memrealtime() - tstart < (u64)max(ri, ci) * STAGGER) __builtin_amdgcn_s_sleep(16);
  if (first && wid == 0) __builtin_amdgcn_fence(__ATOMIC_ACQUIRE, "");
  f32x16 acc[MI] = {};
  struct Frag { bf16x8 w[4], x[MI][4]; };
  auto read = [&](Frag& f) {
    const LAS char* st = lds + rslot;
    rslot = next_slot(rslot);
#pragma unroll
    for (int ks = 0; ks < 4; ++ks) {
      const bf16x4 w0 = __builtin_amdgcn_ds_read_tr16_b64_v4bf16((LAS bf16x4*)(st + wo[0] + ks * 4096));
      const bf16x4 w1 = __builtin_amdgcn_ds_read_tr16_b64_v4bf16((LAS bf16x4*)(st + wo[1] + ks * 4096));
      f.w[ks] = __builtin_shufflevector(w0, w1, 0, 1, 2, 3, 4, 5, 6, 7);
#pragma unroll
      for (int mi = 0; mi < MI; ++mi) f.x[mi][ks] = *(const LAS bf16x8*)(st + xo[ks] + mi * 4096);
    }
  };
  auto mma = [&](const Frag& f) {
#pragma unroll
    for (int ks = 0; ks < 4; ++ks)
#pragma unroll
      for (int mi = 0; mi < MI; ++mi) acc[mi] = __builtin_amdgcn_mfma_f32_32x32x16_bf16(f.w[ks], f.x[mi][ks], acc[mi], 0, 0, 0);
  };
  int next_wait = T_CROSS;
  u64 need = 0xFFFF, have = 0;
  auto waitpoint = [&](int tn) {
    if (tn == T_INTRA) { need = 0xFFF; have = 0; }
    if (wid == 0 && (have & need) != need) {
      bool bad = false;
      have = wave_wait2<SL_GEMM>(tn < T_INTRA ? fcross : fintra, need, seq, tmo, bad);
      if (bad) *lbad = 1;
    }
    if (tn < T_INTRA) { next_wait += 4; need = (need << 16) | 0xFFFF; }
    else { next_wait += 3; need = (need << 12) | 0xFFF; }
  };
  auto step = [&](auto rc, int tn, const Frag& fc, Frag& fn) {
    constexpr int R = decltype(rc)::value;
    if (tn >= 0 && __builtin_expect(tn == next_wait, 0)) waitpoint(tn);
    if constexpr (R >= 0) {
      WAIT_VM(R * NDMA);
      __builtin_amdgcn_s_waitcnt(0xC07F);
      CBAR();
      __builtin_amdgcn_s_barrier();
      CBAR();
      read(fn);
    }
    if (tn >= 0) issue(tn);
    mma(fc);
  };
  Frag fa, fb;
  auto tail = [&](auto nc) {
    constexpr int N = decltype(nc)::value;
    static_for<0, N / 2>([&](auto ic) {
      constexpr int i = decltype(ic)::value;
      step(std::integral_constant<int, N - 2 - 2 * i>{}, -1, fa, fb);
      step(std::integral_constant<int, N - 3 - 2 * i>{}, -1, fb, fa);
    });
  };
  static_assert(NS % 2 == 0 && NS >= T_SPLIT && T_SPLIT + 1 == T_CROSS, "run structure");
  for (int t = 0; t < T_SPLIT; ++t) issue_own(t);
  WAIT_VM((T_SPLIT - 1) * NDMA);
  __builtin_amdgcn_s_barrier();
  CBAR();
  read(fa);
  tail(std::integral_constant<int, T_SPLIT>{});
  islot = rslot = 0;
  __builtin_amdgcn_s_waitcnt(0xC07F);
  __builtin_amdgcn_s_barrier();
  issue_own(T_SPLIT);
  for (int t = T_CROSS; t < T_SPLIT + NS; ++t) {
    if (t == next_wait) { waitpoint(t); __builtin_amdgcn_s_barrier(); }
    issue(t);
  }
  WAIT_VM((NS - 1) * NDMA);
  __builtin_amdgcn_s_barrier();
  CBAR();
  read(fa);
  for (int tn = T_SPLIT + NS; tn < NT; tn += 2) {
    step(std::integral_constant<int, NS - 2>{}, tn, fa, fb);
    step(std::integral_constant<int, NS - 2>{}, tn + 1, fb, fa);
  }
  tail(std::integral_constant<int, NS>{});
  const u32 tmv = __hip_atomic_load((const GAS u32*)tmo, __ATOMIC_RELAXED, __HIP_MEMORY_SCOPE_AGENT);
  __builtin_amdgcn_s_waitcnt(0xC07F);
  __builtin_amdgcn_s_barrier();
  CBAR();
  constexpr u32 EROW = 528;
#pragma unroll
  for (int mi = 0; mi < MI; ++mi) {
    const u32 mr = wm * 32 * MI + mi * 32 + (lane & 31);
#pragma unroll
    for (int gq = 0; gq < 4; ++gq) {
      const f32x4 o = {acc[mi][4 * gq], acc[mi][4 * gq + 1], acc[mi][4 * gq + 2], acc[mi][4 * gq + 3]};
      *(LAS f32x4*)(lds + mr * EROW + (wn * 32 + 8 * gq + 4 * h) * 4) = o;
    }
  }
  __syncthreads();
  const u32 bad = (*lbad || tmv == seq) ? NAN2 : 0u;
  const u32 keep = rank >> 2;
  const bool do_exp = !HALF || hsel != keep, do_keep = !HALF || hsel == keep;
  char* wsq = p.ws[4 | ((rank & 3) ^ sb)];
  if (do_exp) {
    const u32 e16 = P16_OFF + (sb * 64 + m) * 16384 + wid * 1024;
#pragma unroll
    for (int part = 0; part < 2; ++part) {
      const LAS char* se = lds + ((HALF ? 0 : (keep ^ 1) * 64) + part * 32 + (tid >> 4)) * EROW + (tid & 15) * 32;
      const f32x4 c0 = *(const LAS f32x4*)se, c1 = *(const LAS f32x4*)(se + 16);
      const bf16x8 o = {(__bf16)c0[0], (__bf16)c0[1], (__bf16)c0[2], (__bf16)c0[3],
                        (__bf16)c1[0], (__bf16)c1[1], (__bf16)c1[2], (__bf16)c1[3]};
      st128(__builtin_bit_cast(u32x4, o) | bad, rs, lane * 16, e16 + part * 8192);
    }
  }
  if (HALF && do_keep && m < MDIR) {
    f32x4 kc[2][2];
#pragma unroll
    for (int part = 0; part < 2; ++part) {
      const LAS char* sk = lds + (part * 32 + (tid >> 4)) * EROW + (tid & 15) * 32;
      kc[part][0] = *(const LAS f32x4*)sk; kc[part][1] = *(const LAS f32x4*)(sk + 16);
    }
    bool badw = bad != 0;
    wave_wait<2>(ws + FPR_OFF + (sb * 64 + m) * FOS, 1, seq, tmo, badw);
    u32x4 r[2];
#pragma unroll
    for (int part = 0; part < 2; ++part) {
      r[part] = __builtin_amdgcn_raw_buffer_load_b128(mk_rsrc(wsq, DATA_BYTES), lane * 16, P16_OFF + ((rank & 3) * 64 + m) * 16384 + wid * 1024 + part * 8192, AUX_SYS);
      WAIT_VM(0);
    }
    const auto ro = mk_rsrc(p.out[0], OUT_BYTES);
#pragma unroll
    for (int part = 0; part < 2; ++part) {
      float y[8];
#pragma unroll
      for (int e = 0; e < 4; ++e) {
        y[2 * e] = silu(kc[part][e >> 1][(2 * e) & 3] + __uint_as_float(r[part][e] << 16));
        y[2 * e + 1] = silu(kc[part][e >> 1][(2 * e + 1) & 3] + __uint_as_float(r[part][e] & 0xFFFF0000u));
      }
      const bf16x8 o = {(__bf16)y[0], (__bf16)y[1], (__bf16)y[2], (__bf16)y[3],
                        (__bf16)y[4], (__bf16)y[5], (__bf16)y[6], (__bf16)y[7]};
      st128(__builtin_bit_cast(u32x4, o) | (badw ? NAN2 : 0u), ro, (lane >> 4) * 8192 + (lane & 15) * 16,
            (row0 + part * 32 + wid * 4) * 8192 + col0 * 2);
    }
    WAIT_VM(0);
    __syncthreads();
    if (tid >= 1 && tid < NR) st_sys(p.ws[tid] + FF_OFF + (rank * 256 + m * 4 + sb) * FOS, seq);
    return;
  }
  if (do_keep) {
    const u32 e32 = P32_OFF + (sb * 64 + m) * 32768 + wid * 2048;
#pragma unroll
    for (int part = 0; part < 2; ++part) {
      const LAS char* sk = lds + ((HALF ? 0 : keep * 64) + part * 32 + (tid >> 4)) * EROW + (tid & 15) * 32;
      const u32x4 c0 = *(const LAS u32x4*)sk, c1 = *(const LAS u32x4*)(sk + 16);
      st128(c0 | bad, rs, lane * 16, e32 + part * 16384);
      st128(c1 | bad, rs, lane * 16, e32 + part * 16384 + 1024);
    }
  }
  WAIT_VM(0);
  __syncthreads();
  if (tid == 0 && do_exp) st_sys(wsq + FPR_OFF + ((rank & 3) * 64 + m) * FOS, seq);
  if (tid == 64 && do_keep) st_sys(ws + FPL_OFF + (sb * 64 + m) * FOS, seq);
}

constexpr u32 LK0 = NPUSH / 8, NUNIT = 64 - 2 * LK0;
static_assert(NPUSH % 8 == 0 && LK0 < 8 * KC, "staging units");
__device__ __forceinline__ void stage_unit(const Params& p, u32 u, u32 seq) {
  const int tid = threadIdx.x, lane = tid & 63;
  const u32 wid = rfl(tid >> 6);
  constexpr u32 NPAIR = 2 * (8 * KC - LK0);
  const bool cr = u < NPAIR && (u & 1);
  const u32 lk = u < NPAIR ? LK0 + (u >> 1) : 8 * KC + u - NPAIR;
  const u32 lev = cr ? lk / KC : lk / KI, kt = cr ? KI + lk - lev * KC : lk - lev * KI, op = wid >> 2, sup = wid & 3;
  const auto r = mk_rsrc(op ? p.w : p.x, X_BYTES);
  const auto rs = mk_rsrc(p.ws[0], DATA_BYTES);
  const u32 vo = op ? (lane >> 4) * 8192 + (lane & 15) * 16 : (lane >> 3) * 1024 + (lane & 7) * 16;
  const u32 so = op ? kt * 64 * 8192 + (sup * 1024 + lev * 128) * 2 : (sup * 1024 + lev * 128) * 1024 + kt * 128;
  u32x4 v[16];
#pragma unroll
  for (int i = 0; i < 16; ++i) v[i] = __builtin_amdgcn_raw_buffer_load_b128(r, vo, so + i * (op ? 4 * 8192 : 8 * 1024), 0);
#pragma unroll
  for (int i = 0; i < 16; ++i) st128(v[i], rs, lane * 16, EX_OFF + ((lev * 8 + kt) * 8 + wid) * BLK + i * 1024);
  WAIT_VM(0);
  __syncthreads();
  if (tid >= 1 && tid < NR && (tid >= 4) == cr) st_sys(p.ws[tid] + SG_OFF + (p.rank * 40 + lk) * FOS, seq);
}

__device__ __forceinline__ void gemm_role(const Params& p, u32 g, u32 k, u32 seq, LAS char* lds) {
  const u64 tstart = __builtin_amdgcn_s_memrealtime();
  static_assert(GEMM_WGS >= 16 && GEMM_WGS <= 24, "unit schedule below");
  if (g == 0 && k >= 16)
    for (u32 r = 0; r < HELP_ROUNDS; ++r) { stage_unit(p, r * 16 + k - 8, seq); }
  const u32 keep = p.rank >> 2;
  if (k < 16) gemm_tile<true>(p, g >= 4 ? (p.rank ^ g) & 3 : g, k, g >= 4 ? keep : keep ^ 1, seq, tstart, true, lds);
  const u32 arm = g & 1;
  for (u32 n = k < 16 ? GEMM_WGS - 16 + k : k - 16; n < 24; n += GEMM_WGS) {
    u32 lev = 4, idx = n;
    for (;;) {
      const u32 cnt = lev + ((lev & 1) == arm);
      if (idx < cnt) break;
      idx -= cnt; ++lev;
    }
    gemm_tile<false>(p, g >> 1, lev * lev + (idx == lev ? 2 * lev : arm * lev + idx), 0, seq, tstart, k >= 16, lds);
  }
}

__global__ void __launch_bounds__(THREADS, 2) fused_gemm_allreduce_silu(Params p) {
  extern __shared__ __attribute__((aligned(16))) char smem[];
  const int lane = threadIdx.x & 63;
  const u32 wid = rfl(threadIdx.x >> 6);
  LAS u32* ctl = (LAS u32*)smem;
  LAS u32* lctl = (LAS u32*)(smem + LDS_CTL);
  u32 slot = (blockIdx.x & ~7u) | ((u32)__builtin_amdgcn_s_getreg((3 << 11) | 20) & 7);
  auto is_push = [&]() { return slot < 8 * ENG_WGS && (slot & 7) != 0 && wid < PUSH_WAVES; };
  if (__builtin_expect(is_push(), 1)) {
    if (push_role<false>(p, slot, wid, lane, lctl)) return;
  } else {
    if (wid == 7) {
      if (lane < 32) ctl[lane] = 0;
      if (lane == 0) {
        GAS u32* ctr = (GAS u32*)(p.ws[0] + CTR_OFF) + blockIdx.x;
        GAS u32* clm = (GAS u32*)(p.ws[0] + CLM_OFF);
        const u32 c = __hip_atomic_load(ctr, __ATOMIC_RELAXED, __HIP_MEMORY_SCOPE_AGENT);
        u32 sl = slot;
#pragma unroll 1
        for (int n = 0; n < 4 * GRID; ++n) {
          if (__hip_atomic_fetch_add(clm + sl * 16, 1u, __ATOMIC_RELAXED, __HIP_MEMORY_SCOPE_AGENT) == c) break;
          __hip_atomic_fetch_add(clm + sl * 16, ~0u, __ATOMIC_RELAXED, __HIP_MEMORY_SCOPE_AGENT);
          sl = (sl + 1) & (GRID - 1);
        }
        lctl[0] = 0; lctl[1] = sl; lctl[2] = c + 1;
      }
    }
    asm volatile("s_waitcnt lgkmcnt(0)" ::: "memory");
    __builtin_amdgcn_s_barrier();
    slot = rfl(LD_LDS(lctl + 1));
    if (threadIdx.x == THREADS - 1)
      __hip_atomic_store((GAS u32*)(p.ws[0] + CTR_OFF) + blockIdx.x, LD_LDS(lctl + 2), __ATOMIC_RELAXED, __HIP_MEMORY_SCOPE_AGENT);
    WAIT_VM(0);
  }
  if (__builtin_expect(is_push(), 0)) {
    push_role<true>(p, slot, wid, lane, lctl);
    return;
  }
  const u32 seq = rfl(LD_LDS(lctl + 2));
  const u32 g = slot & 7, s = slot >> 3;
  const u32 peer = ((u32)p.rank ^ g) & (NR - 1);
  if (s < ENG_WGS) {
    if (g == 0) {
      for (u32 r = 0; r < HELP_ROUNDS; ++r) stage_unit(p, r * 16 + s, seq);
      for (u32 u = HELP_ROUNDS * 16 + s; u < NUNIT; u += 8) stage_unit(p, u, seq);
      return;
    }
    if (wid < PUSH_WAVES + PULL_WAVES) {
      pull_role(p, peer, (wid - PUSH_WAVES) * ENG_WGS + s, lane, seq, ctl);
    } else if (wid < PUSH_WAVES + PULL_WAVES + FIN_WAVES && g >= 4) {
      fin_role(p, peer, s, wid - PUSH_WAVES - PULL_WAVES, lane, seq, ctl);
    }
  } else {
    gemm_role(p, g, s - ENG_WGS, seq, (LAS char*)smem);
  }
}

}

extern "C" void kernel_launch(int rank, int nranks, ncclComm_t comm,
                              hipStream_t stream,
                              void* const* d_in, const int* in_sizes, int n_in,
                              void* d_out, int out_size,
                              const int* in_dtypes, int out_dtype,
                              void* const* d_out_all,
                              void* const* d_ws_all, size_t ws_bytes) {
  if (nranks != NR || (unsigned)rank >= (unsigned)NR || n_in < 2 || ws_bytes < WS_USED ||
      in_sizes[0] != 4096 * 512 || in_sizes[1] != 512 * 4096 || out_size != 4096 * 4096) {
    fprintf(stderr, "kernel_launch: unexpected problem shape\n");
    return;
  }
  HIP_CHECK(hipFuncSetAttribute((const void*)fused_gemm_allreduce_silu,
                                hipFuncAttributeMaxDynamicSharedMemorySize, LDS_BYTES));
  Params p;
  p.x = (const char*)d_in[0];
  p.w = (const char*)d_in[1];
  const int phys[NR] = {0, 2, 3, 6, 1, 4, 5, 7};
  p.rank = 0; p.pad = 0;
  for (int i = 0; i < NR; ++i) if (phys[i] == rank) p.rank = i;
  for (int i = 0; i < NR; ++i) { p.ws[i] = (char*)d_ws_all[phys[p.rank ^ i]]; p.out[i] = (char*)d_out_all[phys[p.rank ^ i]]; }
  p.pad2 = 0;
  for (u32 g = 0; g < NR; ++g)
    for (u32 job = 0; job < NPUSH; ++job) {
      const u32 peer = p.rank ^ g, kts = g >= 4 ? KC : KI, lk = job >> 3, op = (job >> 2) & 1, uv = (job >> 1) & 1, half = job & 1;
      const u32 lev = lk / kts, kt = (g >= 4 ? KI : 0) + lk % kts;
      p.psrc[g][job & 15][job >> 4] = op ? (kt * 64 + half * 32) * 8192 + (scol(peer, uv) * 1024 + lev * 128) * 2
                                         : (srow(peer, uv) * 1024 + lev * 128 + half * 64) * 1024 + kt * 128;
    }
  hipLaunchKernelGGL(fused_gemm_allreduce_silu, dim3(GRID), dim3(THREADS), LDS_BYTES, stream, p);
  HIP_CHECK(hipGetLastError());
}
```

```cpp
#include <hip/hip_runtime.h>
#include <rccl/rccl.h>
#include <cstdio>
#include <type_traits>

#define HIP_CHECK(expr) do { hipError_t _e = (expr); if (_e != hipSuccess) { \
    fprintf(stderr, "HIP error %s at %s:%d\n", hipGetErrorString(_e), __FILE__, __LINE__); \
    return; } } while (0)

namespace {

typedef unsigned int u32;
typedef unsigned long long u64;
typedef u32 u32x4 __attribute__((ext_vector_type(4)));
typedef float f32x4 __attribute__((ext_vector_type(4)));
typedef __bf16 bf16x4 __attribute__((ext_vector_type(4)));
typedef __bf16 bf16x8 __attribute__((ext_vector_type(8)));
typedef float f32x16 __attribute__((ext_vector_type(16)));
#define LAS __attribute__((address_space(3)))
#define GAS __attribute__((address_space(1)))

constexpr int NR = 8;
constexpr u32 X_BYTES = 4096u * 512u * 2u;
constexpr u32 W_BYTES = 512u * 4096u * 2u;
constexpr u32 OUT_BYTES = 4096u * 4096u * 2u;
constexpr u32 FS = 128;

constexpr u32 BLK = 16384;
constexpr u32 IN_OFF = 0;
constexpr u32 IN_SRC = 128 * BLK;
constexpr u32 P16_OFF = IN_OFF + NR * IN_SRC;
constexpr u32 PIN_OFF = P16_OFF + 256 * 16384;
constexpr u32 P32_OFF = PIN_OFF + 256 * 16384;
constexpr u32 EX_OFF = P32_OFF + 256 * 32768;
constexpr u32 DATA_BYTES = EX_OFF + 512 * BLK;
constexpr u32 FI_OFF = DATA_BYTES;
constexpr u32 FPR_OFF = FI_OFF + NR * 256 * FS;
constexpr u32 FPL_OFF = FPR_OFF + 256 * FS;
constexpr u32 FPI_OFF = FPL_OFF + 256 * FS;
constexpr u32 FF_OFF = FPI_OFF + 512 * FS;
constexpr u32 SG_OFF = FF_OFF + NR * 256 * FS;
constexpr u32 GO_OFF = SG_OFF + NR * 64 * FS;
constexpr u32 RF_OFF = GO_OFF + NR * 16 * FS;
constexpr u32 CTR_OFF = RF_OFF + 64 * FS;
constexpr u32 CLM_OFF = CTR_OFF + 256 * 4;
constexpr u32 TMO_OFF = CLM_OFF + 256 * 64;
constexpr u32 TICK_OFF = TMO_OFF + 128;
constexpr u32 SEG_OFF = TICK_OFF + 32 * 128;
constexpr u32 WS_USED = SEG_OFF + 8 * 128;

constexpr u64 DEADLINE_TICKS = 80000000ull;

constexpr int ENG_WGS = 8, FIN_WGS = 4, GEMM_WGS = 20;
constexpr int PUSH_WAVES = 2, PULL_WAVES = 2;
constexpr int PUSH_D = 4, PUSH_ROUNDS = 4;
constexpr u32 NPUSH = PUSH_ROUNDS * ENG_WGS * PUSH_WAVES;
constexpr u32 PL = 4;
constexpr int GO_AT = 6;
constexpr int NS_HALF = 6;
constexpr u32 NHALF = 64;
constexpr u64 STAGGER = 190;
constexpr u32 FLEV = 3, FLAG_ = 2, TOPK = 48, SWK = 1;
constexpr int SL_PULL = 2, SL_FIN = 4, SL_GEMM = 8;
constexpr u32 NAN2 = 0x7FC07FC0u;
constexpr int GRID = 8 * (ENG_WGS + FIN_WGS + GEMM_WGS);
static_assert(GRID == 256, "slots wrap at a power of two");
constexpr int THREADS = 512;
constexpr u32 LDS_CTL = 144 * 1024;
constexpr u32 LDS_BYTES = LDS_CTL + 64;
constexpr int AUX_SYS = 17;

struct Params {
  const char* x;
  const char* w;
  int rank;
  int pad;
  char* ws[NR];
  char* out[NR];
  u32 psrc[NR][16];
};

__device__ __forceinline__ u32 rfl(u32 v) { return __builtin_amdgcn_readfirstlane(v); }
__device__ __forceinline__ __amdgpu_buffer_rsrc_t mk_rsrc(const void* p, u32 n) {
  const u64 a = (u64)p;
  const u64 u = ((u64)rfl((u32)(a >> 32)) << 32) | rfl((u32)a);
  return __builtin_amdgcn_make_buffer_rsrc((void*)u, 0, n, 0x00020000);
}
__device__ __forceinline__ u32 ld_sys(const char* p) {
  return __hip_atomic_load((const GAS u32*)p, __ATOMIC_RELAXED, __HIP_MEMORY_SCOPE_SYSTEM);
}
__device__ __forceinline__ void st_sys(char* p, u32 v) {
  __hip_atomic_store((GAS u32*)p, v, __ATOMIC_RELAXED, __HIP_MEMORY_SCOPE_SYSTEM);
}
#define WAIT_VM(n) asm volatile("s_waitcnt vmcnt(%0)" ::"n"(n) : "memory")
#define CBAR() asm volatile("" ::: "memory")
#define LD_LDS(q) __hip_atomic_load(q, __ATOMIC_RELAXED, __HIP_MEMORY_SCOPE_WORKGROUP)
#define ST_LDS(q, v) __hip_atomic_store(q, v, __ATOMIC_RELAXED, __HIP_MEMORY_SCOPE_WORKGROUP)

__device__ __forceinline__ void st128(u32x4 v, __amdgpu_buffer_rsrc_t r, u32 voff, u32 soff) {
  __builtin_amdgcn_raw_buffer_store_b128(v, r, voff, soff, AUX_SYS);
  asm volatile("s_nop 1" ::"v"(v) : "memory");
}

template <int SLEEP>
__device__ __forceinline__ u64 wave_wait(const char* flag, u64 need, u32 seq, char* tmo, bool& bad) {
  const u64 t0 = __builtin_amdgcn_s_memrealtime();
  for (u32 it = 1;; ++it) {
    const u64 ok = __ballot(ld_sys(flag) >= seq);
    if ((ok & need) == need) return ok;
    if ((it & 15) == 0 || bad) {
      const u32 t = __hip_atomic_load((const GAS u32*)tmo, __ATOMIC_RELAXED, __HIP_MEMORY_SCOPE_AGENT);
      if (rfl(t) == seq || __builtin_amdgcn_s_memrealtime() - t0 > DEADLINE_TICKS) {
        __hip_atomic_store((GAS u32*)tmo, seq, __ATOMIC_RELAXED, __HIP_MEMORY_SCOPE_AGENT);
        bad = true;
        return ~0ull;
      }
    }
    __builtin_amdgcn_s_sleep(SLEEP);
  }
}

template <int SLEEP>
__device__ __forceinline__ u64 wave_wait2(const char* flag, u64 need, u32 seq, char* tmo, bool& bad) {
  const u64 t0 = __builtin_amdgcn_s_memrealtime();
  const bool needed = (need >> (threadIdx.x & 63)) & 1;
  bool ok = false;
  for (u32 it = 1;; ++it) {
    if (!ok && (it == 1 || needed)) ok = ld_sys(flag) >= seq;
    const u64 have = __ballot(ok);
    if ((have & need) == need) return have;
    if ((it & 15) == 0) {
      const u32 t = __hip_atomic_load((const GAS u32*)tmo, __ATOMIC_RELAXED, __HIP_MEMORY_SCOPE_AGENT);
      if (rfl(t) == seq || __builtin_amdgcn_s_memrealtime() - t0 > DEADLINE_TICKS) {
        __hip_atomic_store((GAS u32*)tmo, seq, __ATOMIC_RELAXED, __HIP_MEMORY_SCOPE_AGENT);
        bad = true;
        return ~0ull;
      }
    }
    __builtin_amdgcn_s_sleep(SLEEP);
  }
}

__device__ __forceinline__ void dma4(u32 dst, __amdgpu_buffer_rsrc_t ra, u32 va0, u32 va1, u32 sa,
                                     __amdgpu_buffer_rsrc_t rb, u32 vb0, u32 vb1, u32 sb) {
  u32 keep;
  dst = rfl(dst); sa = rfl(sa); sb = rfl(sb);
  asm volatile(
      "s_mov_b32 %0, m0\n\t"
      "s_nop 4\n\t"
      "s_mov_b32 m0, %1\n\t"
      "s_nop 0\n\t"
      "buffer_load_dwordx4 %3, %5, %6 offen lds\n\t"
      "buffer_load_dwordx4 %4, %5, %6 offen offset:1024 lds\n\t"
      "s_mov_b32 m0, %2\n\t"
      "s_nop 0\n\t"
      "buffer_load_dwordx4 %7, %9, %10 offen lds\n\t"
      "buffer_load_dwordx4 %8, %9, %10 offen offset:1024 lds\n\t"
      "s_mov_b32 m0, %0"
      : "=&s"(keep)
      : "s"(dst), "s"(dst + BLK), "v"(va0), "v"(va1), "s"(ra), "s"(sa), "v"(vb0), "v"(vb1), "s"(rb), "s"(sb)
      : "memory");
}

__device__ __forceinline__ void dma3(u32 dsta, u32 dstb, __amdgpu_buffer_rsrc_t ra, u32 va0, u32 sa,
                                     __amdgpu_buffer_rsrc_t rb, u32 vb0, u32 vb1, u32 sb) {
  u32 keep;
  dsta = rfl(dsta); dstb = rfl(dstb); sa = rfl(sa); sb = rfl(sb);
  asm volatile(
      "s_mov_b32 %0, m0\n\t"
      "s_nop 4\n\t"
      "s_mov_b32 m0, %1\n\t"
      "s_nop 0\n\t"
      "buffer_load_dwordx4 %3, %4, %5 offen lds\n\t"
      "s_mov_b32 m0, %2\n\t"
      "s_nop 0\n\t"
      "buffer_load_dwordx4 %6, %8, %9 offen lds\n\t"
      "buffer_load_dwordx4 %7, %8, %9 offen offset:1024 lds\n\t"
      "s_mov_b32 m0, %0"
      : "=&s"(keep)
      : "s"(dsta), "s"(dstb), "v"(va0), "s"(ra), "s"(sa), "v"(vb0), "v"(vb1), "s"(rb), "s"(sb)
      : "memory");
}

template <int I, int N, class F>
__device__ __forceinline__ void static_for(F&& f) {
  if constexpr (I < N) { f(std::integral_constant<int, I>{}); static_for<I + 1, N>(f); }
}

__device__ __forceinline__ void tile_rc(u32 m, u32& ri, u32& ci) {
  u32 l = 0;
#pragma unroll
  for (u32 t = 1; t < 16; ++t) l += m >= t * t;
  const u32 idx = m - l * l;
  ri = idx < l ? l : idx - l;
  ci = idx < l ? idx : l;
}
__device__ __forceinline__ u32 fin_base(u32 r) { return (((r >> 1) & 1) * 2048 + (r >> 2) * 64) * 8192 + (r & 1) * 4096; }

template <bool claimed>
__device__ __forceinline__ bool push_role(const Params& p, u32& slot, u32 wid, int lane, LAS u32* lctl) {
  constexpr int D = PUSH_D, JI = 8;
  constexpr u32 NW = ENG_WGS * PUSH_WAVES;
  static_assert(PUSH_ROUNDS >= 3 && NW == 16 && X_BYTES == W_BYTES && GO_AT <= JI, "push schedule");
  const u32 g = slot & 7, w = wid * ENG_WGS + (slot >> 3), op = (slot >> 4) & 1;
  u64 bs = (u64)(&p.x)[op], bw = (u64)p.ws[g];
  u32 src = p.psrc[g][w];
  u32 rank = p.rank;
  asm volatile("" : "+s"(bs), "+s"(bw), "+s"(src), "+s"(rank));
  const auto rs = mk_rsrc((const void*)bs, X_BYTES);
  const u32 vs = op ? (lane >> 4) * 8192 + (lane & 15) * 16 : (lane >> 3) * 1024 + (lane & 7) * 16;
  const u32 str = op ? 4 * 8192 : 8 * 1024;
  auto ld = [&](int r, int i) -> u32x4 { return __builtin_amdgcn_raw_buffer_load_b128(rs, vs, src + r * (op ? 256 : 128 * 1024) + i * str, 0); };
  u32x4 v[D], v2[JI - D];
#pragma unroll
  for (int i = 0; i < D; ++i) { v[i] = ld(0, i); CBAR(); }
#pragma unroll
  for (int i = D; i < JI; ++i) { v2[i - D] = ld(0, i); CBAR(); }
  char* wsp = (char*)bw;
  const auto rd = mk_rsrc(wsp, DATA_BYTES);
  const u32 dst = IN_OFF + rank * IN_SRC + w * 8192, flag = FI_OFF + (rank * 256 + w) * FS;
  u32 seq = 0;
  auto round = [&](auto rc) {
    constexpr int R = decltype(rc)::value;
    constexpr bool FIRST = R == 0, LAST = R == PUSH_ROUNDS - 1;
#pragma unroll
    for (int i = 0; i < JI; ++i) {
      if (!FIRST && i == D - 1) {
        WAIT_VM(2 * D - 1);
        if (R == 1) {
          if (!claimed) __builtin_amdgcn_s_barrier();
          CBAR();
          seq = rfl(LD_LDS(lctl + 2));
        }
        if (lane == 0) st_sys(wsp + flag + (R - 1) * NW * FS, seq);
        CBAR();
      }
      if (LAST && i == JI - GO_AT) {
        if (lane == 0) st_sys(wsp + GO_OFF + (rank * 16 + w) * FS, seq);
        if (lane == 0) ST_LDS(lctl - LDS_CTL / 4 + 24 + wid, 1u);
        CBAR();
      }
      st128(FIRST && i >= D ? v2[i >= D ? i - D : 0] : v[i % D], rd, lane * 16, dst + R * NW * 8192 + i * 1024);
      if (i + D < JI) { if (!FIRST) v[i % D] = ld(R, i + D); }
      else if (!LAST) v[i % D] = ld(LAST ? R : R + 1, i + D - JI);
      CBAR();
    }
  };
  static_assert(JI - D >= D - 1, "the count above also holds in the last round");
  round(std::integral_constant<int, 0>{});
  round(std::integral_constant<int, 1>{});
  const u32 slot2 = rfl(LD_LDS(lctl + 1));
  if (__builtin_expect(slot2 != slot, 0)) { WAIT_VM(0); slot = slot2; return false; }
  static_for<2, PUSH_ROUNDS>(round);
  WAIT_VM(0);
  if (lane == 0) st_sys(wsp + flag + (PUSH_ROUNDS - 1) * NW * FS, seq);
  return true;
}

__device__ __forceinline__ float silu(float a) {
  const float c = fminf(fmaxf(a, -60.f), 60.f);
  return a * __builtin_amdgcn_rcpf(1.f + __expf(-c));
}

constexpr u32 MB_RING = 0, MB_END = 16, MB_PUSHED = 24;

constexpr u32 NE = 4 * FLEV * FLEV, NPOOL = (1024 - NE) / 2;
__device__ __forceinline__ u32 pool_job(u32 pos) {
  return pos < NPOOL - 2 * TOPK ? pos : pos >= NPOOL - TOPK ? pos - TOPK : pos + TOPK;
}

__device__ __forceinline__ void pull_role(const Params& p, u32 peer, u32 w, int lane, u32 seq, LAS u32* ctl) {
  constexpr u32 NW = ENG_WGS * PULL_WAVES, NONE = ~0u, REL = 1u << 31;
  const u32 rank = p.rank, g = rank ^ peer;
  const bool pl = g == PL;
  const u32 NI = pl ? 512 : 256 - NPUSH, NJ = pl ? NI + NE : NI + 1024;
  char* ws = p.ws[0];
  char* tmo = ws + TMO_OFF;
  const u64 bf = (u64)p.out[g], bp = (u64)p.ws[g], bo = (u64)p.out[0];
  const u32 vwide = (lane >> 4) * 8192 + (lane & 15) * 16;
  const auto rtick = mk_rsrc(ws + TICK_OFF + (seq & 1) * 2048, 2048);
  if (w == 0 && (lane == 0 || (pl && lane < 3))) st_sys(ws + TICK_OFF + ((seq & 1) ^ 1) * 2048 + (lane ? 7 + lane : g) * 128, 0);
  const u32 bi = (rank >> 1) & 1, bj = rank & 1;
  const u32 isrc = pl ? P16_OFF : EX_OFF + (NPUSH / 4 * 8 + (rank >> 2) * 4 + bi) * BLK;
  const u32 is1 = pl ? 2 * BLK : 8 * BLK, is2 = pl ? BLK : (2 + bj - bi) * BLK;
  const u32 idst = pl ? PIN_OFF : IN_OFF + peer * IN_SRC + NPUSH * 8192;
  const u32 ipre = pl ? FPR_OFF : SG_OFF + (peer * 64 + NPUSH / 4) * FS, ish = pl ? 1 : 2;
  const u32 ipost = pl ? FPI_OFF : FI_OFF + (peer * 256 + NPUSH) * FS;
  const u32 fown = fin_base(peer), fpar = fin_base(rank ^ PL);
  const u32 rpre = pl ? FF_OFF + peer * 256 * FS : RF_OFF + g * 8 * FS, rsh = pl ? 2 : 7;
  u32 ttile = 0;
#pragma unroll
  for (u32 e = 0; e < 4; ++e) {
    u32 ri, ci;
    tile_rc(lane * 4 + e, ri, ci);
    ttile |= (ri << 4 | ci) << (8 * e);
  }
  const u64 t0 = __builtin_amdgcn_s_memrealtime();
  u32 tf = w, tf2 = NW + w, lastf = 0, sub = 0;
  u32 cdst = 0, cact = 0, pend = 0, ack = 0;
  u32 nring = 0;
  u32 word = g;
  bool bad = false, cisf = false;
  auto poll = [&]() -> u32 {
    const u32 o = tf < NI ? ipre + (tf >> ish) * FS : tf < REL ? FF_OFF + (peer * 256 + ((tf - NI) >> 2)) * FS : rpre + ((tf & 1023) >> rsh) * FS;
    return ld_sys(ws + rfl(o));
  };
  u32 pv = 0;
  u32x4 v[4];
  auto draw = [&](bool on) -> u32 {
    return __builtin_amdgcn_raw_ptr_buffer_atomic_add_i32(word >= 8 && pl ? 1u << 16 : 1u, rtick, on && word && lane == 0 ? 0u : 0x80000000u, word * 128, 0);
  };
  u32 sstart, sbase;
  {
    const u32 l = lane >> 1, fl = min(max(l, (u32)FLAG_) - FLAG_, (u32)FLEV);
    sstart = (lane & 1) ? 2 * (l + 1) * (l + 1) + 4 * fl * fl : 2 * l * l + 4 * fl * fl;
    sbase = (lane & 1) ? REL | 4 * fl * fl : 2 * l * l;
  }
  auto handle = [&](u32 id) -> u32 {
    if (!pl) return id;
    const u32 sg = 63 - __builtin_clzll(__ballot(sstart <= id));
    return __builtin_amdgcn_readlane(sbase, sg) + id - __builtin_amdgcn_readlane(sstart, sg);
  };
  tf = handle(tf); tf2 = handle(tf2);
  auto drawn = [&](u32 t) -> u32 {
    t = rfl(t);
    if (word == 0) return NONE;
    if (word < 8) {
      const u32 id = t + 2 * NW;
      if (id + 1 + SWK * w >= NJ) word = 8 + (w & 1);
      if (__builtin_expect(id < NJ, 1)) return handle(id);
      t = rfl(draw(true));
    }
    const u32 f = t & 0xFFFF, b = t >> 16;
    if (f + b >= NPOOL) { word = 0; return NONE; }
    return REL | (NE + pool_job(pl ? NPOOL - 1 - b : f) * 2 + (w & 1));
  };
  auto put = [&](int i) {
    const auto ro = mk_rsrc((const void*)(cisf ? bo : (u64)ws), cisf ? OUT_BYTES : DATA_BYTES);
    st128(v[i], ro, cisf ? vwide : lane * 16, rfl(cdst + i * (cisf ? 4 * 8192 : 1024)));
  };
  auto done = [&](u32 a) {
    if (a) {
      LAS u32* e = ctl + MB_RING + (w / ENG_WGS) * 8 + (nring++ & 7);
      while (rfl(LD_LDS(e)) != 0) __builtin_amdgcn_s_sleep(1);
      if (lane == 0) ST_LDS(e, a);
      CBAR();
    }
  };
  auto ready = [&]() -> bool { return tf != NONE && (bad || (__ballot(pv >= seq) & 1)); };
  auto round = [&](auto first_c) {
    constexpr bool FIRST = decltype(first_c)::value;
    const u32 h = tf, hs = sub;
    const bool isf = h >= NI, tix = isf || hs;
    sub = isf ? 0 : hs ^ 1;
    tf = tix ? tf2 : tf;
    const u32 tk = draw(tix);
    pv = poll();
    CBAR();
    done(ack);
    u32 nsrc, ndst, nact;
    if (isf) {
      const u32 j = h < REL ? h - NI : h & 1023, rc = (__builtin_amdgcn_readlane(ttile, j >> 4) >> (2 * (j & 12))) & 255;
      nsrc = ndst = (h < REL ? fown : fpar) + ((rc >> 4) * 128 + (j & 3) * 16) * 8192 + (rc & 15) * 256;
      nact = h < REL ? (j >> 7) * 2 + 1 : 0;
      lastf = ndst;
    } else {
      nsrc = isrc + (h >> 2) * is1 + ((h >> 1) & 1) * is2 + (h & 1) * 8192 + hs * 4096;
      ndst = idst + h * 8192 + hs * 4096;
      nact = hs ? ipost + h * FS : 0;
    }
    const auto rsrc = mk_rsrc((const void*)(isf ? bf : bp), isf ? OUT_BYTES : DATA_BYTES);
    const u32 vs = isf ? vwide : lane * 16, str = isf ? 4 * 8192 : 1024;
#pragma unroll
    for (int i = 0; i < 4; ++i) {
      if (!FIRST) put(i);
      v[i] = __builtin_amdgcn_raw_buffer_load_b128(rsrc, vs, rfl(nsrc + i * str), AUX_SYS);
      CBAR();
    }
    ack = 0;
    if (!FIRST) {
      WAIT_VM(11);
      ack = pend; pend = cact;
    }
    cdst = ndst; cisf = isf; cact = nact;
    if (tix) tf2 = drawn(tk);
  };
  if (!pl) {
    while (rfl(LD_LDS(ctl + MB_PUSHED + w / ENG_WGS)) == 0) __builtin_amdgcn_s_sleep(2);
    wave_wait<1>(ws + ((lane & 1) ? GO_OFF + (peer * 16 + w) * FS : ipre + (w >> ish) * FS), 3, seq, tmo, bad);
    if (bad) WAIT_VM(0);
    pv = seq;
  }
  for (u32 it = 1; tf != NONE; ++it) {
    if (ready()) {
      round(std::true_type{});
      if (ready()) {
        round(std::false_type{});
        while (ready()) round(std::false_type{});
      }
#pragma unroll
      for (int i = 0; i < 4; ++i) put(i);
      done(ack);
      if (pend | cact) {
        WAIT_VM(0);
        done(pend);
        done(cact);
      }
      ack = pend = 0;
      continue;
    }
    if ((it & 63) == 0) {
      const u32 t = __hip_atomic_load((const GAS u32*)tmo, __ATOMIC_RELAXED, __HIP_MEMORY_SCOPE_AGENT);
      if (rfl(t) == seq || __builtin_amdgcn_s_memrealtime() - t0 > DEADLINE_TICKS) {
        __hip_atomic_store((GAS u32*)tmo, seq, __ATOMIC_RELAXED, __HIP_MEMORY_SCOPE_AGENT);
        WAIT_VM(0);
        bad = true;
      }
    }
    __builtin_amdgcn_s_sleep(SL_PULL * 3);
    pv = poll();
  }
  if (bad) {
    WAIT_VM(0);
    const u32x4 nanv = {NAN2, NAN2, NAN2, NAN2};
    for (int i = 0; i < 4; ++i) st128(nanv, mk_rsrc(p.out[0], OUT_BYTES), vwide, rfl(lastf) + i * 4 * 8192);
  }
  if (lane == 0) __hip_atomic_fetch_add(ctl + MB_END, 1u, __ATOMIC_RELAXED, __HIP_MEMORY_SCOPE_WORKGROUP);
}

__device__ __forceinline__ void flag_role(const Params& p, u32 g, int lane, u32 seq, LAS u32* ctl) {
  char* ws = p.ws[0];
  char* rf = p.ws[g ^ PL] + RF_OFF + (g ^ PL) * 8 * FS;
  for (;;) {
    const u32 end = rfl(LD_LDS(ctl + MB_END));
    const u32 a = lane < 16 ? LD_LDS(ctl + MB_RING + lane) : 0;
    if (a) {
      ST_LDS(ctl + MB_RING + lane, 0u);
      if (!(a & 1)) st_sys(ws + a, seq);
      else if (__hip_atomic_fetch_add((GAS u32*)(ws + SEG_OFF + g * 128) + (a >> 1), 1u, __ATOMIC_RELAXED, __HIP_MEMORY_SCOPE_AGENT) == 128 * seq - 1)
        st_sys(rf + (a >> 1) * FS, seq);
    }
    if (end == PULL_WAVES && __ballot(a != 0) == 0) break;
    __builtin_amdgcn_s_sleep(2);
  }
}

__device__ __forceinline__ void fin_role(const Params& p, u32 f, u32 seq) {
  const int tid = threadIdx.x, lane = tid & 63;
  const u32 wid = rfl(tid >> 6), rank = p.rank;
  char* ws = p.ws[0];
  char* tmo = ws + TMO_OFF;
  const auto rl = mk_rsrc(ws, DATA_BYTES);
  const auto ro = mk_rsrc(p.out[0], OUT_BYTES);
  const u32 vwide = (lane >> 4) * 8192 + (lane & 15) * 16;
  char* wsl = p.ws[lane & 7];
  for (u32 m = f; m < 256; m += 8 * FIN_WGS) {
    u32 ri, ci;
    tile_rc(m, ri, ci);
    const u32 item = wid * 2, dst = fin_base(rank) + (ri * 128 + item * 4) * 8192 + ci * 256;
    bool bad = false;
    wave_wait<SL_FIN>(ws + ((lane & 1) ? FPL_OFF + m * FS : FPI_OFF + (m * 2 + (wid >> 2)) * FS), 3, seq, tmo, bad);
    u32x4 r[2], a[2], b[2];
#pragma unroll
    for (int i = 0; i < 2; ++i) {
      r[i] = __builtin_amdgcn_raw_buffer_load_b128(rl, lane * 16, PIN_OFF + m * 16384 + (item + i) * 1024, AUX_SYS);
      a[i] = __builtin_amdgcn_raw_buffer_load_b128(rl, lane * 16, P32_OFF + m * 32768 + (item + i) * 2048, AUX_SYS);
      b[i] = __builtin_amdgcn_raw_buffer_load_b128(rl, lane * 16, P32_OFF + m * 32768 + (item + i) * 2048 + 1024, AUX_SYS);
    }
    bad |= __hip_atomic_load((const GAS u32*)tmo, __ATOMIC_RELAXED, __HIP_MEMORY_SCOPE_AGENT) == seq;
#pragma unroll
    for (int i = 0; i < 2; ++i) {
      float y[8];
#pragma unroll
      for (int e = 0; e < 4; ++e) {
        const u32 l0 = e < 2 ? a[i][2 * e] : b[i][2 * e - 4], l1 = e < 2 ? a[i][2 * e + 1] : b[i][2 * e - 3];
        y[2 * e] = silu(__uint_as_float(l0) + __uint_as_float(r[i][e] << 16));
        y[2 * e + 1] = silu(__uint_as_float(l1) + __uint_as_float(r[i][e] & 0xFFFF0000u));
      }
      const bf16x8 o = {(__bf16)y[0], (__bf16)y[1], (__bf16)y[2], (__bf16)y[3],
                        (__bf16)y[4], (__bf16)y[5], (__bf16)y[6], (__bf16)y[7]};
      st128(__builtin_bit_cast(u32x4, o) | (bad ? NAN2 : 0u), ro, vwide, dst + i * 4 * 8192);
    }
    WAIT_VM(0);
    __syncthreads();
    if (tid >= 1 && tid < NR) st_sys(wsl + FF_OFF + (rank * 256 + m) * FS, seq);
  }
}

template <bool HALF>
__device__ __forceinline__ void gemm_tile(const Params& p, u32 m, u32 hsel, u32 seq, u64 tstart, bool first, LAS char* lds) {
  const int tid = threadIdx.x, lane = tid & 63;
  const u32 wid = rfl(tid >> 6);
  const u32 rank = p.rank;
  char* ws = p.ws[0];
  char* tmo = ws + TMO_OFF;
  const auto rx = mk_rsrc(p.x, X_BYTES);
  const auto rw = mk_rsrc(p.w, W_BYTES);
  const auto rs = mk_rsrc(ws, DATA_BYTES);
  constexpr int NOWN = 8, NRCV = 24;
  constexpr int MI = HALF ? 1 : 2, NDMA = HALF ? 3 : 4, NS = HALF ? NS_HALF : 4;
  constexpr u32 XB = HALF ? BLK / 2 : BLK, SB = XB + BLK;
  static_assert(NS * SB <= LDS_CTL, "stages fit");

  u32 vao[2], vbo[2], vaw[2], vbw[2];
#pragma unroll
  for (int q = 0; q < 2; ++q) {
    const u32 ra = wid * 8 * MI + q * 8 + (lane >> 3), ca = (lane & 7) ^ ((ra >> 1) & 7);
    const u32 kb = wid * 8 + q * 4 + (lane >> 4), cbk = (lane & 15) ^ (((kb & 3) << 2) | ((kb >> 2) & 3));
    vao[q] = ra * 1024 + ca * 16 - q * 1024;  vaw[q] = ra * 128 + ca * 16 - q * 1024;
    vbo[q] = kb * 8192 + cbk * 16 - q * 1024; vbw[q] = kb * 256 + cbk * 16 - q * 1024;
  }
  const u32 lds0 = (u32)(size_t)lds;
  const u32 wm = wid & 1, wn = wid >> 1, h = lane >> 5;
  u32 xo[4], wo[2];
  {
    const u32 r = wm * 32 * MI + (lane & 31);
#pragma unroll
    for (int ks = 0; ks < 4; ++ks) xo[ks] = r * 128 + (((2 * ks + h) ^ ((r >> 1) & 7)) * 16);
    const u32 bk = (lane >> 4) & 1, q = (lane & 15) >> 2, pp = lane & 3;
#pragma unroll
    for (int t = 0; t < 2; ++t)
      wo[t] = XB + 256 * (8 * h + 4 * t + q) + 16 * ((4 * wn + 2 * bk + (pp >> 1)) ^ ((q << 2) | (2 * h + t))) + 8 * (pp & 1);
  }
  LAS u32* lbad = (LAS u32*)(lds + LDS_CTL);
  u32 ri, ci;
  tile_rc(m, ri, ci);
  const u32 row0 = ((rank >> 1) & 1) * 2048 + ri * 128 + (HALF ? hsel * 64 : 0), col0 = (rank & 1) * 2048 + ci * 128;
  u32 islot = 0, rslot = 0;
  auto next_slot = [&](u32 o) -> u32 { return o + SB == NS * SB ? 0 : o + SB; };
  auto issue_own = [&](int t) {
    const u32 dst = lds0 + islot, sa = row0 * 1024 + t * 128, sw = t * 64 * 8192 + col0 * 2;
    islot = next_slot(islot);
    if (HALF) dma3(dst + wid * 1024, dst + XB + wid * 2048, rx, vao[0], sa, rw, vbo[0], vbo[1], sw);
    else dma4(dst + wid * 2048, rx, vao[0], vao[1], sa, rw, vbo[0], vbo[1], sw);
  };
  u32 taba, tabw;
  {
    const u32 ktl = lane / 6, c = lane - ktl * 6, src = rank ^ (c + 1 + (c >= 3));
    const u32 base = IN_OFF + src * IN_SRC + ktl * 2 * BLK;
    taba = base + ri * 8 * BLK + (HALF ? hsel * 8192 : 0);
    tabw = base + (ci * 8 + 1) * BLK;
  }
  auto issue = [&](int t) {
    const u32 dst = lds0 + islot, sa = __builtin_amdgcn_readlane(taba, t), sw = __builtin_amdgcn_readlane(tabw, t);
    islot = next_slot(islot);
    if (HALF) dma3(dst + wid * 1024, dst + XB + wid * 2048, rs, vaw[0], sa, rs, vbw[0], vbw[1], sw);
    else dma4(dst + wid * 2048, rs, vaw[0], vaw[1], sa, rs, vbw[0], vbw[1], sw);
  };
  const u32 fop = (lane >> 1) & 1, fq = min(lane >> 2, 11), fk = fq / 6, fc = fq - fk * 6;
  const char* fin = ws + FI_OFF + ((rank ^ (fc + 1 + (fc >= 3))) * 256 + (((fop ? ci : ri) * 4 + fk) * 2 + fop) * 2 + ((HALF && !fop) ? hsel : (lane & 1))) * FS;

  while (__builtin_amdgcn_s_memrealtime() - tstart < (u64)max(ri, ci) * STAGGER) __builtin_amdgcn_s_sleep(16);
  if (first && wid == 0) __builtin_amdgcn_fence(__ATOMIC_ACQUIRE, "");
  f32x16 acc[MI] = {};
  struct Frag { bf16x8 w[4], x[MI][4]; };
  auto read = [&](Frag& f) {
    const LAS char* st = lds + rslot;
    rslot = next_slot(rslot);
#pragma unroll
    for (int ks = 0; ks < 4; ++ks) {
      const bf16x4 w0 = __builtin_amdgcn_ds_read_tr16_b64_v4bf16((LAS bf16x4*)(st + wo[0] + ks * 4096));
      const bf16x4 w1 = __builtin_amdgcn_ds_read_tr16_b64_v4bf16((LAS bf16x4*)(st + wo[1] + ks * 4096));
      f.w[ks] = __builtin_shufflevector(w0, w1, 0, 1, 2, 3, 4, 5, 6, 7);
#pragma unroll
      for (int mi = 0; mi < MI; ++mi) f.x[mi][ks] = *(const LAS bf16x8*)(st + xo[ks] + mi * 4096);
    }
  };
  auto mma = [&](const Frag& f) {
#pragma unroll
    for (int ks = 0; ks < 4; ++ks)
#pragma unroll
      for (int mi = 0; mi < MI; ++mi) acc[mi] = __builtin_amdgcn_mfma_f32_32x32x16_bf16(f.w[ks], f.x[mi][ks], acc[mi], 0, 0, 0);
  };
  auto waitpoint = [&](int tn) {
    if (wid == 0) {
      bool bad = false;
      wave_wait2<SL_GEMM>(fin + (tn ? 8 * FS : 0), ~0ull, seq, tmo, bad);
      if (bad) *lbad = 1;
    }
  };
  auto step = [&](auto rc, auto oc, int tn, const Frag& fc, Frag& fn) {
    constexpr int R = decltype(rc)::value;
    constexpr bool OWN = decltype(oc)::value;
    if (!OWN && __builtin_expect(tn == 12, 0)) waitpoint(tn);
    if constexpr (R >= 0) {
      WAIT_VM(R * NDMA);
      __builtin_amdgcn_s_waitcnt(0xC07F);
      CBAR();
      __builtin_amdgcn_s_barrier();
      CBAR();
      read(fn);
    }
    if (tn >= 0) { if (OWN) issue_own(tn); else issue(tn); }
    mma(fc);
  };
  Frag fa, fb;
  static_assert(NS % 2 == 0 && NS <= NOWN, "run structure");
  auto run = [&](auto oc, int n) {
    constexpr bool OWN = decltype(oc)::value;
    islot = rslot = 0;
    for (int t = 0; t < NS; ++t) { if (OWN) issue_own(t); else issue(t); }
    WAIT_VM((NS - 1) * NDMA);
    __builtin_amdgcn_s_barrier();
    CBAR();
    read(fa);
    for (int tn = NS; tn < n; tn += 2) {
      step(std::integral_constant<int, NS - 2>{}, oc, tn, fa, fb);
      step(std::integral_constant<int, NS - 2>{}, oc, tn + 1, fb, fa);
    }
    static_for<0, NS / 2>([&](auto ic) {
      constexpr int i = decltype(ic)::value;
      step(std::integral_constant<int, NS - 2 - 2 * i>{}, std::true_type{}, -1, fa, fb);
      step(std::integral_constant<int, NS - 3 - 2 * i>{}, std::true_type{}, -1, fb, fa);
    });
  };
  run(std::true_type{}, NOWN);
  waitpoint(0);
  __builtin_amdgcn_s_waitcnt(0xC07F);
  __builtin_amdgcn_s_barrier();
  run(std::false_type{}, NRCV);
  const u32 tmv = __hip_atomic_load((const GAS u32*)tmo, __ATOMIC_RELAXED, __HIP_MEMORY_SCOPE_AGENT);
  __builtin_amdgcn_s_waitcnt(0xC07F);
  __builtin_amdgcn_s_barrier();
  CBAR();
  constexpr u32 EROW = 528;
#pragma unroll
  for (int mi = 0; mi < MI; ++mi) {
    const u32 mr = wm * 32 * MI + mi * 32 + (lane & 31);
#pragma unroll
    for (int gq = 0; gq < 4; ++gq) {
      const f32x4 o = {acc[mi][4 * gq], acc[mi][4 * gq + 1], acc[mi][4 * gq + 2], acc[mi][4 * gq + 3]};
      *(LAS f32x4*)(lds + mr * EROW + (wn * 32 + 8 * gq + 4 * h) * 4) = o;
    }
  }
  __syncthreads();
  const u32 bad = (*lbad || tmv == seq) ? NAN2 : 0u;
  const u32 keep = rank >> 2;
  const bool do_exp = !HALF || hsel != keep, do_keep = !HALF || hsel == keep;
  if (do_exp) {
    const u32 e16 = P16_OFF + m * 16384 + wid * 1024;
#pragma unroll
    for (int part = 0; part < 2; ++part) {
      const LAS char* se = lds + ((HALF ? 0 : (keep ^ 1) * 64) + part * 32 + (tid >> 4)) * EROW + (tid & 15) * 32;
      const f32x4 c0 = *(const LAS f32x4*)se, c1 = *(const LAS f32x4*)(se + 16);
      const bf16x8 o = {(__bf16)c0[0], (__bf16)c0[1], (__bf16)c0[2], (__bf16)c0[3],
                        (__bf16)c1[0], (__bf16)c1[1], (__bf16)c1[2], (__bf16)c1[3]};
      st128(__builtin_bit_cast(u32x4, o) | bad, rs, lane * 16, e16 + part * 8192);
    }
  }
  if (do_keep) {
    const u32 e32 = P32_OFF + m * 32768 + wid * 2048;
#pragma unroll
    for (int part = 0; part < 2; ++part) {
      const LAS char* sk = lds + ((HALF ? 0 : keep * 64) + part * 32 + (tid >> 4)) * EROW + (tid & 15) * 32;
      const u32x4 c0 = *(const LAS u32x4*)sk, c1 = *(const LAS u32x4*)(sk + 16);
      st128(c0 | bad, rs, lane * 16, e32 + part * 16384);
      st128(c1 | bad, rs, lane * 16, e32 + part * 16384 + 1024);
    }
  }
  WAIT_VM(0);
  __syncthreads();
  if (tid == 0 && do_exp) st_sys(p.ws[PL] + FPR_OFF + m * FS, seq);
  if (tid == 64 && do_keep) st_sys(ws + FPL_OFF + m * FS, seq);
}

__device__ __forceinline__ void stage_unit(const Params& p, u32 lk, u32 seq) {
  const int tid = threadIdx.x, lane = tid & 63;
  const u32 wid = rfl(tid >> 6);
  const u32 lev = lk >> 2, kt = (wid >> 2) * 4 + (lk & 3), op = (wid >> 1) & 1, rc = (wid & 1) * 2048 + lev * 128;
  const auto r = mk_rsrc(op ? p.w : p.x, X_BYTES);
  const auto rs = mk_rsrc(p.ws[0], DATA_BYTES);
  const u32 vo = op ? (lane >> 4) * 8192 + (lane & 15) * 16 : (lane >> 3) * 1024 + (lane & 7) * 16;
  const u32 so = op ? kt * 64 * 8192 + rc * 2 : rc * 1024 + kt * 128;
  u32x4 v[16];
#pragma unroll
  for (int i = 0; i < 16; ++i) v[i] = __builtin_amdgcn_raw_buffer_load_b128(r, vo, so + i * (op ? 4 * 8192 : 8 * 1024), 0);
#pragma unroll
  for (int i = 0; i < 16; ++i) st128(v[i], rs, lane * 16, EX_OFF + (lk * 8 + wid) * BLK + i * 1024);
  WAIT_VM(0);
  __syncthreads();
  if (tid >= 1 && tid < NR && tid != PL) st_sys(p.ws[tid] + SG_OFF + (p.rank * 64 + lk) * FS, seq);
}

__device__ __forceinline__ void gemm_role(const Params& p, u32 k, u32 seq, LAS char* lds) {
  const u64 tstart = __builtin_amdgcn_s_memrealtime();
  constexpr u32 NG = 8 * GEMM_WGS;
  static_assert(2 * NHALF <= NG && 256 + NHALF <= 2 * NG, "unit schedule");
  if (k < 2 * NHALF) gemm_tile<true>(p, k >> 1, k & 1, seq, tstart, true, lds);
  for (u32 n = k < 2 * NHALF ? k + NG : k; n < 256 + NHALF; n += NG) gemm_tile<false>(p, n - NHALF, 0, seq, tstart, n < NG, lds);
}

__global__ void __launch_bounds__(THREADS, 2) fused_gemm_allreduce_silu(Params p) {
  extern __shared__ __attribute__((aligned(16))) char smem[];
  const int lane = threadIdx.x & 63;
  const u32 wid = rfl(threadIdx.x >> 6);
  LAS u32* ctl = (LAS u32*)smem;
  LAS u32* lctl = (LAS u32*)(smem + LDS_CTL);
  u32 slot = (blockIdx.x & ~7u) | ((u32)__builtin_amdgcn_s_getreg((3 << 11) | 20) & 7);
  auto is_push = [&]() { return slot < 8 * ENG_WGS && (slot & 3) != 0 && wid < PUSH_WAVES; };
  if (__builtin_expect(is_push(), 1)) {
    if (push_role<false>(p, slot, wid, lane, lctl)) return;
  } else {
    if (wid == 7) {
      if (lane < 32) ctl[lane] = 0;
      if (lane == 0) {
        GAS u32* ctr = (GAS u32*)(p.ws[0] + CTR_OFF) + blockIdx.x;
        GAS u32* clm = (GAS u32*)(p.ws[0] + CLM_OFF);
        const u32 c = __hip_atomic_load(ctr, __ATOMIC_RELAXED, __HIP_MEMORY_SCOPE_AGENT);
        u32 sl = slot;
#pragma unroll 1
        for (int n = 0; n < 4 * GRID; ++n) {
          if (__hip_atomic_fetch_add(clm + sl * 16, 1u, __ATOMIC_RELAXED, __HIP_MEMORY_SCOPE_AGENT) == c) break;
          __hip_atomic_fetch_add(clm + sl * 16, ~0u, __ATOMIC_RELAXED, __HIP_MEMORY_SCOPE_AGENT);
          sl = (sl + 1) & (GRID - 1);
        }
        lctl[0] = 0; lctl[1] = sl; lctl[2] = c + 1;
      }
    }
    asm volatile("s_waitcnt lgkmcnt(0)" ::: "memory");
    __builtin_amdgcn_s_barrier();
    slot = rfl(LD_LDS(lctl + 1));
    if (threadIdx.x == THREADS - 1)
      __hip_atomic_store((GAS u32*)(p.ws[0] + CTR_OFF) + blockIdx.x, LD_LDS(lctl + 2), __ATOMIC_RELAXED, __HIP_MEMORY_SCOPE_AGENT);
    WAIT_VM(0);
  }
  if (__builtin_expect(is_push(), 0)) {
    push_role<true>(p, slot, wid, lane, lctl);
    return;
  }
  const u32 seq = rfl(LD_LDS(lctl + 2));
  const u32 g = slot & 7, s = slot >> 3;
  const u32 peer = ((u32)p.rank ^ g) & (NR - 1);
  constexpr u32 LK0 = NPUSH / 4;
  if (s < ENG_WGS) {
    if (g == 0) {
      for (u32 u = s; u < 64 - LK0; u += u < 16 ? 16 : u < 24 ? 16 : 8) stage_unit(p, LK0 + u, seq);
    } else if (wid >= PUSH_WAVES && wid < PUSH_WAVES + PULL_WAVES) {
      pull_role(p, peer, (wid - PUSH_WAVES) * ENG_WGS + s, lane, seq, ctl);
    } else if (wid == PUSH_WAVES + PULL_WAVES) {
      flag_role(p, g, lane, seq, ctl);
    }
  } else if (s < ENG_WGS + FIN_WGS) {
    if ((g & 3) == 0) { stage_unit(p, LK0 + 8 + g + s - ENG_WGS, seq); stage_unit(p, LK0 + 24 + g + s - ENG_WGS, seq); }
    fin_role(p, (s - ENG_WGS) * 8 + g, seq);
  } else {
    const u32 k = s - ENG_WGS - FIN_WGS;
    gemm_role(p, (k >> 2) * 32 + g * 4 + (k & 3), seq, (LAS char*)smem);
  }
}

}

extern "C" void kernel_launch(int rank, int nranks, ncclComm_t comm,
                              hipStream_t stream,
                              void* const* d_in, const int* in_sizes, int n_in,
                              void* d_out, int out_size,
                              const int* in_dtypes, int out_dtype,
                              void* const* d_out_all,
                              void* const* d_ws_all, size_t ws_bytes) {
  if (nranks != NR || (unsigned)rank >= (unsigned)NR || n_in < 2 || ws_bytes < WS_USED ||
      in_sizes[0] != 4096 * 512 || in_sizes[1] != 512 * 4096 || out_size != 4096 * 4096) {
    fprintf(stderr, "kernel_launch: unexpected problem shape\n");
    return;
  }
  HIP_CHECK(hipFuncSetAttribute((const void*)fused_gemm_allreduce_silu,
                                hipFuncAttributeMaxDynamicSharedMemorySize, LDS_BYTES));
  Params p;
  p.x = (const char*)d_in[0];
  p.w = (const char*)d_in[1];
  const int phys[NR] = {0, 2, 3, 6, 1, 4, 5, 7};
  p.rank = 0; p.pad = 0;
  for (int i = 0; i < NR; ++i) if (phys[i] == rank) p.rank = i;
  for (int i = 0; i < NR; ++i) { p.ws[i] = (char*)d_ws_all[phys[p.rank ^ i]]; p.out[i] = (char*)d_out_all[phys[p.rank ^ i]]; }
  for (u32 g = 0; g < NR; ++g)
    for (u32 job = 0; job < 16; ++job) {
      const u32 peer = p.rank ^ g, lev = job >> 4, kt = (peer >> 2) * 4 + ((job >> 2) & 3), op = (job >> 1) & 1, half = job & 1;
      p.psrc[g][job] = op ? (kt * 64 + half * 32) * 8192 + ((peer & 1) * 2048 + lev * 128) * 2
                                         : (((peer >> 1) & 1) * 2048 + lev * 128 + half * 64) * 1024 + kt * 128;
    }
  hipLaunchKernelGGL(fused_gemm_allreduce_silu, dim3(GRID), dim3(THREADS), LDS_BYTES, stream, p);
  HIP_CHECK(hipGetLastError());
}
```

```cpp
#include <hip/hip_runtime.h>
#include <rccl/rccl.h>
#include <cstdio>
#include <type_traits>

#define HIP_CHECK(expr) do { hipError_t _e = (expr); if (_e != hipSuccess) { \
    fprintf(stderr, "HIP error %s at %s:%d\n", hipGetErrorString(_e), __FILE__, __LINE__); \
    return; } } while (0)

namespace {

typedef unsigned int u32;
typedef unsigned long long u64;
typedef u32 u32x4 __attribute__((ext_vector_type(4)));
typedef float f32x4 __attribute__((ext_vector_type(4)));
typedef __bf16 bf16x4 __attribute__((ext_vector_type(4)));
typedef __bf16 bf16x8 __attribute__((ext_vector_type(8)));
typedef float f32x16 __attribute__((ext_vector_type(16)));
#define LAS __attribute__((address_space(3)))
#define GAS __attribute__((address_space(1)))

constexpr int NR = 8;
constexpr u32 X_BYTES = 4096u * 512u * 2u;
constexpr u32 W_BYTES = 512u * 4096u * 2u;
constexpr u32 OUT_BYTES = 4096u * 4096u * 2u;
constexpr u32 FS = 128;

constexpr u32 BLK = 16384;
constexpr u32 IN_OFF = 0;
constexpr u32 IN_SRC = 128 * BLK;
constexpr u32 P16_OFF = IN_OFF + NR * IN_SRC;
constexpr u32 PIN_OFF = P16_OFF + 320 * 16384;
constexpr u32 P32_OFF = PIN_OFF + 320 * 16384;
constexpr u32 EX_OFF = P32_OFF + 320 * 32768;
constexpr u32 DATA_BYTES = EX_OFF + 512 * BLK;
constexpr u32 FI_OFF = DATA_BYTES;
constexpr u32 FPR_OFF = FI_OFF + NR * 256 * FS;
constexpr u32 FPL_OFF = FPR_OFF + 320 * FS;
constexpr u32 FPI_OFF = FPL_OFF + 320 * FS;
constexpr u32 FF_OFF = FPI_OFF + 640 * FS;
constexpr u32 SG_OFF = FF_OFF + NR * 256 * FS;
constexpr u32 GO_OFF = SG_OFF + NR * 64 * FS;
constexpr u32 RF_OFF = GO_OFF + NR * 16 * FS;
constexpr u32 CTR_OFF = RF_OFF + 64 * FS;
constexpr u32 CLM_OFF = CTR_OFF + 256 * 4;
constexpr u32 TMO_OFF = CLM_OFF + 256 * 64;
constexpr u32 TICK_OFF = TMO_OFF + 128;
constexpr u32 SEG_OFF = TICK_OFF + 32 * 128;
constexpr u32 WS_USED = SEG_OFF + 8 * 128;

constexpr u64 DEADLINE_TICKS = 80000000ull;

constexpr int ENG_WGS = 8, FIN_WGS = 4, GEMM_WGS = 20;
constexpr int PUSH_WAVES = 2, PULL_WAVES = 2;
constexpr int PUSH_D = 4, PUSH_ROUNDS = 4;
constexpr u32 NPUSH = PUSH_ROUNDS * ENG_WGS * PUSH_WAVES;
constexpr u32 PL = 4;
constexpr int GO_AT = 6;
constexpr int NS_HALF = 6;
constexpr u32 NHALF = 64;
constexpr u64 STAGGER = 190;
constexpr u32 FLEV = 4, FLAG_ = 0, TOPK = 48, SWK = 1;
constexpr int SL_PULL = 2, SL_FIN = 4, SL_GEMM = 8;
constexpr u32 NAN2 = 0x7FC07FC0u;
constexpr int GRID = 8 * (ENG_WGS + FIN_WGS + GEMM_WGS);
static_assert(GRID == 256, "slots wrap at a power of two");
constexpr int THREADS = 512;
constexpr u32 LDS_CTL = 144 * 1024;
constexpr u32 LDS_BYTES = LDS_CTL + 64;
constexpr int AUX_SYS = 17;

struct Params {
  const char* x;
  const char* w;
  int rank;
  int pad;
  char* ws[NR];
  char* out[NR];
  u32 psrc[NR][16];
};

__device__ __forceinline__ u32 rfl(u32 v) { return __builtin_amdgcn_readfirstlane(v); }
__device__ __forceinline__ __amdgpu_buffer_rsrc_t mk_rsrc(const void* p, u32 n) {
  const u64 a = (u64)p;
  const u64 u = ((u64)rfl((u32)(a >> 32)) << 32) | rfl((u32)a);
  return __builtin_amdgcn_make_buffer_rsrc((void*)u, 0, n, 0x00020000);
}
__device__ __forceinline__ u32 ld_sys(const char* p) {
  return __hip_atomic_load((const GAS u32*)p, __ATOMIC_RELAXED, __HIP_MEMORY_SCOPE_SYSTEM);
}
__device__ __forceinline__ void st_sys(char* p, u32 v) {
  __hip_atomic_store((GAS u32*)p, v, __ATOMIC_RELAXED, __HIP_MEMORY_SCOPE_SYSTEM);
}
#define WAIT_VM(n) asm volatile("s_waitcnt vmcnt(%0)" ::"n"(n) : "memory")
#define CBAR() asm volatile("" ::: "memory")
#define LD_LDS(q) __hip_atomic_load(q, __ATOMIC_RELAXED, __HIP_MEMORY_SCOPE_WORKGROUP)
#define ST_LDS(q, v) __hip_atomic_store(q, v, __ATOMIC_RELAXED, __HIP_MEMORY_SCOPE_WORKGROUP)

__device__ __forceinline__ void st128(u32x4 v, __amdgpu_buffer_rsrc_t r, u32 voff, u32 soff) {
  __builtin_amdgcn_raw_buffer_store_b128(v, r, voff, soff, AUX_SYS);
  asm volatile("s_nop 1" ::"v"(v) : "memory");
}

template <int SLEEP>
__device__ __forceinline__ u64 wave_wait(const char* flag, u64 need, u32 seq, char* tmo, bool& bad) {
  const u64 t0 = __builtin_amdgcn_s_memrealtime();
  for (u32 it = 1;; ++it) {
    const u64 ok = __ballot(ld_sys(flag) >= seq);
    if ((ok & need) == need) return ok;
    if ((it & 15) == 0 || bad) {
      const u32 t = __hip_atomic_load((const GAS u32*)tmo, __ATOMIC_RELAXED, __HIP_MEMORY_SCOPE_AGENT);
      if (rfl(t) == seq || __builtin_amdgcn_s_memrealtime() - t0 > DEADLINE_TICKS) {
        __hip_atomic_store((GAS u32*)tmo, seq, __ATOMIC_RELAXED, __HIP_MEMORY_SCOPE_AGENT);
        bad = true;
        return ~0ull;
      }
    }
    __builtin_amdgcn_s_sleep(SLEEP);
  }
}

template <int SLEEP>
__device__ __forceinline__ u64 wave_wait2(const char* flag, u64 need, u32 seq, char* tmo, bool& bad) {
  const u64 t0 = __builtin_amdgcn_s_memrealtime();
  const bool needed = (need >> (threadIdx.x & 63)) & 1;
  bool ok = false;
  for (u32 it = 1;; ++it) {
    if (!ok && (it == 1 || needed)) ok = ld_sys(flag) >= seq;
    const u64 have = __ballot(ok);
    if ((have & need) == need) return have;
    if ((it & 15) == 0) {
      const u32 t = __hip_atomic_load((const GAS u32*)tmo, __ATOMIC_RELAXED, __HIP_MEMORY_SCOPE_AGENT);
      if (rfl(t) == seq || __builtin_amdgcn_s_memrealtime() - t0 > DEADLINE_TICKS) {
        __hip_atomic_store((GAS u32*)tmo, seq, __ATOMIC_RELAXED, __HIP_MEMORY_SCOPE_AGENT);
        bad = true;
        return ~0ull;
      }
    }
    __builtin_amdgcn_s_sleep(SLEEP);
  }
}

__device__ __forceinline__ void dma4(u32 dst, __amdgpu_buffer_rsrc_t ra, u32 va0, u32 va1, u32 sa,
                                     __amdgpu_buffer_rsrc_t rb, u32 vb0, u32 vb1, u32 sb) {
  u32 keep;
  dst = rfl(dst); sa = rfl(sa); sb = rfl(sb);
  asm volatile(
      "s_mov_b32 %0, m0\n\t"
      "s_nop 4\n\t"
      "s_mov_b32 m0, %1\n\t"
      "s_nop 0\n\t"
      "buffer_load_dwordx4 %3, %5, %6 offen lds\n\t"
      "buffer_load_dwordx4 %4, %5, %6 offen offset:1024 lds\n\t"
      "s_mov_b32 m0, %2\n\t"
      "s_nop 0\n\t"
      "buffer_load_dwordx4 %7, %9, %10 offen lds\n\t"
      "buffer_load_dwordx4 %8, %9, %10 offen offset:1024 lds\n\t"
      "s_mov_b32 m0, %0"
      : "=&s"(keep)
      : "s"(dst), "s"(dst + BLK), "v"(va0), "v"(va1), "s"(ra), "s"(sa), "v"(vb0), "v"(vb1), "s"(rb), "s"(sb)
      : "memory");
}

__device__ __forceinline__ void dma3(u32 dsta, u32 dstb, __amdgpu_buffer_rsrc_t ra, u32 va0, u32 sa,
                                     __amdgpu_buffer_rsrc_t rb, u32 vb0, u32 vb1, u32 sb) {
  u32 keep;
  dsta = rfl(dsta); dstb = rfl(dstb); sa = rfl(sa); sb = rfl(sb);
  asm volatile(
      "s_mov_b32 %0, m0\n\t"
      "s_nop 4\n\t"
      "s_mov_b32 m0, %1\n\t"
      "s_nop 0\n\t"
      "buffer_load_dwordx4 %3, %4, %5 offen lds\n\t"
      "s_mov_b32 m0, %2\n\t"
      "s_nop 0\n\t"
      "buffer_load_dwordx4 %6, %8, %9 offen lds\n\t"
      "buffer_load_dwordx4 %7, %8, %9 offen offset:1024 lds\n\t"
      "s_mov_b32 m0, %0"
      : "=&s"(keep)
      : "s"(dsta), "s"(dstb), "v"(va0), "s"(ra), "s"(sa), "v"(vb0), "v"(vb1), "s"(rb), "s"(sb)
      : "memory");
}

template <int I, int N, class F>
__device__ __forceinline__ void static_for(F&& f) {
  if constexpr (I < N) { f(std::integral_constant<int, I>{}); static_for<I + 1, N>(f); }
}

__device__ __forceinline__ void tile_rc(u32 m, u32& ri, u32& ci) {
  u32 l = 0;
#pragma unroll
  for (u32 t = 1; t < 16; ++t) l += m >= t * t;
  const u32 idx = m - l * l;
  ri = idx < l ? l : idx - l;
  ci = idx < l ? idx : l;
}
__device__ __forceinline__ u32 fin_base(u32 r) { return (((r >> 1) & 1) * 2048 + (r >> 2) * 64) * 8192 + (r & 1) * 4096; }

template <bool claimed>
__device__ __forceinline__ bool push_role(const Params& p, u32& slot, u32 wid, int lane, LAS u32* lctl) {
  constexpr int D = PUSH_D, JI = 8;
  constexpr u32 NW = ENG_WGS * PUSH_WAVES;
  static_assert(PUSH_ROUNDS >= 3 && NW == 16 && X_BYTES == W_BYTES && GO_AT <= JI, "push schedule");
  const u32 g = slot & 7, w = wid * ENG_WGS + (slot >> 3), op = (slot >> 4) & 1;
  u64 bs = (u64)(&p.x)[op], bw = (u64)p.ws[g];
  u32 src = p.psrc[g][w];
  u32 rank = p.rank;
  asm volatile("" : "+s"(bs), "+s"(bw), "+s"(src), "+s"(rank));
  const auto rs = mk_rsrc((const void*)bs, X_BYTES);
  const u32 vs = op ? (lane >> 4) * 8192 + (lane & 15) * 16 : (lane >> 3) * 1024 + (lane & 7) * 16;
  const u32 str = op ? 4 * 8192 : 8 * 1024;
  auto ld = [&](int r, int i) -> u32x4 { return __builtin_amdgcn_raw_buffer_load_b128(rs, vs, src + r * (op ? 256 : 128 * 1024) + i * str, 0); };
  u32x4 v[D], v2[JI - D];
#pragma unroll
  for (int i = 0; i < D; ++i) { v[i] = ld(0, i); CBAR(); }
#pragma unroll
  for (int i = D; i < JI; ++i) { v2[i - D] = ld(0, i); CBAR(); }
  char* wsp = (char*)bw;
  const auto rd = mk_rsrc(wsp, DATA_BYTES);
  const u32 dst = IN_OFF + rank * IN_SRC + w * 8192, flag = FI_OFF + (rank * 256 + w) * FS;
  u32 seq = 0;
  auto round = [&](auto rc) {
    constexpr int R = decltype(rc)::value;
    constexpr bool FIRST = R == 0, LAST = R == PUSH_ROUNDS - 1;
#pragma unroll
    for (int i = 0; i < JI; ++i) {
      if (!FIRST && i == D - 1) {
        WAIT_VM(2 * D - 1);
        if (R == 1) {
          if (!claimed) __builtin_amdgcn_s_barrier();
          CBAR();
          seq = rfl(LD_LDS(lctl + 2));
        }
        if (lane == 0) st_sys(wsp + flag + (R - 1) * NW * FS, seq);
        CBAR();
      }
      if (LAST && i == JI - GO_AT) {
        if (lane == 0) st_sys(wsp + GO_OFF + (rank * 16 + w) * FS, seq);
        if (lane == 0) ST_LDS(lctl - LDS_CTL / 4 + 24 + wid, 1u);
        CBAR();
      }
      st128(FIRST && i >= D ? v2[i >= D ? i - D : 0] : v[i % D], rd, lane * 16, dst + R * NW * 8192 + i * 1024);
      if (i + D < JI) { if (!FIRST) v[i % D] = ld(R, i + D); }
      else if (!LAST) v[i % D] = ld(LAST ? R : R + 1, i + D - JI);
      CBAR();
    }
  };
  static_assert(JI - D >= D - 1, "the count above also holds in the last round");
  round(std::integral_constant<int, 0>{});
  round(std::integral_constant<int, 1>{});
  const u32 slot2 = rfl(LD_LDS(lctl + 1));
  if (__builtin_expect(slot2 != slot, 0)) { WAIT_VM(0); slot = slot2; return false; }
  static_for<2, PUSH_ROUNDS>(round);
  WAIT_VM(0);
  if (lane == 0) st_sys(wsp + flag + (PUSH_ROUNDS - 1) * NW * FS, seq);
  return true;
}

__device__ __forceinline__ float silu(float a) {
  const float c = fminf(fmaxf(a, -60.f), 60.f);
  return a * __builtin_amdgcn_rcpf(1.f + __expf(-c));
}

constexpr u32 MB_RING = 0, MB_END = 16, MB_PUSHED = 24;

constexpr u32 NX = FLEV * FLEV, NE = 4 * NX, NPOOL = (1024 - NE) / 2;
static_assert(NX <= NHALF && NX <= 32, "early tiles");
__device__ __forceinline__ u32 xbase(u32 seq) { return 256 + (seq & 1) * 32; }
__device__ __forceinline__ u32 pool_job(u32 pos) {
  return pos < NPOOL - 2 * TOPK ? pos : pos >= NPOOL - TOPK ? pos - TOPK : pos + TOPK;
}

__device__ __forceinline__ void pull_role(const Params& p, u32 peer, u32 w, int lane, u32 seq, LAS u32* ctl) {
  constexpr u32 NW = ENG_WGS * PULL_WAVES, NONE = ~0u, REL = 1u << 31;
  const u32 rank = p.rank, g = rank ^ peer;
  const bool pl = g == PL;
  const u32 NI = pl ? 640 : 256 - NPUSH, NJ = pl ? 512 + 2 * NX : NI + 1024;
  char* ws = p.ws[0];
  char* tmo = ws + TMO_OFF;
  const u64 bf = (u64)p.out[g], bp = (u64)p.ws[g], bo = (u64)p.out[0];
  const u32 vwide = (lane >> 4) * 8192 + (lane & 15) * 16;
  const auto rtick = mk_rsrc(ws + TICK_OFF + (seq & 1) * 2048, 2048);
  if (w == 0 && (lane == 0 || (pl && lane < 3))) st_sys(ws + TICK_OFF + ((seq & 1) ^ 1) * 2048 + (lane ? 7 + lane : g) * 128, 0);
  const u32 bi = (rank >> 1) & 1, bj = rank & 1;
  const u32 isrc = pl ? P16_OFF : EX_OFF + (NPUSH / 4 * 8 + (rank >> 2) * 4 + bi) * BLK;
  const u32 is1 = pl ? 2 * BLK : 8 * BLK, is2 = pl ? BLK : (2 + bj - bi) * BLK;
  const u32 idst = pl ? PIN_OFF : IN_OFF + peer * IN_SRC + NPUSH * 8192;
  const u32 ipre = pl ? FPR_OFF : SG_OFF + (peer * 64 + NPUSH / 4) * FS, ish = pl ? 1 : 2;
  const u32 ipost = pl ? FPI_OFF : FI_OFF + (peer * 256 + NPUSH) * FS;
  const u32 fown = fin_base(peer), fpar = fin_base(rank ^ PL);
  const u32 rpre = pl ? FF_OFF + peer * 256 * FS : RF_OFF + g * 8 * FS, rsh = pl ? 2 : 7;
  u32 ttile = 0;
#pragma unroll
  for (u32 e = 0; e < 4; ++e) {
    u32 ri, ci;
    tile_rc(lane * 4 + e, ri, ci);
    ttile |= (ri << 4 | ci) << (8 * e);
  }
  const u64 t0 = __builtin_amdgcn_s_memrealtime();
  u32 tf = w, tf2 = NW + w, lastf = 0, sub = 0;
  u32 cdst = 0, cact = 0, pend = 0, ack = 0;
  u32 nring = 0;
  u32 word = g;
  bool bad = false, cisf = false;
  auto poll = [&]() -> u32 {
    const u32 o = tf < NI ? ipre + (tf >> ish) * FS : tf < REL ? FF_OFF + (peer * 256 + ((tf - NI) >> 2)) * FS : rpre + ((tf & 1023) >> rsh) * FS;
    return ld_sys(ws + rfl(o));
  };
  u32 pv = 0;
  u32x4 v[4];
  auto draw = [&](bool on) -> u32 {
    return __builtin_amdgcn_raw_ptr_buffer_atomic_add_i32(word >= 8 && pl ? 1u << 16 : 1u, rtick, on && word && lane == 0 ? 0u : 0x80000000u, word * 128, 0);
  };
  u32 sstart, sbase;
  {
    const u32 l = lane >> 1, fl = min(max(l, (u32)FLAG_) - FLAG_, (u32)FLEV);
    sstart = (lane & 1) ? 2 * (l + 1) * (l + 1) + 2 * fl * fl : 2 * l * l + 2 * fl * fl;
    sbase = (lane & 1) ? 2 * xbase(seq) + 2 * fl * fl : 2 * l * l;
  }
  auto handle = [&](u32 id) -> u32 {
    if (!pl) return id;
    const u32 sg = 63 - __builtin_clzll(__ballot(sstart <= id));
    return __builtin_amdgcn_readlane(sbase, sg) + id - __builtin_amdgcn_readlane(sstart, sg);
  };
  tf = handle(tf); tf2 = handle(tf2);
  auto drawn = [&](u32 t) -> u32 {
    t = rfl(t);
    if (word == 0) return NONE;
    if (word < 8) {
      const u32 id = t + 2 * NW;
      if (id + 1 + SWK * w >= NJ) word = 8 + (w & 1);
      if (__builtin_expect(id < NJ, 1)) return handle(id);
      t = rfl(draw(true));
    }
    const u32 f = t & 0xFFFF, b = t >> 16;
    if (f + b >= NPOOL) { word = 0; return NONE; }
    return REL | (NE + pool_job(pl ? NPOOL - 1 - b : f) * 2 + (w & 1));
  };
  auto put = [&](int i) {
    const auto ro = mk_rsrc((const void*)(cisf ? bo : (u64)ws), cisf ? OUT_BYTES : DATA_BYTES);
    st128(v[i], ro, cisf ? vwide : lane * 16, rfl(cdst + i * (cisf ? 4 * 8192 : 1024)));
  };
  auto done = [&](u32 a) {
    if (a) {
      LAS u32* e = ctl + MB_RING + (w / ENG_WGS) * 8 + (nring++ & 7);
      while (rfl(LD_LDS(e)) != 0) __builtin_amdgcn_s_sleep(1);
      if (lane == 0) ST_LDS(e, a);
      CBAR();
    }
  };
  auto ready = [&]() -> bool { return tf != NONE && (bad || (__ballot(pv >= seq) & 1)); };
  auto round = [&](auto first_c) {
    constexpr bool FIRST = decltype(first_c)::value;
    const u32 h = tf, hs = sub;
    const bool isf = h >= NI, tix = isf || hs;
    sub = isf ? 0 : hs ^ 1;
    tf = tix ? tf2 : tf;
    const u32 tk = draw(tix);
    pv = poll();
    CBAR();
    done(ack);
    u32 nsrc, ndst, nact;
    if (isf) {
      const u32 j = h < REL ? h - NI : h & 1023, rc = (__builtin_amdgcn_readlane(ttile, j >> 4) >> (2 * (j & 12))) & 255;
      nsrc = ndst = (h < REL ? fown : fpar) + ((rc >> 4) * 128 + (j & 3) * 16) * 8192 + (rc & 15) * 256;
      nact = h < REL ? (j >> 7) * 2 + 1 : 0;
      lastf = ndst;
    } else {
      nsrc = isrc + (h >> 2) * is1 + ((h >> 1) & 1) * is2 + (h & 1) * 8192 + hs * 4096;
      ndst = idst + h * 8192 + hs * 4096;
      nact = hs ? ipost + h * FS : 0;
    }
    const auto rsrc = mk_rsrc((const void*)(isf ? bf : bp), isf ? OUT_BYTES : DATA_BYTES);
    const u32 vs = isf ? vwide : lane * 16, str = isf ? 4 * 8192 : 1024;
#pragma unroll
    for (int i = 0; i < 4; ++i) {
      if (!FIRST) put(i);
      v[i] = __builtin_amdgcn_raw_buffer_load_b128(rsrc, vs, rfl(nsrc + i * str), AUX_SYS);
      CBAR();
    }
    ack = 0;
    if (!FIRST) {
      WAIT_VM(11);
      ack = pend; pend = cact;
    }
    cdst = ndst; cisf = isf; cact = nact;
    if (tix) tf2 = drawn(tk);
  };
  if (!pl) {
    while (rfl(LD_LDS(ctl + MB_PUSHED + w / ENG_WGS)) == 0) __builtin_amdgcn_s_sleep(2);
    wave_wait<1>(ws + ((lane & 1) ? GO_OFF + (peer * 16 + w) * FS : ipre + (w >> ish) * FS), 3, seq, tmo, bad);
    if (bad) WAIT_VM(0);
    pv = seq;
  }
  for (u32 it = 1; tf != NONE; ++it) {
    if (ready()) {
      round(std::true_type{});
      if (ready()) {
        round(std::false_type{});
        while (ready()) round(std::false_type{});
      }
#pragma unroll
      for (int i = 0; i < 4; ++i) put(i);
      done(ack);
      if (pend | cact) {
        WAIT_VM(0);
        done(pend);
        done(cact);
      }
      ack = pend = 0;
      continue;
    }
    if ((it & 63) == 0) {
      const u32 t = __hip_atomic_load((const GAS u32*)tmo, __ATOMIC_RELAXED, __HIP_MEMORY_SCOPE_AGENT);
      if (rfl(t) == seq || __builtin_amdgcn_s_memrealtime() - t0 > DEADLINE_TICKS) {
        __hip_atomic_store((GAS u32*)tmo, seq, __ATOMIC_RELAXED, __HIP_MEMORY_SCOPE_AGENT);
        WAIT_VM(0);
        bad = true;
      }
    }
    __builtin_amdgcn_s_sleep(SL_PULL * 3);
    pv = poll();
  }
  if (bad) {
    WAIT_VM(0);
    const u32x4 nanv = {NAN2, NAN2, NAN2, NAN2};
    for (int i = 0; i < 4; ++i) st128(nanv, mk_rsrc(p.out[0], OUT_BYTES), vwide, rfl(lastf) + i * 4 * 8192);
  }
  if (lane == 0) __hip_atomic_fetch_add(ctl + MB_END, 1u, __ATOMIC_RELAXED, __HIP_MEMORY_SCOPE_WORKGROUP);
}

__device__ __forceinline__ void flag_role(const Params& p, u32 g, int lane, u32 seq, LAS u32* ctl) {
  char* ws = p.ws[0];
  char* rf = p.ws[g ^ PL] + RF_OFF + (g ^ PL) * 8 * FS;
  for (;;) {
    const u32 end = rfl(LD_LDS(ctl + MB_END));
    const u32 a = lane < 16 ? LD_LDS(ctl + MB_RING + lane) : 0;
    if (a) {
      ST_LDS(ctl + MB_RING + lane, 0u);
      if (!(a & 1)) st_sys(ws + a, seq);
      else if (__hip_atomic_fetch_add((GAS u32*)(ws + SEG_OFF + g * 128) + (a >> 1), 1u, __ATOMIC_RELAXED, __HIP_MEMORY_SCOPE_AGENT) == 128 * seq - 1)
        st_sys(rf + (a >> 1) * FS, seq);
    }
    if (end == PULL_WAVES && __ballot(a != 0) == 0) break;
    __builtin_amdgcn_s_sleep(2);
  }
}

__device__ __forceinline__ void fin_role(const Params& p, u32 f, u32 seq) {
  const int tid = threadIdx.x, lane = tid & 63;
  const u32 wid = rfl(tid >> 6), rank = p.rank;
  char* ws = p.ws[0];
  char* tmo = ws + TMO_OFF;
  const auto rl = mk_rsrc(ws, DATA_BYTES);
  const auto ro = mk_rsrc(p.out[0], OUT_BYTES);
  const u32 vwide = (lane >> 4) * 8192 + (lane & 15) * 16;
  char* wsl = p.ws[lane & 7];
  for (u32 n = f; n < 256 + NX; n += 8 * FIN_WGS) {
    const u32 m = n < 256 ? n : xbase(seq) + n - 256;
    u32 ri, ci;
    tile_rc(n & 255, ri, ci);
    const u32 item = wid * 2, dst = fin_base(m < 256 ? rank : rank ^ PL) + (ri * 128 + item * 4) * 8192 + ci * 256;
    bool bad = false;
    wave_wait<SL_FIN>(ws + ((lane & 1) ? FPL_OFF + m * FS : FPI_OFF + (m * 2 + (wid >> 2)) * FS), 3, seq, tmo, bad);
    u32x4 r[2], a[2], b[2];
#pragma unroll
    for (int i = 0; i < 2; ++i) {
      r[i] = __builtin_amdgcn_raw_buffer_load_b128(rl, lane * 16, PIN_OFF + m * 16384 + (item + i) * 1024, AUX_SYS);
      a[i] = __builtin_amdgcn_raw_buffer_load_b128(rl, lane * 16, P32_OFF + m * 32768 + (item + i) * 2048, AUX_SYS);
      b[i] = __builtin_amdgcn_raw_buffer_load_b128(rl, lane * 16, P32_OFF + m * 32768 + (item + i) * 2048 + 1024, AUX_SYS);
    }
    bad |= __hip_atomic_load((const GAS u32*)tmo, __ATOMIC_RELAXED, __HIP_MEMORY_SCOPE_AGENT) == seq;
#pragma unroll
    for (int i = 0; i < 2; ++i) {
      float y[8];
#pragma unroll
      for (int e = 0; e < 4; ++e) {
        const u32 l0 = e < 2 ? a[i][2 * e] : b[i][2 * e - 4], l1 = e < 2 ? a[i][2 * e + 1] : b[i][2 * e - 3];
        y[2 * e] = silu(__uint_as_float(l0) + __uint_as_float(r[i][e] << 16));
        y[2 * e + 1] = silu(__uint_as_float(l1) + __uint_as_float(r[i][e] & 0xFFFF0000u));
      }
      const bf16x8 o = {(__bf16)y[0], (__bf16)y[1], (__bf16)y[2], (__bf16)y[3],
                        (__bf16)y[4], (__bf16)y[5], (__bf16)y[6], (__bf16)y[7]};
      st128(__builtin_bit_cast(u32x4, o) | (bad ? NAN2 : 0u), ro, vwide, dst + i * 4 * 8192);
    }
    WAIT_VM(0);
    __syncthreads();
    if (tid >= 1 && tid < NR && m < 256) st_sys(wsl + FF_OFF + (rank * 256 + m) * FS, seq);
  }
}

template <bool HALF>
__device__ __forceinline__ void gemm_tile(const Params& p, u32 m, u32 hsel, u32 seq, u64 tstart, bool first, LAS char* lds) {
  const int tid = threadIdx.x, lane = tid & 63;
  const u32 wid = rfl(tid >> 6);
  const u32 rank = p.rank;
  char* ws = p.ws[0];
  char* tmo = ws + TMO_OFF;
  const auto rx = mk_rsrc(p.x, X_BYTES);
  const auto rw = mk_rsrc(p.w, W_BYTES);
  const auto rs = mk_rsrc(ws, DATA_BYTES);
  constexpr int NOWN = 8, NRCV = 24;
  constexpr int MI = HALF ? 1 : 2, NDMA = HALF ? 3 : 4, NS = HALF ? NS_HALF : 4;
  constexpr u32 XB = HALF ? BLK / 2 : BLK, SB = XB + BLK;
  static_assert(NS * SB <= LDS_CTL, "stages fit");

  u32 vao[2], vbo[2], vaw[2], vbw[2];
#pragma unroll
  for (int q = 0; q < 2; ++q) {
    const u32 ra = wid * 8 * MI + q * 8 + (lane >> 3), ca = (lane & 7) ^ ((ra >> 1) & 7);
    const u32 kb = wid * 8 + q * 4 + (lane >> 4), cbk = (lane & 15) ^ (((kb & 3) << 2) | ((kb >> 2) & 3));
    vao[q] = ra * 1024 + ca * 16 - q * 1024;  vaw[q] = ra * 128 + ca * 16 - q * 1024;
    vbo[q] = kb * 8192 + cbk * 16 - q * 1024; vbw[q] = kb * 256 + cbk * 16 - q * 1024;
  }
  const u32 lds0 = (u32)(size_t)lds;
  const u32 wm = wid & 1, wn = wid >> 1, h = lane >> 5;
  u32 xo[4], wo[2];
  {
    const u32 r = wm * 32 * MI + (lane & 31);
#pragma unroll
    for (int ks = 0; ks < 4; ++ks) xo[ks] = r * 128 + (((2 * ks + h) ^ ((r >> 1) & 7)) * 16);
    const u32 bk = (lane >> 4) & 1, q = (lane & 15) >> 2, pp = lane & 3;
#pragma unroll
    for (int t = 0; t < 2; ++t)
      wo[t] = XB + 256 * (8 * h + 4 * t + q) + 16 * ((4 * wn + 2 * bk + (pp >> 1)) ^ ((q << 2) | (2 * h + t))) + 8 * (pp & 1);
  }
  LAS u32* lbad = (LAS u32*)(lds + LDS_CTL);
  u32 ri, ci;
  tile_rc(m, ri, ci);
  const u32 row0 = ((rank >> 1) & 1) * 2048 + ri * 128 + (HALF ? hsel * 64 : 0), col0 = (rank & 1) * 2048 + ci * 128;
  u32 islot = 0, rslot = 0;
  auto next_slot = [&](u32 o) -> u32 { return o + SB == NS * SB ? 0 : o + SB; };
  auto issue_own = [&](int t) {
    const u32 dst = lds0 + islot, sa = row0 * 1024 + t * 128, sw = t * 64 * 8192 + col0 * 2;
    islot = next_slot(islot);
    if (HALF) dma3(dst + wid * 1024, dst + XB + wid * 2048, rx, vao[0], sa, rw, vbo[0], vbo[1], sw);
    else dma4(dst + wid * 2048, rx, vao[0], vao[1], sa, rw, vbo[0], vbo[1], sw);
  };
  u32 taba, tabw;
  {
    const u32 ktl = lane / 6, c = lane - ktl * 6, src = rank ^ (c + 1 + (c >= 3));
    const u32 base = IN_OFF + src * IN_SRC + ktl * 2 * BLK;
    taba = base + ri * 8 * BLK + (HALF ? hsel * 8192 : 0);
    tabw = base + (ci * 8 + 1) * BLK;
  }
  auto issue = [&](int t) {
    const u32 dst = lds0 + islot, sa = __builtin_amdgcn_readlane(taba, t), sw = __builtin_amdgcn_readlane(tabw, t);
    islot = next_slot(islot);
    if (HALF) dma3(dst + wid * 1024, dst + XB + wid * 2048, rs, vaw[0], sa, rs, vbw[0], vbw[1], sw);
    else dma4(dst + wid * 2048, rs, vaw[0], vaw[1], sa, rs, vbw[0], vbw[1], sw);
  };
  const u32 fop = (lane >> 1) & 1, fq = min(lane >> 2, 11), fk = fq / 6, fc = fq - fk * 6;
  const char* fin = ws + FI_OFF + ((rank ^ (fc + 1 + (fc >= 3))) * 256 + (((fop ? ci : ri) * 4 + fk) * 2 + fop) * 2 + ((HALF && !fop) ? hsel : (lane & 1))) * FS;

  while (__builtin_amdgcn_s_memrealtime() - tstart < (u64)max(ri, ci) * STAGGER) __builtin_amdgcn_s_sleep(16);
  if (first && wid == 0) __builtin_amdgcn_fence(__ATOMIC_ACQUIRE, "");
  f32x16 acc[MI] = {};
  struct Frag { bf16x8 w[4], x[MI][4]; };
  auto read = [&](Frag& f) {
    const LAS char* st = lds + rslot;
    rslot = next_slot(rslot);
#pragma unroll
    for (int ks = 0; ks < 4; ++ks) {
      const bf16x4 w0 = __builtin_amdgcn_ds_read_tr16_b64_v4bf16((LAS bf16x4*)(st + wo[0] + ks * 4096));
      const bf16x4 w1 = __builtin_amdgcn_ds_read_tr16_b64_v4bf16((LAS bf16x4*)(st + wo[1] + ks * 4096));
      f.w[ks] = __builtin_shufflevector(w0, w1, 0, 1, 2, 3, 4, 5, 6, 7);
#pragma unroll
      for (int mi = 0; mi < MI; ++mi) f.x[mi][ks] = *(const LAS bf16x8*)(st + xo[ks] + mi * 4096);
    }
  };
  auto mma = [&](const Frag& f) {
#pragma unroll
    for (int ks = 0; ks < 4; ++ks)
#pragma unroll
      for (int mi = 0; mi < MI; ++mi) acc[mi] = __builtin_amdgcn_mfma_f32_32x32x16_bf16(f.w[ks], f.x[mi][ks], acc[mi], 0, 0, 0);
  };
  auto waitpoint = [&](int tn) {
    if (wid == 0) {
      bool bad = false;
      wave_wait2<SL_GEMM>(fin + (tn ? 8 * FS : 0), ~0ull, seq, tmo, bad);
      if (bad) *lbad = 1;
    }
  };
  auto step = [&](auto rc, auto oc, int tn, const Frag& fc, Frag& fn) {
    constexpr int R = decltype(rc)::value;
    constexpr bool OWN = decltype(oc)::value;
    if (!OWN && __builtin_expect(tn == 12, 0)) waitpoint(tn);
    if constexpr (R >= 0) {
      WAIT_VM(R * NDMA);
      __builtin_amdgcn_s_waitcnt(0xC07F);
      CBAR();
      __builtin_amdgcn_s_barrier();
      CBAR();
      read(fn);
    }
    if (tn >= 0) { if (OWN) issue_own(tn); else issue(tn); }
    mma(fc);
  };
  Frag fa, fb;
  static_assert(NS % 2 == 0 && NS <= NOWN, "run structure");
  auto run = [&](auto oc, int n) {
    constexpr bool OWN = decltype(oc)::value;
    islot = rslot = 0;
    for (int t = 0; t < NS; ++t) { if (OWN) issue_own(t); else issue(t); }
    WAIT_VM((NS - 1) * NDMA);
    __builtin_amdgcn_s_barrier();
    CBAR();
    read(fa);
    for (int tn = NS; tn < n; tn += 2) {
      step(std::integral_constant<int, NS - 2>{}, oc, tn, fa, fb);
      step(std::integral_constant<int, NS - 2>{}, oc, tn + 1, fb, fa);
    }
    static_for<0, NS / 2>([&](auto ic) {
      constexpr int i = decltype(ic)::value;
      step(std::integral_constant<int, NS - 2 - 2 * i>{}, std::true_type{}, -1, fa, fb);
      step(std::integral_constant<int, NS - 3 - 2 * i>{}, std::true_type{}, -1, fb, fa);
    });
  };
  run(std::true_type{}, NOWN);
  waitpoint(0);
  __builtin_amdgcn_s_waitcnt(0xC07F);
  __builtin_amdgcn_s_barrier();
  run(std::false_type{}, NRCV);
  const u32 tmv = __hip_atomic_load((const GAS u32*)tmo, __ATOMIC_RELAXED, __HIP_MEMORY_SCOPE_AGENT);
  __builtin_amdgcn_s_waitcnt(0xC07F);
  __builtin_amdgcn_s_barrier();
  CBAR();
  constexpr u32 EROW = 528;
#pragma unroll
  for (int mi = 0; mi < MI; ++mi) {
    const u32 mr = wm * 32 * MI + mi * 32 + (lane & 31);
#pragma unroll
    for (int gq = 0; gq < 4; ++gq) {
      const f32x4 o = {acc[mi][4 * gq], acc[mi][4 * gq + 1], acc[mi][4 * gq + 2], acc[mi][4 * gq + 3]};
      *(LAS f32x4*)(lds + mr * EROW + (wn * 32 + 8 * gq + 4 * h) * 4) = o;
    }
  }
  __syncthreads();
  const u32 bad = (*lbad || tmv == seq) ? NAN2 : 0u;
  const u32 keep = rank >> 2;
  const bool do_exp = !HALF || hsel != keep || m < NX, do_keep = !HALF || hsel == keep || m < NX;
  const u32 me = HALF && hsel == keep ? xbase(seq) + m : m, mk = HALF && hsel != keep ? xbase(seq) + m : m;
  if (do_exp) {
    const u32 e16 = P16_OFF + me * 16384 + wid * 1024;
#pragma unroll
    for (int part = 0; part < 2; ++part) {
      const LAS char* se = lds + ((HALF ? 0 : (keep ^ 1) * 64) + part * 32 + (tid >> 4)) * EROW + (tid & 15) * 32;
      const f32x4 c0 = *(const LAS f32x4*)se, c1 = *(const LAS f32x4*)(se + 16);
      const bf16x8 o = {(__bf16)c0[0], (__bf16)c0[1], (__bf16)c0[2], (__bf16)c0[3],
                        (__bf16)c1[0], (__bf16)c1[1], (__bf16)c1[2], (__bf16)c1[3]};
      st128(__builtin_bit_cast(u32x4, o) | bad, rs, lane * 16, e16 + part * 8192);
    }
  }
  if (do_keep) {
    const u32 e32 = P32_OFF + mk * 32768 + wid * 2048;
#pragma unroll
    for (int part = 0; part < 2; ++part) {
      const LAS char* sk = lds + ((HALF ? 0 : keep * 64) + part * 32 + (tid >> 4)) * EROW + (tid & 15) * 32;
      const u32x4 c0 = *(const LAS u32x4*)sk, c1 = *(const LAS u32x4*)(sk + 16);
      st128(c0 | bad, rs, lane * 16, e32 + part * 16384);
      st128(c1 | bad, rs, lane * 16, e32 + part * 16384 + 1024);
    }
  }
  WAIT_VM(0);
  __syncthreads();
  if (tid == 0 && do_exp) st_sys(p.ws[PL] + FPR_OFF + me * FS, seq);
  if (tid == 64 && do_keep) st_sys(ws + FPL_OFF + mk * FS, seq);
}

__device__ __forceinline__ void stage_unit(const Params& p, u32 lk, u32 seq) {
  const int tid = threadIdx.x, lane = tid & 63;
  const u32 wid = rfl(tid >> 6);
  const u32 lev = lk >> 2, kt = (wid >> 2) * 4 + (lk & 3), op = (wid >> 1) & 1, rc = (wid & 1) * 2048 + lev * 128;
  const auto r = mk_rsrc(op ? p.w : p.x, X_BYTES);
  const auto rs = mk_rsrc(p.ws[0], DATA_BYTES);
  const u32 vo = op ? (lane >> 4) * 8192 + (lane & 15) * 16 : (lane >> 3) * 1024 + (lane & 7) * 16;
  const u32 so = op ? kt * 64 * 8192 + rc * 2 : rc * 1024 + kt * 128;
  u32x4 v[16];
#pragma unroll
  for (int i = 0; i < 16; ++i) v[i] = __builtin_amdgcn_raw_buffer_load_b128(r, vo, so + i * (op ? 4 * 8192 : 8 * 1024), 0);
#pragma unroll
  for (int i = 0; i < 16; ++i) st128(v[i], rs, lane * 16, EX_OFF + (lk * 8 + wid) * BLK + i * 1024);
  WAIT_VM(0);
  __syncthreads();
  if (tid >= 1 && tid < NR && tid != PL) st_sys(p.ws[tid] + SG_OFF + (p.rank * 64 + lk) * FS, seq);
}

__device__ __forceinline__ void gemm_role(const Params& p, u32 k, u32 seq, LAS char* lds) {
  const u64 tstart = __builtin_amdgcn_s_memrealtime();
  constexpr u32 NG = 8 * GEMM_WGS;
  static_assert(2 * NHALF <= NG && 256 + NHALF <= 2 * NG, "unit schedule");
  if (k < 2 * NHALF) gemm_tile<true>(p, k >> 1, k & 1, seq, tstart, true, lds);
  for (u32 n = k < 2 * NHALF ? k + NG : k; n < 256 + NHALF; n += NG) gemm_tile<false>(p, n - NHALF, 0, seq, tstart, n < NG, lds);
}

__global__ void __launch_bounds__(THREADS, 2) fused_gemm_allreduce_silu(Params p) {
  extern __shared__ __attribute__((aligned(16))) char smem[];
  const int lane = threadIdx.x & 63;
  const u32 wid = rfl(threadIdx.x >> 6);
  LAS u32* ctl = (LAS u32*)smem;
  LAS u32* lctl = (LAS u32*)(smem + LDS_CTL);
  u32 slot = (blockIdx.x & ~7u) | ((u32)__builtin_amdgcn_s_getreg((3 << 11) | 20) & 7);
  auto is_push = [&]() { return slot < 8 * ENG_WGS && (slot & 3) != 0 && wid < PUSH_WAVES; };
  if (__builtin_expect(is_push(), 1)) {
    if (push_role<false>(p, slot, wid, lane, lctl)) return;
  } else {
    if (wid == 7) {
      if (lane < 32) ctl[lane] = 0;
      if (lane == 0) {
        GAS u32* ctr = (GAS u32*)(p.ws[0] + CTR_OFF) + blockIdx.x;
        GAS u32* clm = (GAS u32*)(p.ws[0] + CLM_OFF);
        const u32 c = __hip_atomic_load(ctr, __ATOMIC_RELAXED, __HIP_MEMORY_SCOPE_AGENT);
        u32 sl = slot;
#pragma unroll 1
        for (int n = 0; n < 4 * GRID; ++n) {
          if (__hip_atomic_fetch_add(clm + sl * 16, 1u, __ATOMIC_RELAXED, __HIP_MEMORY_SCOPE_AGENT) == c) break;
          __hip_atomic_fetch_add(clm + sl * 16, ~0u, __ATOMIC_RELAXED, __HIP_MEMORY_SCOPE_AGENT);
          sl = (sl + 1) & (GRID - 1);
        }
        lctl[0] = 0; lctl[1] = sl; lctl[2] = c + 1;
      }
    }
    asm volatile("s_waitcnt lgkmcnt(0)" ::: "memory");
    __builtin_amdgcn_s_barrier();
    slot = rfl(LD_LDS(lctl + 1));
    if (threadIdx.x == THREADS - 1)
      __hip_atomic_store((GAS u32*)(p.ws[0] + CTR_OFF) + blockIdx.x, LD_LDS(lctl + 2), __ATOMIC_RELAXED, __HIP_MEMORY_SCOPE_AGENT);
    WAIT_VM(0);
  }
  if (__builtin_expect(is_push(), 0)) {
    push_role<true>(p, slot, wid, lane, lctl);
    return;
  }
  const u32 seq = rfl(LD_LDS(lctl + 2));
  const u32 g = slot & 7, s = slot >> 3;
  const u32 peer = ((u32)p.rank ^ g) & (NR - 1);
  constexpr u32 LK0 = NPUSH / 4;
  if (s < ENG_WGS) {
    if (g == 0) {
      for (u32 u = s; u < 64 - LK0; u += u < 16 ? 16 : u < 24 ? 16 : 8) stage_unit(p, LK0 + u, seq);
    } else if (wid >= PUSH_WAVES && wid < PUSH_WAVES + PULL_WAVES) {
      pull_role(p, peer, (wid - PUSH_WAVES) * ENG_WGS + s, lane, seq, ctl);
    } else if (wid == PUSH_WAVES + PULL_WAVES) {
      flag_role(p, g, lane, seq, ctl);
    }
  } else if (s < ENG_WGS + FIN_WGS) {
    if ((g & 3) == 0) { stage_unit(p, LK0 + 8 + g + s - ENG_WGS, seq); stage_unit(p, LK0 + 24 + g + s - ENG_WGS, seq); }
    fin_role(p, (s - ENG_WGS) * 8 + g, seq);
  } else {
    const u32 k = s - ENG_WGS - FIN_WGS;
    gemm_role(p, (k >> 2) * 32 + g * 4 + (k & 3), seq, (LAS char*)smem);
  }
}

}

extern "C" void kernel_launch(int rank, int nranks, ncclComm_t comm,
                              hipStream_t stream,
                              void* const* d_in, const int* in_sizes, int n_in,
                              void* d_out, int out_size,
                              const int* in_dtypes, int out_dtype,
                              void* const* d_out_all,
                              void* const* d_ws_all, size_t ws_bytes) {
  if (nranks != NR || (unsigned)rank >= (unsigned)NR || n_in < 2 || ws_bytes < WS_USED ||
      in_sizes[0] != 4096 * 512 || in_sizes[1] != 512 * 4096 || out_size != 4096 * 4096) {
    fprintf(stderr, "kernel_launch: unexpected problem shape\n");
    return;
  }
  HIP_CHECK(hipFuncSetAttribute((const void*)fused_gemm_allreduce_silu,
                                hipFuncAttributeMaxDynamicSharedMemorySize, LDS_BYTES));
  Params p;
  p.x = (const char*)d_in[0];
  p.w = (const char*)d_in[1];
  const int phys[NR] = {0, 2, 3, 6, 1, 4, 5, 7};
  p.rank = 0; p.pad = 0;
  for (int i = 0; i < NR; ++i) if (phys[i] == rank) p.rank = i;
  for (int i = 0; i < NR; ++i) { p.ws[i] = (char*)d_ws_all[phys[p.rank ^ i]]; p.out[i] = (char*)d_out_all[phys[p.rank ^ i]]; }
  for (u32 g = 0; g < NR; ++g)
    for (u32 job = 0; job < 16; ++job) {
      const u32 peer = p.rank ^ g, lev = job >> 4, kt = (peer >> 2) * 4 + ((job >> 2) & 3), op = (job >> 1) & 1, half = job & 1;
      p.psrc[g][job] = op ? (kt * 64 + half * 32) * 8192 + ((peer & 1) * 2048 + lev * 128) * 2
                                         : (((peer >> 1) & 1) * 2048 + lev * 128 + half * 64) * 1024 + kt * 128;
    }
  hipLaunchKernelGGL(fused_gemm_allreduce_silu, dim3(GRID), dim3(THREADS), LDS_BYTES, stream, p);
  HIP_CHECK(hipGetLastError());
}
```

```cpp
#include <hip/hip_runtime.h>
#include <rccl/rccl.h>
#include <cstdio>
#include <type_traits>

#define HIP_CHECK(expr) do { hipError_t _e = (expr); if (_e != hipSuccess) { \
    fprintf(stderr, "HIP error %s at %s:%d\n", hipGetErrorString(_e), __FILE__, __LINE__); \
    return; } } while (0)

namespace {

typedef unsigned int u32;
typedef unsigned long long u64;
typedef u32 u32x4 __attribute__((ext_vector_type(4)));
typedef float f32x4 __attribute__((ext_vector_type(4)));
typedef __bf16 bf16x4 __attribute__((ext_vector_type(4)));
typedef __bf16 bf16x8 __attribute__((ext_vector_type(8)));
typedef float f32x16 __attribute__((ext_vector_type(16)));
#define LAS __attribute__((address_space(3)))
#define GAS __attribute__((address_space(1)))

constexpr int NR = 8;
constexpr u32 X_BYTES = 4096u * 512u * 2u;
constexpr u32 W_BYTES = 512u * 4096u * 2u;
constexpr u32 OUT_BYTES = 4096u * 4096u * 2u;
constexpr u32 FS = 128;

constexpr u32 BLK = 16384;
constexpr u32 IN_OFF = 0;
constexpr u32 IN_SRC = 128 * BLK;
constexpr u32 P16_OFF = IN_OFF + NR * IN_SRC;
constexpr u32 PIN_OFF = P16_OFF + 320 * 16384;
constexpr u32 P32_OFF = PIN_OFF + 320 * 16384;
constexpr u32 EX_OFF = P32_OFF + 320 * 32768;
constexpr u32 DATA_BYTES = EX_OFF + 512 * BLK;
constexpr u32 FI_OFF = DATA_BYTES;
constexpr u32 FPR_OFF = FI_OFF + NR * 256 * FS;
constexpr u32 FPL_OFF = FPR_OFF + 320 * FS;
constexpr u32 FPI_OFF = FPL_OFF + 320 * FS;
constexpr u32 FF_OFF = FPI_OFF + 1280 * FS;
constexpr u32 SG_OFF = FF_OFF + NR * 256 * FS;
constexpr u32 GO_OFF = SG_OFF + NR * 64 * FS;
constexpr u32 RF_OFF = GO_OFF + NR * 16 * FS;
constexpr u32 QC_OFF = RF_OFF + 64 * FS;
constexpr u32 CTR_OFF = QC_OFF + 128 * FS;
constexpr u32 CLM_OFF = CTR_OFF + 256 * 4;
constexpr u32 TMO_OFF = CLM_OFF + 256 * 64;
constexpr u32 TICK_OFF = TMO_OFF + 128;
constexpr u32 SEG_OFF = TICK_OFF + 32 * 128;
constexpr u32 WS_USED = SEG_OFF + 8 * 128;

constexpr u64 DEADLINE_TICKS = 80000000ull;

constexpr int ENG_WGS = 8, FIN_WGS = 4, GEMM_WGS = 20;
constexpr int PUSH_WAVES = 2, PULL_WAVES = 2;
constexpr int PUSH_D = 4, PUSH_ROUNDS = 4;
constexpr u32 NPUSH = PUSH_ROUNDS * ENG_WGS * PUSH_WAVES;
constexpr u32 PL = 4;
constexpr int GO_AT = 6;
constexpr int NS_HALF = 6;
constexpr u32 NQUART = 25, NHALF = 55;
constexpr u64 STAGGER = 190;
constexpr u32 NX = 20, TOPK = 64, SWK = 1;
constexpr int SL_PULL = 2, SL_FIN = 4, SL_GEMM = 8;
constexpr u32 NAN2 = 0x7FC07FC0u;
constexpr int GRID = 8 * (ENG_WGS + FIN_WGS + GEMM_WGS);
static_assert(GRID == 256, "slots wrap at a power of two");
constexpr int THREADS = 512;
constexpr u32 LDS_CTL = 144 * 1024;
constexpr u32 LDS_BYTES = LDS_CTL + 64;
constexpr int AUX_SYS = 17;

struct Params {
  const char* x;
  const char* w;
  int rank;
  int pad;
  char* ws[NR];
  char* out[NR];
  u32 psrc[NR][16];
};

__device__ __forceinline__ u32 rfl(u32 v) { return __builtin_amdgcn_readfirstlane(v); }
__device__ __forceinline__ __amdgpu_buffer_rsrc_t mk_rsrc(const void* p, u32 n) {
  const u64 a = (u64)p;
  const u64 u = ((u64)rfl((u32)(a >> 32)) << 32) | rfl((u32)a);
  return __builtin_amdgcn_make_buffer_rsrc((void*)u, 0, n, 0x00020000);
}
__device__ __forceinline__ u32 ld_sys(const char* p) {
  return __hip_atomic_load((const GAS u32*)p, __ATOMIC_RELAXED, __HIP_MEMORY_SCOPE_SYSTEM);
}
__device__ __forceinline__ void st_sys(char* p, u32 v) {
  __hip_atomic_store((GAS u32*)p, v, __ATOMIC_RELAXED, __HIP_MEMORY_SCOPE_SYSTEM);
}
#define WAIT_VM(n) asm volatile("s_waitcnt vmcnt(%0)" ::"n"(n) : "memory")
#define CBAR() asm volatile("" ::: "memory")
#define LD_LDS(q) __hip_atomic_load(q, __ATOMIC_RELAXED, __HIP_MEMORY_SCOPE_WORKGROUP)
#define ST_LDS(q, v) __hip_atomic_store(q, v, __ATOMIC_RELAXED, __HIP_MEMORY_SCOPE_WORKGROUP)

__device__ __forceinline__ void st128(u32x4 v, __amdgpu_buffer_rsrc_t r, u32 voff, u32 soff) {
  __builtin_amdgcn_raw_buffer_store_b128(v, r, voff, soff, AUX_SYS);
  asm volatile("s_nop 1" ::"v"(v) : "memory");
}

template <int SLEEP>
__device__ __forceinline__ u64 wave_wait(const char* flag, u64 need, u32 seq, char* tmo, bool& bad) {
  const u64 t0 = __builtin_amdgcn_s_memrealtime();
  for (u32 it = 1;; ++it) {
    const u64 ok = __ballot(ld_sys(flag) >= seq);
    if ((ok & need) == need) return ok;
    if ((it & 15) == 0 || bad) {
      const u32 t = __hip_atomic_load((const GAS u32*)tmo, __ATOMIC_RELAXED, __HIP_MEMORY_SCOPE_AGENT);
      if (rfl(t) == seq || __builtin_amdgcn_s_memrealtime() - t0 > DEADLINE_TICKS) {
        __hip_atomic_store((GAS u32*)tmo, seq, __ATOMIC_RELAXED, __HIP_MEMORY_SCOPE_AGENT);
        bad = true;
        return ~0ull;
      }
    }
    __builtin_amdgcn_s_sleep(SLEEP);
  }
}

template <int SLEEP>
__device__ __forceinline__ u64 wave_wait2(const char* flag, u64 need, u32 seq, char* tmo, bool& bad) {
  const u64 t0 = __builtin_amdgcn_s_memrealtime();
  const bool needed = (need >> (threadIdx.x & 63)) & 1;
  bool ok = false;
  for (u32 it = 1;; ++it) {
    if (!ok && (it == 1 || needed)) ok = ld_sys(flag) >= seq;
    const u64 have = __ballot(ok);
    if ((have & need) == need) return have;
    if ((it & 15) == 0) {
      const u32 t = __hip_atomic_load((const GAS u32*)tmo, __ATOMIC_RELAXED, __HIP_MEMORY_SCOPE_AGENT);
      if (rfl(t) == seq || __builtin_amdgcn_s_memrealtime() - t0 > DEADLINE_TICKS) {
        __hip_atomic_store((GAS u32*)tmo, seq, __ATOMIC_RELAXED, __HIP_MEMORY_SCOPE_AGENT);
        bad = true;
        return ~0ull;
      }
    }
    __builtin_amdgcn_s_sleep(SLEEP);
  }
}

__device__ __forceinline__ void dma4(u32 dst, __amdgpu_buffer_rsrc_t ra, u32 va0, u32 va1, u32 sa,
                                     __amdgpu_buffer_rsrc_t rb, u32 vb0, u32 vb1, u32 sb) {
  u32 keep;
  dst = rfl(dst); sa = rfl(sa); sb = rfl(sb);
  asm volatile(
      "s_mov_b32 %0, m0\n\t"
      "s_nop 4\n\t"
      "s_mov_b32 m0, %1\n\t"
      "s_nop 0\n\t"
      "buffer_load_dwordx4 %3, %5, %6 offen lds\n\t"
      "buffer_load_dwordx4 %4, %5, %6 offen offset:1024 lds\n\t"
      "s_mov_b32 m0, %2\n\t"
      "s_nop 0\n\t"
      "buffer_load_dwordx4 %7, %9, %10 offen lds\n\t"
      "buffer_load_dwordx4 %8, %9, %10 offen offset:1024 lds\n\t"
      "s_mov_b32 m0, %0"
      : "=&s"(keep)
      : "s"(dst), "s"(dst + BLK), "v"(va0), "v"(va1), "s"(ra), "s"(sa), "v"(vb0), "v"(vb1), "s"(rb), "s"(sb)
      : "memory");
}

__device__ __forceinline__ void dma3(u32 dsta, u32 dstb, __amdgpu_buffer_rsrc_t ra, u32 va0, u32 sa,
                                     __amdgpu_buffer_rsrc_t rb, u32 vb0, u32 vb1, u32 sb) {
  u32 keep;
  dsta = rfl(dsta); dstb = rfl(dstb); sa = rfl(sa); sb = rfl(sb);
  asm volatile(
      "s_mov_b32 %0, m0\n\t"
      "s_nop 4\n\t"
      "s_mov_b32 m0, %1\n\t"
      "s_nop 0\n\t"
      "buffer_load_dwordx4 %3, %4, %5 offen lds\n\t"
      "s_mov_b32 m0, %2\n\t"
      "s_nop 0\n\t"
      "buffer_load_dwordx4 %6, %8, %9 offen lds\n\t"
      "buffer_load_dwordx4 %7, %8, %9 offen offset:1024 lds\n\t"
      "s_mov_b32 m0, %0"
      : "=&s"(keep)
      : "s"(dsta), "s"(dstb), "v"(va0), "s"(ra), "s"(sa), "v"(vb0), "v"(vb1), "s"(rb), "s"(sb)
      : "memory");
}

template <int I, int N, class F>
__device__ __forceinline__ void static_for(F&& f) {
  if constexpr (I < N) { f(std::integral_constant<int, I>{}); static_for<I + 1, N>(f); }
}

__device__ __forceinline__ void tile_rc(u32 m, u32& ri, u32& ci) {
  u32 l = 0;
#pragma unroll
  for (u32 t = 1; t < 16; ++t) l += m >= t * t;
  const u32 idx = m - l * l;
  ri = idx < l ? l : idx - l;
  ci = idx < l ? idx : l;
}
__device__ __forceinline__ u32 fin_base(u32 r) { return (((r >> 1) & 1) * 2048 + (r >> 2) * 64) * 8192 + (r & 1) * 4096; }

template <bool claimed>
__device__ __forceinline__ bool push_role(const Params& p, u32& slot, u32 wid, int lane, LAS u32* lctl) {
  constexpr int D = PUSH_D, JI = 8;
  constexpr u32 NW = ENG_WGS * PUSH_WAVES;
  static_assert(PUSH_ROUNDS >= 3 && NW == 16 && X_BYTES == W_BYTES && GO_AT <= JI, "push schedule");
  const u32 g = slot & 7, w = wid * ENG_WGS + (slot >> 3), op = (slot >> 4) & 1;
  u64 bs = (u64)(&p.x)[op], bw = (u64)p.ws[g];
  u32 src = p.psrc[g][w];
  u32 rank = p.rank;
  asm volatile("" : "+s"(bs), "+s"(bw), "+s"(src), "+s"(rank));
  const auto rs = mk_rsrc((const void*)bs, X_BYTES);
  const u32 vs = op ? (lane >> 4) * 8192 + (lane & 15) * 16 : (lane >> 3) * 1024 + (lane & 7) * 16;
  const u32 str = op ? 4 * 8192 : 8 * 1024;
  auto ld = [&](int r, int i) -> u32x4 { return __builtin_amdgcn_raw_buffer_load_b128(rs, vs, src + r * (op ? 256 : 128 * 1024) + i * str, 0); };
  u32x4 v[D], v2[JI - D];
#pragma unroll
  for (int i = 0; i < D; ++i) { v[i] = ld(0, i); CBAR(); }
#pragma unroll
  for (int i = D; i < JI; ++i) { v2[i - D] = ld(0, i); CBAR(); }
  char* wsp = (char*)bw;
  const auto rd = mk_rsrc(wsp, DATA_BYTES);
  const u32 dst = IN_OFF + rank * IN_SRC + w * 8192, flag = FI_OFF + (rank * 256 + w) * FS;
  u32 seq = 0;
  auto round = [&](auto rc) {
    constexpr int R = decltype(rc)::value;
    constexpr bool FIRST = R == 0, LAST = R == PUSH_ROUNDS - 1;
#pragma unroll
    for (int i = 0; i < JI; ++i) {
      if (!FIRST && i == D - 1) {
        WAIT_VM(2 * D - 1);
        if (R == 1) {
          if (!claimed) __builtin_amdgcn_s_barrier();
          CBAR();
          seq = rfl(LD_LDS(lctl + 2));
        }
        if (lane == 0) st_sys(wsp + flag + (R - 1) * NW * FS, seq);
        CBAR();
      }
      if (LAST && i == JI - GO_AT) {
        if (lane == 0) st_sys(wsp + GO_OFF + (rank * 16 + w) * FS, seq);
        if (lane == 0) ST_LDS(lctl - LDS_CTL / 4 + 24 + wid, 1u);
        CBAR();
      }
      st128(FIRST && i >= D ? v2[i >= D ? i - D : 0] : v[i % D], rd, lane * 16, dst + R * NW * 8192 + i * 1024);
      if (i + D < JI) { if (!FIRST) v[i % D] = ld(R, i + D); }
      else if (!LAST) v[i % D] = ld(LAST ? R : R + 1, i + D - JI);
      CBAR();
    }
  };
  static_assert(JI - D >= D - 1, "the count above also holds in the last round");
  round(std::integral_constant<int, 0>{});
  round(std::integral_constant<int, 1>{});
  const u32 slot2 = rfl(LD_LDS(lctl + 1));
  if (__builtin_expect(slot2 != slot, 0)) { WAIT_VM(0); slot = slot2; return false; }
  static_for<2, PUSH_ROUNDS>(round);
  WAIT_VM(0);
  if (lane == 0) st_sys(wsp + flag + (PUSH_ROUNDS - 1) * NW * FS, seq);
  return true;
}

__device__ __forceinline__ float silu(float a) {
  const float c = fminf(fmaxf(a, -60.f), 60.f);
  return a * __builtin_amdgcn_rcpf(1.f + __expf(-c));
}

constexpr u32 MB_RING = 0, MB_END = 16, MB_PUSHED = 24;

constexpr u32 NE = 4 * NX, NPOOL = (1024 - NE) / 2;
static_assert(NX <= NHALF && NX <= 32, "early tiles");
__device__ __forceinline__ u32 xbase(u32 seq) { return 256 + (seq & 1) * 32; }
__device__ __forceinline__ u32 pool_job(u32 pos) {
  return pos < NPOOL - 2 * TOPK ? pos : pos >= NPOOL - TOPK ? pos - TOPK : pos + TOPK;
}

__device__ __forceinline__ void pull_role(const Params& p, u32 peer, u32 w, int lane, u32 seq, LAS u32* ctl) {
  constexpr u32 NW = ENG_WGS * PULL_WAVES, NONE = ~0u, REL = 1u << 31;
  const u32 rank = p.rank, g = rank ^ peer;
  const bool pl = g == PL;
  const u32 NI = pl ? 1280 : 256 - NPUSH, NJ = pl ? 1024 + 4 * NX : NI + 1024;
  char* ws = p.ws[0];
  char* tmo = ws + TMO_OFF;
  const u64 bf = (u64)p.out[g], bp = (u64)p.ws[g], bo = (u64)p.out[0];
  const u32 vwide = (lane >> 4) * 8192 + (lane & 15) * 16;
  const auto rtick = mk_rsrc(ws + TICK_OFF + (seq & 1) * 2048, 2048);
  if (w == 0 && (lane == 0 || (pl && lane < 3))) st_sys(ws + TICK_OFF + ((seq & 1) ^ 1) * 2048 + (lane ? 7 + lane : g) * 128, 0);
  const u32 bi = (rank >> 1) & 1, bj = rank & 1;
  const u32 isrc = pl ? P16_OFF : EX_OFF + (NPUSH / 4 * 8 + (rank >> 2) * 4 + bi) * BLK;
  const u32 is1 = pl ? BLK : 8 * BLK, is2 = pl ? 8192 : (2 + bj - bi) * BLK, isz = pl ? 4096 : 8192;
  const u32 idst = pl ? PIN_OFF : IN_OFF + peer * IN_SRC + NPUSH * 8192;
  const u32 ipre = pl ? FPR_OFF : SG_OFF + (peer * 64 + NPUSH / 4) * FS, ish = 2;
  const u32 ipost = pl ? FPI_OFF : FI_OFF + (peer * 256 + NPUSH) * FS;
  const u32 fown = fin_base(peer), fpar = fin_base(rank ^ PL);
  const u32 rpre = pl ? FF_OFF + peer * 256 * FS : RF_OFF + g * 8 * FS, rsh = pl ? 2 : 7;
  u32 ttile = 0;
#pragma unroll
  for (u32 e = 0; e < 4; ++e) {
    u32 ri, ci;
    tile_rc(lane * 4 + e, ri, ci);
    ttile |= (ri << 4 | ci) << (8 * e);
  }
  const u64 t0 = __builtin_amdgcn_s_memrealtime();
  u32 tf = w, tf2 = NW + w, lastf = 0, sub = 0;
  u32 cdst = 0, cact = 0, pend = 0, ack = 0;
  u32 nring = 0;
  u32 word = g;
  bool bad = false, cisf = false;
  auto poll = [&]() -> u32 {
    const u32 o = tf < NI ? ipre + (tf >> ish) * FS : tf < REL ? FF_OFF + (peer * 256 + ((tf - NI) >> 2)) * FS : rpre + ((tf & 1023) >> rsh) * FS;
    return ld_sys(ws + rfl(o));
  };
  u32 pv = 0;
  u32x4 v[4];
  auto draw = [&](bool on) -> u32 {
    return __builtin_amdgcn_raw_ptr_buffer_atomic_add_i32(word >= 8 && pl ? 1u << 16 : 1u, rtick, on && word && lane == 0 ? 0u : 0x80000000u, word * 128, 0);
  };
  u32 sstart, sbase;
  {
    const u32 l = lane >> 1, xb = min(l * l, NX);
    sstart = 4 * (((lane & 1) ? (l + 1) * (l + 1) : l * l) + xb);
    sbase = 4 * ((lane & 1) ? xbase(seq) + xb : l * l);
  }
  auto handle = [&](u32 id) -> u32 {
    if (!pl) return id;
    const u32 sg = 63 - __builtin_clzll(__ballot(sstart <= id));
    return __builtin_amdgcn_readlane(sbase, sg) + id - __builtin_amdgcn_readlane(sstart, sg);
  };
  tf = handle(tf); tf2 = handle(tf2);
  auto drawn = [&](u32 t) -> u32 {
    t = rfl(t);
    if (word == 0) return NONE;
    if (word < 8) {
      const u32 id = t + 2 * NW;
      if (id + 1 + SWK * w >= NJ) word = 8 + (w & 1);
      if (__builtin_expect(id < NJ, 1)) return handle(id);
      t = rfl(draw(true));
    }
    const u32 f = t & 0xFFFF, b = t >> 16;
    if (f + b >= NPOOL) { word = 0; return NONE; }
    return REL | (NE + pool_job(pl ? NPOOL - 1 - b : f) * 2 + (w & 1));
  };
  auto put = [&](int i) {
    const auto ro = mk_rsrc((const void*)(cisf ? bo : (u64)ws), cisf ? OUT_BYTES : DATA_BYTES);
    st128(v[i], ro, cisf ? vwide : lane * 16, rfl(cdst + i * (cisf ? 4 * 8192 : 1024)));
  };
  auto done = [&](u32 a) {
    if (a) {
      LAS u32* e = ctl + MB_RING + (w / ENG_WGS) * 8 + (nring++ & 7);
      while (rfl(LD_LDS(e)) != 0) __builtin_amdgcn_s_sleep(1);
      if (lane == 0) ST_LDS(e, a);
      CBAR();
    }
  };
  auto ready = [&]() -> bool { return tf != NONE && (bad || (__ballot(pv >= seq) & 1)); };
  auto round = [&](auto first_c) {
    constexpr bool FIRST = decltype(first_c)::value;
    const u32 h = tf, hs = sub;
    const bool isf = h >= NI, tix = isf || hs || pl;
    sub = tix ? 0 : 1;
    tf = tix ? tf2 : tf;
    const u32 tk = draw(tix);
    pv = poll();
    CBAR();
    done(ack);
    u32 nsrc, ndst, nact;
    if (isf) {
      const u32 j = h < REL ? h - NI : h & 1023, rc = (__builtin_amdgcn_readlane(ttile, j >> 4) >> (2 * (j & 12))) & 255;
      nsrc = ndst = (h < REL ? fown : fpar) + ((rc >> 4) * 128 + (j & 3) * 16) * 8192 + (rc & 15) * 256;
      nact = h < REL ? (j >> 7) * 2 + 1 : 0;
      lastf = ndst;
    } else {
      nsrc = isrc + (h >> 2) * is1 + ((h >> 1) & 1) * is2 + (h & 1) * isz + hs * 4096;
      ndst = idst + h * isz + hs * 4096;
      nact = tix ? ipost + h * FS : 0;
    }
    const auto rsrc = mk_rsrc((const void*)(isf ? bf : bp), isf ? OUT_BYTES : DATA_BYTES);
    const u32 vs = isf ? vwide : lane * 16, str = isf ? 4 * 8192 : 1024;
#pragma unroll
    for (int i = 0; i < 4; ++i) {
      if (!FIRST) put(i);
      v[i] = __builtin_amdgcn_raw_buffer_load_b128(rsrc, vs, rfl(nsrc + i * str), AUX_SYS);
      CBAR();
    }
    ack = 0;
    if (!FIRST) {
      WAIT_VM(11);
      ack = pend; pend = cact;
    }
    cdst = ndst; cisf = isf; cact = nact;
    if (tix) tf2 = drawn(tk);
  };
  if (!pl) {
    while (rfl(LD_LDS(ctl + MB_PUSHED + w / ENG_WGS)) == 0) __builtin_amdgcn_s_sleep(2);
    wave_wait<1>(ws + ((lane & 1) ? GO_OFF + (peer * 16 + w) * FS : ipre + (w >> ish) * FS), 3, seq, tmo, bad);
    if (bad) WAIT_VM(0);
    pv = seq;
  }
  for (u32 it = 1; tf != NONE; ++it) {
    if (ready()) {
      round(std::true_type{});
      if (ready()) {
        round(std::false_type{});
        while (ready()) round(std::false_type{});
      }
#pragma unroll
      for (int i = 0; i < 4; ++i) put(i);
      done(ack);
      if (pend | cact) {
        WAIT_VM(0);
        done(pend);
        done(cact);
      }
      ack = pend = 0;
      continue;
    }
    if ((it & 63) == 0) {
      const u32 t = __hip_atomic_load((const GAS u32*)tmo, __ATOMIC_RELAXED, __HIP_MEMORY_SCOPE_AGENT);
      if (rfl(t) == seq || __builtin_amdgcn_s_memrealtime() - t0 > DEADLINE_TICKS) {
        __hip_atomic_store((GAS u32*)tmo, seq, __ATOMIC_RELAXED, __HIP_MEMORY_SCOPE_AGENT);
        WAIT_VM(0);
        bad = true;
      }
    }
    __builtin_amdgcn_s_sleep(SL_PULL * 3);
    pv = poll();
  }
  if (bad) {
    WAIT_VM(0);
    const u32x4 nanv = {NAN2, NAN2, NAN2, NAN2};
    for (int i = 0; i < 4; ++i) st128(nanv, mk_rsrc(p.out[0], OUT_BYTES), vwide, rfl(lastf) + i * 4 * 8192);
  }
  if (lane == 0) __hip_atomic_fetch_add(ctl + MB_END, 1u, __ATOMIC_RELAXED, __HIP_MEMORY_SCOPE_WORKGROUP);
}

__device__ __forceinline__ void flag_role(const Params& p, u32 g, int lane, u32 seq, LAS u32* ctl) {
  char* ws = p.ws[0];
  char* rf = p.ws[g ^ PL] + RF_OFF + (g ^ PL) * 8 * FS;
  for (;;) {
    const u32 end = rfl(LD_LDS(ctl + MB_END));
    const u32 a = lane < 16 ? LD_LDS(ctl + MB_RING + lane) : 0;
    if (a) {
      ST_LDS(ctl + MB_RING + lane, 0u);
      if (!(a & 1)) st_sys(ws + a, seq);
      else if (__hip_atomic_fetch_add((GAS u32*)(ws + SEG_OFF + g * 128) + (a >> 1), 1u, __ATOMIC_RELAXED, __HIP_MEMORY_SCOPE_AGENT) == 128 * seq - 1)
        st_sys(rf + (a >> 1) * FS, seq);
    }
    if (end == PULL_WAVES && __ballot(a != 0) == 0) break;
    __builtin_amdgcn_s_sleep(2);
  }
}

__device__ __forceinline__ void fin_role(const Params& p, u32 f, u32 seq) {
  const int tid = threadIdx.x, lane = tid & 63;
  const u32 wid = rfl(tid >> 6), rank = p.rank;
  char* ws = p.ws[0];
  char* tmo = ws + TMO_OFF;
  const auto rl = mk_rsrc(ws, DATA_BYTES);
  const auto ro = mk_rsrc(p.out[0], OUT_BYTES);
  const u32 vwide = (lane >> 4) * 8192 + (lane & 15) * 16;
  char* wsl = p.ws[lane & 7];
  for (u32 n = f; n < 256 + NX; n += 8 * FIN_WGS) {
    const u32 m = n < 256 ? n : xbase(seq) + n - 256;
    u32 ri, ci;
    tile_rc(n & 255, ri, ci);
    const u32 item = wid * 2, dst = fin_base(m < 256 ? rank : rank ^ PL) + (ri * 128 + item * 4) * 8192 + ci * 256;
    bool bad = false;
    wave_wait<SL_FIN>(ws + ((lane & 1) ? FPL_OFF + m * FS : FPI_OFF + (m * 4 + (wid >> 1)) * FS), 3, seq, tmo, bad);
    u32x4 r[2], a[2], b[2];
#pragma unroll
    for (int i = 0; i < 2; ++i) {
      r[i] = __builtin_amdgcn_raw_buffer_load_b128(rl, lane * 16, PIN_OFF + m * 16384 + (item + i) * 1024, AUX_SYS);
      a[i] = __builtin_amdgcn_raw_buffer_load_b128(rl, lane * 16, P32_OFF + m * 32768 + (item + i) * 2048, AUX_SYS);
      b[i] = __builtin_amdgcn_raw_buffer_load_b128(rl, lane * 16, P32_OFF + m * 32768 + (item + i) * 2048 + 1024, AUX_SYS);
    }
    bad |= __hip_atomic_load((const GAS u32*)tmo, __ATOMIC_RELAXED, __HIP_MEMORY_SCOPE_AGENT) == seq;
#pragma unroll
    for (int i = 0; i < 2; ++i) {
      float y[8];
#pragma unroll
      for (int e = 0; e < 4; ++e) {
        const u32 l0 = e < 2 ? a[i][2 * e] : b[i][2 * e - 4], l1 = e < 2 ? a[i][2 * e + 1] : b[i][2 * e - 3];
        y[2 * e] = silu(__uint_as_float(l0) + __uint_as_float(r[i][e] << 16));
        y[2 * e + 1] = silu(__uint_as_float(l1) + __uint_as_float(r[i][e] & 0xFFFF0000u));
      }
      const bf16x8 o = {(__bf16)y[0], (__bf16)y[1], (__bf16)y[2], (__bf16)y[3],
                        (__bf16)y[4], (__bf16)y[5], (__bf16)y[6], (__bf16)y[7]};
      st128(__builtin_bit_cast(u32x4, o) | (bad ? NAN2 : 0u), ro, vwide, dst + i * 4 * 8192);
    }
    WAIT_VM(0);
    __syncthreads();
    if (tid >= 1 && tid < NR && m < 256) st_sys(wsl + FF_OFF + (rank * 256 + m) * FS, seq);
  }
}

template <int KIND>
__device__ __forceinline__ void gemm_tile(const Params& p, u32 m, u32 hsel, u32 csel, u32 seq, u64 tstart, bool first, LAS char* lds) {
  const int tid = threadIdx.x, lane = tid & 63;
  const u32 wid = rfl(tid >> 6);
  const u32 rank = p.rank;
  char* ws = p.ws[0];
  char* tmo = ws + TMO_OFF;
  const auto rx = mk_rsrc(p.x, X_BYTES);
  const auto rw = mk_rsrc(p.w, W_BYTES);
  const auto rs = mk_rsrc(ws, DATA_BYTES);
  constexpr bool HALF = KIND == 1, QUART = KIND == 2;
  constexpr int KS = QUART ? 2 : 1, NOWN = 8 / KS, NRCV = 24 / KS;
  constexpr int MI = KIND ? 1 : 2, NDMA = HALF ? 3 : 4, NS = HALF ? NS_HALF : 4;
  constexpr u32 XB = HALF ? BLK / 2 : BLK, SB = XB + BLK;
  static_assert(NS * SB <= LDS_CTL, "stages fit");

  u32 vao[2], vbo[2], vaw[2], vbw[2];
#pragma unroll
  for (int q = 0; q < 2; ++q) {
    const u32 ra = wid * (HALF ? 8 : 16) + q * 8 + (lane >> 3), ca = (lane & 7) ^ ((ra >> 1) & 7);
    const u32 kb = wid * 8 + q * 4 + (lane >> 4), cbk = (lane & 15) ^ (((kb & 3) << 2) | ((kb >> 2) & 3));
    const u32 ap = QUART ? ra >> 6 : 0, ar = ra - ap * 64, bp = QUART ? cbk >> 3 : 0, bc = cbk - bp * 8;
    vao[q] = ar * 1024 + ca * 16 + ap * 128 - q * 1024;        vaw[q] = ar * 128 + ca * 16 + ap * 2 * BLK - q * 1024;
    vbo[q] = kb * 8192 + bc * 16 + bp * 64 * 8192 - q * 1024;  vbw[q] = kb * 256 + bc * 16 + bp * 2 * BLK - q * 1024;
  }
  const u32 lds0 = (u32)(size_t)lds;
  const u32 wm = wid & 1, wn = wid >> 1, h = lane >> 5;
  u32 xo[4], wo[2];
  {
    const u32 r = wm * 32 * MI + (lane & 31) + (QUART ? (wid >> 2) * 64 : 0);
#pragma unroll
    for (int ks = 0; ks < 4; ++ks) xo[ks] = r * 128 + (((2 * ks + h) ^ ((r >> 1) & 7)) * 16);
    const u32 bk = (lane >> 4) & 1, q = (lane & 15) >> 2, pp = lane & 3;
#pragma unroll
    for (int t = 0; t < 2; ++t)
      wo[t] = XB + 256 * (8 * h + 4 * t + q) + 16 * ((4 * wn + 2 * bk + (pp >> 1)) ^ ((q << 2) | (2 * h + t))) + 8 * (pp & 1);
  }
  LAS u32* lbad = (LAS u32*)(lds + LDS_CTL);
  u32 ri, ci;
  tile_rc(m, ri, ci);
  const u32 row0 = ((rank >> 1) & 1) * 2048 + ri * 128 + hsel * 64, col0 = (rank & 1) * 2048 + ci * 128 + csel * 64;
  u32 islot = 0, rslot = 0;
  auto next_slot = [&](u32 o) -> u32 { return o + SB == NS * SB ? 0 : o + SB; };
  auto issue_own = [&](int t) {
    const u32 dst = lds0 + islot, sa = row0 * 1024 + t * KS * 128, sw = t * KS * 64 * 8192 + col0 * 2;
    islot = next_slot(islot);
    if (HALF) dma3(dst + wid * 1024, dst + XB + wid * 2048, rx, vao[0], sa, rw, vbo[0], vbo[1], sw);
    else dma4(dst + wid * 2048, rx, vao[0], vao[1], sa, rw, vbo[0], vbo[1], sw);
  };
  u32 taba, tabw;
  {
    const u32 ktl = lane / 6, c = lane - ktl * 6, src = rank ^ (c + 1 + (c >= 3));
    const u32 base = IN_OFF + src * IN_SRC + ktl * KS * 2 * BLK;
    taba = base + ri * 8 * BLK + hsel * 8192;
    tabw = base + (ci * 8 + 1) * BLK + csel * 128;
  }
  auto issue = [&](int t) {
    const u32 dst = lds0 + islot, sa = __builtin_amdgcn_readlane(taba, t), sw = __builtin_amdgcn_readlane(tabw, t);
    islot = next_slot(islot);
    if (HALF) dma3(dst + wid * 1024, dst + XB + wid * 2048, rs, vaw[0], sa, rs, vbw[0], vbw[1], sw);
    else dma4(dst + wid * 2048, rs, vaw[0], vaw[1], sa, rs, vbw[0], vbw[1], sw);
  };
  const u32 fop = (lane >> 1) & 1, fq = min(lane >> 2, 11), fk = fq / 6, fc = fq - fk * 6;
  const char* fin = ws + FI_OFF + ((rank ^ (fc + 1 + (fc >= 3))) * 256 + (((fop ? ci : ri) * 4 + fk) * 2 + fop) * 2 + ((KIND && !fop) ? hsel : (lane & 1))) * FS;

  while (__builtin_amdgcn_s_memrealtime() - tstart < (u64)max(ri, ci) * STAGGER) __builtin_amdgcn_s_sleep(16);
  if (first && wid == 0) __builtin_amdgcn_fence(__ATOMIC_ACQUIRE, "");
  f32x16 acc[MI] = {};
  struct Frag { bf16x8 w[4], x[MI][4]; };
  auto read = [&](Frag& f) {
    const LAS char* st = lds + rslot;
    rslot = next_slot(rslot);
#pragma unroll
    for (int ks = 0; ks < 4; ++ks) {
      const bf16x4 w0 = __builtin_amdgcn_ds_read_tr16_b64_v4bf16((LAS bf16x4*)(st + wo[0] + ks * 4096));
      const bf16x4 w1 = __builtin_amdgcn_ds_read_tr16_b64_v4bf16((LAS bf16x4*)(st + wo[1] + ks * 4096));
      f.w[ks] = __builtin_shufflevector(w0, w1, 0, 1, 2, 3, 4, 5, 6, 7);
#pragma unroll
      for (int mi = 0; mi < MI; ++mi) f.x[mi][ks] = *(const LAS bf16x8*)(st + xo[ks] + mi * 4096);
    }
  };
  auto mma = [&](const Frag& f) {
#pragma unroll
    for (int ks = 0; ks < 4; ++ks)
#pragma unroll
      for (int mi = 0; mi < MI; ++mi) acc[mi] = __builtin_amdgcn_mfma_f32_32x32x16_bf16(f.w[ks], f.x[mi][ks], acc[mi], 0, 0, 0);
  };
  auto step = [&](auto rc, auto oc, int tn, const Frag& fc, Frag& fn) {
    constexpr int R = decltype(rc)::value;
    constexpr bool OWN = decltype(oc)::value;
    if constexpr (R >= 0) {
      WAIT_VM(R * NDMA);
      __builtin_amdgcn_s_waitcnt(0xC07F);
      CBAR();
      __builtin_amdgcn_s_barrier();
      CBAR();
      read(fn);
    }
    if (tn >= 0) { if (OWN) issue_own(tn); else issue(tn); }
    mma(fc);
  };
  Frag fa, fb;
  static_assert(NS % 2 == 0 && NS <= NOWN, "run structure");
  auto run = [&](auto oc, int n) {
    constexpr bool OWN = decltype(oc)::value;
    islot = rslot = 0;
    for (int t = 0; t < NS; ++t) { if (OWN) issue_own(t); else issue(t); }
    WAIT_VM((NS - 1) * NDMA);
    __builtin_amdgcn_s_barrier();
    CBAR();
    read(fa);
    for (int tn = NS; tn < n; tn += 2) {
      step(std::integral_constant<int, NS - 2>{}, oc, tn, fa, fb);
      step(std::integral_constant<int, NS - 2>{}, oc, tn + 1, fb, fa);
    }
    static_for<0, NS / 2>([&](auto ic) {
      constexpr int i = decltype(ic)::value;
      step(std::integral_constant<int, NS - 2 - 2 * i>{}, std::true_type{}, -1, fa, fb);
      step(std::integral_constant<int, NS - 3 - 2 * i>{}, std::true_type{}, -1, fb, fa);
    });
  };
  run(std::true_type{}, NOWN);
  if (wid < 2) {
    bool bad = false;
    wave_wait2<SL_GEMM>(fin + wid * 8 * FS, ~0ull, seq, tmo, bad);
    if (bad) *lbad = 1;
  }
  __builtin_amdgcn_s_waitcnt(0xC07F);
  __builtin_amdgcn_s_barrier();
  run(std::false_type{}, NRCV);
  const u32 tmv = __hip_atomic_load((const GAS u32*)tmo, __ATOMIC_RELAXED, __HIP_MEMORY_SCOPE_AGENT);
  __builtin_amdgcn_s_waitcnt(0xC07F);
  __builtin_amdgcn_s_barrier();
  CBAR();
  constexpr u32 EROW = 528;
#pragma unroll
  for (int mi = 0; mi < MI; ++mi) {
    const u32 mr = wm * 32 * MI + mi * 32 + (lane & 31);
#pragma unroll
    for (int gq = 0; gq < 4; ++gq) {
      const f32x4 o = {acc[mi][4 * gq], acc[mi][4 * gq + 1], acc[mi][4 * gq + 2], acc[mi][4 * gq + 3]};
      *(LAS f32x4*)(lds + mr * EROW + (wn * 32 + 8 * gq + 4 * h) * 4) = o;
    }
  }
  __syncthreads();
  const u32 bad = (*lbad || tmv == seq) ? NAN2 : 0u;
  const u32 keep = rank >> 2;
  const bool do_exp = !KIND || hsel != keep || m < NX, do_keep = !KIND || hsel == keep || m < NX;
  const u32 me = KIND && hsel == keep ? xbase(seq) + m : m, mk = KIND && hsel != keep ? xbase(seq) + m : m;
  if constexpr (QUART) {
    const u32 row = tid >> 3, ch = tid & 7;
    const LAS char* se = lds + row * EROW + ch * 32;
    const f32x4 c0 = *(const LAS f32x4*)se + *(const LAS f32x4*)(se + 256), c1 = *(const LAS f32x4*)(se + 16) + *(const LAS f32x4*)(se + 272);
    GAS u32* qc = (GAS u32*)(ws + QC_OFF + (m * 2 + hsel) * FS);
    if (do_exp) {
      const bf16x8 o = {(__bf16)c0[0], (__bf16)c0[1], (__bf16)c0[2], (__bf16)c0[3],
                        (__bf16)c1[0], (__bf16)c1[1], (__bf16)c1[2], (__bf16)c1[3]};
      st128(__builtin_bit_cast(u32x4, o) | bad, rs, row * 256 + ch * 16, P16_OFF + me * 16384 + csel * 128);
      WAIT_VM(0);
      __builtin_amdgcn_s_barrier();
      if (tid == 0 && __hip_atomic_fetch_add(qc, 1u, __ATOMIC_RELAXED, __HIP_MEMORY_SCOPE_AGENT) == 2 * seq - 1)
        st_sys(p.ws[PL] + FPR_OFF + me * FS, seq);
    }
    if (do_keep) {
      const u32 vo = (row >> 2) * 2048 + (row & 3) * 256 + ch * 16, e32 = P32_OFF + mk * 32768 + csel * 128;
      st128(__builtin_bit_cast(u32x4, c0) | bad, rs, vo, e32);
      st128(__builtin_bit_cast(u32x4, c1) | bad, rs, vo, e32 + 1024);
    }
    WAIT_VM(0);
    __syncthreads();
    if (tid == 64 && do_keep && __hip_atomic_fetch_add(qc + 1, 1u, __ATOMIC_RELAXED, __HIP_MEMORY_SCOPE_AGENT) == 2 * seq - 1)
      st_sys(ws + FPL_OFF + mk * FS, seq);
    return;
  }
  if (do_exp) {
    const u32 e16 = P16_OFF + me * 16384 + wid * 1024;
#pragma unroll
    for (int part = 0; part < 2; ++part) {
      const LAS char* se = lds + ((HALF ? 0 : (keep ^ 1) * 64) + part * 32 + (tid >> 4)) * EROW + (tid & 15) * 32;
      const f32x4 c0 = *(const LAS f32x4*)se, c1 = *(const LAS f32x4*)(se + 16);
      const bf16x8 o = {(__bf16)c0[0], (__bf16)c0[1], (__bf16)c0[2], (__bf16)c0[3],
                        (__bf16)c1[0], (__bf16)c1[1], (__bf16)c1[2], (__bf16)c1[3]};
      st128(__builtin_bit_cast(u32x4, o) | bad, rs, lane * 16, e16 + part * 8192);
    }
  }
  if (HALF && do_exp && do_keep) {
    WAIT_VM(0);
    __builtin_amdgcn_s_barrier();
    if (tid == 0) st_sys(p.ws[PL] + FPR_OFF + me * FS, seq);
  }
  if (do_keep) {
    const u32 e32 = P32_OFF + mk * 32768 + wid * 2048;
#pragma unroll
    for (int part = 0; part < 2; ++part) {
      const LAS char* sk = lds + ((HALF ? 0 : keep * 64) + part * 32 + (tid >> 4)) * EROW + (tid & 15) * 32;
      const u32x4 c0 = *(const LAS u32x4*)sk, c1 = *(const LAS u32x4*)(sk + 16);
      st128(c0 | bad, rs, lane * 16, e32 + part * 16384);
      st128(c1 | bad, rs, lane * 16, e32 + part * 16384 + 1024);
    }
  }
  WAIT_VM(0);
  __syncthreads();
  if (tid == 0 && do_exp && !(HALF && do_keep)) st_sys(p.ws[PL] + FPR_OFF + me * FS, seq);
  if (tid == 64 && do_keep) st_sys(ws + FPL_OFF + mk * FS, seq);
}

__device__ __forceinline__ void stage_unit(const Params& p, u32 lk, u32 seq) {
  const int tid = threadIdx.x, lane = tid & 63;
  const u32 wid = rfl(tid >> 6);
  const u32 lev = lk >> 2, kt = (wid >> 2) * 4 + (lk & 3), op = (wid >> 1) & 1, rc = (wid & 1) * 2048 + lev * 128;
  const auto r = mk_rsrc(op ? p.w : p.x, X_BYTES);
  const auto rs = mk_rsrc(p.ws[0], DATA_BYTES);
  const u32 vo = op ? (lane >> 4) * 8192 + (lane & 15) * 16 : (lane >> 3) * 1024 + (lane & 7) * 16;
  const u32 so = op ? kt * 64 * 8192 + rc * 2 : rc * 1024 + kt * 128;
  u32x4 v[16];
#pragma unroll
  for (int i = 0; i < 16; ++i) v[i] = __builtin_amdgcn_raw_buffer_load_b128(r, vo, so + i * (op ? 4 * 8192 : 8 * 1024), 0);
#pragma unroll
  for (int i = 0; i < 16; ++i) st128(v[i], rs, lane * 16, EX_OFF + (lk * 8 + wid) * BLK + i * 1024);
  WAIT_VM(0);
  __syncthreads();
  if (tid >= 1 && tid < NR && tid != PL) st_sys(p.ws[tid] + SG_OFF + (p.rank * 64 + lk) * FS, seq);
}

__device__ __forceinline__ void gemm_role(const Params& p, u32 k, u32 seq, LAS char* lds) {
  const u64 tstart = __builtin_amdgcn_s_memrealtime();
  constexpr u32 NG = 8 * GEMM_WGS, NSMALL = 2 * NQUART + 2 * NHALF;
  static_assert(NQUART <= NHALF && NSMALL <= NG, "unit schedule");
  if (k < 4 * NQUART) gemm_tile<2>(p, k >> 2, (k >> 1) & 1, k & 1, seq, tstart, true, lds);
  else if (k < NSMALL) gemm_tile<1>(p, (k >> 1) - NQUART, k & 1, 0, seq, tstart, true, lds);
  for (u32 n = k < NSMALL ? k + NG : k; n < 256 - NHALF + NSMALL; n += NG) gemm_tile<0>(p, n - NSMALL + NHALF, 0, 0, seq, tstart, n < NG, lds);
}

__global__ void __launch_bounds__(THREADS, 2) fused_gemm_allreduce_silu(Params p) {
  extern __shared__ __attribute__((aligned(16))) char smem[];
  const int lane = threadIdx.x & 63;
  const u32 wid = rfl(threadIdx.x >> 6);
  LAS u32* ctl = (LAS u32*)smem;
  LAS u32* lctl = (LAS u32*)(smem + LDS_CTL);
  u32 slot = (blockIdx.x & ~7u) | ((u32)__builtin_amdgcn_s_getreg((3 << 11) | 20) & 7);
  auto is_push = [&]() { return slot < 8 * ENG_WGS && (slot & 3) != 0 && wid < PUSH_WAVES; };
  if (__builtin_expect(is_push(), 1)) {
    if (push_role<false>(p, slot, wid, lane, lctl)) return;
  } else {
    if (wid == 7) {
      if (lane < 32) ctl[lane] = 0;
      if (lane == 0) {
        GAS u32* ctr = (GAS u32*)(p.ws[0] + CTR_OFF) + blockIdx.x;
        GAS u32* clm = (GAS u32*)(p.ws[0] + CLM_OFF);
        const u32 c = __hip_atomic_load(ctr, __ATOMIC_RELAXED, __HIP_MEMORY_SCOPE_AGENT);
        u32 sl = slot;
#pragma unroll 1
        for (int n = 0; n < 4 * GRID; ++n) {
          if (__hip_atomic_fetch_add(clm + sl * 16, 1u, __ATOMIC_RELAXED, __HIP_MEMORY_SCOPE_AGENT) == c) break;
          __hip_atomic_fetch_add(clm + sl * 16, ~0u, __ATOMIC_RELAXED, __HIP_MEMORY_SCOPE_AGENT);
          sl = (sl + 1) & (GRID - 1);
        }
        lctl[0] = 0; lctl[1] = sl; lctl[2] = c + 1;
      }
    }
    asm volatile("s_waitcnt lgkmcnt(0)" ::: "memory");
    __builtin_amdgcn_s_barrier();
    slot = rfl(LD_LDS(lctl + 1));
    if (threadIdx.x == THREADS - 1)
      __hip_atomic_store((GAS u32*)(p.ws[0] + CTR_OFF) + blockIdx.x, LD_LDS(lctl + 2), __ATOMIC_RELAXED, __HIP_MEMORY_SCOPE_AGENT);
    WAIT_VM(0);
  }
  if (__builtin_expect(is_push(), 0)) {
    push_role<true>(p, slot, wid, lane, lctl);
    return;
  }
  const u32 seq = rfl(LD_LDS(lctl + 2));
  const u32 g = slot & 7, s = slot >> 3;
  const u32 peer = ((u32)p.rank ^ g) & (NR - 1);
  constexpr u32 LK0 = NPUSH / 4;
  if (s < ENG_WGS) {
    if (g == 0) {
      for (u32 u = s; u < 64 - LK0; u += u < 16 ? 16 : u < 24 ? 16 : 8) stage_unit(p, LK0 + u, seq);
    } else if (wid >= PUSH_WAVES && wid < PUSH_WAVES + PULL_WAVES) {
      pull_role(p, peer, (wid - PUSH_WAVES) * ENG_WGS + s, lane, seq, ctl);
    } else if (wid == PUSH_WAVES + PULL_WAVES) {
      flag_role(p, g, lane, seq, ctl);
    }
  } else if (s < ENG_WGS + FIN_WGS) {
    if ((g & 3) == 0) { stage_unit(p, LK0 + 8 + g + s - ENG_WGS, seq); stage_unit(p, LK0 + 24 + g + s - ENG_WGS, seq); }
    fin_role(p, (s - ENG_WGS) * 8 + g, seq);
  } else {
    const u32 k = s - ENG_WGS - FIN_WGS;
    gemm_role(p, (k >> 2) * 32 + g * 4 + (k & 3), seq, (LAS char*)smem);
  }
}

}

extern "C" void kernel_launch(int rank, int nranks, ncclComm_t comm,
                              hipStream_t stream,
                              void* const* d_in, const int* in_sizes, int n_in,
                              void* d_out, int out_size,
                              const int* in_dtypes, int out_dtype,
                              void* const* d_out_all,
                              void* const* d_ws_all, size_t ws_bytes) {
  if (nranks != NR || (unsigned)rank >= (unsigned)NR || n_in < 2 || ws_bytes < WS_USED ||
      in_sizes[0] != 4096 * 512 || in_sizes[1] != 512 * 4096 || out_size != 4096 * 4096) {
    fprintf(stderr, "kernel_launch: unexpected problem shape\n");
    return;
  }
  HIP_CHECK(hipFuncSetAttribute((const void*)fused_gemm_allreduce_silu,
                                hipFuncAttributeMaxDynamicSharedMemorySize, LDS_BYTES));
  Params p;
  p.x = (const char*)d_in[0];
  p.w = (const char*)d_in[1];
  const int phys[NR] = {0, 2, 3, 6, 1, 4, 5, 7};
  p.rank = 0; p.pad = 0;
  for (int i = 0; i < NR; ++i) if (phys[i] == rank) p.rank = i;
  for (int i = 0; i < NR; ++i) { p.ws[i] = (char*)d_ws_all[phys[p.rank ^ i]]; p.out[i] = (char*)d_out_all[phys[p.rank ^ i]]; }
  for (u32 g = 0; g < NR; ++g)
    for (u32 job = 0; job < 16; ++job) {
      const u32 peer = p.rank ^ g, lev = job >> 4, kt = (peer >> 2) * 4 + ((job >> 2) & 3), op = (job >> 1) & 1, half = job & 1;
      p.psrc[g][job] = op ? (kt * 64 + half * 32) * 8192 + ((peer & 1) * 2048 + lev * 128) * 2
                                         : (((peer >> 1) & 1) * 2048 + lev * 128 + half * 64) * 1024 + kt * 128;
    }
  hipLaunchKernelGGL(fused_gemm_allreduce_silu, dim3(GRID), dim3(THREADS), LDS_BYTES, stream, p);
  HIP_CHECK(hipGetLastError());
}
```

```cpp
#include <hip/hip_runtime.h>
#include <rccl/rccl.h>
#include <cstdio>
#include <type_traits>

#define HIP_CHECK(expr) do { hipError_t _e = (expr); if (_e != hipSuccess) { \
    fprintf(stderr, "HIP error %s at %s:%d\n", hipGetErrorString(_e), __FILE__, __LINE__); \
    return; } } while (0)

namespace {

typedef unsigned int u32;
typedef unsigned long long u64;
typedef u32 u32x4 __attribute__((ext_vector_type(4)));
typedef float f32x4 __attribute__((ext_vector_type(4)));
typedef __bf16 bf16x4 __attribute__((ext_vector_type(4)));
typedef __bf16 bf16x8 __attribute__((ext_vector_type(8)));
typedef float f32x16 __attribute__((ext_vector_type(16)));
#define LAS __attribute__((address_space(3)))
#define GAS __attribute__((address_space(1)))

constexpr int NR = 8;
constexpr u32 X_BYTES = 4096u * 512u * 2u;
constexpr u32 W_BYTES = 512u * 4096u * 2u;
constexpr u32 OUT_BYTES = 4096u * 4096u * 2u;
constexpr u32 FS = 128;

constexpr u32 BLK = 16384;
constexpr u32 IN_OFF = 0;
constexpr u32 IN_SRC = 128 * BLK;
constexpr u32 P16_OFF = IN_OFF + NR * IN_SRC;
constexpr u32 PIN_OFF = P16_OFF + 320 * 16384;
constexpr u32 P32_OFF = PIN_OFF + 320 * 16384;
constexpr u32 EX_OFF = P32_OFF + 320 * 32768;
constexpr u32 DATA_BYTES = EX_OFF + 512 * BLK;
constexpr u32 FI_OFF = DATA_BYTES;
constexpr u32 FPR_OFF = FI_OFF + NR * 256 * FS;
constexpr u32 FPL_OFF = FPR_OFF + 320 * FS;
constexpr u32 FPI_OFF = FPL_OFF + 320 * FS;
constexpr u32 FF_OFF = FPI_OFF + 1280 * FS;
constexpr u32 SG_OFF = FF_OFF + NR * 256 * FS;
constexpr u32 GO_OFF = SG_OFF + NR * 64 * FS;
constexpr u32 RF_OFF = GO_OFF + NR * 16 * FS;
constexpr u32 QC_OFF = RF_OFF + 64 * FS;
constexpr u32 CTR_OFF = QC_OFF + 128 * FS;
constexpr u32 CLM_OFF = CTR_OFF + 256 * 4;
constexpr u32 TMO_OFF = CLM_OFF + 256 * 64;
constexpr u32 TICK_OFF = TMO_OFF + 128;
constexpr u32 SEG_OFF = TICK_OFF + 32 * 128;
constexpr u32 WS_USED = SEG_OFF + 8 * 128;

constexpr u64 DEADLINE_TICKS = 80000000ull;

constexpr int ENG_WGS = 8, FIN_WGS = 4, GEMM_WGS = 20;
constexpr int PUSH_WAVES = 2, PULL_WAVES = 2;
constexpr int PUSH_D = 4, PUSH_ROUNDS = 4;
constexpr u32 NPUSH = PUSH_ROUNDS * ENG_WGS * PUSH_WAVES;
constexpr u32 PL = 4;
constexpr int GO_AT = 6;
constexpr int NS_HALF = 6;
constexpr u32 NQUART = 25, NHALF = 55;
constexpr u64 STAGGER = 190;
constexpr u32 NX = 22, TOPK = 44;
constexpr int SL_PULL = 2, SL_FIN = 4, SL_GEMM = 8;
constexpr u32 NAN2 = 0x7FC07FC0u;
constexpr int GRID = 8 * (ENG_WGS + FIN_WGS + GEMM_WGS);
static_assert(GRID == 256, "slots wrap at a power of two");
constexpr int THREADS = 512;
constexpr u32 LDS_CTL = 144 * 1024;
constexpr u32 LDS_BYTES = LDS_CTL + 64;
constexpr int AUX_SYS = 17;

struct Params {
  const char* x;
  const char* w;
  int rank;
  int pad;
  char* ws[NR];
  char* out[NR];
  u32 psrc[NR][16];
};

__device__ __forceinline__ u32 rfl(u32 v) { return __builtin_amdgcn_readfirstlane(v); }
__device__ __forceinline__ __amdgpu_buffer_rsrc_t mk_rsrc(const void* p, u32 n) {
  const u64 a = (u64)p;
  const u64 u = ((u64)rfl((u32)(a >> 32)) << 32) | rfl((u32)a);
  return __builtin_amdgcn_make_buffer_rsrc((void*)u, 0, n, 0x00020000);
}
__device__ __forceinline__ u32 ld_sys(const char* p) {
  return __hip_atomic_load((const GAS u32*)p, __ATOMIC_RELAXED, __HIP_MEMORY_SCOPE_SYSTEM);
}
__device__ __forceinline__ void st_sys(char* p, u32 v) {
  __hip_atomic_store((GAS u32*)p, v, __ATOMIC_RELAXED, __HIP_MEMORY_SCOPE_SYSTEM);
}
#define WAIT_VM(n) asm volatile("s_waitcnt vmcnt(%0)" ::"n"(n) : "memory")
#define CBAR() asm volatile("" ::: "memory")
#define LD_LDS(q) __hip_atomic_load(q, __ATOMIC_RELAXED, __HIP_MEMORY_SCOPE_WORKGROUP)
#define ST_LDS(q, v) __hip_atomic_store(q, v, __ATOMIC_RELAXED, __HIP_MEMORY_SCOPE_WORKGROUP)

__device__ __forceinline__ void st128(u32x4 v, __amdgpu_buffer_rsrc_t r, u32 voff, u32 soff) {
  __builtin_amdgcn_raw_buffer_store_b128(v, r, voff, soff, AUX_SYS);
  asm volatile("s_nop 1" ::"v"(v) : "memory");
}

template <int SLEEP>
__device__ __forceinline__ u64 wave_wait(const char* flag, u64 need, u32 seq, char* tmo, bool& bad) {
  const u64 t0 = __builtin_amdgcn_s_memrealtime();
  for (u32 it = 1;; ++it) {
    const u64 ok = __ballot(ld_sys(flag) >= seq);
    if ((ok & need) == need) return ok;
    if ((it & 15) == 0 || bad) {
      const u32 t = __hip_atomic_load((const GAS u32*)tmo, __ATOMIC_RELAXED, __HIP_MEMORY_SCOPE_AGENT);
      if (rfl(t) == seq || __builtin_amdgcn_s_memrealtime() - t0 > DEADLINE_TICKS) {
        __hip_atomic_store((GAS u32*)tmo, seq, __ATOMIC_RELAXED, __HIP_MEMORY_SCOPE_AGENT);
        bad = true;
        return ~0ull;
      }
    }
    __builtin_amdgcn_s_sleep(SLEEP);
  }
}

template <int SLEEP>
__device__ __forceinline__ u64 wave_wait2(const char* flag, u64 need, u32 seq, char* tmo, bool& bad) {
  const u64 t0 = __builtin_amdgcn_s_memrealtime();
  const bool needed = (need >> (threadIdx.x & 63)) & 1;
  bool ok = false;
  for (u32 it = 1;; ++it) {
    if (!ok && (it == 1 || needed)) ok = ld_sys(flag) >= seq;
    const u64 have = __ballot(ok);
    if ((have & need) == need) return have;
    if ((it & 15) == 0) {
      const u32 t = __hip_atomic_load((const GAS u32*)tmo, __ATOMIC_RELAXED, __HIP_MEMORY_SCOPE_AGENT);
      if (rfl(t) == seq || __builtin_amdgcn_s_memrealtime() - t0 > DEADLINE_TICKS) {
        __hip_atomic_store((GAS u32*)tmo, seq, __ATOMIC_RELAXED, __HIP_MEMORY_SCOPE_AGENT);
        bad = true;
        return ~0ull;
      }
    }
    __builtin_amdgcn_s_sleep(SLEEP);
  }
}

__device__ __forceinline__ void dma4(u32 dst, __amdgpu_buffer_rsrc_t ra, u32 va0, u32 va1, u32 sa,
                                     __amdgpu_buffer_rsrc_t rb, u32 vb0, u32 vb1, u32 sb) {
  u32 keep;
  dst = rfl(dst); sa = rfl(sa); sb = rfl(sb);
  asm volatile(
      "s_mov_b32 %0, m0\n\t"
      "s_nop 4\n\t"
      "s_mov_b32 m0, %1\n\t"
      "s_nop 0\n\t"
      "buffer_load_dwordx4 %3, %5, %6 offen lds\n\t"
      "buffer_load_dwordx4 %4, %5, %6 offen offset:1024 lds\n\t"
      "s_mov_b32 m0, %2\n\t"
      "s_nop 0\n\t"
      "buffer_load_dwordx4 %7, %9, %10 offen lds\n\t"
      "buffer_load_dwordx4 %8, %9, %10 offen offset:1024 lds\n\t"
      "s_mov_b32 m0, %0"
      : "=&s"(keep)
      : "s"(dst), "s"(dst + BLK), "v"(va0), "v"(va1), "s"(ra), "s"(sa), "v"(vb0), "v"(vb1), "s"(rb), "s"(sb)
      : "memory");
}

__device__ __forceinline__ void dma3(u32 dsta, u32 dstb, __amdgpu_buffer_rsrc_t ra, u32 va0, u32 sa,
                                     __amdgpu_buffer_rsrc_t rb, u32 vb0, u32 vb1, u32 sb) {
  u32 keep;
  dsta = rfl(dsta); dstb = rfl(dstb); sa = rfl(sa); sb = rfl(sb);
  asm volatile(
      "s_mov_b32 %0, m0\n\t"
      "s_nop 4\n\t"
      "s_mov_b32 m0, %1\n\t"
      "s_nop 0\n\t"
      "buffer_load_dwordx4 %3, %4, %5 offen lds\n\t"
      "s_mov_b32 m0, %2\n\t"
      "s_nop 0\n\t"
      "buffer_load_dwordx4 %6, %8, %9 offen lds\n\t"
      "buffer_load_dwordx4 %7, %8, %9 offen offset:1024 lds\n\t"
      "s_mov_b32 m0, %0"
      : "=&s"(keep)
      : "s"(dsta), "s"(dstb), "v"(va0), "s"(ra), "s"(sa), "v"(vb0), "v"(vb1), "s"(rb), "s"(sb)
      : "memory");
}

template <int I, int N, class F>
__device__ __forceinline__ void static_for(F&& f) {
  if constexpr (I < N) { f(std::integral_constant<int, I>{}); static_for<I + 1, N>(f); }
}

__device__ __forceinline__ void tile_rc(u32 m, u32& ri, u32& ci) {
  u32 l = 0;
#pragma unroll
  for (u32 t = 1; t < 16; ++t) l += m >= t * t;
  const u32 idx = m - l * l;
  ri = idx < l ? l : idx - l;
  ci = idx < l ? idx : l;
}
__device__ __forceinline__ u32 fin_base(u32 r) { return (((r >> 1) & 1) * 2048 + (r >> 2) * 64) * 8192 + (r & 1) * 4096; }

template <bool claimed>
__device__ __forceinline__ bool push_role(const Params& p, u32& slot, u32 wid, int lane, LAS u32* lctl) {
  constexpr int D = PUSH_D, JI = 8;
  constexpr u32 NW = ENG_WGS * PUSH_WAVES;
  static_assert(PUSH_ROUNDS >= 3 && NW == 16 && X_BYTES == W_BYTES && GO_AT <= JI, "push schedule");
  const u32 g = slot & 7, w = wid * ENG_WGS + (slot >> 3), op = (slot >> 4) & 1;
  u64 bs = (u64)(&p.x)[op], bw = (u64)p.ws[g];
  u32 src = p.psrc[g][w];
  u32 rank = p.rank;
  asm volatile("" : "+s"(bs), "+s"(bw), "+s"(src), "+s"(rank));
  const auto rs = mk_rsrc((const void*)bs, X_BYTES);
  const u32 vs = op ? (lane >> 4) * 8192 + (lane & 15) * 16 : (lane >> 3) * 1024 + (lane & 7) * 16;
  const u32 str = op ? 4 * 8192 : 8 * 1024;
  auto ld = [&](int r, int i) -> u32x4 { return __builtin_amdgcn_raw_buffer_load_b128(rs, vs, src + r * (op ? 256 : 128 * 1024) + i * str, 0); };
  u32x4 v[D], v2[JI - D];
#pragma unroll
  for (int i = 0; i < D; ++i) { v[i] = ld(0, i); CBAR(); }
#pragma unroll
  for (int i = D; i < JI; ++i) { v2[i - D] = ld(0, i); CBAR(); }
  char* wsp = (char*)bw;
  const auto rd = mk_rsrc(wsp, DATA_BYTES);
  const u32 dst = IN_OFF + rank * IN_SRC + w * 8192, flag = FI_OFF + (rank * 256 + w) * FS;
  u32 seq = 0;
  auto round = [&](auto rc) {
    constexpr int R = decltype(rc)::value;
    constexpr bool FIRST = R == 0, LAST = R == PUSH_ROUNDS - 1;
#pragma unroll
    for (int i = 0; i < JI; ++i) {
      if (!FIRST && i == D - 1) {
        WAIT_VM(2 * D - 1);
        if (R == 1) {
          if (!claimed) __builtin_amdgcn_s_barrier();
          CBAR();
          seq = rfl(LD_LDS(lctl + 2));
        }
        if (lane == 0) st_sys(wsp + flag + (R - 1) * NW * FS, seq);
        CBAR();
      }
      if (LAST && i == JI - GO_AT) {
        if (lane == 0) st_sys(wsp + GO_OFF + (rank * 16 + w) * FS, seq);
        if (lane == 0) ST_LDS(lctl - LDS_CTL / 4 + 24 + wid, 1u);
        CBAR();
      }
      st128(FIRST && i >= D ? v2[i >= D ? i - D : 0] : v[i % D], rd, lane * 16, dst + R * NW * 8192 + i * 1024);
      if (i + D < JI) { if (!FIRST) v[i % D] = ld(R, i + D); }
      else if (!LAST) v[i % D] = ld(LAST ? R : R + 1, i + D - JI);
      CBAR();
    }
  };
  static_assert(JI - D >= D - 1, "the count above also holds in the last round");
  round(std::integral_constant<int, 0>{});
  round(std::integral_constant<int, 1>{});
  const u32 slot2 = rfl(LD_LDS(lctl + 1));
  if (__builtin_expect(slot2 != slot, 0)) { WAIT_VM(0); slot = slot2; return false; }
  static_for<2, PUSH_ROUNDS>(round);
  WAIT_VM(0);
  if (lane == 0) st_sys(wsp + flag + (PUSH_ROUNDS - 1) * NW * FS, seq);
  return true;
}

__device__ __forceinline__ float silu(float a) {
  const float c = fminf(fmaxf(a, -60.f), 60.f);
  return a * __builtin_amdgcn_rcpf(1.f + __expf(-c));
}

constexpr u32 MB_RING = 0, MB_END = 16, MB_PUSHED = 24;

constexpr u32 NE = 4 * NX, NPOOL = (1024 - NE) / 2;
static_assert(NX <= NHALF && NX <= 32, "early tiles");
__device__ __forceinline__ u32 xbase(u32 seq) { return 256 + (seq & 1) * 32; }
__device__ __forceinline__ u32 pool_job(u32 pos) {
  return pos < NPOOL - 2 * TOPK ? pos : pos >= NPOOL - TOPK ? pos - TOPK : pos + TOPK;
}

__device__ __forceinline__ void pull_role(const Params& p, u32 peer, u32 w, int lane, u32 seq, LAS u32* ctl) {
  constexpr u32 NW = ENG_WGS * PULL_WAVES, NONE = ~0u, REL = 1u << 31;
  const u32 rank = p.rank, g = rank ^ peer;
  const bool pl = g == PL;
  const u32 NI = pl ? 1280 : 256 - NPUSH, NJ = pl ? 1024 + 4 * NX : NI + 1024;
  char* ws = p.ws[0];
  char* tmo = ws + TMO_OFF;
  const u64 bf = (u64)p.out[g], bp = (u64)p.ws[g], bo = (u64)p.out[0];
  const u32 vwide = (lane >> 4) * 8192 + (lane & 15) * 16;
  const auto rtick = mk_rsrc(ws + TICK_OFF + (seq & 1) * 2048, 2048);
  if (w == 0 && (lane == 0 || (pl && lane < 3))) st_sys(ws + TICK_OFF + ((seq & 1) ^ 1) * 2048 + (lane ? 7 + lane : g) * 128, 0);
  const u32 bi = (rank >> 1) & 1, bj = rank & 1;
  const u32 isrc = pl ? P16_OFF : EX_OFF + (NPUSH / 4 * 8 + (rank >> 2) * 4 + bi) * BLK;
  const u32 is1 = pl ? BLK : 8 * BLK, is2 = pl ? 8192 : (2 + bj - bi) * BLK, isz = pl ? 4096 : 8192;
  const u32 idst = pl ? PIN_OFF : IN_OFF + peer * IN_SRC + NPUSH * 8192;
  const u32 ipre = pl ? FPR_OFF : SG_OFF + (peer * 64 + NPUSH / 4) * FS, ish = 2;
  const u32 ipost = pl ? FPI_OFF : FI_OFF + (peer * 256 + NPUSH) * FS;
  const u32 fown = fin_base(peer), fpar = fin_base(rank ^ PL);
  const u32 rpre = pl ? FF_OFF + peer * 256 * FS : RF_OFF + g * 8 * FS, rsh = pl ? 2 : 7;
  u32 ttile = 0;
#pragma unroll
  for (u32 e = 0; e < 4; ++e) {
    u32 ri, ci;
    tile_rc(lane * 4 + e, ri, ci);
    ttile |= (ri << 4 | ci) << (8 * e);
  }
  const u64 t0 = __builtin_amdgcn_s_memrealtime();
  u32 tf = w, tf2 = NW + w, lastf = 0, sub = 0;
  u32 cdst = 0, cact = 0, pend = 0, ack = 0;
  u32 nring = 0;
  u32 from = 1, spare = NONE;
  bool bad = false, cisf = false;
  auto poll = [&]() -> u32 {
    const u32 o = tf < NI ? ipre + (tf >> ish) * FS : tf < REL ? FF_OFF + (peer * 256 + ((tf - NI) >> 2)) * FS : rpre + ((tf & 1023) >> rsh) * FS;
    return ld_sys(ws + rfl(o));
  };
  u32 pv = 0;
  u32x4 v[4];
  const u32 tvo = lane == 0 ? g * 128 : (8 + (w & 1)) * 128, tinc = lane && pl ? 1u << 16 : 1u, tbit = lane < 2 ? lane + 1 : 0;
  auto draw = [&](u32 m) -> u32 {
    return __builtin_amdgcn_raw_ptr_buffer_atomic_add_i32(tinc, rtick, (m & tbit) ? tvo : 0x80000000u, 0, 0);
  };
  u32 sstart, sbase;
  {
    const u32 l = lane >> 1, xb = min(l * l, NX);
    sstart = 4 * (((lane & 1) ? (l + 1) * (l + 1) : l * l) + xb);
    sbase = 4 * ((lane & 1) ? xbase(seq) + xb : l * l);
  }
  auto handle = [&](u32 id) -> u32 {
    if (!pl) return id;
    const u32 sg = 63 - __builtin_clzll(__ballot(sstart <= id));
    return __builtin_amdgcn_readlane(sbase, sg) + id - __builtin_amdgcn_readlane(sstart, sg);
  };
  tf = handle(tf); tf2 = handle(tf2);
  auto drawn = [&](u32 m, u32 t) -> u32 {
    u32 own = NONE, tp = __builtin_amdgcn_readlane(t, 1);
    if (m & 1) {
      const u32 id = __builtin_amdgcn_readlane(t, 0) + 2 * NW;
      if (id < NJ) own = handle(id);
      if (m == 1) {
        if (__builtin_expect(id < NJ, 1)) { from = id + NW >= NJ ? 3 : 1; return own; }
        tp = __builtin_amdgcn_readlane(draw(2), 1);
        asm volatile("" : "+s"(tp));
      }
    } else if (m == 0) {
      own = spare; spare = NONE;
      return own;
    }
    const u32 f = tp & 0xFFFF, b = tp >> 16;
    const u32 hp = f + b < NPOOL ? REL | (NE + pool_job(pl ? NPOOL - 1 - b : f) * 2 + (w & 1)) : NONE;
    from = hp == NONE ? 0 : 2;
    spare = own == NONE ? NONE : hp;
    return own == NONE ? hp : own;
  };
  auto put = [&](int i) {
    const auto ro = mk_rsrc((const void*)(cisf ? bo : (u64)ws), cisf ? OUT_BYTES : DATA_BYTES);
    st128(v[i], ro, cisf ? vwide : lane * 16, rfl(cdst + i * (cisf ? 4 * 8192 : 1024)));
  };
  auto done = [&](u32 a) {
    if (a) {
      LAS u32* e = ctl + MB_RING + (w / ENG_WGS) * 8 + (nring++ & 7);
      while (rfl(LD_LDS(e)) != 0) __builtin_amdgcn_s_sleep(1);
      if (lane == 0) ST_LDS(e, a);
      CBAR();
    }
  };
  auto ready = [&]() -> bool { return tf != NONE && (bad || (__ballot(pv >= seq) & 1)); };
  auto round = [&](auto first_c) {
    constexpr bool FIRST = decltype(first_c)::value;
    const u32 h = tf, hs = sub;
    const bool isf = h >= NI, tix = isf || hs || pl;
    sub = tix ? 0 : 1;
    tf = tix ? tf2 : tf;
    const u32 dm = tix && spare == NONE ? from : 0, tk = draw(dm);
    pv = poll();
    CBAR();
    done(ack);
    u32 nsrc, ndst, nact;
    if (isf) {
      const u32 j = h < REL ? h - NI : h & 1023, rc = (__builtin_amdgcn_readlane(ttile, j >> 4) >> (2 * (j & 12))) & 255;
      nsrc = ndst = (h < REL ? fown : fpar) + ((rc >> 4) * 128 + (j & 3) * 16) * 8192 + (rc & 15) * 256;
      nact = h < REL ? (j >> 7) * 2 + 1 : 0;
      lastf = ndst;
    } else {
      nsrc = isrc + (h >> 2) * is1 + ((h >> 1) & 1) * is2 + (h & 1) * isz + hs * 4096;
      ndst = idst + h * isz + hs * 4096;
      nact = tix ? ipost + h * FS : 0;
    }
    const auto rsrc = mk_rsrc((const void*)(isf ? bf : bp), isf ? OUT_BYTES : DATA_BYTES);
    const u32 vs = isf ? vwide : lane * 16, str = isf ? 4 * 8192 : 1024;
#pragma unroll
    for (int i = 0; i < 4; ++i) {
      if (!FIRST) put(i);
      v[i] = __builtin_amdgcn_raw_buffer_load_b128(rsrc, vs, rfl(nsrc + i * str), AUX_SYS);
      CBAR();
    }
    ack = 0;
    if (!FIRST) {
      WAIT_VM(11);
      ack = pend; pend = cact;
    }
    cdst = ndst; cisf = isf; cact = nact;
    if (tix) tf2 = drawn(dm, tk);
  };
  if (!pl) {
    while (rfl(LD_LDS(ctl + MB_PUSHED + w / ENG_WGS)) == 0) __builtin_amdgcn_s_sleep(2);
    wave_wait<1>(ws + ((lane & 1) ? GO_OFF + (peer * 16 + w) * FS : ipre + (w >> ish) * FS), 3, seq, tmo, bad);
    if (bad) WAIT_VM(0);
    pv = seq;
  }
  for (u32 it = 1; tf != NONE; ++it) {
    if (ready()) {
      round(std::true_type{});
      if (ready()) {
        round(std::false_type{});
        while (ready()) round(std::false_type{});
      }
#pragma unroll
      for (int i = 0; i < 4; ++i) put(i);
      done(ack);
      if (pend | cact) {
        WAIT_VM(0);
        done(pend);
        done(cact);
      }
      ack = pend = 0;
      continue;
    }
    if ((it & 63) == 0) {
      const u32 t = __hip_atomic_load((const GAS u32*)tmo, __ATOMIC_RELAXED, __HIP_MEMORY_SCOPE_AGENT);
      if (rfl(t) == seq || __builtin_amdgcn_s_memrealtime() - t0 > DEADLINE_TICKS) {
        __hip_atomic_store((GAS u32*)tmo, seq, __ATOMIC_RELAXED, __HIP_MEMORY_SCOPE_AGENT);
        WAIT_VM(0);
        bad = true;
      }
    }
    __builtin_amdgcn_s_sleep(SL_PULL * 3);
    pv = poll();
  }
  if (bad) {
    WAIT_VM(0);
    const u32x4 nanv = {NAN2, NAN2, NAN2, NAN2};
    for (int i = 0; i < 4; ++i) st128(nanv, mk_rsrc(p.out[0], OUT_BYTES), vwide, rfl(lastf) + i * 4 * 8192);
  }
  if (lane == 0) __hip_atomic_fetch_add(ctl + MB_END, 1u, __ATOMIC_RELAXED, __HIP_MEMORY_SCOPE_WORKGROUP);
}

__device__ __forceinline__ void flag_role(const Params& p, u32 g, int lane, u32 seq, LAS u32* ctl) {
  char* ws = p.ws[0];
  char* rf = p.ws[g ^ PL] + RF_OFF + (g ^ PL) * 8 * FS;
  for (;;) {
    const u32 end = rfl(LD_LDS(ctl + MB_END));
    const u32 a = lane < 16 ? LD_LDS(ctl + MB_RING + lane) : 0;
    if (a) {
      ST_LDS(ctl + MB_RING + lane, 0u);
      if (!(a & 1)) st_sys(ws + a, seq);
      else if (__hip_atomic_fetch_add((GAS u32*)(ws + SEG_OFF + g * 128) + (a >> 1), 1u, __ATOMIC_RELAXED, __HIP_MEMORY_SCOPE_AGENT) == 128 * seq - 1)
        st_sys(rf + (a >> 1) * FS, seq);
    }
    if (end == PULL_WAVES && __ballot(a != 0) == 0) break;
    __builtin_amdgcn_s_sleep(2);
  }
}

__device__ __forceinline__ void fin_role(const Params& p, u32 f, u32 seq) {
  const int tid = threadIdx.x, lane = tid & 63;
  const u32 wid = rfl(tid >> 6), rank = p.rank;
  char* ws = p.ws[0];
  char* tmo = ws + TMO_OFF;
  const auto rl = mk_rsrc(ws, DATA_BYTES);
  const auto ro = mk_rsrc(p.out[0], OUT_BYTES);
  const u32 vwide = (lane >> 4) * 8192 + (lane & 15) * 16;
  char* wsl = p.ws[lane & 7];
  for (u32 n = f; n < 256 + NX; n += 8 * FIN_WGS) {
    const u32 m = n < 256 ? n : xbase(seq) + n - 256;
    u32 ri, ci;
    tile_rc(n & 255, ri, ci);
    const u32 item = wid * 2, dst = fin_base(m < 256 ? rank : rank ^ PL) + (ri * 128 + item * 4) * 8192 + ci * 256;
    bool bad = false;
    wave_wait<SL_FIN>(ws + ((lane & 1) ? FPL_OFF + m * FS : FPI_OFF + (m * 4 + (wid >> 1)) * FS), 3, seq, tmo, bad);
    u32x4 r[2], a[2], b[2];
#pragma unroll
    for (int i = 0; i < 2; ++i) {
      r[i] = __builtin_amdgcn_raw_buffer_load_b128(rl, lane * 16, PIN_OFF + m * 16384 + (item + i) * 1024, AUX_SYS);
      a[i] = __builtin_amdgcn_raw_buffer_load_b128(rl, lane * 16, P32_OFF + m * 32768 + (item + i) * 2048, AUX_SYS);
      b[i] = __builtin_amdgcn_raw_buffer_load_b128(rl, lane * 16, P32_OFF + m * 32768 + (item + i) * 2048 + 1024, AUX_SYS);
    }
    bad |= __hip_atomic_load((const GAS u32*)tmo, __ATOMIC_RELAXED, __HIP_MEMORY_SCOPE_AGENT) == seq;
#pragma unroll
    for (int i = 0; i < 2; ++i) {
      float y[8];
#pragma unroll
      for (int e = 0; e < 4; ++e) {
        const u32 l0 = e < 2 ? a[i][2 * e] : b[i][2 * e - 4], l1 = e < 2 ? a[i][2 * e + 1] : b[i][2 * e - 3];
        y[2 * e] = silu(__uint_as_float(l0) + __uint_as_float(r[i][e] << 16));
        y[2 * e + 1] = silu(__uint_as_float(l1) + __uint_as_float(r[i][e] & 0xFFFF0000u));
      }
      const bf16x8 o = {(__bf16)y[0], (__bf16)y[1], (__bf16)y[2], (__bf16)y[3],
                        (__bf16)y[4], (__bf16)y[5], (__bf16)y[6], (__bf16)y[7]};
      st128(__builtin_bit_cast(u32x4, o) | (bad ? NAN2 : 0u), ro, vwide, dst + i * 4 * 8192);
    }
    WAIT_VM(0);
    __syncthreads();
    if (tid >= 1 && tid < NR && m < 256) st_sys(wsl + FF_OFF + (rank * 256 + m) * FS, seq);
  }
}

template <int KIND>
__device__ __forceinline__ void gemm_tile(const Params& p, u32 m, u32 hsel, u32 csel, u32 seq, u64 tstart, bool first, LAS char* lds) {
  const int tid = threadIdx.x, lane = tid & 63;
  const u32 wid = rfl(tid >> 6);
  const u32 rank = p.rank;
  char* ws = p.ws[0];
  char* tmo = ws + TMO_OFF;
  const auto rx = mk_rsrc(p.x, X_BYTES);
  const auto rw = mk_rsrc(p.w, W_BYTES);
  const auto rs = mk_rsrc(ws, DATA_BYTES);
  constexpr bool HALF = KIND == 1, QUART = KIND == 2;
  constexpr int KS = QUART ? 2 : 1, NOWN = 8 / KS, NRCV = 24 / KS;
  constexpr int MI = KIND ? 1 : 2, NDMA = HALF ? 3 : 4, NS = HALF ? NS_HALF : 4;
  constexpr u32 XB = HALF ? BLK / 2 : BLK, SB = XB + BLK;
  static_assert(NS * SB <= LDS_CTL, "stages fit");

  u32 vao[2], vbo[2], vaw[2], vbw[2];
#pragma unroll
  for (int q = 0; q < 2; ++q) {
    const u32 ra = wid * (HALF ? 8 : 16) + q * 8 + (lane >> 3), ca = (lane & 7) ^ ((ra >> 1) & 7);
    const u32 kb = wid * 8 + q * 4 + (lane >> 4), cbk = (lane & 15) ^ (((kb & 3) << 2) | ((kb >> 2) & 3));
    const u32 ap = QUART ? ra >> 6 : 0, ar = ra - ap * 64, bp = QUART ? cbk >> 3 : 0, bc = cbk - bp * 8;
    vao[q] = ar * 1024 + ca * 16 + ap * 128 - q * 1024;        vaw[q] = ar * 128 + ca * 16 + ap * 2 * BLK - q * 1024;
    vbo[q] = kb * 8192 + bc * 16 + bp * 64 * 8192 - q * 1024;  vbw[q] = kb * 256 + bc * 16 + bp * 2 * BLK - q * 1024;
  }
  const u32 lds0 = (u32)(size_t)lds;
  const u32 wm = wid & 1, wn = wid >> 1, h = lane >> 5;
  u32 xo[4], wo[2];
  {
    const u32 r = wm * 32 * MI + (lane & 31) + (QUART ? (wid >> 2) * 64 : 0);
#pragma unroll
    for (int ks = 0; ks < 4; ++ks) xo[ks] = r * 128 + (((2 * ks + h) ^ ((r >> 1) & 7)) * 16);
    const u32 bk = (lane >> 4) & 1, q = (lane & 15) >> 2, pp = lane & 3;
#pragma unroll
    for (int t = 0; t < 2; ++t)
      wo[t] = XB + 256 * (8 * h + 4 * t + q) + 16 * ((4 * wn + 2 * bk + (pp >> 1)) ^ ((q << 2) | (2 * h + t))) + 8 * (pp & 1);
  }
  LAS u32* lbad = (LAS u32*)(lds + LDS_CTL);
  u32 ri, ci;
  tile_rc(m, ri, ci);
  const u32 row0 = ((rank >> 1) & 1) * 2048 + ri * 128 + hsel * 64, col0 = (rank & 1) * 2048 + ci * 128 + csel * 64;
  u32 islot = 0, rslot = 0;
  auto next_slot = [&](u32 o) -> u32 { return o + SB == NS * SB ? 0 : o + SB; };
  auto issue_own = [&](int t) {
    const u32 dst = lds0 + islot, sa = row0 * 1024 + t * KS * 128, sw = t * KS * 64 * 8192 + col0 * 2;
    islot = next_slot(islot);
    if (HALF) dma3(dst + wid * 1024, dst + XB + wid * 2048, rx, vao[0], sa, rw, vbo[0], vbo[1], sw);
    else dma4(dst + wid * 2048, rx, vao[0], vao[1], sa, rw, vbo[0], vbo[1], sw);
  };
  u32 taba, tabw;
  {
    const u32 ktl = lane / 6, c = lane - ktl * 6, src = rank ^ (c + 1 + (c >= 3));
    const u32 base = IN_OFF + src * IN_SRC + ktl * KS * 2 * BLK;
    taba = base + ri * 8 * BLK + hsel * 8192;
    tabw = base + (ci * 8 + 1) * BLK + csel * 128;
  }
  auto issue = [&](int t) {
    const u32 dst = lds0 + islot, sa = __builtin_amdgcn_readlane(taba, t), sw = __builtin_amdgcn_readlane(tabw, t);
    islot = next_slot(islot);
    if (HALF) dma3(dst + wid * 1024, dst + XB + wid * 2048, rs, vaw[0], sa, rs, vbw[0], vbw[1], sw);
    else dma4(dst + wid * 2048, rs, vaw[0], vaw[1], sa, rs, vbw[0], vbw[1], sw);
  };
  const u32 fop = (lane >> 1) & 1, fq = min(lane >> 2, 11), fk = fq / 6, fc = fq - fk * 6;
  const char* fin = ws + FI_OFF + ((rank ^ (fc + 1 + (fc >= 3))) * 256 + (((fop ? ci : ri) * 4 + fk) * 2 + fop) * 2 + ((KIND && !fop) ? hsel : (lane & 1))) * FS;

  while (__builtin_amdgcn_s_memrealtime() - tstart < (u64)max(ri, ci) * STAGGER) __builtin_amdgcn_s_sleep(16);
  if (first && wid == 0) __builtin_amdgcn_fence(__ATOMIC_ACQUIRE, "");
  f32x16 acc[MI] = {};
  struct Frag { bf16x8 w[4], x[MI][4]; };
  auto read = [&](Frag& f) {
    const LAS char* st = lds + rslot;
    rslot = next_slot(rslot);
#pragma unroll
    for (int ks = 0; ks < 4; ++ks) {
      const bf16x4 w0 = __builtin_amdgcn_ds_read_tr16_b64_v4bf16((LAS bf16x4*)(st + wo[0] + ks * 4096));
      const bf16x4 w1 = __builtin_amdgcn_ds_read_tr16_b64_v4bf16((LAS bf16x4*)(st + wo[1] + ks * 4096));
      f.w[ks] = __builtin_shufflevector(w0, w1, 0, 1, 2, 3, 4, 5, 6, 7);
#pragma unroll
      for (int mi = 0; mi < MI; ++mi) f.x[mi][ks] = *(const LAS bf16x8*)(st + xo[ks] + mi * 4096);
    }
  };
  auto mma = [&](const Frag& f) {
#pragma unroll
    for (int ks = 0; ks < 4; ++ks)
#pragma unroll
      for (int mi = 0; mi < MI; ++mi) acc[mi] = __builtin_amdgcn_mfma_f32_32x32x16_bf16(f.w[ks], f.x[mi][ks], acc[mi], 0, 0, 0);
  };
  auto step = [&](auto rc, auto oc, int tn, const Frag& fc, Frag& fn) {
    constexpr int R = decltype(rc)::value;
    constexpr bool OWN = decltype(oc)::value;
    if constexpr (R >= 0) {
      WAIT_VM(R * NDMA);
      __builtin_amdgcn_s_waitcnt(0xC07F);
      CBAR();
      __builtin_amdgcn_s_barrier();
      CBAR();
      read(fn);
    }
    if (tn >= 0) { if (OWN) issue_own(tn); else issue(tn); }
    mma(fc);
  };
  Frag fa, fb;
  static_assert(NS % 2 == 0 && NS <= NOWN, "run structure");
  auto run = [&](auto oc, int n) {
    constexpr bool OWN = decltype(oc)::value;
    islot = rslot = 0;
    for (int t = 0; t < NS; ++t) { if (OWN) issue_own(t); else issue(t); }
    WAIT_VM((NS - 1) * NDMA);
    __builtin_amdgcn_s_barrier();
    CBAR();
    read(fa);
    for (int tn = NS; tn < n; tn += 2) {
      step(std::integral_constant<int, NS - 2>{}, oc, tn, fa, fb);
      step(std::integral_constant<int, NS - 2>{}, oc, tn + 1, fb, fa);
    }
    static_for<0, NS / 2>([&](auto ic) {
      constexpr int i = decltype(ic)::value;
      step(std::integral_constant<int, NS - 2 - 2 * i>{}, std::true_type{}, -1, fa, fb);
      step(std::integral_constant<int, NS - 3 - 2 * i>{}, std::true_type{}, -1, fb, fa);
    });
  };
  run(std::true_type{}, NOWN);
  if (wid < 2) {
    bool bad = false;
    wave_wait2<SL_GEMM>(fin + wid * 8 * FS, ~0ull, seq, tmo, bad);
    if (bad) *lbad = 1;
  }
  __builtin_amdgcn_s_waitcnt(0xC07F);
  __builtin_amdgcn_s_barrier();
  run(std::false_type{}, NRCV);
  const u32 tmv = __hip_atomic_load((const GAS u32*)tmo, __ATOMIC_RELAXED, __HIP_MEMORY_SCOPE_AGENT);
  __builtin_amdgcn_s_waitcnt(0xC07F);
  __builtin_amdgcn_s_barrier();
  CBAR();
  constexpr u32 EROW = 528;
#pragma unroll
  for (int mi = 0; mi < MI; ++mi) {
    const u32 mr = wm * 32 * MI + mi * 32 + (lane & 31);
#pragma unroll
    for (int gq = 0; gq < 4; ++gq) {
      const f32x4 o = {acc[mi][4 * gq], acc[mi][4 * gq + 1], acc[mi][4 * gq + 2], acc[mi][4 * gq + 3]};
      *(LAS f32x4*)(lds + mr * EROW + (wn * 32 + 8 * gq + 4 * h) * 4) = o;
    }
  }
  __syncthreads();
  const u32 bad = (*lbad || tmv == seq) ? NAN2 : 0u;
  const u32 keep = rank >> 2;
  const bool do_exp = !KIND || hsel != keep || m < NX, do_keep = !KIND || hsel == keep || m < NX;
  const u32 me = KIND && hsel == keep ? xbase(seq) + m : m, mk = KIND && hsel != keep ? xbase(seq) + m : m;
  if constexpr (QUART) {
    const u32 row = tid >> 3, ch = tid & 7;
    const LAS char* se = lds + row * EROW + ch * 32;
    const f32x4 c0 = *(const LAS f32x4*)se + *(const LAS f32x4*)(se + 256), c1 = *(const LAS f32x4*)(se + 16) + *(const LAS f32x4*)(se + 272);
    GAS u32* qc = (GAS u32*)(ws + QC_OFF + (m * 2 + hsel) * FS);
    if (do_exp) {
      const bf16x8 o = {(__bf16)c0[0], (__bf16)c0[1], (__bf16)c0[2], (__bf16)c0[3],
                        (__bf16)c1[0], (__bf16)c1[1], (__bf16)c1[2], (__bf16)c1[3]};
      st128(__builtin_bit_cast(u32x4, o) | bad, rs, row * 256 + ch * 16, P16_OFF + me * 16384 + csel * 128);
      WAIT_VM(0);
      __builtin_amdgcn_s_barrier();
      if (tid == 0 && __hip_atomic_fetch_add(qc, 1u, __ATOMIC_RELAXED, __HIP_MEMORY_SCOPE_AGENT) == 2 * seq - 1)
        st_sys(p.ws[PL] + FPR_OFF + me * FS, seq);
    }
    if (do_keep) {
      const u32 vo = (row >> 2) * 2048 + (row & 3) * 256 + ch * 16, e32 = P32_OFF + mk * 32768 + csel * 128;
      st128(__builtin_bit_cast(u32x4, c0) | bad, rs, vo, e32);
      st128(__builtin_bit_cast(u32x4, c1) | bad, rs, vo, e32 + 1024);
    }
    WAIT_VM(0);
    __syncthreads();
    if (tid == 64 && do_keep && __hip_atomic_fetch_add(qc + 1, 1u, __ATOMIC_RELAXED, __HIP_MEMORY_SCOPE_AGENT) == 2 * seq - 1)
      st_sys(ws + FPL_OFF + mk * FS, seq);
    return;
  }
  if (do_exp) {
    const u32 e16 = P16_OFF + me * 16384 + wid * 1024;
#pragma unroll
    for (int part = 0; part < 2; ++part) {
      const LAS char* se = lds + ((HALF ? 0 : (keep ^ 1) * 64) + part * 32 + (tid >> 4)) * EROW + (tid & 15) * 32;
      const f32x4 c0 = *(const LAS f32x4*)se, c1 = *(const LAS f32x4*)(se + 16);
      const bf16x8 o = {(__bf16)c0[0], (__bf16)c0[1], (__bf16)c0[2], (__bf16)c0[3],
                        (__bf16)c1[0], (__bf16)c1[1], (__bf16)c1[2], (__bf16)c1[3]};
      st128(__builtin_bit_cast(u32x4, o) | bad, rs, lane * 16, e16 + part * 8192);
    }
  }
  if (HALF && do_exp && do_keep) {
    WAIT_VM(0);
    __builtin_amdgcn_s_barrier();
    if (tid == 0) st_sys(p.ws[PL] + FPR_OFF + me * FS, seq);
  }
  if (do_keep) {
    const u32 e32 = P32_OFF + mk * 32768 + wid * 2048;
#pragma unroll
    for (int part = 0; part < 2; ++part) {
      const LAS char* sk = lds + ((HALF ? 0 : keep * 64) + part * 32 + (tid >> 4)) * EROW + (tid & 15) * 32;
      const u32x4 c0 = *(const LAS u32x4*)sk, c1 = *(const LAS u32x4*)(sk + 16);
      st128(c0 | bad, rs, lane * 16, e32 + part * 16384);
      st128(c1 | bad, rs, lane * 16, e32 + part * 16384 + 1024);
    }
  }
  WAIT_VM(0);
  __syncthreads();
  if (tid == 0 && do_exp && !(HALF && do_keep)) st_sys(p.ws[PL] + FPR_OFF + me * FS, seq);
  if (tid == 64 && do_keep) st_sys(ws + FPL_OFF + mk * FS, seq);
}

__device__ __forceinline__ void stage_unit(const Params& p, u32 lk, u32 seq) {
  const int tid = threadIdx.x, lane = tid & 63;
  const u32 wid = rfl(tid >> 6);
  const u32 lev = lk >> 2, kt = (wid >> 2) * 4 + (lk & 3), op = (wid >> 1) & 1, rc = (wid & 1) * 2048 + lev * 128;
  const auto r = mk_rsrc(op ? p.w : p.x, X_BYTES);
  const auto rs = mk_rsrc(p.ws[0], DATA_BYTES);
  const u32 vo = op ? (lane >> 4) * 8192 + (lane & 15) * 16 : (lane >> 3) * 1024 + (lane & 7) * 16;
  const u32 so = op ? kt * 64 * 8192 + rc * 2 : rc * 1024 + kt * 128;
  u32x4 v[16];
#pragma unroll
  for (int i = 0; i < 16; ++i) v[i] = __builtin_amdgcn_raw_buffer_load_b128(r, vo, so + i * (op ? 4 * 8192 : 8 * 1024), 0);
#pragma unroll
  for (int i = 0; i < 16; ++i) st128(v[i], rs, lane * 16, EX_OFF + (lk * 8 + wid) * BLK + i * 1024);
  WAIT_VM(0);
  __syncthreads();
  if (tid >= 1 && tid < NR && tid != PL) st_sys(p.ws[tid] + SG_OFF + (p.rank * 64 + lk) * FS, seq);
}

__device__ __forceinline__ void gemm_role(const Params& p, u32 k, u32 seq, LAS char* lds) {
  const u64 tstart = __builtin_amdgcn_s_memrealtime();
  constexpr u32 NG = 8 * GEMM_WGS, NSMALL = 2 * NQUART + 2 * NHALF;
  static_assert(NQUART <= NHALF && NSMALL <= NG, "unit schedule");
  if (k < 4 * NQUART) gemm_tile<2>(p, k >> 2, (k >> 1) & 1, k & 1, seq, tstart, true, lds);
  else if (k < NSMALL) gemm_tile<1>(p, (k >> 1) - NQUART, k & 1, 0, seq, tstart, true, lds);
  for (u32 n = k < NSMALL ? k + NG : k; n < 256 - NHALF + NSMALL; n += NG) gemm_tile<0>(p, n - NSMALL + NHALF, 0, 0, seq, tstart, n < NG, lds);
}

__global__ void __launch_bounds__(THREADS, 2) fused_gemm_allreduce_silu(Params p) {
  extern __shared__ __attribute__((aligned(16))) char smem[];
  const int lane = threadIdx.x & 63;
  const u32 wid = rfl(threadIdx.x >> 6);
  LAS u32* ctl = (LAS u32*)smem;
  LAS u32* lctl = (LAS u32*)(smem + LDS_CTL);
  u32 slot = (blockIdx.x & ~7u) | ((u32)__builtin_amdgcn_s_getreg((3 << 11) | 20) & 7);
  auto is_push = [&]() { return slot < 8 * ENG_WGS && (slot & 3) != 0 && wid < PUSH_WAVES; };
  if (__builtin_expect(is_push(), 1)) {
    if (push_role<false>(p, slot, wid, lane, lctl)) return;
  } else {
    if (wid == 7) {
      if (lane < 32) ctl[lane] = 0;
      if (lane == 0) {
        GAS u32* ctr = (GAS u32*)(p.ws[0] + CTR_OFF) + blockIdx.x;
        GAS u32* clm = (GAS u32*)(p.ws[0] + CLM_OFF);
        const u32 c = __hip_atomic_load(ctr, __ATOMIC_RELAXED, __HIP_MEMORY_SCOPE_AGENT);
        u32 sl = slot;
#pragma unroll 1
        for (int n = 0; n < 4 * GRID; ++n) {
          if (__hip_atomic_fetch_add(clm + sl * 16, 1u, __ATOMIC_RELAXED, __HIP_MEMORY_SCOPE_AGENT) == c) break;
          __hip_atomic_fetch_add(clm + sl * 16, ~0u, __ATOMIC_RELAXED, __HIP_MEMORY_SCOPE_AGENT);
          sl = (sl + 1) & (GRID - 1);
        }
        lctl[0] = 0; lctl[1] = sl; lctl[2] = c + 1;
      }
    }
    asm volatile("s_waitcnt lgkmcnt(0)" ::: "memory");
    __builtin_amdgcn_s_barrier();
    slot = rfl(LD_LDS(lctl + 1));
    if (threadIdx.x == THREADS - 1)
      __hip_atomic_store((GAS u32*)(p.ws[0] + CTR_OFF) + blockIdx.x, LD_LDS(lctl + 2), __ATOMIC_RELAXED, __HIP_MEMORY_SCOPE_AGENT);
    WAIT_VM(0);
  }
  if (__builtin_expect(is_push(), 0)) {
    push_role<true>(p, slot, wid, lane, lctl);
    return;
  }
  const u32 seq = rfl(LD_LDS(lctl + 2));
  const u32 g = slot & 7, s = slot >> 3;
  const u32 peer = ((u32)p.rank ^ g) & (NR - 1);
  constexpr u32 LK0 = NPUSH / 4;
  if (s < ENG_WGS) {
    if (g == 0) {
      for (u32 u = s; u < 64 - LK0; u += u < 16 ? 16 : u < 24 ? 16 : 8) stage_unit(p, LK0 + u, seq);
    } else if (wid >= PUSH_WAVES && wid < PUSH_WAVES + PULL_WAVES) {
      pull_role(p, peer, (wid - PUSH_WAVES) * ENG_WGS + s, lane, seq, ctl);
    } else if (wid == PUSH_WAVES + PULL_WAVES) {
      flag_role(p, g, lane, seq, ctl);
    }
  } else if (s < ENG_WGS + FIN_WGS) {
    if ((g & 3) == 0) { stage_unit(p, LK0 + 8 + g + s - ENG_WGS, seq); stage_unit(p, LK0 + 24 + g + s - ENG_WGS, seq); }
    fin_role(p, (s - ENG_WGS) * 8 + g, seq);
  } else {
    const u32 k = s - ENG_WGS - FIN_WGS;
    gemm_role(p, (k >> 2) * 32 + g * 4 + (k & 3), seq, (LAS char*)smem);
  }
}

}

extern "C" void kernel_launch(int rank, int nranks, ncclComm_t comm,
                              hipStream_t stream,
                              void* const* d_in, const int* in_sizes, int n_in,
                              void* d_out, int out_size,
                              const int* in_dtypes, int out_dtype,
                              void* const* d_out_all,
                              void* const* d_ws_all, size_t ws_bytes) {
  if (nranks != NR || (unsigned)rank >= (unsigned)NR || n_in < 2 || ws_bytes < WS_USED ||
      in_sizes[0] != 4096 * 512 || in_sizes[1] != 512 * 4096 || out_size != 4096 * 4096) {
    fprintf(stderr, "kernel_launch: unexpected problem shape\n");
    return;
  }
  HIP_CHECK(hipFuncSetAttribute((const void*)fused_gemm_allreduce_silu,
                                hipFuncAttributeMaxDynamicSharedMemorySize, LDS_BYTES));
  Params p;
  p.x = (const char*)d_in[0];
  p.w = (const char*)d_in[1];
  const int phys[NR] = {0, 2, 3, 6, 1, 4, 5, 7};
  p.rank = 0; p.pad = 0;
  for (int i = 0; i < NR; ++i) if (phys[i] == rank) p.rank = i;
  for (int i = 0; i < NR; ++i) { p.ws[i] = (char*)d_ws_all[phys[p.rank ^ i]]; p.out[i] = (char*)d_out_all[phys[p.rank ^ i]]; }
  for (u32 g = 0; g < NR; ++g)
    for (u32 job = 0; job < 16; ++job) {
      const u32 peer = p.rank ^ g, lev = job >> 4, kt = (peer >> 2) * 4 + ((job >> 2) & 3), op = (job >> 1) & 1, half = job & 1;
      p.psrc[g][job] = op ? (kt * 64 + half * 32) * 8192 + ((peer & 1) * 2048 + lev * 128) * 2
                                         : (((peer >> 1) & 1) * 2048 + lev * 128 + half * 64) * 1024 + kt * 128;
    }
  hipLaunchKernelGGL(fused_gemm_allreduce_silu, dim3(GRID), dim3(THREADS), LDS_BYTES, stream, p);
  HIP_CHECK(hipGetLastError());
}
```

```cpp
#include <hip/hip_runtime.h>
#include <rccl/rccl.h>
#include <cstdio>
#include <type_traits>

#define HIP_CHECK(expr) do { hipError_t _e = (expr); if (_e != hipSuccess) { \
    fprintf(stderr, "HIP error %s at %s:%d\n", hipGetErrorString(_e), __FILE__, __LINE__); \
    return; } } while (0)

namespace {

typedef unsigned int u32;
typedef unsigned long long u64;
typedef u32 u32x4 __attribute__((ext_vector_type(4)));
typedef float f32x4 __attribute__((ext_vector_type(4)));
typedef __bf16 bf16x4 __attribute__((ext_vector_type(4)));
typedef __bf16 bf16x8 __attribute__((ext_vector_type(8)));
typedef float f32x16 __attribute__((ext_vector_type(16)));
#define LAS __attribute__((address_space(3)))
#define GAS __attribute__((address_space(1)))

constexpr int NR = 8;
constexpr u32 X_BYTES = 4096u * 512u * 2u;
constexpr u32 W_BYTES = 512u * 4096u * 2u;
constexpr u32 OUT_BYTES = 4096u * 4096u * 2u;
constexpr u32 FS = 128;

constexpr u32 BLK = 16384;
constexpr u32 IN_OFF = 0;
constexpr u32 IN_SRC = 128 * BLK;
constexpr u32 P16_OFF = IN_OFF + NR * IN_SRC;
constexpr u32 PIN_OFF = P16_OFF + 320 * 16384;
constexpr u32 P32_OFF = PIN_OFF + 320 * 16384;
constexpr u32 EX_OFF = P32_OFF + 320 * 32768;
constexpr u32 DATA_BYTES = EX_OFF + 512 * BLK;
constexpr u32 FI_OFF = DATA_BYTES;
constexpr u32 FPR_OFF = FI_OFF + NR * 256 * FS;
constexpr u32 FPL_OFF = FPR_OFF + 320 * FS;
constexpr u32 FPI_OFF = FPL_OFF + 320 * FS;
constexpr u32 FF_OFF = FPI_OFF + 1280 * FS;
constexpr u32 SG_OFF = FF_OFF + NR * 256 * FS;
constexpr u32 GO_OFF = SG_OFF + NR * 64 * FS;
constexpr u32 RF_OFF = GO_OFF + NR * 16 * FS;
constexpr u32 QC_OFF = RF_OFF + 64 * FS;
constexpr u32 CTR_OFF = QC_OFF + 128 * FS;
constexpr u32 CLM_OFF = CTR_OFF + 256 * 4;
constexpr u32 TMO_OFF = CLM_OFF + 256 * 64;
constexpr u32 TICK_OFF = TMO_OFF + 128;
constexpr u32 SEG_OFF = TICK_OFF + 32 * 128;
constexpr u32 WS_USED = SEG_OFF + 8 * 128;

constexpr u64 DEADLINE_TICKS = 80000000ull;

constexpr int ENG_WGS = 8, FIN_WGS = 4, GEMM_WGS = 20;
constexpr int PUSH_WAVES = 2, PULL_WAVES = 2;
constexpr int PUSH_D = 4, PUSH_ROUNDS = 4;
constexpr u32 NPUSH = PUSH_ROUNDS * ENG_WGS * PUSH_WAVES;
constexpr u32 PL = 4;
constexpr int GO_AT = 6;
constexpr int NS_HALF = 6;
constexpr u32 NQUART = 36, NHALF = 100;
constexpr u64 STAGGER = 190;
constexpr u32 NX = 20, TOPK = 44;
constexpr int SL_PULL = 2, SL_FIN = 4, SL_GEMM = 8;
constexpr u32 NAN2 = 0x7FC07FC0u;
constexpr int GRID = 8 * (ENG_WGS + FIN_WGS + GEMM_WGS);
static_assert(GRID == 256, "slots wrap at a power of two");
constexpr int THREADS = 512;
constexpr u32 LDS_CTL = 144 * 1024;
constexpr u32 LDS_BYTES = LDS_CTL + 64;
constexpr int AUX_SYS = 17;

struct Params {
  const char* x;
  const char* w;
  int rank;
  int pad;
  char* ws[NR];
  char* out[NR];
  u32 psrc[NR][16];
};

__device__ __forceinline__ u32 rfl(u32 v) { return __builtin_amdgcn_readfirstlane(v); }
__device__ __forceinline__ __amdgpu_buffer_rsrc_t mk_rsrc(const void* p, u32 n) {
  const u64 a = (u64)p;
  const u64 u = ((u64)rfl((u32)(a >> 32)) << 32) | rfl((u32)a);
  return __builtin_amdgcn_make_buffer_rsrc((void*)u, 0, n, 0x00020000);
}
__device__ __forceinline__ u32 ld_sys(const char* p) {
  return __hip_atomic_load((const GAS u32*)p, __ATOMIC_RELAXED, __HIP_MEMORY_SCOPE_SYSTEM);
}
__device__ __forceinline__ void st_sys(char* p, u32 v) {
  __hip_atomic_store((GAS u32*)p, v, __ATOMIC_RELAXED, __HIP_MEMORY_SCOPE_SYSTEM);
}
#define WAIT_VM(n) asm volatile("s_waitcnt vmcnt(%0)" ::"n"(n) : "memory")
#define CBAR() asm volatile("" ::: "memory")
#define LD_LDS(q) __hip_atomic_load(q, __ATOMIC_RELAXED, __HIP_MEMORY_SCOPE_WORKGROUP)
#define ST_LDS(q, v) __hip_atomic_store(q, v, __ATOMIC_RELAXED, __HIP_MEMORY_SCOPE_WORKGROUP)

__device__ __forceinline__ void st128(u32x4 v, __amdgpu_buffer_rsrc_t r, u32 voff, u32 soff) {
  __builtin_amdgcn_raw_buffer_store_b128(v, r, voff, soff, AUX_SYS);
  asm volatile("s_nop 1" ::"v"(v) : "memory");
}

template <int SLEEP>
__device__ __forceinline__ u64 wave_wait(const char* flag, u64 need, u32 seq, char* tmo, bool& bad) {
  const u64 t0 = __builtin_amdgcn_s_memrealtime();
  for (u32 it = 1;; ++it) {
    const u64 ok = __ballot(ld_sys(flag) >= seq);
    if ((ok & need) == need) return ok;
    if ((it & 15) == 0 || bad) {
      const u32 t = __hip_atomic_load((const GAS u32*)tmo, __ATOMIC_RELAXED, __HIP_MEMORY_SCOPE_AGENT);
      if (rfl(t) == seq || __builtin_amdgcn_s_memrealtime() - t0 > DEADLINE_TICKS) {
        __hip_atomic_store((GAS u32*)tmo, seq, __ATOMIC_RELAXED, __HIP_MEMORY_SCOPE_AGENT);
        bad = true;
        return ~0ull;
      }
    }
    __builtin_amdgcn_s_sleep(SLEEP);
  }
}

template <int SLEEP>
__device__ __forceinline__ u64 wave_wait2(const char* flag, u64 need, u32 seq, char* tmo, bool& bad) {
  const u64 t0 = __builtin_amdgcn_s_memrealtime();
  const bool needed = (need >> (threadIdx.x & 63)) & 1;
  bool ok = false;
  for (u32 it = 1;; ++it) {
    if (!ok && (it == 1 || needed)) ok = ld_sys(flag) >= seq;
    const u64 have = __ballot(ok);
    if ((have & need) == need) return have;
    if ((it & 15) == 0) {
      const u32 t = __hip_atomic_load((const GAS u32*)tmo, __ATOMIC_RELAXED, __HIP_MEMORY_SCOPE_AGENT);
      if (rfl(t) == seq || __builtin_amdgcn_s_memrealtime() - t0 > DEADLINE_TICKS) {
        __hip_atomic_store((GAS u32*)tmo, seq, __ATOMIC_RELAXED, __HIP_MEMORY_SCOPE_AGENT);
        bad = true;
        return ~0ull;
      }
    }
    __builtin_amdgcn_s_sleep(SLEEP);
  }
}

__device__ __forceinline__ void dma4(u32 dst, __amdgpu_buffer_rsrc_t ra, u32 va0, u32 va1, u32 sa,
                                     __amdgpu_buffer_rsrc_t rb, u32 vb0, u32 vb1, u32 sb) {
  u32 keep;
  dst = rfl(dst); sa = rfl(sa); sb = rfl(sb);
  asm volatile(
      "s_mov_b32 %0, m0\n\t"
      "s_nop 4\n\t"
      "s_mov_b32 m0, %1\n\t"
      "s_nop 0\n\t"
      "buffer_load_dwordx4 %3, %5, %6 offen lds\n\t"
      "buffer_load_dwordx4 %4, %5, %6 offen offset:1024 lds\n\t"
      "s_mov_b32 m0, %2\n\t"
      "s_nop 0\n\t"
      "buffer_load_dwordx4 %7, %9, %10 offen lds\n\t"
      "buffer_load_dwordx4 %8, %9, %10 offen offset:1024 lds\n\t"
      "s_mov_b32 m0, %0"
      : "=&s"(keep)
      : "s"(dst), "s"(dst + BLK), "v"(va0), "v"(va1), "s"(ra), "s"(sa), "v"(vb0), "v"(vb1), "s"(rb), "s"(sb)
      : "memory");
}

__device__ __forceinline__ void dma3(u32 dsta, u32 dstb, __amdgpu_buffer_rsrc_t ra, u32 va0, u32 sa,
                                     __amdgpu_buffer_rsrc_t rb, u32 vb0, u32 vb1, u32 sb) {
  u32 keep;
  dsta = rfl(dsta); dstb = rfl(dstb); sa = rfl(sa); sb = rfl(sb);
  asm volatile(
      "s_mov_b32 %0, m0\n\t"
      "s_nop 4\n\t"
      "s_mov_b32 m0, %1\n\t"
      "s_nop 0\n\t"
      "buffer_load_dwordx4 %3, %4, %5 offen lds\n\t"
      "s_mov_b32 m0, %2\n\t"
      "s_nop 0\n\t"
      "buffer_load_dwordx4 %6, %8, %9 offen lds\n\t"
      "buffer_load_dwordx4 %7, %8, %9 offen offset:1024 lds\n\t"
      "s_mov_b32 m0, %0"
      : "=&s"(keep)
      : "s"(dsta), "s"(dstb), "v"(va0), "s"(ra), "s"(sa), "v"(vb0), "v"(vb1), "s"(rb), "s"(sb)
      : "memory");
}

template <int I, int N, class F>
__device__ __forceinline__ void static_for(F&& f) {
  if constexpr (I < N) { f(std::integral_constant<int, I>{}); static_for<I + 1, N>(f); }
}

__device__ __forceinline__ void tile_rc(u32 m, u32& ri, u32& ci) {
  u32 l = 0;
#pragma unroll
  for (u32 t = 1; t < 16; ++t) l += m >= t * t;
  const u32 idx = m - l * l;
  ri = idx < l ? l : idx - l;
  ci = idx < l ? idx : l;
}
__device__ __forceinline__ u32 fin_base(u32 r) { return (((r >> 1) & 1) * 2048 + (r >> 2) * 64) * 8192 + (r & 1) * 4096; }

template <bool claimed>
__device__ __forceinline__ bool push_role(const Params& p, u32& slot, u32 wid, int lane, LAS u32* lctl) {
  constexpr int D = PUSH_D, JI = 8;
  constexpr u32 NW = ENG_WGS * PUSH_WAVES;
  static_assert(PUSH_ROUNDS >= 3 && NW == 16 && X_BYTES == W_BYTES && GO_AT <= JI, "push schedule");
  const u32 g = slot & 7, w = wid * ENG_WGS + (slot >> 3), op = (slot >> 4) & 1;
  u64 bs = (u64)(&p.x)[op], bw = (u64)p.ws[g];
  u32 src = p.psrc[g][w];
  u32 rank = p.rank;
  asm volatile("" : "+s"(bs), "+s"(bw), "+s"(src), "+s"(rank));
  const auto rs = mk_rsrc((const void*)bs, X_BYTES);
  const u32 vs = op ? (lane >> 4) * 8192 + (lane & 15) * 16 : (lane >> 3) * 1024 + (lane & 7) * 16;
  const u32 str = op ? 4 * 8192 : 8 * 1024;
  auto ld = [&](int r, int i) -> u32x4 { return __builtin_amdgcn_raw_buffer_load_b128(rs, vs, src + r * (op ? 256 : 128 * 1024) + i * str, 0); };
  u32x4 v[D], v2[JI - D];
#pragma unroll
  for (int i = 0; i < D; ++i) { v[i] = ld(0, i); CBAR(); }
#pragma unroll
  for (int i = D; i < JI; ++i) { v2[i - D] = ld(0, i); CBAR(); }
  char* wsp = (char*)bw;
  const auto rd = mk_rsrc(wsp, DATA_BYTES);
  const u32 dst = IN_OFF + rank * IN_SRC + w * 8192, flag = FI_OFF + (rank * 256 + w) * FS;
  u32 seq = 0;
  auto round = [&](auto rc) {
    constexpr int R = decltype(rc)::value;
    constexpr bool FIRST = R == 0, LAST = R == PUSH_ROUNDS - 1;
#pragma unroll
    for (int i = 0; i < JI; ++i) {
      if (!FIRST && i == D - 1) {
        WAIT_VM(2 * D - 1);
        if (R == 1) {
          if (!claimed) __builtin_amdgcn_s_barrier();
          CBAR();
          seq = rfl(LD_LDS(lctl + 2));
        }
        if (lane == 0) st_sys(wsp + flag + (R - 1) * NW * FS, seq);
        CBAR();
      }
      if (LAST && i == JI - GO_AT) {
        if (lane == 0) st_sys(wsp + GO_OFF + (rank * 16 + w) * FS, seq);
        if (lane == 0) ST_LDS(lctl - LDS_CTL / 4 + 24 + wid, 1u);
        CBAR();
      }
      st128(FIRST && i >= D ? v2[i >= D ? i - D : 0] : v[i % D], rd, lane * 16, dst + R * NW * 8192 + i * 1024);
      if (i + D < JI) { if (!FIRST) v[i % D] = ld(R, i + D); }
      else if (!LAST) v[i % D] = ld(LAST ? R : R + 1, i + D - JI);
      CBAR();
    }
  };
  static_assert(JI - D >= D - 1, "the count above also holds in the last round");
  round(std::integral_constant<int, 0>{});
  round(std::integral_constant<int, 1>{});
  const u32 slot2 = rfl(LD_LDS(lctl + 1));
  if (__builtin_expect(slot2 != slot, 0)) { WAIT_VM(0); slot = slot2; return false; }
  static_for<2, PUSH_ROUNDS>(round);
  WAIT_VM(0);
  if (lane == 0) st_sys(wsp + flag + (PUSH_ROUNDS - 1) * NW * FS, seq);
  return true;
}

__device__ __forceinline__ float silu(float a) {
  const float c = fminf(fmaxf(a, -60.f), 60.f);
  return a * __builtin_amdgcn_rcpf(1.f + __expf(-c));
}

constexpr u32 MB_RING = 0, MB_END = 16, MB_PUSHED = 24;

constexpr u32 NE = 4 * NX, NPOOL = (1024 - NE) / 2;
static_assert(NX <= NHALF && NX <= 32, "early tiles");
__device__ __forceinline__ u32 xbase(u32 seq) { return 256 + (seq & 1) * 32; }
__device__ __forceinline__ u32 pool_job(u32 pos) {
  return pos < NPOOL - 2 * TOPK ? pos : pos >= NPOOL - TOPK ? pos - TOPK : pos + TOPK;
}

__device__ __forceinline__ void pull_role(const Params& p, u32 peer, u32 w, int lane, u32 seq, LAS u32* ctl) {
  constexpr u32 NW = ENG_WGS * PULL_WAVES, NONE = ~0u, REL = 1u << 31;
  const u32 rank = p.rank, g = rank ^ peer;
  const bool pl = g == PL;
  const u32 NI = pl ? 1280 : 256 - NPUSH, NJ = pl ? 1024 + 4 * NX : NI + 1024;
  char* ws = p.ws[0];
  char* tmo = ws + TMO_OFF;
  const u64 bf = (u64)p.out[g], bp = (u64)p.ws[g], bo = (u64)p.out[0];
  const u32 vwide = (lane >> 4) * 8192 + (lane & 15) * 16;
  const auto rtick = mk_rsrc(ws + TICK_OFF + (seq & 1) * 2048, 2048);
  if (w == 0 && (lane == 0 || (pl && lane < 3))) st_sys(ws + TICK_OFF + ((seq & 1) ^ 1) * 2048 + (lane ? 7 + lane : g) * 128, 0);
  const u32 bi = (rank >> 1) & 1, bj = rank & 1;
  const u32 isrc = pl ? P16_OFF : EX_OFF + (NPUSH / 4 * 8 + (rank >> 2) * 4 + bi) * BLK;
  const u32 is1 = pl ? BLK : 8 * BLK, is2 = pl ? 8192 : (2 + bj - bi) * BLK, isz = pl ? 4096 : 8192;
  const u32 idst = pl ? PIN_OFF : IN_OFF + peer * IN_SRC + NPUSH * 8192;
  const u32 ipre = pl ? FPR_OFF : SG_OFF + (peer * 64 + NPUSH / 4) * FS, ish = 2;
  const u32 ipost = pl ? FPI_OFF : FI_OFF + (peer * 256 + NPUSH) * FS;
  const u32 fown = fin_base(peer), fpar = fin_base(rank ^ PL);
  const u32 rpre = pl ? FF_OFF + peer * 256 * FS : RF_OFF + g * 8 * FS, rsh = pl ? 2 : 7;
  u32 ttile = 0;
#pragma unroll
  for (u32 e = 0; e < 4; ++e) {
    u32 ri, ci;
    tile_rc(lane * 4 + e, ri, ci);
    ttile |= (ri << 4 | ci) << (8 * e);
  }
  const u64 t0 = __builtin_amdgcn_s_memrealtime();
  u32 tf = w, tf2 = NW + w, lastf = 0, sub = 0;
  u32 cdst = 0, cact = 0, pend = 0, ack = 0;
  u32 nring = 0;
  u32 from = 1, spare = NONE;
  bool bad = false, cisf = false;
  auto poll = [&]() -> u32 {
    const u32 o = tf < NI ? ipre + (tf >> ish) * FS : tf < REL ? FF_OFF + (peer * 256 + ((tf - NI) >> 2)) * FS : rpre + ((tf & 1023) >> rsh) * FS;
    return ld_sys(ws + rfl(o));
  };
  u32 pv = 0;
  u32x4 v[4];
  const u32 tvo = lane == 0 ? g * 128 : (8 + (w & 1)) * 128, tinc = lane && pl ? 1u << 16 : 1u, tbit = lane < 2 ? lane + 1 : 0;
  auto draw = [&](u32 m) -> u32 {
    return __builtin_amdgcn_raw_ptr_buffer_atomic_add_i32(tinc, rtick, (m & tbit) ? tvo : 0x80000000u, 0, 0);
  };
  u32 sstart, sbase;
  {
    const u32 l = lane >> 1, xb = min(l * l, NX);
    sstart = 4 * (((lane & 1) ? (l + 1) * (l + 1) : l * l) + xb);
    sbase = 4 * ((lane & 1) ? xbase(seq) + xb : l * l);
  }
  auto handle = [&](u32 id) -> u32 {
    if (!pl) return id;
    const u32 sg = 63 - __builtin_clzll(__ballot(sstart <= id));
    return __builtin_amdgcn_readlane(sbase, sg) + id - __builtin_amdgcn_readlane(sstart, sg);
  };
  tf = handle(tf); tf2 = handle(tf2);
  auto drawn = [&](u32 m, u32 t) -> u32 {
    u32 own = NONE, tp = __builtin_amdgcn_readlane(t, 1);
    if (m & 1) {
      const u32 id = __builtin_amdgcn_readlane(t, 0) + 2 * NW;
      if (id < NJ) own = handle(id);
      if (m == 1) {
        if (__builtin_expect(id < NJ, 1)) { from = id + NW >= NJ ? 3 : 1; return own; }
        tp = __builtin_amdgcn_readlane(draw(2), 1);
        asm volatile("" : "+s"(tp));
      }
    } else if (m == 0) {
      own = spare; spare = NONE;
      return own;
    }
    const u32 f = tp & 0xFFFF, b = tp >> 16;
    const u32 hp = f + b < NPOOL ? REL | (NE + pool_job(pl ? NPOOL - 1 - b : f) * 2 + (w & 1)) : NONE;
    from = hp == NONE ? 0 : 2;
    spare = own == NONE ? NONE : hp;
    return own == NONE ? hp : own;
  };
  auto put = [&](int i) {
    const auto ro = mk_rsrc((const void*)(cisf ? bo : (u64)ws), cisf ? OUT_BYTES : DATA_BYTES);
    st128(v[i], ro, cisf ? vwide : lane * 16, rfl(cdst + i * (cisf ? 4 * 8192 : 1024)));
  };
  auto done = [&](u32 a) {
    if (a) {
      LAS u32* e = ctl + MB_RING + (w / ENG_WGS) * 8 + (nring++ & 7);
      while (rfl(LD_LDS(e)) != 0) __builtin_amdgcn_s_sleep(1);
      if (lane == 0) ST_LDS(e, a);
      CBAR();
    }
  };
  auto ready = [&]() -> bool { return tf != NONE && (bad || (__ballot(pv >= seq) & 1)); };
  auto round = [&](auto first_c) {
    constexpr bool FIRST = decltype(first_c)::value;
    const u32 h = tf, hs = sub;
    const bool isf = h >= NI, tix = isf || hs || pl;
    sub = tix ? 0 : 1;
    tf = tix ? tf2 : tf;
    const u32 dm = tix && spare == NONE ? from : 0, tk = draw(dm);
    pv = poll();
    CBAR();
    done(ack);
    u32 nsrc, ndst, nact;
    if (isf) {
      const u32 j = h < REL ? h - NI : h & 1023, rc = (__builtin_amdgcn_readlane(ttile, j >> 4) >> (2 * (j & 12))) & 255;
      nsrc = ndst = (h < REL ? fown : fpar) + ((rc >> 4) * 128 + (j & 3) * 16) * 8192 + (rc & 15) * 256;
      nact = h < REL ? (j >> 7) * 2 + 1 : 0;
      lastf = ndst;
    } else {
      nsrc = isrc + (h >> 2) * is1 + ((h >> 1) & 1) * is2 + (h & 1) * isz + hs * 4096;
      ndst = idst + h * isz + hs * 4096;
      nact = tix ? ipost + h * FS : 0;
    }
    const auto rsrc = mk_rsrc((const void*)(isf ? bf : bp), isf ? OUT_BYTES : DATA_BYTES);
    const u32 vs = isf ? vwide : lane * 16, str = isf ? 4 * 8192 : 1024;
#pragma unroll
    for (int i = 0; i < 4; ++i) {
      if (!FIRST) put(i);
      v[i] = __builtin_amdgcn_raw_buffer_load_b128(rsrc, vs, rfl(nsrc + i * str), AUX_SYS);
      CBAR();
    }
    ack = 0;
    if (!FIRST) {
      WAIT_VM(11);
      ack = pend; pend = cact;
    }
    cdst = ndst; cisf = isf; cact = nact;
    if (tix) tf2 = drawn(dm, tk);
  };
  if (!pl) {
    while (rfl(LD_LDS(ctl + MB_PUSHED + w / ENG_WGS)) == 0) __builtin_amdgcn_s_sleep(2);
    wave_wait<1>(ws + ((lane & 1) ? GO_OFF + (peer * 16 + w) * FS : ipre + (w >> ish) * FS), 3, seq, tmo, bad);
    if (bad) WAIT_VM(0);
    pv = seq;
  }
  for (u32 it = 1; tf != NONE; ++it) {
    if (ready()) {
      round(std::true_type{});
      if (ready()) {
        round(std::false_type{});
        while (ready()) round(std::false_type{});
      }
#pragma unroll
      for (int i = 0; i < 4; ++i) put(i);
      done(ack);
      if (pend | cact) {
        WAIT_VM(0);
        done(pend);
        done(cact);
      }
      ack = pend = 0;
      continue;
    }
    if ((it & 63) == 0) {
      const u32 t = __hip_atomic_load((const GAS u32*)tmo, __ATOMIC_RELAXED, __HIP_MEMORY_SCOPE_AGENT);
      if (rfl(t) == seq || __builtin_amdgcn_s_memrealtime() - t0 > DEADLINE_TICKS) {
        __hip_atomic_store((GAS u32*)tmo, seq, __ATOMIC_RELAXED, __HIP_MEMORY_SCOPE_AGENT);
        WAIT_VM(0);
        bad = true;
      }
    }
    __builtin_amdgcn_s_sleep(SL_PULL * 3);
    pv = poll();
  }
  if (bad) {
    WAIT_VM(0);
    const u32x4 nanv = {NAN2, NAN2, NAN2, NAN2};
    for (int i = 0; i < 4; ++i) st128(nanv, mk_rsrc(p.out[0], OUT_BYTES), vwide, rfl(lastf) + i * 4 * 8192);
  }
  if (lane == 0) __hip_atomic_fetch_add(ctl + MB_END, 1u, __ATOMIC_RELAXED, __HIP_MEMORY_SCOPE_WORKGROUP);
}

__device__ __forceinline__ void flag_role(const Params& p, u32 g, int lane, u32 seq, LAS u32* ctl) {
  char* ws = p.ws[0];
  char* rf = p.ws[g ^ PL] + RF_OFF + (g ^ PL) * 8 * FS;
  for (;;) {
    const u32 end = rfl(LD_LDS(ctl + MB_END));
    const u32 a = lane < 16 ? LD_LDS(ctl + MB_RING + lane) : 0;
    if (a) {
      ST_LDS(ctl + MB_RING + lane, 0u);
      if (!(a & 1)) st_sys(ws + a, seq);
      else if (__hip_atomic_fetch_add((GAS u32*)(ws + SEG_OFF + g * 128) + (a >> 1), 1u, __ATOMIC_RELAXED, __HIP_MEMORY_SCOPE_AGENT) == 128 * seq - 1)
        st_sys(rf + (a >> 1) * FS, seq);
    }
    if (end == PULL_WAVES && __ballot(a != 0) == 0) break;
    __builtin_amdgcn_s_sleep(2);
  }
}

__device__ __forceinline__ void fin_role(const Params& p, u32 f, u32 seq) {
  const int tid = threadIdx.x, lane = tid & 63;
  const u32 wid = rfl(tid >> 6), rank = p.rank;
  char* ws = p.ws[0];
  char* tmo = ws + TMO_OFF;
  const auto rl = mk_rsrc(ws, DATA_BYTES);
  const auto ro = mk_rsrc(p.out[0], OUT_BYTES);
  const u32 vwide = (lane >> 4) * 8192 + (lane & 15) * 16;
  char* wsl = p.ws[lane & 7];
  for (u32 n = f; n < 256 + NX; n += 8 * FIN_WGS) {
    const u32 m = n < 256 ? n : xbase(seq) + n - 256;
    u32 ri, ci;
    tile_rc(n & 255, ri, ci);
    const u32 item = wid * 2, dst = fin_base(m < 256 ? rank : rank ^ PL) + (ri * 128 + item * 4) * 8192 + ci * 256;
    bool bad = false;
    wave_wait<SL_FIN>(ws + ((lane & 1) ? FPL_OFF + m * FS : FPI_OFF + (m * 4 + (wid >> 1)) * FS), 3, seq, tmo, bad);
    u32x4 r[2], a[2], b[2];
#pragma unroll
    for (int i = 0; i < 2; ++i) {
      r[i] = __builtin_amdgcn_raw_buffer_load_b128(rl, lane * 16, PIN_OFF + m * 16384 + (item + i) * 1024, AUX_SYS);
      a[i] = __builtin_amdgcn_raw_buffer_load_b128(rl, lane * 16, P32_OFF + m * 32768 + (item + i) * 2048, AUX_SYS);
      b[i] = __builtin_amdgcn_raw_buffer_load_b128(rl, lane * 16, P32_OFF + m * 32768 + (item + i) * 2048 + 1024, AUX_SYS);
    }
    bad |= __hip_atomic_load((const GAS u32*)tmo, __ATOMIC_RELAXED, __HIP_MEMORY_SCOPE_AGENT) == seq;
#pragma unroll
    for (int i = 0; i < 2; ++i) {
      float y[8];
#pragma unroll
      for (int e = 0; e < 4; ++e) {
        const u32 l0 = e < 2 ? a[i][2 * e] : b[i][2 * e - 4], l1 = e < 2 ? a[i][2 * e + 1] : b[i][2 * e - 3];
        y[2 * e] = silu(__uint_as_float(l0) + __uint_as_float(r[i][e] << 16));
        y[2 * e + 1] = silu(__uint_as_float(l1) + __uint_as_float(r[i][e] & 0xFFFF0000u));
      }
      const bf16x8 o = {(__bf16)y[0], (__bf16)y[1], (__bf16)y[2], (__bf16)y[3],
                        (__bf16)y[4], (__bf16)y[5], (__bf16)y[6], (__bf16)y[7]};
      st128(__builtin_bit_cast(u32x4, o) | (bad ? NAN2 : 0u), ro, vwide, dst + i * 4 * 8192);
    }
    WAIT_VM(0);
    __syncthreads();
    if (tid >= 1 && tid < NR && m < 256) st_sys(wsl + FF_OFF + (rank * 256 + m) * FS, seq);
  }
}

template <int KIND>
__device__ __forceinline__ void gemm_tile(const Params& p, u32 m, u32 hsel, u32 csel, u32 seq, u64 tstart, bool first, LAS char* lds) {
  const int tid = threadIdx.x, lane = tid & 63;
  const u32 wid = rfl(tid >> 6);
  const u32 rank = p.rank;
  char* ws = p.ws[0];
  char* tmo = ws + TMO_OFF;
  const auto rx = mk_rsrc(p.x, X_BYTES);
  const auto rw = mk_rsrc(p.w, W_BYTES);
  const auto rs = mk_rsrc(ws, DATA_BYTES);
  constexpr bool HALF = KIND == 1, QUART = KIND == 2;
  constexpr int KS = QUART ? 2 : 1, NOWN = 8 / KS, NRCV = 24 / KS;
  constexpr int MI = KIND ? 1 : 2, NDMA = HALF ? 3 : 4, NS = HALF ? NS_HALF : 4;
  constexpr u32 XB = HALF ? BLK / 2 : BLK, SB = XB + BLK;
  static_assert(NS * SB <= LDS_CTL, "stages fit");

  u32 vao[2], vbo[2], vaw[2], vbw[2];
#pragma unroll
  for (int q = 0; q < 2; ++q) {
    const u32 ra = wid * (HALF ? 8 : 16) + q * 8 + (lane >> 3), ca = (lane & 7) ^ ((ra >> 1) & 7);
    const u32 kb = wid * 8 + q * 4 + (lane >> 4), cbk = (lane & 15) ^ (((kb & 3) << 2) | ((kb >> 2) & 3));
    const u32 ap = QUART ? ra >> 6 : 0, ar = ra - ap * 64, bp = QUART ? cbk >> 3 : 0, bc = cbk - bp * 8;
    vao[q] = ar * 1024 + ca * 16 + ap * 128 - q * 1024;        vaw[q] = ar * 128 + ca * 16 + ap * 2 * BLK - q * 1024;
    vbo[q] = kb * 8192 + bc * 16 + bp * 64 * 8192 - q * 1024;  vbw[q] = kb * 256 + bc * 16 + bp * 2 * BLK - q * 1024;
  }
  const u32 lds0 = (u32)(size_t)lds;
  const u32 wm = wid & 1, wn = wid >> 1, h = lane >> 5;
  u32 xo[4], wo[2];
  {
    const u32 r = wm * 32 * MI + (lane & 31) + (QUART ? (wid >> 2) * 64 : 0);
#pragma unroll
    for (int ks = 0; ks < 4; ++ks) xo[ks] = r * 128 + (((2 * ks + h) ^ ((r >> 1) & 7)) * 16);
    const u32 bk = (lane >> 4) & 1, q = (lane & 15) >> 2, pp = lane & 3;
#pragma unroll
    for (int t = 0; t < 2; ++t)
      wo[t] = XB + 256 * (8 * h + 4 * t + q) + 16 * ((4 * wn + 2 * bk + (pp >> 1)) ^ ((q << 2) | (2 * h + t))) + 8 * (pp & 1);
  }
  LAS u32* lbad = (LAS u32*)(lds + LDS_CTL);
  u32 ri, ci;
  tile_rc(m, ri, ci);
  const u32 row0 = ((rank >> 1) & 1) * 2048 + ri * 128 + hsel * 64, col0 = (rank & 1) * 2048 + ci * 128 + csel * 64;
  u32 islot = 0, rslot = 0;
  auto next_slot = [&](u32 o) -> u32 { return o + SB == NS * SB ? 0 : o + SB; };
  auto issue_own = [&](int t) {
    const u32 dst = lds0 + islot, sa = row0 * 1024 + t * KS * 128, sw = t * KS * 64 * 8192 + col0 * 2;
    islot = next_slot(islot);
    if (HALF) dma3(dst + wid * 1024, dst + XB + wid * 2048, rx, vao[0], sa, rw, vbo[0], vbo[1], sw);
    else dma4(dst + wid * 2048, rx, vao[0], vao[1], sa, rw, vbo[0], vbo[1], sw);
  };
  u32 taba, tabw;
  {
    const u32 ktl = lane / 6, c = lane - ktl * 6, src = rank ^ (c + 1 + (c >= 3));
    const u32 base = IN_OFF + src * IN_SRC + ktl * KS * 2 * BLK;
    taba = base + ri * 8 * BLK + hsel * 8192;
    tabw = base + (ci * 8 + 1) * BLK + csel * 128;
  }
  auto issue = [&](int t) {
    const u32 dst = lds0 + islot, sa = __builtin_amdgcn_readlane(taba, t), sw = __builtin_amdgcn_readlane(tabw, t);
    islot = next_slot(islot);
    if (HALF) dma3(dst + wid * 1024, dst + XB + wid * 2048, rs, vaw[0], sa, rs, vbw[0], vbw[1], sw);
    else dma4(dst + wid * 2048, rs, vaw[0], vaw[1], sa, rs, vbw[0], vbw[1], sw);
  };
  const u32 fop = (lane >> 1) & 1, fq = min(lane >> 2, 11), fk = fq / 6, fc = fq - fk * 6;
  const char* fin = ws + FI_OFF + ((rank ^ (fc + 1 + (fc >= 3))) * 256 + (((fop ? ci : ri) * 4 + fk) * 2 + fop) * 2 + ((KIND && !fop) ? hsel : (lane & 1))) * FS;

  while (__builtin_amdgcn_s_memrealtime() - tstart < (u64)max(ri, ci) * STAGGER) __builtin_amdgcn_s_sleep(16);
  if (first && wid == 0) __builtin_amdgcn_fence(__ATOMIC_ACQUIRE, "");
  f32x16 acc[MI] = {};
  struct Frag { bf16x8 w[4], x[MI][4]; };
  auto read = [&](Frag& f) {
    const LAS char* st = lds + rslot;
    rslot = next_slot(rslot);
#pragma unroll
    for (int ks = 0; ks < 4; ++ks) {
      const bf16x4 w0 = __builtin_amdgcn_ds_read_tr16_b64_v4bf16((LAS bf16x4*)(st + wo[0] + ks * 4096));
      const bf16x4 w1 = __builtin_amdgcn_ds_read_tr16_b64_v4bf16((LAS bf16x4*)(st + wo[1] + ks * 4096));
      f.w[ks] = __builtin_shufflevector(w0, w1, 0, 1, 2, 3, 4, 5, 6, 7);
#pragma unroll
      for (int mi = 0; mi < MI; ++mi) f.x[mi][ks] = *(const LAS bf16x8*)(st + xo[ks] + mi * 4096);
    }
  };
  auto mma = [&](const Frag& f) {
#pragma unroll
    for (int ks = 0; ks < 4; ++ks)
#pragma unroll
      for (int mi = 0; mi < MI; ++mi) acc[mi] = __builtin_amdgcn_mfma_f32_32x32x16_bf16(f.w[ks], f.x[mi][ks], acc[mi], 0, 0, 0);
  };
  auto step = [&](auto rc, auto oc, int tn, const Frag& fc, Frag& fn) {
    constexpr int R = decltype(rc)::value;
    constexpr bool OWN = decltype(oc)::value;
    if constexpr (R >= 0) {
      WAIT_VM(R * NDMA);
      __builtin_amdgcn_s_waitcnt(0xC07F);
      CBAR();
      __builtin_amdgcn_s_barrier();
      CBAR();
      read(fn);
    }
    if (tn >= 0) { if (OWN) issue_own(tn); else issue(tn); }
    mma(fc);
  };
  Frag fa, fb;
  static_assert(NS % 2 == 0 && NS <= NOWN, "run structure");
  auto run = [&](auto oc, int n) {
    constexpr bool OWN = decltype(oc)::value;
    islot = rslot = 0;
    for (int t = 0; t < NS; ++t) { if (OWN) issue_own(t); else issue(t); }
    WAIT_VM((NS - 1) * NDMA);
    __builtin_amdgcn_s_barrier();
    CBAR();
    read(fa);
    for (int tn = NS; tn < n; tn += 2) {
      step(std::integral_constant<int, NS - 2>{}, oc, tn, fa, fb);
      step(std::integral_constant<int, NS - 2>{}, oc, tn + 1, fb, fa);
    }
    static_for<0, NS / 2>([&](auto ic) {
      constexpr int i = decltype(ic)::value;
      step(std::integral_constant<int, NS - 2 - 2 * i>{}, std::true_type{}, -1, fa, fb);
      step(std::integral_constant<int, NS - 3 - 2 * i>{}, std::true_type{}, -1, fb, fa);
    });
  };
  run(std::true_type{}, NOWN);
  if (wid < 2) {
    bool bad = false;
    wave_wait2<SL_GEMM>(fin + wid * 8 * FS, ~0ull, seq, tmo, bad);
    if (bad) *lbad = 1;
  }
  __builtin_amdgcn_s_waitcnt(0xC07F);
  __builtin_amdgcn_s_barrier();
  run(std::false_type{}, NRCV);
  const u32 tmv = __hip_atomic_load((const GAS u32*)tmo, __ATOMIC_RELAXED, __HIP_MEMORY_SCOPE_AGENT);
  __builtin_amdgcn_s_waitcnt(0xC07F);
  __builtin_amdgcn_s_barrier();
  CBAR();
  constexpr u32 EROW = 528;
#pragma unroll
  for (int mi = 0; mi < MI; ++mi) {
    const u32 mr = wm * 32 * MI + mi * 32 + (lane & 31);
#pragma unroll
    for (int gq = 0; gq < 4; ++gq) {
      const f32x4 o = {acc[mi][4 * gq], acc[mi][4 * gq + 1], acc[mi][4 * gq + 2], acc[mi][4 * gq + 3]};
      *(LAS f32x4*)(lds + mr * EROW + (wn * 32 + 8 * gq + 4 * h) * 4) = o;
    }
  }
  __syncthreads();
  const u32 bad = (*lbad || tmv == seq) ? NAN2 : 0u;
  const u32 keep = rank >> 2;
  const bool do_exp = !KIND || hsel != keep || m < NX, do_keep = !KIND || hsel == keep || m < NX;
  const u32 me = KIND && hsel == keep ? xbase(seq) + m : m, mk = KIND && hsel != keep ? xbase(seq) + m : m;
  if constexpr (QUART) {
    const u32 row = tid >> 3, ch = tid & 7;
    const LAS char* se = lds + row * EROW + ch * 32;
    const f32x4 c0 = *(const LAS f32x4*)se + *(const LAS f32x4*)(se + 256), c1 = *(const LAS f32x4*)(se + 16) + *(const LAS f32x4*)(se + 272);
    GAS u32* qc = (GAS u32*)(ws + QC_OFF + (m * 2 + hsel) * FS);
    if (do_exp) {
      const bf16x8 o = {(__bf16)c0[0], (__bf16)c0[1], (__bf16)c0[2], (__bf16)c0[3],
                        (__bf16)c1[0], (__bf16)c1[1], (__bf16)c1[2], (__bf16)c1[3]};
      st128(__builtin_bit_cast(u32x4, o) | bad, rs, row * 256 + ch * 16, P16_OFF + me * 16384 + csel * 128);
      WAIT_VM(0);
      __builtin_amdgcn_s_barrier();
      if (tid == 0 && __hip_atomic_fetch_add(qc, 1u, __ATOMIC_RELAXED, __HIP_MEMORY_SCOPE_AGENT) == 2 * seq - 1)
        st_sys(p.ws[PL] + FPR_OFF + me * FS, seq);
    }
    if (do_keep) {
      const u32 vo = (row >> 2) * 2048 + (row & 3) * 256 + ch * 16, e32 = P32_OFF + mk * 32768 + csel * 128;
      st128(__builtin_bit_cast(u32x4, c0) | bad, rs, vo, e32);
      st128(__builtin_bit_cast(u32x4, c1) | bad, rs, vo, e32 + 1024);
    }
    WAIT_VM(0);
    __syncthreads();
    if (tid == 64 && do_keep && __hip_atomic_fetch_add(qc + 1, 1u, __ATOMIC_RELAXED, __HIP_MEMORY_SCOPE_AGENT) == 2 * seq - 1)
      st_sys(ws + FPL_OFF + mk * FS, seq);
    return;
  }
  if (do_exp) {
    const u32 e16 = P16_OFF + me * 16384 + wid * 1024;
#pragma unroll
    for (int part = 0; part < 2; ++part) {
      const LAS char* se = lds + ((HALF ? 0 : (keep ^ 1) * 64) + part * 32 + (tid >> 4)) * EROW + (tid & 15) * 32;
      const f32x4 c0 = *(const LAS f32x4*)se, c1 = *(const LAS f32x4*)(se + 16);
      const bf16x8 o = {(__bf16)c0[0], (__bf16)c0[1], (__bf16)c0[2], (__bf16)c0[3],
                        (__bf16)c1[0], (__bf16)c1[1], (__bf16)c1[2], (__bf16)c1[3]};
      st128(__builtin_bit_cast(u32x4, o) | bad, rs, lane * 16, e16 + part * 8192);
    }
  }
  if (HALF && do_exp && do_keep) {
    WAIT_VM(0);
    __builtin_amdgcn_s_barrier();
    if (tid == 0) st_sys(p.ws[PL] + FPR_OFF + me * FS, seq);
  }
  if (do_keep) {
    const u32 e32 = P32_OFF + mk * 32768 + wid * 2048;
#pragma unroll
    for (int part = 0; part < 2; ++part) {
      const LAS char* sk = lds + ((HALF ? 0 : keep * 64) + part * 32 + (tid >> 4)) * EROW + (tid & 15) * 32;
      const u32x4 c0 = *(const LAS u32x4*)sk, c1 = *(const LAS u32x4*)(sk + 16);
      st128(c0 | bad, rs, lane * 16, e32 + part * 16384);
      st128(c1 | bad, rs, lane * 16, e32 + part * 16384 + 1024);
    }
  }
  WAIT_VM(0);
  __syncthreads();
  if (tid == 0 && do_exp && !(HALF && do_keep)) st_sys(p.ws[PL] + FPR_OFF + me * FS, seq);
  if (tid == 64 && do_keep) st_sys(ws + FPL_OFF + mk * FS, seq);
}

__device__ __forceinline__ void stage_unit(const Params& p, u32 lk, u32 seq) {
  const int tid = threadIdx.x, lane = tid & 63;
  const u32 wid = rfl(tid >> 6);
  const u32 lev = lk >> 2, kt = (wid >> 2) * 4 + (lk & 3), op = (wid >> 1) & 1, rc = (wid & 1) * 2048 + lev * 128;
  const auto r = mk_rsrc(op ? p.w : p.x, X_BYTES);
  const auto rs = mk_rsrc(p.ws[0], DATA_BYTES);
  const u32 vo = op ? (lane >> 4) * 8192 + (lane & 15) * 16 : (lane >> 3) * 1024 + (lane & 7) * 16;
  const u32 so = op ? kt * 64 * 8192 + rc * 2 : rc * 1024 + kt * 128;
  u32x4 v[16];
#pragma unroll
  for (int i = 0; i < 16; ++i) v[i] = __builtin_amdgcn_raw_buffer_load_b128(r, vo, so + i * (op ? 4 * 8192 : 8 * 1024), 0);
#pragma unroll
  for (int i = 0; i < 16; ++i) st128(v[i], rs, lane * 16, EX_OFF + (lk * 8 + wid) * BLK + i * 1024);
  WAIT_VM(0);
  __syncthreads();
  if (tid >= 1 && tid < NR && tid != PL) st_sys(p.ws[tid] + SG_OFF + (p.rank * 64 + lk) * FS, seq);
}

__device__ __forceinline__ void gemm_role(const Params& p, u32 k, u32 seq, LAS char* lds) {
  const u64 tstart = __builtin_amdgcn_s_memrealtime();
  constexpr u32 NG = 8 * GEMM_WGS, NSMALL = 2 * NQUART + 2 * NHALF;
  static_assert(NQUART <= NHALF && NQUART <= 64, "unit schedule");
  for (u32 n = k; n < 256 - NHALF + NSMALL; n += NG) {
    if (n < 4 * NQUART) gemm_tile<2>(p, n >> 2, (n >> 1) & 1, n & 1, seq, tstart, n < NG, lds);
    else if (n < NSMALL) gemm_tile<1>(p, (n >> 1) - NQUART, n & 1, 0, seq, tstart, n < NG, lds);
    else gemm_tile<0>(p, n - NSMALL + NHALF, 0, 0, seq, tstart, n < NG, lds);
  }
}

__global__ void __launch_bounds__(THREADS, 2) fused_gemm_allreduce_silu(Params p) {
  extern __shared__ __attribute__((aligned(16))) char smem[];
  const int lane = threadIdx.x & 63;
  const u32 wid = rfl(threadIdx.x >> 6);
  LAS u32* ctl = (LAS u32*)smem;
  LAS u32* lctl = (LAS u32*)(smem + LDS_CTL);
  u32 slot = (blockIdx.x & ~7u) | ((u32)__builtin_amdgcn_s_getreg((3 << 11) | 20) & 7);
  auto is_push = [&]() { return slot < 8 * ENG_WGS && (slot & 3) != 0 && wid < PUSH_WAVES; };
  if (__builtin_expect(is_push(), 1)) {
    if (push_role<false>(p, slot, wid, lane, lctl)) return;
  } else {
    if (wid == 7) {
      if (lane < 32) ctl[lane] = 0;
      if (lane == 0) {
        GAS u32* ctr = (GAS u32*)(p.ws[0] + CTR_OFF) + blockIdx.x;
        GAS u32* clm = (GAS u32*)(p.ws[0] + CLM_OFF);
        const u32 c = __hip_atomic_load(ctr, __ATOMIC_RELAXED, __HIP_MEMORY_SCOPE_AGENT);
        u32 sl = slot;
#pragma unroll 1
        for (int n = 0; n < 4 * GRID; ++n) {
          if (__hip_atomic_fetch_add(clm + sl * 16, 1u, __ATOMIC_RELAXED, __HIP_MEMORY_SCOPE_AGENT) == c) break;
          __hip_atomic_fetch_add(clm + sl * 16, ~0u, __ATOMIC_RELAXED, __HIP_MEMORY_SCOPE_AGENT);
          sl = (sl + 1) & (GRID - 1);
        }
        lctl[0] = 0; lctl[1] = sl; lctl[2] = c + 1;
      }
    }
    asm volatile("s_waitcnt lgkmcnt(0)" ::: "memory");
    __builtin_amdgcn_s_barrier();
    slot = rfl(LD_LDS(lctl + 1));
    if (threadIdx.x == THREADS - 1)
      __hip_atomic_store((GAS u32*)(p.ws[0] + CTR_OFF) + blockIdx.x, LD_LDS(lctl + 2), __ATOMIC_RELAXED, __HIP_MEMORY_SCOPE_AGENT);
    WAIT_VM(0);
  }
  if (__builtin_expect(is_push(), 0)) {
    push_role<true>(p, slot, wid, lane, lctl);
    return;
  }
  const u32 seq = rfl(LD_LDS(lctl + 2));
  const u32 g = slot & 7, s = slot >> 3;
  const u32 peer = ((u32)p.rank ^ g) & (NR - 1);
  constexpr u32 LK0 = NPUSH / 4;
  if (s < ENG_WGS) {
    if (g == 0) {
      for (u32 u = s; u < 64 - LK0; u += u < 16 ? 16 : u < 24 ? 16 : 8) stage_unit(p, LK0 + u, seq);
    } else if (wid >= PUSH_WAVES && wid < PUSH_WAVES + PULL_WAVES) {
      pull_role(p, peer, (wid - PUSH_WAVES) * ENG_WGS + s, lane, seq, ctl);
    } else if (wid == PUSH_WAVES + PULL_WAVES) {
      flag_role(p, g, lane, seq, ctl);
    }
  } else if (s < ENG_WGS + FIN_WGS) {
    if ((g & 3) == 0) { stage_unit(p, LK0 + 8 + g + s - ENG_WGS, seq); stage_unit(p, LK0 + 24 + g + s - ENG_WGS, seq); }
    fin_role(p, (s - ENG_WGS) * 8 + g, seq);
  } else {
    const u32 k = s - ENG_WGS - FIN_WGS;
    gemm_role(p, (k >> 2) * 32 + g * 4 + (k & 3), seq, (LAS char*)smem);
  }
}

}

extern "C" void kernel_launch(int rank, int nranks, ncclComm_t comm,
                              hipStream_t stream,
                              void* const* d_in, const int* in_sizes, int n_in,
                              void* d_out, int out_size,
                              const int* in_dtypes, int out_dtype,
                              void* const* d_out_all,
                              void* const* d_ws_all, size_t ws_bytes) {
  if (nranks != NR || (unsigned)rank >= (unsigned)NR || n_in < 2 || ws_bytes < WS_USED ||
      in_sizes[0] != 4096 * 512 || in_sizes[1] != 512 * 4096 || out_size != 4096 * 4096) {
    fprintf(stderr, "kernel_launch: unexpected problem shape\n");
    return;
  }
  HIP_CHECK(hipFuncSetAttribute((const void*)fused_gemm_allreduce_silu,
                                hipFuncAttributeMaxDynamicSharedMemorySize, LDS_BYTES));
  Params p;
  p.x = (const char*)d_in[0];
  p.w = (const char*)d_in[1];
  const int phys[NR] = {0, 2, 3, 6, 1, 4, 5, 7};
  p.rank = 0; p.pad = 0;
  for (int i = 0; i < NR; ++i) if (phys[i] == rank) p.rank = i;
  for (int i = 0; i < NR; ++i) { p.ws[i] = (char*)d_ws_all[phys[p.rank ^ i]]; p.out[i] = (char*)d_out_all[phys[p.rank ^ i]]; }
  for (u32 g = 0; g < NR; ++g)
    for (u32 job = 0; job < 16; ++job) {
      const u32 peer = p.rank ^ g, lev = job >> 4, kt = (peer >> 2) * 4 + ((job >> 2) & 3), op = (job >> 1) & 1, half = job & 1;
      p.psrc[g][job] = op ? (kt * 64 + half * 32) * 8192 + ((peer & 1) * 2048 + lev * 128) * 2
                                         : (((peer >> 1) & 1) * 2048 + lev * 128 + half * 64) * 1024 + kt * 128;
    }
  hipLaunchKernelGGL(fused_gemm_allreduce_silu, dim3(GRID), dim3(THREADS), LDS_BYTES, stream, p);
  HIP_CHECK(hipGetLastError());
}
```

```cpp
#include <hip/hip_runtime.h>
#include <rccl/rccl.h>
#include <cstdio>
#include <type_traits>

#define HIP_CHECK(expr) do { hipError_t _e = (expr); if (_e != hipSuccess) { \
    fprintf(stderr, "HIP error %s at %s:%d\n", hipGetErrorString(_e), __FILE__, __LINE__); \
    return; } } while (0)

namespace {

typedef unsigned int u32;
typedef unsigned long long u64;
typedef u32 u32x4 __attribute__((ext_vector_type(4)));
typedef float f32x4 __attribute__((ext_vector_type(4)));
typedef __bf16 bf16x4 __attribute__((ext_vector_type(4)));
typedef __bf16 bf16x8 __attribute__((ext_vector_type(8)));
typedef float f32x16 __attribute__((ext_vector_type(16)));
#define LAS __attribute__((address_space(3)))
#define GAS __attribute__((address_space(1)))

constexpr int NR = 8;
constexpr u32 X_BYTES = 4096u * 512u * 2u;
constexpr u32 W_BYTES = 512u * 4096u * 2u;
constexpr u32 OUT_BYTES = 4096u * 4096u * 2u;
constexpr u32 FS = 128;

constexpr u32 BLK = 16384;
constexpr u32 IN_OFF = 0;
constexpr u32 IN_SRC = 128 * BLK;
constexpr u32 P16_OFF = IN_OFF + NR * IN_SRC;
constexpr u32 PIN_OFF = P16_OFF + 320 * 16384;
constexpr u32 P32_OFF = PIN_OFF + 320 * 16384;
constexpr u32 EX_OFF = P32_OFF + 320 * 32768;
constexpr u32 DATA_BYTES = EX_OFF + 512 * BLK;
constexpr u32 FI_OFF = DATA_BYTES;
constexpr u32 FPR_OFF = FI_OFF + NR * 256 * FS;
constexpr u32 FPL_OFF = FPR_OFF + 320 * FS;
constexpr u32 FPI_OFF = FPL_OFF + 320 * FS;
constexpr u32 FF_OFF = FPI_OFF + 1280 * FS;
constexpr u32 SG_OFF = FF_OFF + NR * 256 * FS;
constexpr u32 GO_OFF = SG_OFF + NR * 64 * FS;
constexpr u32 RF_OFF = GO_OFF + NR * 16 * FS;
constexpr u32 QC_OFF = RF_OFF + 64 * FS;
constexpr u32 FC_OFF = QC_OFF + 128 * FS;
constexpr u32 CTR_OFF = FC_OFF + 256 * FS;
constexpr u32 CLM_OFF = CTR_OFF + 256 * 4;
constexpr u32 TMO_OFF = CLM_OFF + 256 * 64;
constexpr u32 TICK_OFF = TMO_OFF + 128;
constexpr u32 SEG_OFF = TICK_OFF + 32 * 128;
constexpr u32 WS_USED = SEG_OFF + 8 * 128;

constexpr u64 DEADLINE_TICKS = 80000000ull;

constexpr int ENG_WGS = 8, FIN_WGS = 4, GEMM_WGS = 20;
constexpr int PUSH_WAVES = 2, PULL_WAVES = 2;
constexpr int PUSH_D = 4, PUSH_ROUNDS = 4;
constexpr u32 NPUSH = PUSH_ROUNDS * ENG_WGS * PUSH_WAVES;
constexpr u32 PL = 4;
constexpr int GO_AT = 6;
constexpr int NS_HALF = 6;
constexpr u32 NQUART = 36, NHALF = 100;
constexpr u64 STAGGER = 190;
constexpr u32 NX = 20, TOPK = 52;
constexpr u32 RETIRE = 2, RET_AT = 40;
constexpr u32 FG = 2;
constexpr int SL_PULL = 2, SL_FIN = 4, SL_GEMM = 8;
constexpr u32 NAN2 = 0x7FC07FC0u;
constexpr int GRID = 8 * (ENG_WGS + FIN_WGS + GEMM_WGS);
static_assert(GRID == 256, "slots wrap at a power of two");
constexpr int THREADS = 512;
constexpr u32 LDS_CTL = 144 * 1024;
constexpr u32 LDS_BYTES = LDS_CTL + 64;
constexpr int AUX_SYS = 17;

struct Params {
  const char* x;
  const char* w;
  int rank;
  int pad;
  char* ws[NR];
  char* out[NR];
  u32 psrc[NR][16];
};

__device__ __forceinline__ u32 rfl(u32 v) { return __builtin_amdgcn_readfirstlane(v); }
__device__ __forceinline__ __amdgpu_buffer_rsrc_t mk_rsrc(const void* p, u32 n) {
  const u64 a = (u64)p;
  const u64 u = ((u64)rfl((u32)(a >> 32)) << 32) | rfl((u32)a);
  return __builtin_amdgcn_make_buffer_rsrc((void*)u, 0, n, 0x00020000);
}
__device__ __forceinline__ u32 ld_sys(const char* p) {
  return __hip_atomic_load((const GAS u32*)p, __ATOMIC_RELAXED, __HIP_MEMORY_SCOPE_SYSTEM);
}
__device__ __forceinline__ void st_sys(char* p, u32 v) {
  __hip_atomic_store((GAS u32*)p, v, __ATOMIC_RELAXED, __HIP_MEMORY_SCOPE_SYSTEM);
}
#define WAIT_VM(n) asm volatile("s_waitcnt vmcnt(%0)" ::"n"(n) : "memory")
#define CBAR() asm volatile("" ::: "memory")
#define LD_LDS(q) __hip_atomic_load(q, __ATOMIC_RELAXED, __HIP_MEMORY_SCOPE_WORKGROUP)
#define ST_LDS(q, v) __hip_atomic_store(q, v, __ATOMIC_RELAXED, __HIP_MEMORY_SCOPE_WORKGROUP)

__device__ __forceinline__ void st128(u32x4 v, __amdgpu_buffer_rsrc_t r, u32 voff, u32 soff) {
  __builtin_amdgcn_raw_buffer_store_b128(v, r, voff, soff, AUX_SYS);
  asm volatile("s_nop 1" ::"v"(v) : "memory");
}

template <int SLEEP>
__device__ __forceinline__ u64 wave_wait(const char* flag, u64 need, u32 seq, char* tmo, bool& bad) {
  const u64 t0 = __builtin_amdgcn_s_memrealtime();
  for (u32 it = 1;; ++it) {
    const u64 ok = __ballot(ld_sys(flag) >= seq);
    if ((ok & need) == need) return ok;
    if ((it & 15) == 0 || bad) {
      const u32 t = __hip_atomic_load((const GAS u32*)tmo, __ATOMIC_RELAXED, __HIP_MEMORY_SCOPE_AGENT);
      if (rfl(t) == seq || __builtin_amdgcn_s_memrealtime() - t0 > DEADLINE_TICKS) {
        __hip_atomic_store((GAS u32*)tmo, seq, __ATOMIC_RELAXED, __HIP_MEMORY_SCOPE_AGENT);
        bad = true;
        return ~0ull;
      }
    }
    __builtin_amdgcn_s_sleep(SLEEP);
  }
}

template <int SLEEP>
__device__ __forceinline__ u64 wave_wait2(const char* flag, u64 need, u32 seq, char* tmo, bool& bad) {
  const u64 t0 = __builtin_amdgcn_s_memrealtime();
  const bool needed = (need >> (threadIdx.x & 63)) & 1;
  bool ok = false;
  for (u32 it = 1;; ++it) {
    if (!ok && (it == 1 || needed)) ok = ld_sys(flag) >= seq;
    const u64 have = __ballot(ok);
    if ((have & need) == need) return have;
    if ((it & 15) == 0) {
      const u32 t = __hip_atomic_load((const GAS u32*)tmo, __ATOMIC_RELAXED, __HIP_MEMORY_SCOPE_AGENT);
      if (rfl(t) == seq || __builtin_amdgcn_s_memrealtime() - t0 > DEADLINE_TICKS) {
        __hip_atomic_store((GAS u32*)tmo, seq, __ATOMIC_RELAXED, __HIP_MEMORY_SCOPE_AGENT);
        bad = true;
        return ~0ull;
      }
    }
    __builtin_amdgcn_s_sleep(SLEEP);
  }
}

__device__ __forceinline__ void dma4(u32 dst, __amdgpu_buffer_rsrc_t ra, u32 va0, u32 va1, u32 sa,
                                     __amdgpu_buffer_rsrc_t rb, u32 vb0, u32 vb1, u32 sb) {
  u32 keep;
  dst = rfl(dst); sa = rfl(sa); sb = rfl(sb);
  asm volatile(
      "s_mov_b32 %0, m0\n\t"
      "s_nop 4\n\t"
      "s_mov_b32 m0, %1\n\t"
      "s_nop 0\n\t"
      "buffer_load_dwordx4 %3, %5, %6 offen lds\n\t"
      "buffer_load_dwordx4 %4, %5, %6 offen offset:1024 lds\n\t"
      "s_mov_b32 m0, %2\n\t"
      "s_nop 0\n\t"
      "buffer_load_dwordx4 %7, %9, %10 offen lds\n\t"
      "buffer_load_dwordx4 %8, %9, %10 offen offset:1024 lds\n\t"
      "s_mov_b32 m0, %0"
      : "=&s"(keep)
      : "s"(dst), "s"(dst + BLK), "v"(va0), "v"(va1), "s"(ra), "s"(sa), "v"(vb0), "v"(vb1), "s"(rb), "s"(sb)
      : "memory");
}

__device__ __forceinline__ void dma3(u32 dsta, u32 dstb, __amdgpu_buffer_rsrc_t ra, u32 va0, u32 sa,
                                     __amdgpu_buffer_rsrc_t rb, u32 vb0, u32 vb1, u32 sb) {
  u32 keep;
  dsta = rfl(dsta); dstb = rfl(dstb); sa = rfl(sa); sb = rfl(sb);
  asm volatile(
      "s_mov_b32 %0, m0\n\t"
      "s_nop 4\n\t"
      "s_mov_b32 m0, %1\n\t"
      "s_nop 0\n\t"
      "buffer_load_dwordx4 %3, %4, %5 offen lds\n\t"
      "s_mov_b32 m0, %2\n\t"
      "s_nop 0\n\t"
      "buffer_load_dwordx4 %6, %8, %9 offen lds\n\t"
      "buffer_load_dwordx4 %7, %8, %9 offen offset:1024 lds\n\t"
      "s_mov_b32 m0, %0"
      : "=&s"(keep)
      : "s"(dsta), "s"(dstb), "v"(va0), "s"(ra), "s"(sa), "v"(vb0), "v"(vb1), "s"(rb), "s"(sb)
      : "memory");
}

template <int I, int N, class F>
__device__ __forceinline__ void static_for(F&& f) {
  if constexpr (I < N) { f(std::integral_constant<int, I>{}); static_for<I + 1, N>(f); }
}

__device__ __forceinline__ void tile_rc(u32 m, u32& ri, u32& ci) {
  u32 l = 0;
#pragma unroll
  for (u32 t = 1; t < 16; ++t) l += m >= t * t;
  const u32 idx = m - l * l;
  ri = idx < l ? l : idx - l;
  ci = idx < l ? idx : l;
}
__device__ __forceinline__ u32 fin_base(u32 r) { return (((r >> 1) & 1) * 2048 + (r >> 2) * 64) * 8192 + (r & 1) * 4096; }

template <bool claimed>
__device__ __forceinline__ bool push_role(const Params& p, u32& slot, u32 wid, int lane, LAS u32* lctl) {
  constexpr int D = PUSH_D, JI = 8;
  constexpr u32 NW = ENG_WGS * PUSH_WAVES;
  static_assert(PUSH_ROUNDS >= 3 && NW == 16 && X_BYTES == W_BYTES && GO_AT <= JI, "push schedule");
  const u32 g = slot & 7, w = wid * ENG_WGS + (slot >> 3), op = (slot >> 4) & 1;
  u64 bs = (u64)(&p.x)[op], bw = (u64)p.ws[g];
  u32 src = p.psrc[g][w];
  u32 rank = p.rank;
  asm volatile("" : "+s"(bs), "+s"(bw), "+s"(src), "+s"(rank));
  const auto rs = mk_rsrc((const void*)bs, X_BYTES);
  const u32 vs = op ? (lane >> 4) * 8192 + (lane & 15) * 16 : (lane >> 3) * 1024 + (lane & 7) * 16;
  const u32 str = op ? 4 * 8192 : 8 * 1024;
  auto ld = [&](int r, int i) -> u32x4 { return __builtin_amdgcn_raw_buffer_load_b128(rs, vs, src + r * (op ? 256 : 128 * 1024) + i * str, 0); };
  u32x4 v[D], v2[JI - D];
#pragma unroll
  for (int i = 0; i < D; ++i) { v[i] = ld(0, i); CBAR(); }
#pragma unroll
  for (int i = D; i < JI; ++i) { v2[i - D] = ld(0, i); CBAR(); }
  char* wsp = (char*)bw;
  const auto rd = mk_rsrc(wsp, DATA_BYTES);
  const u32 dst = IN_OFF + rank * IN_SRC + w * 8192, flag = FI_OFF + (rank * 256 + w) * FS;
  u32 seq = 0;
  auto round = [&](auto rc) {
    constexpr int R = decltype(rc)::value;
    constexpr bool FIRST = R == 0, LAST = R == PUSH_ROUNDS - 1;
#pragma unroll
    for (int i = 0; i < JI; ++i) {
      if (!FIRST && i == D - 1) {
        WAIT_VM(2 * D - 1);
        if (R == 1) {
          if (!claimed) __builtin_amdgcn_s_barrier();
          CBAR();
          seq = rfl(LD_LDS(lctl + 2));
        }
        if (lane == 0) st_sys(wsp + flag + (R - 1) * NW * FS, seq);
        CBAR();
      }
      if (LAST && i == JI - GO_AT) {
        if (lane == 0) st_sys(wsp + GO_OFF + (rank * 16 + w) * FS, seq);
        if (lane == 0) ST_LDS(lctl - LDS_CTL / 4 + 24 + wid, 1u);
        CBAR();
      }
      st128(FIRST && i >= D ? v2[i >= D ? i - D : 0] : v[i % D], rd, lane * 16, dst + R * NW * 8192 + i * 1024);
      if (i + D < JI) { if (!FIRST) v[i % D] = ld(R, i + D); }
      else if (!LAST) v[i % D] = ld(LAST ? R : R + 1, i + D - JI);
      CBAR();
    }
  };
  static_assert(JI - D >= D - 1, "the count above also holds in the last round");
  round(std::integral_constant<int, 0>{});
  round(std::integral_constant<int, 1>{});
  const u32 slot2 = rfl(LD_LDS(lctl + 1));
  if (__builtin_expect(slot2 != slot, 0)) { WAIT_VM(0); slot = slot2; return false; }
  static_for<2, PUSH_ROUNDS>(round);
  WAIT_VM(0);
  if (lane == 0) st_sys(wsp + flag + (PUSH_ROUNDS - 1) * NW * FS, seq);
  return true;
}

__device__ __forceinline__ float silu(float a) {
  const float c = fminf(fmaxf(a, -60.f), 60.f);
  return a * __builtin_amdgcn_rcpf(1.f + __expf(-c));
}

constexpr u32 MB_RING = 0, MB_END = 16, MB_PUSHED = 24;

constexpr u32 NE = 4 * NX, NPOOL = (1024 - NE) / 2;
static_assert(NX <= NHALF && NX <= 32, "early tiles");
__device__ __forceinline__ u32 xbase(u32 seq) { return 256 + (seq & 1) * 32; }
__device__ __forceinline__ u32 pool_job(u32 pos) {
  return pos < NPOOL - 2 * TOPK ? pos : pos >= NPOOL - TOPK ? pos - TOPK : pos + TOPK;
}

__device__ __forceinline__ void pull_role(const Params& p, u32 peer, u32 w, int lane, u32 seq, LAS u32* ctl) {
  constexpr u32 NW = ENG_WGS * PULL_WAVES, NONE = ~0u, REL = 1u << 31;
  const u32 rank = p.rank, g = rank ^ peer;
  const bool pl = g == PL;
  const u32 NI = pl ? 1280 : 256 - NPUSH, NJ = pl ? 1024 + 4 * NX : NI + 1024;
  char* ws = p.ws[0];
  char* tmo = ws + TMO_OFF;
  const u64 bf = (u64)p.out[g], bp = (u64)p.ws[g], bo = (u64)p.out[0];
  const u32 vwide = (lane >> 4) * 8192 + (lane & 15) * 16;
  const auto rtick = mk_rsrc(ws + TICK_OFF + (seq & 1) * 2048, 2048);
  if (w == 0 && (lane == 0 || (pl && lane < 3))) st_sys(ws + TICK_OFF + ((seq & 1) ^ 1) * 2048 + (lane ? 7 + lane : g) * 128, 0);
  const u32 bi = (rank >> 1) & 1, bj = rank & 1;
  const u32 isrc = pl ? P16_OFF : EX_OFF + (NPUSH / 4 * 8 + (rank >> 2) * 4 + bi) * BLK;
  const u32 is1 = pl ? BLK : 8 * BLK, is2 = pl ? 8192 : (2 + bj - bi) * BLK, isz = pl ? 4096 : 8192;
  const u32 idst = pl ? PIN_OFF : IN_OFF + peer * IN_SRC + NPUSH * 8192;
  const u32 ipre = pl ? FPR_OFF : SG_OFF + (peer * 64 + NPUSH / 4) * FS, ish = 2;
  const u32 ipost = pl ? FPI_OFF : FI_OFF + (peer * 256 + NPUSH) * FS;
  const u32 fown = fin_base(peer), fpar = fin_base(rank ^ PL);
  const u32 rpre = pl ? FF_OFF + peer * 256 * FS : RF_OFF + g * 8 * FS, rsh = pl ? 2 + FG : 7;
  u32 ttile = 0;
#pragma unroll
  for (u32 e = 0; e < 4; ++e) {
    u32 ri, ci;
    tile_rc(lane * 4 + e, ri, ci);
    ttile |= (ri << 4 | ci) << (8 * e);
  }
  const u64 t0 = __builtin_amdgcn_s_memrealtime();
  u32 tf = w, tf2 = NW + w, lastf = 0, sub = 0;
  u32 cdst = 0, cact = 0, pend = 0, ack = 0;
  u32 nring = 0;
  const u32 retire = w >= NW - RETIRE ? (pl ? 2 * RET_AT : RET_AT) : NONE;
  u32 from = 1, spare = NONE;
  bool bad = false, cisf = false;
  auto poll = [&]() -> u32 {
    const u32 o = tf < NI ? ipre + (tf >> ish) * FS : tf < REL ? FF_OFF + (peer * 256 + ((tf - NI) >> (2 + FG))) * FS : rpre + ((tf & 1023) >> rsh) * FS;
    return ld_sys(ws + rfl(o));
  };
  u32 pv = 0;
  u32x4 v[4];
  const u32 tvo = lane == 0 ? g * 128 : (8 + (w & 1)) * 128, tinc = lane && pl ? 1u << 16 : 1u, tbit = lane < 2 ? lane + 1 : 0;
  auto draw = [&](u32 m) -> u32 {
    return __builtin_amdgcn_raw_ptr_buffer_atomic_add_i32(tinc, rtick, (m & tbit) ? tvo : 0x80000000u, 0, 0);
  };
  u32 sstart, sbase;
  {
    const u32 l = lane >> 1, xb = min(l * l, NX);
    sstart = 4 * (((lane & 1) ? (l + 1) * (l + 1) : l * l) + xb);
    sbase = 4 * ((lane & 1) ? xbase(seq) + xb : l * l);
  }
  auto handle = [&](u32 id) -> u32 {
    if (!pl) return id;
    const u32 sg = 63 - __builtin_clzll(__ballot(sstart <= id));
    return __builtin_amdgcn_readlane(sbase, sg) + id - __builtin_amdgcn_readlane(sstart, sg);
  };
  tf = handle(tf); tf2 = handle(tf2);
  auto drawn = [&](u32 m, u32 t) -> u32 {
    u32 own = NONE, tp = __builtin_amdgcn_readlane(t, 1);
    if (m & 1) {
      const u32 id = __builtin_amdgcn_readlane(t, 0) + 2 * NW;
      if (id < NJ) own = handle(id);
      if (m == 1) {
        if (__builtin_expect(id < NJ, 1)) { from = id >= retire ? 0 : id + NW - RETIRE >= NJ ? 3 : 1; return own; }
        tp = __builtin_amdgcn_readlane(draw(2), 1);
        asm volatile("" : "+s"(tp));
      }
    } else if (m == 0) {
      own = spare; spare = NONE;
      return own;
    }
    const u32 f = tp & 0xFFFF, b = tp >> 16;
    const u32 hp = f + b < NPOOL ? REL | (NE + pool_job(pl ? NPOOL - 1 - b : f) * 2 + (w & 1)) : NONE;
    from = hp == NONE ? 0 : 2;
    spare = own == NONE ? NONE : hp;
    return own == NONE ? hp : own;
  };
  auto put = [&](int i) {
    const auto ro = mk_rsrc((const void*)(cisf ? bo : (u64)ws), cisf ? OUT_BYTES : DATA_BYTES);
    st128(v[i], ro, cisf ? vwide : lane * 16, rfl(cdst + i * (cisf ? 4 * 8192 : 1024)));
  };
  auto done = [&](u32 a) {
    if (a) {
      LAS u32* e = ctl + MB_RING + (w / ENG_WGS) * 8 + (nring++ & 7);
      while (rfl(LD_LDS(e)) != 0) __builtin_amdgcn_s_sleep(1);
      if (lane == 0) ST_LDS(e, a);
      CBAR();
    }
  };
  auto ready = [&]() -> bool { return tf != NONE && (bad || (__ballot(pv >= seq) & 1)); };
  auto round = [&](auto first_c) {
    constexpr bool FIRST = decltype(first_c)::value;
    const u32 h = tf, hs = sub;
    const bool isf = h >= NI, tix = isf || hs || pl;
    sub = tix ? 0 : 1;
    tf = tix ? tf2 : tf;
    const u32 dm = tix && spare == NONE ? from : 0, tk = draw(dm);
    pv = poll();
    CBAR();
    done(ack);
    u32 nsrc, ndst, nact;
    if (isf) {
      const u32 j = h < REL ? h - NI : h & 1023, rc = (__builtin_amdgcn_readlane(ttile, j >> 4) >> (2 * (j & 12))) & 255;
      nsrc = ndst = (h < REL ? fown : fpar) + ((rc >> 4) * 128 + (j & 3) * 16) * 8192 + (rc & 15) * 256;
      nact = h < REL ? (j >> 7) * 2 + 1 : 0;
      lastf = ndst;
    } else {
      nsrc = isrc + (h >> 2) * is1 + ((h >> 1) & 1) * is2 + (h & 1) * isz + hs * 4096;
      ndst = idst + h * isz + hs * 4096;
      nact = tix ? ipost + h * FS : 0;
    }
    const auto rsrc = mk_rsrc((const void*)(isf ? bf : bp), isf ? OUT_BYTES : DATA_BYTES);
    const u32 vs = isf ? vwide : lane * 16, str = isf ? 4 * 8192 : 1024;
#pragma unroll
    for (int i = 0; i < 4; ++i) {
      if (!FIRST) put(i);
      v[i] = __builtin_amdgcn_raw_buffer_load_b128(rsrc, vs, rfl(nsrc + i * str), AUX_SYS);
      CBAR();
    }
    ack = 0;
    if (!FIRST) {
      WAIT_VM(11);
      ack = pend; pend = cact;
    }
    cdst = ndst; cisf = isf; cact = nact;
    if (tix) tf2 = drawn(dm, tk);
  };
  if (!pl) {
    while (rfl(LD_LDS(ctl + MB_PUSHED + w / ENG_WGS)) == 0) __builtin_amdgcn_s_sleep(2);
    wave_wait<1>(ws + ((lane & 1) ? GO_OFF + (peer * 16 + w) * FS : ipre + (w >> ish) * FS), 3, seq, tmo, bad);
    if (bad) WAIT_VM(0);
    pv = seq;
  }
  for (u32 it = 1; tf != NONE; ++it) {
    if (ready()) {
      round(std::true_type{});
      if (ready()) {
        round(std::false_type{});
        while (ready()) round(std::false_type{});
      }
#pragma unroll
      for (int i = 0; i < 4; ++i) put(i);
      done(ack);
      if (pend | cact) {
        WAIT_VM(0);
        done(pend);
        done(cact);
      }
      ack = pend = 0;
      continue;
    }
    if ((it & 63) == 0) {
      const u32 t = __hip_atomic_load((const GAS u32*)tmo, __ATOMIC_RELAXED, __HIP_MEMORY_SCOPE_AGENT);
      if (rfl(t) == seq || __builtin_amdgcn_s_memrealtime() - t0 > DEADLINE_TICKS) {
        __hip_atomic_store((GAS u32*)tmo, seq, __ATOMIC_RELAXED, __HIP_MEMORY_SCOPE_AGENT);
        WAIT_VM(0);
        bad = true;
      }
    }
    __builtin_amdgcn_s_sleep(SL_PULL * 3);
    pv = poll();
  }
  if (bad) {
    WAIT_VM(0);
    const u32x4 nanv = {NAN2, NAN2, NAN2, NAN2};
    for (int i = 0; i < 4; ++i) st128(nanv, mk_rsrc(p.out[0], OUT_BYTES), vwide, rfl(lastf) + i * 4 * 8192);
  }
  if (lane == 0) __hip_atomic_fetch_add(ctl + MB_END, 1u, __ATOMIC_RELAXED, __HIP_MEMORY_SCOPE_WORKGROUP);
}

__device__ __forceinline__ void flag_role(const Params& p, u32 g, int lane, u32 seq, LAS u32* ctl) {
  char* ws = p.ws[0];
  char* rf = p.ws[g ^ PL] + RF_OFF + (g ^ PL) * 8 * FS;
  for (;;) {
    const u32 end = rfl(LD_LDS(ctl + MB_END));
    const u32 a = lane < 16 ? LD_LDS(ctl + MB_RING + lane) : 0;
    if (a) {
      ST_LDS(ctl + MB_RING + lane, 0u);
      if (!(a & 1)) st_sys(ws + a, seq);
      else if (__hip_atomic_fetch_add((GAS u32*)(ws + SEG_OFF + g * 128) + (a >> 1), 1u, __ATOMIC_RELAXED, __HIP_MEMORY_SCOPE_AGENT) == 128 * seq - 1)
        st_sys(rf + (a >> 1) * FS, seq);
    }
    if (end == PULL_WAVES && __ballot(a != 0) == 0) break;
    __builtin_amdgcn_s_sleep(2);
  }
}

__device__ __forceinline__ void fin_role(const Params& p, u32 f, u32 seq) {
  const int tid = threadIdx.x, lane = tid & 63;
  const u32 wid = rfl(tid >> 6), rank = p.rank;
  char* ws = p.ws[0];
  char* tmo = ws + TMO_OFF;
  const auto rl = mk_rsrc(ws, DATA_BYTES);
  const auto ro = mk_rsrc(p.out[0], OUT_BYTES);
  const u32 vwide = (lane >> 4) * 8192 + (lane & 15) * 16;
  char* wsl = p.ws[lane & 7];
  for (u32 n = f; n < 256 + NX; n += 8 * FIN_WGS) {
    const u32 m = n < 256 ? n : xbase(seq) + n - 256;
    u32 ri, ci;
    tile_rc(n & 255, ri, ci);
    const u32 item = wid * 2, dst = fin_base(m < 256 ? rank : rank ^ PL) + (ri * 128 + item * 4) * 8192 + ci * 256;
    bool bad = false;
    wave_wait<SL_FIN>(ws + ((lane & 1) ? FPL_OFF + m * FS : FPI_OFF + (m * 4 + (wid >> 1)) * FS), 3, seq, tmo, bad);
    u32x4 r[2], a[2], b[2];
#pragma unroll
    for (int i = 0; i < 2; ++i) {
      r[i] = __builtin_amdgcn_raw_buffer_load_b128(rl, lane * 16, PIN_OFF + m * 16384 + (item + i) * 1024, AUX_SYS);
      a[i] = __builtin_amdgcn_raw_buffer_load_b128(rl, lane * 16, P32_OFF + m * 32768 + (item + i) * 2048, AUX_SYS);
      b[i] = __builtin_amdgcn_raw_buffer_load_b128(rl, lane * 16, P32_OFF + m * 32768 + (item + i) * 2048 + 1024, AUX_SYS);
    }
    bad |= __hip_atomic_load((const GAS u32*)tmo, __ATOMIC_RELAXED, __HIP_MEMORY_SCOPE_AGENT) == seq;
#pragma unroll
    for (int i = 0; i < 2; ++i) {
      float y[8];
#pragma unroll
      for (int e = 0; e < 4; ++e) {
        const u32 l0 = e < 2 ? a[i][2 * e] : b[i][2 * e - 4], l1 = e < 2 ? a[i][2 * e + 1] : b[i][2 * e - 3];
        y[2 * e] = silu(__uint_as_float(l0) + __uint_as_float(r[i][e] << 16));
        y[2 * e + 1] = silu(__uint_as_float(l1) + __uint_as_float(r[i][e] & 0xFFFF0000u));
      }
      const bf16x8 o = {(__bf16)y[0], (__bf16)y[1], (__bf16)y[2], (__bf16)y[3],
                        (__bf16)y[4], (__bf16)y[5], (__bf16)y[6], (__bf16)y[7]};
      st128(__builtin_bit_cast(u32x4, o) | (bad ? NAN2 : 0u), ro, vwide, dst + i * 4 * 8192);
    }
    WAIT_VM(0);
    __syncthreads();
    if (wid == 0 && m < 256) {
      u32 c = 0;
      if (lane == 0) c = __hip_atomic_fetch_add((GAS u32*)(ws + FC_OFF + (m >> FG) * FS), 1u, __ATOMIC_RELAXED, __HIP_MEMORY_SCOPE_AGENT);
      if (rfl(c) == (seq << FG) - 1 && lane >= 1 && lane < NR) st_sys(wsl + FF_OFF + (rank * 256 + (m >> FG)) * FS, seq);
    }
  }
}

template <int KIND>
__device__ __forceinline__ void gemm_tile(const Params& p, u32 m, u32 hsel, u32 csel, u32 seq, u64 tstart, bool first, LAS char* lds) {
  const int tid = threadIdx.x, lane = tid & 63;
  const u32 wid = rfl(tid >> 6);
  const u32 rank = p.rank;
  char* ws = p.ws[0];
  char* tmo = ws + TMO_OFF;
  const auto rx = mk_rsrc(p.x, X_BYTES);
  const auto rw = mk_rsrc(p.w, W_BYTES);
  const auto rs = mk_rsrc(ws, DATA_BYTES);
  constexpr bool HALF = KIND == 1, QUART = KIND == 2;
  constexpr int KS = QUART ? 2 : 1, NOWN = 8 / KS, NRCV = 24 / KS;
  constexpr int MI = KIND ? 1 : 2, NDMA = HALF ? 3 : 4, NS = HALF ? NS_HALF : 4;
  constexpr u32 XB = HALF ? BLK / 2 : BLK, SB = XB + BLK;
  static_assert(NS * SB <= LDS_CTL, "stages fit");

  u32 vao[2], vbo[2], vaw[2], vbw[2];
#pragma unroll
  for (int q = 0; q < 2; ++q) {
    const u32 ra = wid * (HALF ? 8 : 16) + q * 8 + (lane >> 3), ca = (lane & 7) ^ ((ra >> 1) & 7);
    const u32 kb = wid * 8 + q * 4 + (lane >> 4), cbk = (lane & 15) ^ (((kb & 3) << 2) | ((kb >> 2) & 3));
    const u32 ap = QUART ? ra >> 6 : 0, ar = ra - ap * 64, bp = QUART ? cbk >> 3 : 0, bc = cbk - bp * 8;
    vao[q] = ar * 1024 + ca * 16 + ap * 128 - q * 1024;        vaw[q] = ar * 128 + ca * 16 + ap * 2 * BLK - q * 1024;
    vbo[q] = kb * 8192 + bc * 16 + bp * 64 * 8192 - q * 1024;  vbw[q] = kb * 256 + bc * 16 + bp * 2 * BLK - q * 1024;
  }
  const u32 lds0 = (u32)(size_t)lds;
  const u32 wm = wid & 1, wn = wid >> 1, h = lane >> 5;
  u32 xo[4], wo[2];
  {
    const u32 r = wm * 32 * MI + (lane & 31) + (QUART ? (wid >> 2) * 64 : 0);
#pragma unroll
    for (int ks = 0; ks < 4; ++ks) xo[ks] = r * 128 + (((2 * ks + h) ^ ((r >> 1) & 7)) * 16);
    const u32 bk = (lane >> 4) & 1, q = (lane & 15) >> 2, pp = lane & 3;
#pragma unroll
    for (int t = 0; t < 2; ++t)
      wo[t] = XB + 256 * (8 * h + 4 * t + q) + 16 * ((4 * wn + 2 * bk + (pp >> 1)) ^ ((q << 2) | (2 * h + t))) + 8 * (pp & 1);
  }
  LAS u32* lbad = (LAS u32*)(lds + LDS_CTL);
  u32 ri, ci;
  tile_rc(m, ri, ci);
  const u32 row0 = ((rank >> 1) & 1) * 2048 + ri * 128 + hsel * 64, col0 = (rank & 1) * 2048 + ci * 128 + csel * 64;
  u32 islot = 0, rslot = 0;
  auto next_slot = [&](u32 o) -> u32 { return o + SB == NS * SB ? 0 : o + SB; };
  auto issue_own = [&](int t) {
    const u32 dst = lds0 + islot, sa = row0 * 1024 + t * KS * 128, sw = t * KS * 64 * 8192 + col0 * 2;
    islot = next_slot(islot);
    if (HALF) dma3(dst + wid * 1024, dst + XB + wid * 2048, rx, vao[0], sa, rw, vbo[0], vbo[1], sw);
    else dma4(dst + wid * 2048, rx, vao[0], vao[1], sa, rw, vbo[0], vbo[1], sw);
  };
  u32 taba, tabw;
  {
    const u32 ktl = lane / 6, c = lane - ktl * 6, src = rank ^ (c + 1 + (c >= 3));
    const u32 base = IN_OFF + src * IN_SRC + ktl * KS * 2 * BLK;
    taba = base + ri * 8 * BLK + hsel * 8192;
    tabw = base + (ci * 8 + 1) * BLK + csel * 128;
  }
  auto issue = [&](int t) {
    const u32 dst = lds0 + islot, sa = __builtin_amdgcn_readlane(taba, t), sw = __builtin_amdgcn_readlane(tabw, t);
    islot = next_slot(islot);
    if (HALF) dma3(dst + wid * 1024, dst + XB + wid * 2048, rs, vaw[0], sa, rs, vbw[0], vbw[1], sw);
    else dma4(dst + wid * 2048, rs, vaw[0], vaw[1], sa, rs, vbw[0], vbw[1], sw);
  };
  const u32 fop = (lane >> 1) & 1, fq = min(lane >> 2, 11), fk = fq / 6, fc = fq - fk * 6;
  const char* fin = ws + FI_OFF + ((rank ^ (fc + 1 + (fc >= 3))) * 256 + (((fop ? ci : ri) * 4 + fk) * 2 + fop) * 2 + ((KIND && !fop) ? hsel : (lane & 1))) * FS;

  while (__builtin_amdgcn_s_memrealtime() - tstart < (u64)max(ri, ci) * STAGGER) __builtin_amdgcn_s_sleep(16);
  if (first && wid == 0) __builtin_amdgcn_fence(__ATOMIC_ACQUIRE, "");
  f32x16 acc[MI] = {};
  struct Frag { bf16x8 w[4], x[MI][4]; };
  auto read = [&](Frag& f) {
    const LAS char* st = lds + rslot;
    rslot = next_slot(rslot);
#pragma unroll
    for (int ks = 0; ks < 4; ++ks) {
      const bf16x4 w0 = __builtin_amdgcn_ds_read_tr16_b64_v4bf16((LAS bf16x4*)(st + wo[0] + ks * 4096));
      const bf16x4 w1 = __builtin_amdgcn_ds_read_tr16_b64_v4bf16((LAS bf16x4*)(st + wo[1] + ks * 4096));
      f.w[ks] = __builtin_shufflevector(w0, w1, 0, 1, 2, 3, 4, 5, 6, 7);
#pragma unroll
      for (int mi = 0; mi < MI; ++mi) f.x[mi][ks] = *(const LAS bf16x8*)(st + xo[ks] + mi * 4096);
    }
  };
  auto mma = [&](const Frag& f) {
#pragma unroll
    for (int ks = 0; ks < 4; ++ks)
#pragma unroll
      for (int mi = 0; mi < MI; ++mi) acc[mi] = __builtin_amdgcn_mfma_f32_32x32x16_bf16(f.w[ks], f.x[mi][ks], acc[mi], 0, 0, 0);
  };
  auto step = [&](auto rc, auto oc, int tn, const Frag& fc, Frag& fn) {
    constexpr int R = decltype(rc)::value;
    constexpr bool OWN = decltype(oc)::value;
    if constexpr (R >= 0) {
      WAIT_VM(R * NDMA);
      __builtin_amdgcn_s_waitcnt(0xC07F);
      CBAR();
      __builtin_amdgcn_s_barrier();
      CBAR();
      read(fn);
    }
    if (tn >= 0) { if (OWN) issue_own(tn); else issue(tn); }
    mma(fc);
  };
  Frag fa, fb;
  static_assert(NS % 2 == 0 && NS <= NOWN, "run structure");
  auto run = [&](auto oc, int n) {
    constexpr bool OWN = decltype(oc)::value;
    islot = rslot = 0;
    for (int t = 0; t < NS; ++t) { if (OWN) issue_own(t); else issue(t); }
    WAIT_VM((NS - 1) * NDMA);
    __builtin_amdgcn_s_barrier();
    CBAR();
    read(fa);
    for (int tn = NS; tn < n; tn += 2) {
      step(std::integral_constant<int, NS - 2>{}, oc, tn, fa, fb);
      step(std::integral_constant<int, NS - 2>{}, oc, tn + 1, fb, fa);
    }
    static_for<0, NS / 2>([&](auto ic) {
      constexpr int i = decltype(ic)::value;
      step(std::integral_constant<int, NS - 2 - 2 * i>{}, std::true_type{}, -1, fa, fb);
      step(std::integral_constant<int, NS - 3 - 2 * i>{}, std::true_type{}, -1, fb, fa);
    });
  };
  run(std::true_type{}, NOWN);
  if (wid < 2) {
    bool bad = false;
    wave_wait2<SL_GEMM>(fin + wid * 8 * FS, ~0ull, seq, tmo, bad);
    if (bad) *lbad = 1;
  }
  __builtin_amdgcn_s_waitcnt(0xC07F);
  __builtin_amdgcn_s_barrier();
  run(std::false_type{}, NRCV);
  const u32 tmv = __hip_atomic_load((const GAS u32*)tmo, __ATOMIC_RELAXED, __HIP_MEMORY_SCOPE_AGENT);
  __builtin_amdgcn_s_waitcnt(0xC07F);
  __builtin_amdgcn_s_barrier();
  CBAR();
  constexpr u32 EROW = 528;
#pragma unroll
  for (int mi = 0; mi < MI; ++mi) {
    const u32 mr = wm * 32 * MI + mi * 32 + (lane & 31);
#pragma unroll
    for (int gq = 0; gq < 4; ++gq) {
      const f32x4 o = {acc[mi][4 * gq], acc[mi][4 * gq + 1], acc[mi][4 * gq + 2], acc[mi][4 * gq + 3]};
      *(LAS f32x4*)(lds + mr * EROW + (wn * 32 + 8 * gq + 4 * h) * 4) = o;
    }
  }
  __syncthreads();
  const u32 bad = (*lbad || tmv == seq) ? NAN2 : 0u;
  const u32 keep = rank >> 2;
  const bool do_exp = !KIND || hsel != keep || m < NX, do_keep = !KIND || hsel == keep || m < NX;
  const u32 me = KIND && hsel == keep ? xbase(seq) + m : m, mk = KIND && hsel != keep ? xbase(seq) + m : m;
  if constexpr (QUART) {
    const u32 row = tid >> 3, ch = tid & 7;
    const LAS char* se = lds + row * EROW + ch * 32;
    const f32x4 c0 = *(const LAS f32x4*)se + *(const LAS f32x4*)(se + 256), c1 = *(const LAS f32x4*)(se + 16) + *(const LAS f32x4*)(se + 272);
    GAS u32* qc = (GAS u32*)(ws + QC_OFF + (m * 2 + hsel) * FS);
    if (do_exp) {
      const bf16x8 o = {(__bf16)c0[0], (__bf16)c0[1], (__bf16)c0[2], (__bf16)c0[3],
                        (__bf16)c1[0], (__bf16)c1[1], (__bf16)c1[2], (__bf16)c1[3]};
      st128(__builtin_bit_cast(u32x4, o) | bad, rs, row * 256 + ch * 16, P16_OFF + me * 16384 + csel * 128);
      WAIT_VM(0);
      __builtin_amdgcn_s_barrier();
      if (tid == 0 && __hip_atomic_fetch_add(qc, 1u, __ATOMIC_RELAXED, __HIP_MEMORY_SCOPE_AGENT) == 2 * seq - 1)
        st_sys(p.ws[PL] + FPR_OFF + me * FS, seq);
    }
    if (do_keep) {
      const u32 vo = (row >> 2) * 2048 + (row & 3) * 256 + ch * 16, e32 = P32_OFF + mk * 32768 + csel * 128;
      st128(__builtin_bit_cast(u32x4, c0) | bad, rs, vo, e32);
      st128(__builtin_bit_cast(u32x4, c1) | bad, rs, vo, e32 + 1024);
    }
    WAIT_VM(0);
    __syncthreads();
    if (tid == 64 && do_keep && __hip_atomic_fetch_add(qc + 1, 1u, __ATOMIC_RELAXED, __HIP_MEMORY_SCOPE_AGENT) == 2 * seq - 1)
      st_sys(ws + FPL_OFF + mk * FS, seq);
    return;
  }
  if (do_exp) {
    const u32 e16 = P16_OFF + me * 16384 + wid * 1024;
#pragma unroll
    for (int part = 0; part < 2; ++part) {
      const LAS char* se = lds + ((HALF ? 0 : (keep ^ 1) * 64) + part * 32 + (tid >> 4)) * EROW + (tid & 15) * 32;
      const f32x4 c0 = *(const LAS f32x4*)se, c1 = *(const LAS f32x4*)(se + 16);
      const bf16x8 o = {(__bf16)c0[0], (__bf16)c0[1], (__bf16)c0[2], (__bf16)c0[3],
                        (__bf16)c1[0], (__bf16)c1[1], (__bf16)c1[2], (__bf16)c1[3]};
      st128(__builtin_bit_cast(u32x4, o) | bad, rs, lane * 16, e16 + part * 8192);
    }
  }
  if (HALF && do_exp && do_keep) {
    WAIT_VM(0);
    __builtin_amdgcn_s_barrier();
    if (tid == 0) st_sys(p.ws[PL] + FPR_OFF + me * FS, seq);
  }
  if (do_keep) {
    const u32 e32 = P32_OFF + mk * 32768 + wid * 2048;
#pragma unroll
    for (int part = 0; part < 2; ++part) {
      const LAS char* sk = lds + ((HALF ? 0 : keep * 64) + part * 32 + (tid >> 4)) * EROW + (tid & 15) * 32;
      const u32x4 c0 = *(const LAS u32x4*)sk, c1 = *(const LAS u32x4*)(sk + 16);
      st128(c0 | bad, rs, lane * 16, e32 + part * 16384);
      st128(c1 | bad, rs, lane * 16, e32 + part * 16384 + 1024);
    }
  }
  WAIT_VM(0);
  __syncthreads();
  if (tid == 0 && do_exp && !(HALF && do_keep)) st_sys(p.ws[PL] + FPR_OFF + me * FS, seq);
  if (tid == 64 && do_keep) st_sys(ws + FPL_OFF + mk * FS, seq);
}

__device__ __forceinline__ void stage_unit(const Params& p, u32 lk, u32 seq) {
  const int tid = threadIdx.x, lane = tid & 63;
  const u32 wid = rfl(tid >> 6);
  const u32 lev = lk >> 2, kt = (wid >> 2) * 4 + (lk & 3), op = (wid >> 1) & 1, rc = (wid & 1) * 2048 + lev * 128;
  const auto r = mk_rsrc(op ? p.w : p.x, X_BYTES);
  const auto rs = mk_rsrc(p.ws[0], DATA_BYTES);
  const u32 vo = op ? (lane >> 4) * 8192 + (lane & 15) * 16 : (lane >> 3) * 1024 + (lane & 7) * 16;
  const u32 so = op ? kt * 64 * 8192 + rc * 2 : rc * 1024 + kt * 128;
  u32x4 v[16];
#pragma unroll
  for (int i = 0; i < 16; ++i) v[i] = __builtin_amdgcn_raw_buffer_load_b128(r, vo, so + i * (op ? 4 * 8192 : 8 * 1024), 0);
#pragma unroll
  for (int i = 0; i < 16; ++i) st128(v[i], rs, lane * 16, EX_OFF + (lk * 8 + wid) * BLK + i * 1024);
  WAIT_VM(0);
  __syncthreads();
  if (tid >= 1 && tid < NR && tid != PL) st_sys(p.ws[tid] + SG_OFF + (p.rank * 64 + lk) * FS, seq);
}

__device__ __forceinline__ void gemm_role(const Params& p, u32 k, u32 seq, LAS char* lds) {
  const u64 tstart = __builtin_amdgcn_s_memrealtime();
  constexpr u32 NG = 8 * GEMM_WGS, NSMALL = 2 * NQUART + 2 * NHALF;
  static_assert(NQUART <= NHALF && NQUART <= 64, "unit schedule");
  for (u32 n = k; n < 256 - NHALF + NSMALL; n += NG) {
    if (n < 4 * NQUART) gemm_tile<2>(p, n >> 2, (n >> 1) & 1, n & 1, seq, tstart, n < NG, lds);
    else if (n < NSMALL) gemm_tile<1>(p, (n >> 1) - NQUART, n & 1, 0, seq, tstart, n < NG, lds);
    else gemm_tile<0>(p, n - NSMALL + NHALF, 0, 0, seq, tstart, n < NG, lds);
  }
}

__global__ void __launch_bounds__(THREADS, 2) fused_gemm_allreduce_silu(Params p) {
  extern __shared__ __attribute__((aligned(16))) char smem[];
  const int lane = threadIdx.x & 63;
  const u32 wid = rfl(threadIdx.x >> 6);
  LAS u32* ctl = (LAS u32*)smem;
  LAS u32* lctl = (LAS u32*)(smem + LDS_CTL);
  u32 slot = (blockIdx.x & ~7u) | ((u32)__builtin_amdgcn_s_getreg((3 << 11) | 20) & 7);
  auto is_push = [&]() { return slot < 8 * ENG_WGS && (slot & 3) != 0 && wid < PUSH_WAVES; };
  if (__builtin_expect(is_push(), 1)) {
    if (push_role<false>(p, slot, wid, lane, lctl)) return;
  } else {
    if (wid == 7) {
      if (lane < 32) ctl[lane] = 0;
      if (lane == 0) {
        GAS u32* ctr = (GAS u32*)(p.ws[0] + CTR_OFF) + blockIdx.x;
        GAS u32* clm = (GAS u32*)(p.ws[0] + CLM_OFF);
        const u32 c = __hip_atomic_load(ctr, __ATOMIC_RELAXED, __HIP_MEMORY_SCOPE_AGENT);
        u32 sl = slot;
#pragma unroll 1
        for (int n = 0; n < 4 * GRID; ++n) {
          if (__hip_atomic_fetch_add(clm + sl * 16, 1u, __ATOMIC_RELAXED, __HIP_MEMORY_SCOPE_AGENT) == c) break;
          __hip_atomic_fetch_add(clm + sl * 16, ~0u, __ATOMIC_RELAXED, __HIP_MEMORY_SCOPE_AGENT);
          sl = (sl + 1) & (GRID - 1);
        }
        lctl[0] = 0; lctl[1] = sl; lctl[2] = c + 1;
      }
    }
    asm volatile("s_waitcnt lgkmcnt(0)" ::: "memory");
    __builtin_amdgcn_s_barrier();
    slot = rfl(LD_LDS(lctl + 1));
    if (threadIdx.x == THREADS - 1)
      __hip_atomic_store((GAS u32*)(p.ws[0] + CTR_OFF) + blockIdx.x, LD_LDS(lctl + 2), __ATOMIC_RELAXED, __HIP_MEMORY_SCOPE_AGENT);
    WAIT_VM(0);
  }
  if (__builtin_expect(is_push(), 0)) {
    push_role<true>(p, slot, wid, lane, lctl);
    return;
  }
  const u32 seq = rfl(LD_LDS(lctl + 2));
  const u32 g = slot & 7, s = slot >> 3;
  const u32 peer = ((u32)p.rank ^ g) & (NR - 1);
  constexpr u32 LK0 = NPUSH / 4;
  if (s < ENG_WGS) {
    if (g == 0) {
      for (u32 u = s; u < 64 - LK0; u += u < 16 ? 16 : u < 24 ? 16 : 8) stage_unit(p, LK0 + u, seq);
    } else if (wid >= PUSH_WAVES && wid < PUSH_WAVES + PULL_WAVES) {
      pull_role(p, peer, (wid - PUSH_WAVES) * ENG_WGS + s, lane, seq, ctl);
    } else if (wid == PUSH_WAVES + PULL_WAVES) {
      flag_role(p, g, lane, seq, ctl);
    }
  } else if (s < ENG_WGS + FIN_WGS) {
    if ((g & 3) == 0) { stage_unit(p, LK0 + 8 + g + s - ENG_WGS, seq); stage_unit(p, LK0 + 24 + g + s - ENG_WGS, seq); }
    fin_role(p, (s - ENG_WGS) * 8 + g, seq);
  } else {
    const u32 k = s - ENG_WGS - FIN_WGS;
    gemm_role(p, (k >> 2) * 32 + g * 4 + (k & 3), seq, (LAS char*)smem);
  }
}

}

extern "C" void kernel_launch(int rank, int nranks, ncclComm_t comm,
                              hipStream_t stream,
                              void* const* d_in, const int* in_sizes, int n_in,
                              void* d_out, int out_size,
                              const int* in_dtypes, int out_dtype,
                              void* const* d_out_all,
                              void* const* d_ws_all, size_t ws_bytes) {
  if (nranks != NR || (unsigned)rank >= (unsigned)NR || n_in < 2 || ws_bytes < WS_USED ||
      in_sizes[0] != 4096 * 512 || in_sizes[1] != 512 * 4096 || out_size != 4096 * 4096) {
    fprintf(stderr, "kernel_launch: unexpected problem shape\n");
    return;
  }
  HIP_CHECK(hipFuncSetAttribute((const void*)fused_gemm_allreduce_silu,
                                hipFuncAttributeMaxDynamicSharedMemorySize, LDS_BYTES));
  Params p;
  p.x = (const char*)d_in[0];
  p.w = (const char*)d_in[1];
  const int phys[NR] = {0, 2, 3, 6, 1, 4, 5, 7};
  p.rank = 0; p.pad = 0;
  for (int i = 0; i < NR; ++i) if (phys[i] == rank) p.rank = i;
  for (int i = 0; i < NR; ++i) { p.ws[i] = (char*)d_ws_all[phys[p.rank ^ i]]; p.out[i] = (char*)d_out_all[phys[p.rank ^ i]]; }
  for (u32 g = 0; g < NR; ++g)
    for (u32 job = 0; job < 16; ++job) {
      const u32 peer = p.rank ^ g, lev = job >> 4, kt = (peer >> 2) * 4 + ((job >> 2) & 3), op = (job >> 1) & 1, half = job & 1;
      p.psrc[g][job] = op ? (kt * 64 + half * 32) * 8192 + ((peer & 1) * 2048 + lev * 128) * 2
                                         : (((peer >> 1) & 1) * 2048 + lev * 128 + half * 64) * 1024 + kt * 128;
    }
  hipLaunchKernelGGL(fused_gemm_allreduce_silu, dim3(GRID), dim3(THREADS), LDS_BYTES, stream, p);
  HIP_CHECK(hipGetLastError());
}
```
